```python
import numpy as np

kernel_name = "gpu_gemm_a2a_m4096_k4096_n16384_bf16_none"


ABSMAX_RTOL = 0.008

COMPUTE_DTYPE = "bfloat16"
EPILOGUE = "none"
IN_QUANT = None
OUT_QUANT = None
IN_DTYPES = {"x": "bfloat16", "w_mat": "bfloat16"}
WORLD_SIZE = 8

def _to_bf16(x):
    u = np.ascontiguousarray(x, dtype=np.float32).view(np.uint32)
    return ((u + 0x7FFF + ((u >> 16) & 1)) & 0xFFFF0000).view(np.float32)

def setup_inputs():

    x = _to_bf16(np.random.randn(4096, 4096).astype(np.float32))
    w_mat = _to_bf16((np.random.randn(4096, 16384) / np.sqrt(4096)).astype(np.float32))
    return {"x": x, "w_mat": w_mat}

def _epilogue(y):
    return y

def reference(x, w_mat):
    y = (x @ w_mat).astype(np.float32)
    return _epilogue(y)
```

```cpp
#include <hip/hip_runtime.h>
#include <rccl/rccl.h>
#include <cstdio>
#include <stdint.h>

#define HIP_CHECK(expr) do { hipError_t _e = (expr); if (_e != hipSuccess) { \
    fprintf(stderr, "HIP error %s at %s:%d\n", hipGetErrorString(_e), __FILE__, __LINE__); \
    return; } } while (0)

#define LAS __attribute__((address_space(3)))
#define GAS __attribute__((address_space(1)))
typedef short s16x4 __attribute__((ext_vector_type(4)));
typedef __bf16 bf16x2 __attribute__((ext_vector_type(2)));
typedef __bf16 bf16x8 __attribute__((ext_vector_type(8)));
typedef float f32x4 __attribute__((ext_vector_type(4)));
typedef float f32x2 __attribute__((ext_vector_type(2)));
typedef unsigned u32x4 __attribute__((ext_vector_type(4)));

constexpr int NR = 8, ML = 512, KD = 4096, ND = 16384, NCOL = ND / NR;
constexpr int BM = 256, BN = 128, BK = 64, NT = KD / BK;
constexpr int XB = BM * BK * 2;
constexpr int WB = BK * BN * 2;
constexpr int STG = XB + WB;
constexpr int NSTG = 3;
constexpr int LDS_BYTES = NSTG * STG + 64;
constexpr int NBLK = 256;
constexpr unsigned long long DEADLINE = 50000000ull;
constexpr int WS_TICKET = 0, WS_TMO = 32, WS_READY = 64  , WS_DONE = 1024  ;

struct Params {
  const unsigned short* x;
  const unsigned short* w;
  unsigned short* out[NR];
  unsigned* ws[NR];
  int rank;
};

#define WAITV(n) asm volatile("s_waitcnt vmcnt(" #n ")" ::: "memory")
#define WAITL(n) asm volatile("s_waitcnt lgkmcnt(" #n ")" ::: "memory")
#define BAR() __builtin_amdgcn_s_barrier()
#define SCHED() __builtin_amdgcn_sched_barrier(0)

__device__ __forceinline__ bf16x8 ldx(LAS const char* p) { return *(LAS const bf16x8*)p; }
__device__ __forceinline__ s16x4 ldw(LAS const char* p) { return __builtin_amdgcn_ds_read_tr16_b64_v4i16((LAS s16x4*)p); }
__device__ __forceinline__ unsigned pk(float a, float b) { f32x2 v = {a, b}; return __builtin_bit_cast(unsigned, __builtin_convertvector(v, bf16x2)); }
__device__ __forceinline__ unsigned ld_sys(const unsigned* p) { return __hip_atomic_load(p, __ATOMIC_RELAXED, __HIP_MEMORY_SCOPE_SYSTEM); }
__device__ __forceinline__ unsigned ld_agent(const unsigned* p) { return __hip_atomic_load(p, __ATOMIC_RELAXED, __HIP_MEMORY_SCOPE_AGENT); }
__device__ __forceinline__ void st_sys(unsigned* p, unsigned v) { __hip_atomic_store(p, v, __ATOMIC_RELAXED, __HIP_MEMORY_SCOPE_SYSTEM); }
__device__ __forceinline__ void st_agent(unsigned* p, unsigned v) { __hip_atomic_store(p, v, __ATOMIC_RELAXED, __HIP_MEMORY_SCOPE_AGENT); }

__global__ __launch_bounds__(512, 2) void gemm_a2a(Params p) {
  extern __shared__ __attribute__((aligned(16))) char smem[];
  LAS char* lds = (LAS char*)smem;
  const int tid = threadIdx.x;
  const int wid = __builtin_amdgcn_readfirstlane(tid >> 6);
  const int lane = tid & 63;
  const int b = blockIdx.x;
  const int xcd = b & 7, idx = b >> 3;
  const int mt = idx & 1, ntl = idx >> 1;
  const int dest = (p.rank + xcd) & 7;
  const int ncol0 = dest * NCOL + ntl * BN;
  unsigned* myws = p.ws[p.rank];
  LAS volatile unsigned* lflag = (LAS volatile unsigned*)(lds + NSTG * STG);
  if (wid >= 4) {
    const int h = wid - 4;
    unsigned xoff[2], woff[4];
#pragma unroll
    for (int par = 0; par < 2; ++par) {
      const int r = par * 8 + (lane >> 3), c = (lane & 7) ^ ((r >> 1) & 7);
      xoff[par] = (unsigned)(r * KD * 2 + c * 16);
    }
#pragma unroll
    for (int i = 0; i < 4; ++i) {
      const int k = (h * 4 + i) * 4 + (lane >> 4);
      const int sw = (k & 3) | (((k >> 3) & 1) << 2);
      const int c = (lane & 15) ^ (sw << 1);
      woff[i] = (unsigned)(k * ND * 2 + c * 16);
    }
    const char* xg = (const char*)p.x + (size_t)(mt * BM + h * 64) * KD * 2;
    const char* wg = (const char*)p.w + (size_t)ncol0 * 2;
    auto issue = [&](int t, int stg) {
      const char* xs = xg + (size_t)t * (BK * 2);
      const char* ws = wg + (size_t)t * ((size_t)BK * ND * 2);
      LAS char* xl = lds + stg * STG + h * 8192;
      LAS char* wl = lds + stg * STG + XB + h * 4096;
#pragma unroll
      for (int i = 0; i < 4; ++i)
        __builtin_amdgcn_global_load_lds((const GAS unsigned*)(ws + woff[i]), (LAS unsigned*)(wl + i * 1024), 16, 0, 0);
#pragma unroll
      for (int i = 0; i < 8; ++i)
        __builtin_amdgcn_global_load_lds((const GAS unsigned*)(xs + (size_t)(i >> 1) * (16 * KD * 2) + xoff[i & 1]), (LAS unsigned*)(xl + i * 1024), 16, 0, 0);
    };
    issue(0, 0);
    issue(1, 1);
    WAITV(12);
    BAR();
    int stg = 2;
    for (int t = 0; t < NT - 2; ++t) {
      issue(t + 2, stg);
      stg = (stg == 2) ? 0 : stg + 1;
      WAITV(12);
      BAR();
    }
    WAITV(0);
    unsigned call = 0, ticket = 0;
    if (h == 3) {
      if (lane == 0) ticket = __hip_atomic_fetch_add(myws + WS_TICKET, 1u, __ATOMIC_RELAXED, __HIP_MEMORY_SCOPE_AGENT);
      ticket = __builtin_amdgcn_readfirstlane(ticket);
      call = ticket / NBLK + 1u;
      if (ticket % NBLK == 0 && lane < NR && lane != p.rank) st_sys(p.ws[lane] + WS_READY + p.rank * 32, call);
    }
    BAR();
    if (h == 3) {
      unsigned bad = 0;
      if (dest != p.rank) {
        const unsigned long long t0 = __builtin_amdgcn_s_memrealtime();
        while (ld_sys(myws + WS_READY + dest * 32) < call) {
          if (ld_agent(myws + WS_TMO) == call || __builtin_amdgcn_s_memrealtime() - t0 > DEADLINE) { st_agent(myws + WS_TMO, call); break; }
          __builtin_amdgcn_s_sleep(2);
        }
      }
      bad = (ld_agent(myws + WS_TMO) == call) ? 1u : 0u;
      if (lane == 0) lflag[0] = bad;
      WAITL(0);
    }
    BAR();
    BAR();
    if (h == 3) {
      if (dest != p.rank && lane == 0) st_sys(p.ws[dest] + WS_DONE + p.rank * 32 + idx, call);
      if (ticket % NBLK == NBLK - 1) {
        const unsigned long long t0 = __builtin_amdgcn_s_memrealtime();
        const bool mine = (lane >> 3) == p.rank;
        bool timed_out = false;
        for (;;) {
          bool ok = true;
#pragma unroll
          for (int j = 0; j < 4; ++j) ok &= ld_sys(myws + WS_DONE + lane * 4 + j) >= call;
          if (__all(ok || mine)) break;
          if (ld_agent(myws + WS_TMO) == call || __builtin_amdgcn_s_memrealtime() - t0 > DEADLINE) { st_agent(myws + WS_TMO, call); timed_out = true; break; }
          __builtin_amdgcn_s_sleep(1);
        }
        __builtin_amdgcn_fence(__ATOMIC_ACQUIRE, "");
        if (timed_out && lane == 0) {
          p.out[dest][(size_t)(p.rank * ML + mt * BM) * NCOL + ntl * BN] = 0x7fc0;
          p.out[p.rank][(size_t)((p.rank ^ 1) * ML) * NCOL] = 0x7fc0;
        }
      }
    }
  } else {
    const int wr = wid >> 1, wc = wid & 1;
    const int fr = lane & 15, fq = lane >> 4;
    const int q = (lane & 15) >> 2, pp = lane & 3;
    const unsigned xa0 = (unsigned)((wr * 128 + fr) * 128 + ((fq ^ ((fr >> 1) & 7)) * 16));
    const unsigned xa1 = xa0 ^ 64u;
    unsigned wa[4];
#pragma unroll
    for (int nf = 0; nf < 4; ++nf)
      wa[nf] = (unsigned)(XB + (8 * fq + q) * 256 + ((((wc ^ (fq & 1)) << 2) | (nf ^ q)) * 32) + 8 * pp);
    f32x4 acc[8][4];
#pragma unroll
    for (int i = 0; i < 8; ++i)
#pragma unroll
      for (int j = 0; j < 4; ++j) acc[i][j] = (f32x4){0.f, 0.f, 0.f, 0.f};
    bf16x8 Xf[8];
    s16x4 Wl[2][4], Wh[2][4];
#define WFRAG(bu, nf) __builtin_bit_cast(bf16x8, __builtin_shufflevector(Wl[bu][nf], Wh[bu][nf], 0, 1, 2, 3, 4, 5, 6, 7))
#define RDW(bu, sb, ks, i) do { if ((i) & 1) Wh[bu][(i) >> 1] = ldw((sb) + wa[(i) >> 1] + (ks) * 8192 + 1024); \
      else Wl[bu][(i) >> 1] = ldw((sb) + wa[(i) >> 1] + (ks) * 8192); } while (0)
#define MMA1(mf, bu, nf) do { acc[mf][nf] = __builtin_amdgcn_mfma_f32_16x16x32_bf16(WFRAG(bu, nf), Xf[mf], acc[mf][nf], 0, 0, 0); SCHED(); } while (0)
#define NOP_ do {} while (0)
#define GRP(mf, bu, F0, F1, F2) do { MMA1(mf, bu, 0); F0; SCHED(); MMA1(mf, bu, 1); F1; SCHED(); MMA1(mf, bu, 2); F2; SCHED(); MMA1(mf, bu, 3); } while (0)
    BAR();
    {
      LAS const char* sb = lds;
#pragma unroll
      for (int i = 0; i < 8; ++i) RDW(0, sb, 0, i);
#pragma unroll
      for (int mf = 0; mf < 7; ++mf) Xf[mf] = ldx(sb + xa0 + mf * 2048);
    }
    int stg = 0;
    for (int t = 0; t < NT; ++t) {
      LAS const char* sb = lds + stg * STG;
      stg = (stg == 2) ? 0 : stg + 1;
      LAS const char* sn = lds + stg * STG;
      GRP(0, 0, Xf[7] = ldx(sb + xa0 + 7 * 2048), RDW(1, sb, 1, 0), RDW(1, sb, 1, 1));
      GRP(1, 0, Xf[0] = ldx(sb + xa1 + 0 * 2048), RDW(1, sb, 1, 2), RDW(1, sb, 1, 3));
      GRP(2, 0, Xf[1] = ldx(sb + xa1 + 1 * 2048), RDW(1, sb, 1, 4), NOP_);
      GRP(3, 0, Xf[2] = ldx(sb + xa1 + 2 * 2048), RDW(1, sb, 1, 5), NOP_);
      GRP(4, 0, Xf[3] = ldx(sb + xa1 + 3 * 2048), RDW(1, sb, 1, 6), NOP_);
      GRP(5, 0, Xf[4] = ldx(sb + xa1 + 4 * 2048), RDW(1, sb, 1, 7), NOP_);
      GRP(6, 0, Xf[5] = ldx(sb + xa1 + 5 * 2048), NOP_, NOP_);
      GRP(7, 0, Xf[6] = ldx(sb + xa1 + 6 * 2048), NOP_, NOP_);
      GRP(0, 1, Xf[7] = ldx(sb + xa1 + 7 * 2048), NOP_, NOP_);
      GRP(1, 1, NOP_, NOP_, NOP_);
      WAITL(0);
      BAR();
      SCHED();
      GRP(2, 1, RDW(0, sn, 0, 0), RDW(0, sn, 0, 1), Xf[0] = ldx(sn + xa0 + 0 * 2048));
      GRP(3, 1, RDW(0, sn, 0, 2), RDW(0, sn, 0, 3), Xf[1] = ldx(sn + xa0 + 1 * 2048));
      GRP(4, 1, RDW(0, sn, 0, 4), RDW(0, sn, 0, 5), Xf[2] = ldx(sn + xa0 + 2 * 2048));
      GRP(5, 1, RDW(0, sn, 0, 6), RDW(0, sn, 0, 7), Xf[3] = ldx(sn + xa0 + 3 * 2048));
      GRP(6, 1, Xf[4] = ldx(sn + xa0 + 4 * 2048), Xf[5] = ldx(sn + xa0 + 5 * 2048), NOP_);
      GRP(7, 1, Xf[6] = ldx(sn + xa0 + 6 * 2048), NOP_, NOP_);
    }
    const bool bad = lflag[0] != 0u;
    auto ro = __builtin_amdgcn_make_buffer_rsrc((void*)p.out[dest], 0, ML * NR * NCOL * 2, 0x00020000);
    const unsigned vo = (unsigned)(((wr * 128 + fr) * NCOL + wc * 64 + (fq & 1) * 16 + (fq >> 1) * 8) * 2);
    const unsigned so = (unsigned)(((p.rank * ML + mt * BM) * NCOL + ntl * BN) * 2);
#pragma unroll
    for (int mf = 0; mf < 8; ++mf) {
#pragma unroll
      for (int np = 0; np < 2; ++np) {
        f32x4 a = acc[mf][2 * np], c = acc[mf][2 * np + 1];
        unsigned a0 = pk(a[0], a[1]), a1 = pk(a[2], a[3]), c0 = pk(c[0], c[1]), c1 = pk(c[2], c[3]);
        auto r0 = __builtin_amdgcn_permlane16_swap(a0, c0, false, false);
        auto r1 = __builtin_amdgcn_permlane16_swap(a1, c1, false, false);
        u32x4 v = {r0[0], r1[0], r0[1], r1[1]};
        if (bad) v = (u32x4){0x7fc07fc0u, 0x7fc07fc0u, 0x7fc07fc0u, 0x7fc07fc0u};
        __builtin_amdgcn_raw_buffer_store_b128(v, ro, vo + np * 64, so + mf * 16 * NCOL * 2, 17);
      }
    }
    WAITV(0);
    BAR();
  }
}

extern "C" void kernel_launch(int rank, int nranks, ncclComm_t comm,
                              hipStream_t stream,
                              void* const* d_in, const int* in_sizes, int n_in,
                              void* d_out, int out_size,
                              const int* in_dtypes, int out_dtype,
                              void* const* d_out_all,
                              void* const* d_ws_all, size_t ws_bytes) {
  if (nranks != NR || (unsigned)rank >= (unsigned)NR) { fprintf(stderr, "unexpected world size\n"); return; }
  Params p;
  p.x = (const unsigned short*)d_in[0];
  p.w = (const unsigned short*)d_in[1];
  for (int i = 0; i < NR; ++i) { p.out[i] = (unsigned short*)d_out_all[i]; p.ws[i] = (unsigned*)d_ws_all[i]; }
  p.rank = rank;
  static bool init = false;
  if (!init) {
    HIP_CHECK(hipFuncSetAttribute((const void*)gemm_a2a, hipFuncAttributeMaxDynamicSharedMemorySize, LDS_BYTES));
    init = true;
  }
  gemm_a2a<<<NBLK, 512, LDS_BYTES, stream>>>(p);
  HIP_CHECK(hipGetLastError());
}
```

```cpp
#include <hip/hip_runtime.h>
#include <rccl/rccl.h>
#include <cstdio>
#include <stdint.h>

#define HIP_CHECK(expr) do { hipError_t _e = (expr); if (_e != hipSuccess) { \
    fprintf(stderr, "HIP error %s at %s:%d\n", hipGetErrorString(_e), __FILE__, __LINE__); \
    return; } } while (0)

#define LAS __attribute__((address_space(3)))
#define GAS __attribute__((address_space(1)))
typedef short s16x4 __attribute__((ext_vector_type(4)));
typedef __bf16 bf16x2 __attribute__((ext_vector_type(2)));
typedef __bf16 bf16x8 __attribute__((ext_vector_type(8)));
typedef float f32x4 __attribute__((ext_vector_type(4)));
typedef float f32x2 __attribute__((ext_vector_type(2)));
typedef unsigned u32x4 __attribute__((ext_vector_type(4)));

constexpr int NR = 8, ML = 512, KD = 4096, ND = 16384, NCOL = ND / NR;
constexpr int BN = 128, BK = 64, NT = KD / BK;
constexpr int MFA = 8, MFB = 9;
constexpr int AROWS = MFA * 32, BROWS = ML - AROWS;
constexpr int XROWS = 7 * BROWS;
constexpr int WB = BK * BN * 2;
constexpr int NSTG = 3;
constexpr int LDS_CTL = NSTG * (WB + MFB * 4096);
constexpr int C_GO = LDS_CTL;
constexpr int C_CALL = LDS_CTL + 16;
constexpr int LDS_BYTES = LDS_CTL + 64;
static_assert(LDS_BYTES <= 163840, "LDS");
constexpr int NBLK = 256;
constexpr unsigned long long DEADLINE = 50000000ull;
constexpr int NCK = 21;
constexpr int WS_TMO = 32, WS_READY = 64  , WS_CNT = 512  , WS_DONE = 1024  ;
constexpr int WS_XFLAG = 2048;
constexpr int WS_XCNT = 4096;
constexpr size_t WS_XBUF = 1u << 20;
constexpr size_t XT_BYTES = (size_t)XROWS * 128;
constexpr size_t XPAR_BYTES = XT_BYTES * NT;
constexpr size_t WS_NEED = WS_XBUF + 2 * XPAR_BYTES;
#define POLL_MARGIN 4
#define TILE_NS 620
#define PUSH_LEAD 3
#define KSKEW 8
#define AMAP 1
#define ACK_NS 800
#define PWIN 2
#define AWIN 2
#define WAUX 2
#define WAUXB 0

struct Params {
  const unsigned short* x;
  const unsigned short* w;
  unsigned short* out[NR];
  unsigned* ws[NR];
  int rank;
};

#define WAITL(n) asm volatile("s_waitcnt lgkmcnt(" #n ")" ::: "memory")
template <int N> __device__ __forceinline__ void waitv() { asm volatile("s_waitcnt vmcnt(%0)" :: "n"(N) : "memory"); }
#define BAR() __builtin_amdgcn_s_barrier()
#define SCHED() __builtin_amdgcn_sched_barrier(0)
#define CFENCE() asm volatile("" ::: "memory")
#define STAMP(i) do {} while (0)

__device__ __forceinline__ bf16x8 ldx(LAS const char* p) { return *(LAS const bf16x8*)p; }
__device__ __forceinline__ s16x4 ldw(LAS const char* p) { return __builtin_amdgcn_ds_read_tr16_b64_v4i16((LAS s16x4*)p); }
__device__ __forceinline__ unsigned pk(float a, float b) { f32x2 v = {a, b}; return __builtin_bit_cast(unsigned, __builtin_convertvector(v, bf16x2)); }
__device__ __forceinline__ unsigned ld_sys(const unsigned* p) { return __hip_atomic_load(p, __ATOMIC_RELAXED, __HIP_MEMORY_SCOPE_SYSTEM); }
__device__ __forceinline__ unsigned ld_agent(const unsigned* p) { return __hip_atomic_load(p, __ATOMIC_RELAXED, __HIP_MEMORY_SCOPE_AGENT); }
__device__ __forceinline__ void st_sys(unsigned* p, unsigned v) { __hip_atomic_store(p, v, __ATOMIC_RELAXED, __HIP_MEMORY_SCOPE_SYSTEM); }
__device__ __forceinline__ void st_agent(unsigned* p, unsigned v) { __hip_atomic_store(p, v, __ATOMIC_RELAXED, __HIP_MEMORY_SCOPE_AGENT); }
__device__ __forceinline__ unsigned lds_rd(LAS char* lds, int off) { return *(LAS volatile unsigned*)(lds + off); }
__device__ __forceinline__ void lds_wr(LAS char* lds, int off, unsigned v) { *(LAS volatile unsigned*)(lds + off) = v; }
__device__ __forceinline__ int ck_end(int c) { return c < 4 ? c + 1 : (c < 8 ? 2 * c - 2 : 4 * c - 16); }
__device__ __forceinline__ int ck_of(int t) { return t < 4 ? t : (t < 12 ? (t >> 1) + 2 : (t >> 2) + 5); }
__device__ __forceinline__ void lds_wait(LAS char* lds, int off, unsigned v) {
  while (!__all(lds_rd(lds, off) >= v)) __builtin_amdgcn_s_sleep(1);
  CFENCE();
}

struct Tile {
  int kind;
  int peer;
  int panel;
  int xrow0;
  int orow0;
  int owner;
  int lrow0;
  bool gathered;
  int koff;
};

template <int MF, int H, int NWS, int AUXW>
__device__ __forceinline__ void loader(const Params& p, const Tile& tl, LAS char* lds, int lane) {
  constexpr int SLOT = WB + MF * 4096;
  constexpr int NW_ = (16 - H + 2) / 3;
  constexpr int X0 = (H + 3 * NW_) - 16;
  constexpr int NX_ = (4 * MF - X0 + 2) / 3;
  constexpr int NPER = NW_ + NX_;
  constexpr bool XWS = NWS > 0;
  static_assert(NPER <= 31, "");
  const auto rw = __builtin_amdgcn_make_buffer_rsrc((void*)p.w, 0, KD * ND * 2, 0x00020000);
  const auto rx = __builtin_amdgcn_make_buffer_rsrc((void*)p.x, 0, ML * KD * 2, 0x00020000);
  unsigned wv[2], xv[2];
#pragma unroll
  for (int v = 0; v < 2; ++v) {
    const int sw = (lane >> 4) | (v << 2);
    wv[v] = (unsigned)((lane >> 4) * ND * 2 + (((lane & 15) ^ (sw << 1)) * 16));
    const int r = v * 8 + (lane >> 3);
    xv[v] = (unsigned)((lane >> 3) * KD * 2 + (((lane & 7) ^ ((r >> 1) & 7)) * 16));
  }
  const unsigned iv = (unsigned)(lane * 16);
  const unsigned wbase = (unsigned)((tl.owner * NCOL + tl.panel * BN) * 2);
  const unsigned xbase = (unsigned)((tl.lrow0 - NWS * 8) * KD * 2);
  auto issue_w = [&](int t, int stg) {
#pragma unroll
    for (int i = 0; i < NW_; ++i) {
      const int pc = H + 3 * i;
      __builtin_amdgcn_raw_ptr_buffer_load_lds(rw, (LAS void*)(lds + stg * SLOT + pc * 1024), 16, wv[(pc >> 1) & 1],
                                               wbase + (unsigned)((t + tl.koff) & (NT - 1)) * (BK * ND * 2) + pc * (4 * ND * 2), 0, AUXW);
    }
  };
  auto issue_xl = [&](int t, int stg) {
#pragma unroll
    for (int i = 0; i < NX_; ++i) {
      const int pc = X0 + 3 * i;
      if (pc >= NWS)
        __builtin_amdgcn_raw_ptr_buffer_load_lds(rx, (LAS void*)(lds + stg * SLOT + WB + pc * 1024), 16, xv[pc & 1],
                                                 xbase + (unsigned)((t + tl.koff) & (NT - 1)) * 128u + pc * (8 * KD * 2), 0, 0);
    }
  };
  issue_w(0, 0);
  issue_xl(0, 0);
  unsigned call = 0;
  if (XWS) {
    BAR();
    call = (unsigned)__builtin_amdgcn_readfirstlane((int)lds_rd(lds, C_CALL));
  }
  const auto ri = __builtin_amdgcn_make_buffer_rsrc((char*)p.ws[p.rank] + WS_XBUF + (call & 1) * XPAR_BYTES, 0, (int)XPAR_BYTES, 0x00020000);
  const unsigned ibase = (unsigned)(tl.xrow0 * 128);
  auto issue_xi = [&](int t, int stg) {
#pragma unroll
    for (int i = 0; i < NX_; ++i) {
      const int pc = X0 + 3 * i;
      if (pc < NWS)
        __builtin_amdgcn_raw_ptr_buffer_load_lds(ri, (LAS void*)(lds + stg * SLOT + WB + pc * 1024), 16, iv,
                                                 ibase + (unsigned)t * (unsigned)XT_BYTES + pc * 1024, 0, 0);
    }
  };
  issue_xi(0, 0);
  issue_w(1, 1);
  issue_xl(1, 1);
  issue_xi(1, 1);
  waitv<NPER>();
  BAR();
  int stg = 2;
  for (int t = 0; t < NT - 2; ++t) {
    issue_w(t + 2, stg);
    issue_xl(t + 2, stg);
    issue_xi(t + 2, stg);
    stg = (stg == NSTG - 1) ? 0 : stg + 1;
    waitv<NPER>();
    BAR();
  }
  waitv<0>();
  BAR();
  BAR();
}

template <int MF, bool REMOTE>
__device__ __forceinline__ void mfma_wave(const Params& p, const Tile& tl, LAS char* lds, int wid, int lane) {
  constexpr int SLOT = WB + MF * 4096;
  const int wr = wid >> 1, wc = wid & 1;
  const int fr = lane & 15, fq = lane >> 4;
  const int q = (lane & 15) >> 2, pp = lane & 3;
  const unsigned xa0 = (unsigned)(WB + (wr * MF * 16 + fr) * 128 + ((fq ^ ((fr >> 1) & 7)) * 16));
  const unsigned xa1 = xa0 ^ 64u;
  unsigned wa[4];
#pragma unroll
  for (int nf = 0; nf < 4; ++nf)
    wa[nf] = (unsigned)((8 * fq + q) * 256 + ((((wc ^ (fq & 1)) << 2) | (nf ^ q)) * 32) + 8 * pp);
  f32x4 acc[MF][4];
#pragma unroll
  for (int i = 0; i < MF; ++i)
#pragma unroll
    for (int j = 0; j < 4; ++j) acc[i][j] = (f32x4){0.f, 0.f, 0.f, 0.f};
  bf16x8 Xf[MF];
  s16x4 Wl[2][4], Wh[2][4];
#define WFRAG(bu, nf) __builtin_bit_cast(bf16x8, __builtin_shufflevector(Wl[bu][nf], Wh[bu][nf], 0, 1, 2, 3, 4, 5, 6, 7))
#define RDW(bu, sb, ks, i) do { if ((i) & 1) Wh[bu][(i) >> 1] = ldw((sb) + wa[(i) >> 1] + (ks) * 8192 + 1024); \
    else Wl[bu][(i) >> 1] = ldw((sb) + wa[(i) >> 1] + (ks) * 8192); } while (0)
#define MMA1(mf, bu, nf) do { acc[mf][nf] = __builtin_amdgcn_mfma_f32_16x16x32_bf16(WFRAG(bu, nf), Xf[mf], acc[mf][nf], 0, 0, 0); SCHED(); } while (0)
#define NOP_ do {} while (0)
#define GRP(mf, bu, F0, F1, F2) do { MMA1(mf, bu, 0); F0; SCHED(); MMA1(mf, bu, 1); F1; SCHED(); MMA1(mf, bu, 2); F2; SCHED(); MMA1(mf, bu, 3); } while (0)
#define RDX(i, sb, xa) Xf[i] = ldx((sb) + (xa) + (i) * 2048)
  if (tl.gathered) BAR();
  BAR();
  STAMP(1);
  {
    LAS const char* sb = lds;
#pragma unroll
    for (int i = 0; i < 8; ++i) RDW(0, sb, 0, i);
#pragma unroll
    for (int mf = 0; mf < MF - 1; ++mf) RDX(mf, sb, xa0);
  }
  int stg = 0;
  for (int t = 0; t < NT; ++t) {
    LAS const char* sb = lds + stg * SLOT;
    stg = (stg == NSTG - 1) ? 0 : stg + 1;
    LAS const char* sn = lds + stg * SLOT;
    GRP(0, 0, RDX(MF - 1, sb, xa0), RDW(1, sb, 1, 0), RDW(1, sb, 1, 1));
    GRP(1, 0, RDX(0, sb, xa1), RDW(1, sb, 1, 2), RDW(1, sb, 1, 3));
    GRP(2, 0, RDX(1, sb, xa1), RDW(1, sb, 1, 4), NOP_);
    GRP(3, 0, RDX(2, sb, xa1), RDW(1, sb, 1, 5), NOP_);
    GRP(4, 0, RDX(3, sb, xa1), RDW(1, sb, 1, 6), NOP_);
    GRP(5, 0, RDX(4, sb, xa1), RDW(1, sb, 1, 7), NOP_);
#pragma unroll
    for (int g = 6; g < MF; ++g) GRP(g, 0, RDX(g - 1, sb, xa1), NOP_, NOP_);
    GRP(0, 1, RDX(MF - 1, sb, xa1), NOP_, NOP_);
    GRP(1, 1, NOP_, NOP_, NOP_);
    WAITL(0);
    BAR();
    CFENCE();
    SCHED();
    GRP(2, 1, RDW(0, sn, 0, 0), RDW(0, sn, 0, 1), RDX(0, sn, xa0));
    GRP(3, 1, RDW(0, sn, 0, 2), RDW(0, sn, 0, 3), RDX(1, sn, xa0));
    GRP(4, 1, RDW(0, sn, 0, 4), RDW(0, sn, 0, 5), RDX(2, sn, xa0));
    GRP(5, 1, RDW(0, sn, 0, 6), RDW(0, sn, 0, 7), RDX(3, sn, xa0));
    GRP(6, 1, RDX(4, sn, xa0), RDX(5, sn, xa0), NOP_);
#pragma unroll
    for (int g = 7; g < MF; ++g) GRP(g, 1, RDX(g - 1, sn, xa0), NOP_, NOP_);
  }
  STAMP(2);
  const unsigned go = (unsigned)__builtin_amdgcn_readfirstlane((int)lds_rd(lds, C_GO));
  const bool bad = (go & 1u) != 0u;
  const auto ro = __builtin_amdgcn_make_buffer_rsrc((void*)p.out[tl.owner], 0, ML * NR * NCOL * 2, 0x00020000);
  const unsigned vo = (unsigned)((fr * NCOL + wc * 64 + (fq & 1) * 16 + (fq >> 1) * 8) * 2);
  const unsigned so = (unsigned)(tl.panel * BN * 2);
#pragma unroll
  for (int mf = 0; mf < MF; ++mf) {
    int orow;
    if (REMOTE) orow = p.rank * ML + wr * MF * 16 + mf * 16;
    else { const int ir = tl.xrow0 + wr * MF * 16 + mf * 16;
      orow = ir < 7 * BROWS ? ((p.rank + 1 + ir / BROWS) & 7) * ML + AROWS + ir % BROWS : p.rank * ML + ir - 7 * BROWS; }
#pragma unroll
    for (int np = 0; np < 2; ++np) {
      f32x4 a = acc[mf][2 * np], c = acc[mf][2 * np + 1];
      unsigned a0 = pk(a[0], a[1]), a1 = pk(a[2], a[3]), c0 = pk(c[0], c[1]), c1 = pk(c[2], c[3]);
      auto r0 = __builtin_amdgcn_permlane16_swap(a0, c0, false, false);
      auto r1 = __builtin_amdgcn_permlane16_swap(a1, c1, false, false);
      u32x4 v = {r0[0], r1[0], r0[1], r1[1]};
      if (bad) v = (u32x4){0x7fc07fc0u, 0x7fc07fc0u, 0x7fc07fc0u, 0x7fc07fc0u};
      __builtin_amdgcn_raw_buffer_store_b128(v, ro, vo + np * 64, so + (unsigned)orow * (NCOL * 2), REMOTE ? 17 : 0);
    }
    if (REMOTE) waitv<AWIN>();
  }
  if (REMOTE) {
    waitv<0>();
    if (lane == 0) st_sys(p.ws[tl.owner] + WS_DONE + p.rank * 64 + tl.panel * 4 + wid, go >> 1);
  }
  STAMP(3);
#undef WFRAG
#undef RDW
#undef MMA1
#undef GRP
#undef RDX
}

__device__ __forceinline__ bool wait_words(const unsigned* word, bool use, unsigned call, unsigned* tmo) {
  const unsigned long long t0 = __builtin_amdgcn_s_memrealtime();
  for (;;) {
    const unsigned v = ld_sys(word);
    if (__all(!use || v >= call)) return true;
    if (ld_agent(tmo) == call || __builtin_amdgcn_s_memrealtime() - t0 > DEADLINE) { st_agent(tmo, call); return false; }
    __builtin_amdgcn_s_sleep(2);
  }
}

__device__ __forceinline__ void comm_wave(const Params& p, const Tile& tl, LAS char* lds, int lane) {
  unsigned* myws = p.ws[p.rank];
  unsigned* tmo = myws + WS_TMO;
  const int b = blockIdx.x;
  const unsigned cnt = ld_agent(myws + WS_CNT + b);
  if (!tl.gathered) BAR();
  const unsigned call = (unsigned)__builtin_amdgcn_readfirstlane((int)cnt) + 1u;
  if (lane == 0) st_agent(myws + WS_CNT + b, call);
  bool ok = true;
  int nb = 1;
  if (tl.gathered) {
    if (lane == 0) lds_wr(lds, C_CALL, call);
    const int s0 = tl.xrow0 / BROWS;
    const int e1 = (tl.xrow0 + MFB * 32 - 1) / BROWS;
    const int s1 = e1 < 7 ? e1 : 6;
    const unsigned* sflag = myws + WS_XFLAG + ((lane & 16) ? s1 : s0) * 32 + (lane & 15);
    auto decode = [&](unsigned v) -> int {
#pragma unroll
      for (int o = 1; o < 32; o <<= 1) { const unsigned u = (unsigned)__shfl_xor((int)v, o); v = u < v ? u : v; }
      const unsigned vm = (unsigned)__builtin_amdgcn_readfirstlane((int)v);
      const int n = vm > call * 128u ? (int)(vm - call * 128u) : 0;
      return n > NT ? NT : n;
    };
    int seen = 0;
    nb = -1;
    while (seen < NT) {
      const int need = nb + 3 < NT ? nb + 3 : NT;
      if (seen < need) {
        const unsigned long long t0 = __builtin_amdgcn_s_memrealtime();
        while ((seen = decode(ld_sys(sflag))) < need) {
          if (ld_agent(tmo) == call || __builtin_amdgcn_s_memrealtime() - t0 > DEADLINE) { st_agent(tmo, call); ok = false; seen = NT; break; }
          __builtin_amdgcn_s_sleep(1);
        }
        __builtin_amdgcn_fence(__ATOMIC_ACQUIRE, "");
        waitv<0>();
        if (nb == -1) STAMP(5);
        WAITL(0);
        BAR(); ++nb;
      } else if (seen < need + POLL_MARGIN) {
        unsigned v;
        asm volatile("global_load_dword %0, %1, off sc0 sc1" : "=v"(v) : "v"(sflag) : "memory");
        BAR(); ++nb;
        asm volatile("s_waitcnt vmcnt(0)" : "+v"(v) :: "memory");
        const int n = decode(v);
        if (n > seen) {
          __builtin_amdgcn_fence(__ATOMIC_ACQUIRE, "");
          if (seen >= (nb + 3 < NT ? nb + 3 : NT)) { BAR(); ++nb; }
          waitv<0>();
          seen = n;
        }
      } else {
        BAR(); ++nb;
      }
    }
    STAMP(7);
  } else if (tl.kind == 0) {
    while (nb < NT / 2) { BAR(); ++nb; }
    ok = wait_words(myws + WS_READY + tl.peer * 32, true, call, tmo);
  }
  while (nb < NT - 1) { BAR(); ++nb; }
  if (ld_agent(tmo) == call) ok = false;
  if (lane == 0) lds_wr(lds, C_GO, call * 2u + (ok ? 0u : 1u));
  WAITL(0);
  while (nb <= NT) { BAR(); ++nb; }
}

__device__ __forceinline__ void copy_group(const Params& p, LAS char* lds, int c, int wid, int lane) {
  unsigned* myws = p.ws[p.rank];
  unsigned* tmo = myws + WS_TMO;
  const int b = blockIdx.x;
  const unsigned long long tstart = __builtin_amdgcn_s_memrealtime();
  const unsigned cnt = ld_agent(myws + WS_CNT + b);
  const int gw = wid * 16 + c, rp = gw & 31, ph = gw >> 5;
  const auto rx = __builtin_amdgcn_make_buffer_rsrc((void*)p.x, 0, ML * KD * 2, 0x00020000);
  const unsigned xv = (unsigned)((AROWS + rp * 8 + (lane >> 3)) * KD * 2 + (((lane & 7) ^ (((rp & 1) * 4 + (lane >> 4)) & 7)) * 16));
  u32x4 d[16];
#pragma unroll
  for (int k = 0; k < 2; ++k) d[k] = __builtin_amdgcn_raw_buffer_load_b128(rx, xv, (unsigned)((ph + 4 * k) * 128), 0);
  LAS unsigned* tcnt = (LAS unsigned*)lds;
  if (wid == 0) { lds_wr(lds, lane * 4, 0u); lds_wr(lds, 256 + lane * 4, 0u); WAITL(0); }
  const unsigned call = (unsigned)__builtin_amdgcn_readfirstlane((int)cnt) + 1u;
  BAR();
  if (wid == 0 && lane == 0) st_agent(myws + WS_CNT + b, call);
  if (gw == 127 && lane < NR && lane != p.rank) st_sys(p.ws[lane] + WS_READY + p.rank * 32, call);
  const size_t xo = WS_XBUF + (call & 1) * XPAR_BYTES;
  unsigned* pflag = p.ws[(p.rank + NR - 1 - (lane < 7 ? lane : 0)) & 7] + WS_XFLAG + (lane < 7 ? lane : 0) * 32 + c;
  auto finish = [&](int t) {
    if (lane == 0) __hip_atomic_fetch_add(tcnt + t, 1u, __ATOMIC_RELAXED, __HIP_MEMORY_SCOPE_WORKGROUP);
    const unsigned long long open = __ballot(lds_rd(lds, lane * 4) < 2u);
    const int pre = open ? __builtin_ctzll(open) : NT;
    const int lvl = pre < 4 ? pre : (pre < 12 ? (pre & ~1) : (pre & ~3));
    unsigned old = 0;
    if (lane == 0) old = __hip_atomic_fetch_max(tcnt + 64, (unsigned)lvl, __ATOMIC_RELAXED, __HIP_MEMORY_SCOPE_WORKGROUP);
    old = (unsigned)__builtin_amdgcn_readfirstlane((int)old);
    if ((unsigned)lvl > old && lane < 7)
      __hip_atomic_fetch_max(pflag, call * 128u + (unsigned)lvl, __ATOMIC_RELAXED, __HIP_MEMORY_SCOPE_SYSTEM);
  };
  bool pending = false;
  long long tissue = 0;
#pragma unroll
  for (int k = 0; k < 16; ++k) {
    const int t = ph + 4 * k;
    for (;;) {
      const long long now = (long long)(__builtin_amdgcn_s_memrealtime() - tstart) * 10;
      if (now >= (long long)(t - PUSH_LEAD) * TILE_NS) break;
      if (pending && now - tissue > ACK_NS) { waitv<0>(); finish(t - 4); pending = false; }
      else __builtin_amdgcn_s_sleep(2);
    }
    for (int i = 0; i < NR - 1; ++i) {
      const int j = (i + gw) % (NR - 1) + 1;
      const auto rd = __builtin_amdgcn_make_buffer_rsrc((char*)p.ws[(p.rank + NR - j) & 7] + xo, 0, (int)XPAR_BYTES, 0x00020000);
      __builtin_amdgcn_raw_buffer_store_b128(d[k], rd, (unsigned)(lane * 16), (unsigned)t * (unsigned)XT_BYTES + (unsigned)(((j - 1) * BROWS + rp * 8) * 128), 17);
    }
    tissue = (long long)(__builtin_amdgcn_s_memrealtime() - tstart) * 10;
    if (pending) { waitv<7>(); finish(t - 4); }
    pending = true;
    if (k + 2 < 16) d[k + 2] = __builtin_amdgcn_raw_buffer_load_b128(rx, xv, (unsigned)((t + 8) * 128), 0);
  }
  waitv<0>();
  finish(ph + 60);
  STAMP(4);
  if (c == 15) {
    if (wid != p.rank) {
      const bool fine = wait_words(myws + WS_DONE + wid * 64 + lane, true, call, tmo);
      __builtin_amdgcn_fence(__ATOMIC_ACQUIRE, "");
      if (!fine && lane == 0) p.out[p.rank][(size_t)(wid * ML) * NCOL] = 0x7fc0;
    }
    STAMP(6);
  }
}

__global__ __launch_bounds__(512, 2) void gemm_a2a(Params p) {
  extern __shared__ __attribute__((aligned(16))) char smem[];
  LAS char* lds = (LAS char*)smem;
  const int wid = __builtin_amdgcn_readfirstlane(threadIdx.x >> 6);
  const int lane = threadIdx.x & 63;
  const int b = blockIdx.x;
  const int xcd = b & 7, idx = b >> 3;
  STAMP(0);
  if (idx < 2) { copy_group(p, lds, xcd * 2 + idx, wid, lane); return; }
  Tile tl;
  int rt = 0;
  if (idx < 16) {
    const int a = xcd * 14 + idx - 2;
    tl.kind = 0; tl.peer = (p.rank + (AMAP ? a % 7 : a >> 4) + 1) & 7; tl.panel = AMAP ? a / 7 : a & 15; tl.xrow0 = 0; tl.owner = tl.peer; tl.lrow0 = 0; tl.gathered = false; tl.koff = KSKEW * xcd;
  } else {
    const int e = idx - 16;
    rt = (xcd & 1) * 4 + (e >> 2);
    tl.kind = 1; tl.panel = (xcd >> 1) * 4 + (e & 3); tl.owner = p.rank; tl.peer = p.rank;
    tl.xrow0 = rt * (MFB * 32);
    tl.lrow0 = rt == 7 ? 7 * MFB * 32 - XROWS : 0;
    tl.gathered = rt < 7; tl.koff = 0;
  }
  if (wid < 4) {
    if (tl.kind == 0) mfma_wave<MFA, true>(p, tl, lds, wid, lane);
    else mfma_wave<MFB, false>(p, tl, lds, wid, lane);
  } else if (wid == 7) {
    comm_wave(p, tl, lds, lane);
  } else {
#define LOADERS(MF, NWS, AUX) do { if (wid == 4) loader<MF, 0, NWS, AUX>(p, tl, lds, lane); else if (wid == 5) loader<MF, 1, NWS, AUX>(p, tl, lds, lane); \
    else loader<MF, 2, NWS, AUX>(p, tl, lds, lane); } while (0)
    if (tl.kind == 0) LOADERS(MFA, 0, WAUX);
    else if (rt < 6) LOADERS(MFB, 4 * MFB, WAUXB);
    else if (rt == 6) LOADERS(MFB, (XROWS - 6 * MFB * 32) / 8, WAUXB);
    else LOADERS(MFB, 0, WAUXB);
  }
}

extern "C" void kernel_launch(int rank, int nranks, ncclComm_t comm,
                              hipStream_t stream,
                              void* const* d_in, const int* in_sizes, int n_in,
                              void* d_out, int out_size,
                              const int* in_dtypes, int out_dtype,
                              void* const* d_out_all,
                              void* const* d_ws_all, size_t ws_bytes) {
  if (nranks != NR || (unsigned)rank >= (unsigned)NR || ws_bytes < WS_NEED) { fprintf(stderr, "unexpected configuration\n"); return; }
  Params p;
  p.x = (const unsigned short*)d_in[0];
  p.w = (const unsigned short*)d_in[1];
  for (int i = 0; i < NR; ++i) { p.out[i] = (unsigned short*)d_out_all[i]; p.ws[i] = (unsigned*)d_ws_all[i]; }
  p.rank = rank;
  static bool init[NR] = {};
  if (!init[rank]) {
    HIP_CHECK(hipFuncSetAttribute((const void*)gemm_a2a, hipFuncAttributeMaxDynamicSharedMemorySize, LDS_BYTES));
    init[rank] = true;
  }
  gemm_a2a<<<NBLK, 512, LDS_BYTES, stream>>>(p);
  HIP_CHECK(hipGetLastError());
}
```

```cpp
#include <hip/hip_runtime.h>
#include <rccl/rccl.h>
#include <cstdio>
#include <stdint.h>

#define HIP_CHECK(expr) do { hipError_t _e = (expr); if (_e != hipSuccess) { \
    fprintf(stderr, "HIP error %s at %s:%d\n", hipGetErrorString(_e), __FILE__, __LINE__); \
    return; } } while (0)

#define LAS __attribute__((address_space(3)))
typedef short s16x4 __attribute__((ext_vector_type(4)));
typedef __bf16 bf16x2 __attribute__((ext_vector_type(2)));
typedef __bf16 bf16x8 __attribute__((ext_vector_type(8)));
typedef float f32x4 __attribute__((ext_vector_type(4)));
typedef float f32x2 __attribute__((ext_vector_type(2)));
typedef unsigned u32x4 __attribute__((ext_vector_type(4)));

constexpr int NR = 8, ML = 512, KD = 4096, ND = 16384, NCOL = ND / NR;
constexpr int BK = 64, NT = KD / BK;
constexpr int MFA = 7;
constexpr int MFN = 5;
constexpr int AROWS = MFA * 32, BROWS = ML - AROWS;
constexpr int BRP = BROWS / 8;
constexpr int XROWS = 7 * BROWS;
constexpr int NROWS = XROWS + ML;
constexpr int NM = MFN * 32;
constexpr int WODD = 8192 + 128;
constexpr int WIMG = WODD + 8192;
#define WPAIR 1
constexpr int SWA = WPAIR ? 4 : 3;
constexpr int LDS_CTL = 3 * (2 * WIMG + MFN * 4096);
static_assert(SWA * WIMG + 3 * MFA * 4096 + 3072 <= LDS_CTL, "LDS");
constexpr int C_GO = LDS_CTL;
constexpr int C_CALL = LDS_CTL + 16;
constexpr int LDS_BYTES = LDS_CTL + 64;
static_assert(LDS_BYTES <= 163840, "LDS");
constexpr int NBLK = 256;
constexpr unsigned long long DEADLINE = 50000000ull;
constexpr int WS_TMO = 32, WS_READY = 64  , WS_CNT = 512  , WS_DONE = 1024  ;
constexpr int WS_XFLAG = 2048;
constexpr size_t WS_XBUF = 1u << 20;
constexpr size_t XT_BYTES = (size_t)XROWS * 128;
constexpr size_t XPAR_BYTES = XT_BYTES * NT;
constexpr size_t WS_NEED = WS_XBUF + 2 * XPAR_BYTES;
constexpr int NUNIT = BRP * NT / 128;
#define POLL_MARGIN 4
#define TILE_NS 640
#define PUSH_LEAD 1
#define ACK_NS 800
#define KSKEW 8
#define XPD 4
#define ACQ 2
#define KSTRIDE 11
#define NAUX 0
#define AMAP 1
#define AWIN 2
#define WAUX 2

struct Params {
  const unsigned short* x;
  const unsigned short* w;
  unsigned short* out[NR];
  unsigned* ws[NR];
  int rank;
};

#define WAITL(n) asm volatile("s_waitcnt lgkmcnt(" #n ")" ::: "memory")
template <int N> __device__ __forceinline__ void waitv() { asm volatile("s_waitcnt vmcnt(%0)" :: "n"(N) : "memory"); }
#define BAR() __builtin_amdgcn_s_barrier()
#define SCHED() __builtin_amdgcn_sched_barrier(0)
#define CFENCE() asm volatile("" ::: "memory")
#define STAMP(i) do {} while (0)
#define TRACE_DECL do {} while (0)
#define TRACE(t) do {} while (0)
#define TRACE_OUT do {} while (0)

__device__ __forceinline__ bf16x8 ldx(LAS const char* p) { return *(LAS const bf16x8*)p; }
__device__ __forceinline__ s16x4 ldw(LAS const char* p) { return __builtin_amdgcn_ds_read_tr16_b64_v4i16((LAS s16x4*)p); }
__device__ __forceinline__ unsigned pk(float a, float b) { f32x2 v = {a, b}; return __builtin_bit_cast(unsigned, __builtin_convertvector(v, bf16x2)); }
__device__ __forceinline__ unsigned ld_sys(const unsigned* p) { return __hip_atomic_load(p, __ATOMIC_RELAXED, __HIP_MEMORY_SCOPE_SYSTEM); }
__device__ __forceinline__ unsigned ld_agent(const unsigned* p) { return __hip_atomic_load(p, __ATOMIC_RELAXED, __HIP_MEMORY_SCOPE_AGENT); }
__device__ __forceinline__ void st_sys(unsigned* p, unsigned v) { __hip_atomic_store(p, v, __ATOMIC_RELAXED, __HIP_MEMORY_SCOPE_SYSTEM); }
__device__ __forceinline__ void st_agent(unsigned* p, unsigned v) { __hip_atomic_store(p, v, __ATOMIC_RELAXED, __HIP_MEMORY_SCOPE_AGENT); }
__device__ __forceinline__ unsigned lds_rd(LAS char* lds, int off) { return *(LAS volatile unsigned*)(lds + off); }
__device__ __forceinline__ void lds_wr(LAS char* lds, int off, unsigned v) { *(LAS volatile unsigned*)(lds + off) = v; }
__device__ __forceinline__ bf16x8 wfrag(s16x4 lo, s16x4 hi) { return __builtin_bit_cast(bf16x8, __builtin_shufflevector(lo, hi, 0, 1, 2, 3, 4, 5, 6, 7)); }
__device__ __forceinline__ u32x4 pack2(f32x4 a, f32x4 c, bool bad) {
  const unsigned a0 = pk(a[0], a[1]), a1 = pk(a[2], a[3]), c0 = pk(c[0], c[1]), c1 = pk(c[2], c[3]);
  auto r0 = __builtin_amdgcn_permlane16_swap(a0, c0, false, false);
  auto r1 = __builtin_amdgcn_permlane16_swap(a1, c1, false, false);
  return bad ? (u32x4){0x7fc07fc0u, 0x7fc07fc0u, 0x7fc07fc0u, 0x7fc07fc0u} : (u32x4){r0[0], r1[0], r0[1], r1[1]};
}
__device__ __forceinline__ unsigned out_voff(int lane) { const int fr = lane & 15, fq = lane >> 4; return (unsigned)((fr * NCOL + (fq & 1) * 16 + (fq >> 1) * 8) * 2); }
__host__ __device__ constexpr int wpiece(int pc) { return ((pc >> 1) & 1) * WODD + (pc >> 2) * 2048 + (pc & 1) * 1024; }

struct Tile {
  int kind;
  int owner;
  int col0;
  int nrow0;
  bool gathered;
  int koff;
  int ai;
};

template <int MF, int NIMG, int SW, bool PAIR, int H, int NWS, int AUXW>
__device__ __forceinline__ void loader(const Params& p, const Tile& tl, LAS char* lds, int lane) {
  constexpr int NWP = 16 * NIMG;
  constexpr int WSLOT = NIMG * WIMG, XSLOT = MF * 4096, LX = SW * WSLOT;
  constexpr int NW_ = (NWP - H + 2) / 3;
  constexpr int X0 = (H + 3 * NW_) - NWP;
  constexpr int NX_ = (4 * MF - X0 + 2) / 3;
  constexpr bool XWS = NWS > 0;
  static_assert(2 * NW_ + 2 * NX_ <= 63, "");
  const auto rw = __builtin_amdgcn_make_buffer_rsrc((void*)p.w, 0, KD * ND * 2, 0x00020000);
  const auto rx = __builtin_amdgcn_make_buffer_rsrc((void*)p.x, 0, ML * KD * 2, 0x00020000);
  const unsigned wv = (unsigned)((lane >> 4) * ND * 2 + (((lane & 15) ^ ((lane >> 4) << 1)) * 16));
  unsigned xv[2];
#pragma unroll
  for (int v = 0; v < 2; ++v) xv[v] = (unsigned)((lane >> 3) * KD * 2 + (((lane & 7) ^ ((v * 4 + (lane >> 4)) & 7)) * 16));
  const unsigned iv = (unsigned)(lane * 16);
  const unsigned wbase = (unsigned)((tl.owner * NCOL + tl.col0) * 2);
  const unsigned xbase = tl.kind == 0 ? 0u : (unsigned)((tl.nrow0 - XROWS) * KD * 2);
  int ws_ = 0, xs_ = 0;
  auto kmap = [&](int t) -> unsigned { return PAIR ? (unsigned)(((((t >> 1) * KSTRIDE) << 1) + (t & 1) + tl.koff) & (NT - 1)) : (unsigned)((t + tl.koff) & (NT - 1)); };
  auto issue_w = [&](int t) {
    const unsigned tb = wbase + kmap(t) * (BK * ND * 2);
#pragma unroll
    for (int i = 0; i < NW_; ++i) {
      const int pc = H + 3 * i, img = pc >> 4, q = pc & 15;
      __builtin_amdgcn_raw_ptr_buffer_load_lds(rw, (LAS void*)(lds + ws_ * WSLOT + img * WIMG + wpiece(q)), 16, wv, tb + q * (4 * ND * 2) + img * 256, 0, AUXW);
    }
    ws_ = ws_ == SW - 1 ? 0 : ws_ + 1;
  };
  auto issue_xl = [&](int t) {
    const unsigned tb = xbase + kmap(t) * 128u;
#pragma unroll
    for (int i = 0; i < NX_; ++i) {
      const int pc = X0 + 3 * i;
      if (pc >= NWS) __builtin_amdgcn_raw_ptr_buffer_load_lds(rx, (LAS void*)(lds + LX + xs_ * XSLOT + pc * 1024), 16, xv[pc & 1], tb + pc * (8 * KD * 2), 0, 0);
    }
  };
  issue_w(0);
  issue_xl(0);
  unsigned call = 0;
  if (XWS) {
    BAR();
    call = (unsigned)__builtin_amdgcn_readfirstlane((int)lds_rd(lds, C_CALL));
  }
  const auto ri = __builtin_amdgcn_make_buffer_rsrc((char*)p.ws[p.rank] + WS_XBUF + (call & 1) * XPAR_BYTES, 0, (int)XPAR_BYTES, 0x00020000);
  const unsigned ibase = (unsigned)(tl.nrow0 * 128);
  auto issue_xi = [&](int t) {
#pragma unroll
    for (int i = 0; i < NX_; ++i) {
      const int pc = X0 + 3 * i;
      if (pc < NWS) __builtin_amdgcn_raw_ptr_buffer_load_lds(ri, (LAS void*)(lds + LX + xs_ * XSLOT + pc * 1024), 16, iv, ibase + (unsigned)t * (unsigned)XT_BYTES + pc * 1024, 0, 0);
    }
    xs_ = xs_ == 2 ? 0 : xs_ + 1;
  };
  issue_xi(0);
  issue_w(1);
  issue_xl(1);
  issue_xi(1);
  waitv<NW_ + NX_>();
  BAR();
  if (PAIR) {
    constexpr int NPF = (XPD && H > 0) ? 2 : 0;
    auto prefetch = [&](int t) {
      const unsigned tb = kmap(t) * 128u + (unsigned)tl.ai * (16 * KD * 2);
#pragma unroll
      for (int i = 0; i < NPF; ++i)
        __builtin_amdgcn_raw_ptr_buffer_load_lds(rx, (LAS void*)(lds + LX + 3 * XSLOT + H * 1024), 16, xv[0], tb + i * (8 * KD * 2), 0, 0);
    };
    for (int t = 0; t < NT - 2; t += 2) {
      issue_w(t + 2);
      issue_w(t + 3);
      prefetch(t + XPD + H - 1);
      issue_xl(t + 2);
      issue_xi(t + 2);
      waitv<2 * NW_ + NPF + NX_>();
      BAR();
      issue_xl(t + 3);
      issue_xi(t + 3);
      waitv<NX_>();
      BAR();
    }
  } else {
    for (int t = 0; t < NT - 2; ++t) {
      issue_w(t + 2);
      issue_xl(t + 2);
      issue_xi(t + 2);
      waitv<NW_ + NX_>();
      BAR();
    }
  }
  waitv<0>();
  BAR();
  BAR();
}

#define NOP_ do {} while (0)
template <int MF>
__device__ __forceinline__ void mfma_a(const Params& p, const Tile& tl, LAS char* lds, int wid, int lane) {
  constexpr int XSLOT = MF * 4096, LX = SWA * WIMG;
  const int wr = wid >> 1, wc = wid & 1;
  const int fr = lane & 15, fq = lane >> 4;
  const int q = (lane & 15) >> 2, pp = lane & 3;
  const unsigned xa0 = (unsigned)(LX + (wr * MF * 16 + fr) * 128 + ((fq ^ ((fr >> 1) & 7)) * 16));
  const unsigned xa1 = xa0 ^ 64u;
  unsigned wa[4];
#pragma unroll
  for (int nf = 0; nf < 4; ++nf)
    wa[nf] = (unsigned)((fq & 1) * WODD + (fq >> 1) * 2048 + q * 256 + wc * 128 + ((nf ^ q) * 32) + 8 * pp);
  f32x4 acc[MF][4];
#pragma unroll
  for (int i = 0; i < MF; ++i)
#pragma unroll
    for (int j = 0; j < 4; ++j) acc[i][j] = (f32x4){0.f, 0.f, 0.f, 0.f};
  bf16x8 Xf[MF];
  s16x4 Wl[2][4], Wh[2][4];
#define RDW(bu, sb, ks, i) do { if ((i) & 1) Wh[bu][(i) >> 1] = ldw((sb) + wa[(i) >> 1] + (ks) * 4096 + 1024); \
    else Wl[bu][(i) >> 1] = ldw((sb) + wa[(i) >> 1] + (ks) * 4096); } while (0)
#define MMA1(mf, bu, nf) do { acc[mf][nf] = __builtin_amdgcn_mfma_f32_16x16x32_bf16(wfrag(Wl[bu][nf], Wh[bu][nf]), Xf[mf], acc[mf][nf], 0, 0, 0); SCHED(); } while (0)
#define GRP(mf, bu, F0, F1, F2) do { MMA1(mf, bu, 0); F0; SCHED(); MMA1(mf, bu, 1); F1; SCHED(); MMA1(mf, bu, 2); F2; SCHED(); MMA1(mf, bu, 3); } while (0)
#define RDX(i, sb, xa) Xf[i] = ldx((sb) + (xa) + (i) * 2048)
  BAR();
  STAMP(1);
#pragma unroll
  for (int i = 0; i < 8; ++i) RDW(0, lds, 0, i);
#pragma unroll
  for (int mf = 0; mf < MF - 1; ++mf) RDX(mf, lds, xa0);
  int sw = 0, sx = 0;
  TRACE_DECL;
  for (int t = 0; t < NT; ++t) {
    TRACE(t);
    LAS const char* wb = lds + sw * WIMG;
    LAS const char* xb = lds + sx * XSLOT;
    sw = (sw == SWA - 1) ? 0 : sw + 1;
    sx = (sx == 2) ? 0 : sx + 1;
    LAS const char* wn = lds + sw * WIMG;
    LAS const char* xn = lds + sx * XSLOT;
    GRP(0, 0, RDX(MF - 1, xb, xa0), RDW(1, wb, 1, 0), RDW(1, wb, 1, 1));
    GRP(1, 0, RDX(0, xb, xa1), RDW(1, wb, 1, 2), RDW(1, wb, 1, 3));
    GRP(2, 0, RDX(1, xb, xa1), RDW(1, wb, 1, 4), NOP_);
    GRP(3, 0, RDX(2, xb, xa1), RDW(1, wb, 1, 5), NOP_);
    GRP(4, 0, RDX(3, xb, xa1), RDW(1, wb, 1, 6), NOP_);
    GRP(5, 0, RDX(4, xb, xa1), RDW(1, wb, 1, 7), NOP_);
#pragma unroll
    for (int g = 6; g < MF; ++g) GRP(g, 0, RDX(g - 1, xb, xa1), NOP_, NOP_);
    GRP(0, 1, RDX(MF - 1, xb, xa1), NOP_, NOP_);
    GRP(1, 1, NOP_, NOP_, NOP_);
    WAITL(0);
    BAR();
    CFENCE();
    SCHED();
    GRP(2, 1, RDW(0, wn, 0, 0), RDW(0, wn, 0, 1), RDX(0, xn, xa0));
    GRP(3, 1, RDW(0, wn, 0, 2), RDW(0, wn, 0, 3), RDX(1, xn, xa0));
    GRP(4, 1, RDW(0, wn, 0, 4), RDW(0, wn, 0, 5), RDX(2, xn, xa0));
    GRP(5, 1, RDW(0, wn, 0, 6), RDW(0, wn, 0, 7), RDX(3, xn, xa0));
    GRP(6, 1, RDX(4, xn, xa0), RDX(5, xn, xa0), NOP_);
#pragma unroll
    for (int g = 7; g < MF; ++g) GRP(g, 1, RDX(g - 1, xn, xa0), NOP_, NOP_);
  }
  STAMP(2);
  TRACE_OUT;
  const unsigned go = (unsigned)__builtin_amdgcn_readfirstlane((int)lds_rd(lds, C_GO));
  const bool bad = (go & 1u) != 0u;
  const auto ro = __builtin_amdgcn_make_buffer_rsrc((void*)p.out[tl.owner], 0, ML * NR * NCOL * 2, 0x00020000);
  const unsigned vo = out_voff(lane);
  const unsigned so = (unsigned)(((p.rank * ML + wr * MF * 16) * NCOL + tl.col0 + wc * 64) * 2);
#pragma unroll
  for (int mf = 0; mf < MF; ++mf) {
#pragma unroll
    for (int np = 0; np < 2; ++np)
      __builtin_amdgcn_raw_buffer_store_b128(pack2(acc[mf][2 * np], acc[mf][2 * np + 1], bad), ro, vo + np * 64, so + mf * (16 * NCOL * 2), 17);
    waitv<AWIN>();
  }
  waitv<0>();
  if (lane == 0) st_sys(p.ws[tl.owner] + WS_DONE + p.rank * 64 + (tl.col0 >> 7) * 4 + wid, go >> 1);
  STAMP(3);
#undef RDW
#undef MMA1
#undef GRP
#undef RDX
}

__device__ __forceinline__ void mfma_n(const Params& p, const Tile& tl, LAS char* lds, int wid, int lane) {
  constexpr int MF = MFN;
  constexpr int WSLOT = 2 * WIMG, XSLOT = MF * 4096, LX = 3 * WSLOT;
  const int wr = wid >> 1, wc = wid & 1;
  const int fr = lane & 15, fq = lane >> 4;
  const int q = (lane & 15) >> 2, pp = lane & 3;
  const unsigned xa0 = (unsigned)(LX + (wr * MF * 16 + fr) * 128 + ((fq ^ ((fr >> 1) & 7)) * 16));
  const unsigned xa1 = xa0 ^ 64u;
  unsigned wa[4];
#pragma unroll
  for (int nf = 0; nf < 4; ++nf)
    wa[nf] = (unsigned)(wc * WIMG + (fq & 1) * WODD + (fq >> 1) * 2048 + q * 256 + ((nf ^ q) * 32) + 8 * pp);
  f32x4 acc[MF][8];
#pragma unroll
  for (int i = 0; i < MF; ++i)
#pragma unroll
    for (int j = 0; j < 8; ++j) acc[i][j] = (f32x4){0.f, 0.f, 0.f, 0.f};
  bf16x8 Xf[2][MF];
  s16x4 Wl[8], Wh[8];
#define RWL(nf, sb, ks) Wl[nf] = ldw((sb) + wa[(nf) & 3] + ((nf) >> 2) * 128 + (ks) * 4096)
#define RWH(nf, sb, ks) Wh[nf] = ldw((sb) + wa[(nf) & 3] + ((nf) >> 2) * 128 + (ks) * 4096 + 1024)
#define RDX(bu, i, sb, xa) Xf[bu][i] = ldx((sb) + (xa) + (i) * 2048)
#define MMA1(nf, bu, mf) do { acc[mf][nf] = __builtin_amdgcn_mfma_f32_16x16x32_bf16(wfrag(Wl[nf], Wh[nf]), Xf[bu][mf], acc[mf][nf], 0, 0, 0); SCHED(); } while (0)
#define GRP(nf, bu, F0, F1, F2, F3, F4) do { MMA1(nf, bu, 0); F0; SCHED(); MMA1(nf, bu, 1); F1; SCHED(); MMA1(nf, bu, 2); F2; SCHED(); \
    MMA1(nf, bu, 3); F3; SCHED(); MMA1(nf, bu, 4); F4; SCHED(); } while (0)
  if (tl.gathered) BAR();
  BAR();
  STAMP(1);
#pragma unroll
  for (int mf = 0; mf < MF; ++mf) RDX(0, mf, lds, xa0);
#pragma unroll
  for (int nf = 0; nf < 7; ++nf) { RWL(nf, lds, 0); RWH(nf, lds, 0); }
  int st = 0;
  TRACE_DECL;
  for (int t = 0; t < NT; ++t) {
    TRACE(t);
    LAS const char* wb = lds + st * WSLOT;
    LAS const char* xb = lds + st * XSLOT;
    st = (st == 2) ? 0 : st + 1;
    LAS const char* wn = lds + st * WSLOT;
    LAS const char* xn = lds + st * XSLOT;
    GRP(0, 0, RWL(7, wb, 0), RWH(7, wb, 0), RDX(1, 0, xb, xa1), NOP_, NOP_);
    GRP(1, 0, RWL(0, wb, 1), RWH(0, wb, 1), RDX(1, 1, xb, xa1), NOP_, NOP_);
    GRP(2, 0, RWL(1, wb, 1), RWH(1, wb, 1), RDX(1, 2, xb, xa1), NOP_, NOP_);
    GRP(3, 0, RWL(2, wb, 1), RWH(2, wb, 1), RDX(1, 3, xb, xa1), NOP_, NOP_);
    GRP(4, 0, RWL(3, wb, 1), RWH(3, wb, 1), RDX(1, 4, xb, xa1), NOP_, NOP_);
    GRP(5, 0, RWL(4, wb, 1), RWH(4, wb, 1), NOP_, NOP_, NOP_);
    GRP(6, 0, RWL(5, wb, 1), RWH(5, wb, 1), NOP_, NOP_, NOP_);
    GRP(7, 0, RWL(6, wb, 1), RWH(6, wb, 1), NOP_, NOP_, NOP_);
    GRP(0, 1, RWL(7, wb, 1), RWH(7, wb, 1), NOP_, NOP_, NOP_);
    GRP(1, 1, NOP_, NOP_, NOP_, NOP_, NOP_);
    WAITL(0);
    BAR();
    CFENCE();
    SCHED();
    GRP(2, 1, RWL(0, wn, 0), RWH(0, wn, 0), RDX(0, 0, xn, xa0), RWL(1, wn, 0), RWH(1, wn, 0));
    GRP(3, 1, RWL(2, wn, 0), RWH(2, wn, 0), RDX(0, 1, xn, xa0), RDX(0, 2, xn, xa0), NOP_);
    GRP(4, 1, RWL(3, wn, 0), RWH(3, wn, 0), RDX(0, 3, xn, xa0), RDX(0, 4, xn, xa0), NOP_);
    GRP(5, 1, RWL(4, wn, 0), RWH(4, wn, 0), NOP_, NOP_, NOP_);
    GRP(6, 1, RWL(5, wn, 0), RWH(5, wn, 0), NOP_, NOP_, NOP_);
    GRP(7, 1, RWL(6, wn, 0), RWH(6, wn, 0), NOP_, NOP_, NOP_);
  }
  STAMP(2);
  TRACE_OUT;
  const unsigned go = (unsigned)__builtin_amdgcn_readfirstlane((int)lds_rd(lds, C_GO));
  const bool bad = (go & 1u) != 0u;
  const auto ro = __builtin_amdgcn_make_buffer_rsrc((void*)p.out[p.rank], 0, ML * NR * NCOL * 2, 0x00020000);
  const unsigned vo = out_voff(lane);
  const unsigned so = (unsigned)((tl.col0 + wc * 128) * 2);
#pragma unroll
  for (int mf = 0; mf < MF; ++mf) {
    const int nr = tl.nrow0 + wr * MF * 16 + mf * 16;
    if (nr < NROWS) {
      const int orow = nr < XROWS ? ((p.rank + 1 + nr / BROWS) & 7) * ML + AROWS + nr % BROWS : p.rank * ML + nr - XROWS;
#pragma unroll
      for (int np = 0; np < 4; ++np)
        __builtin_amdgcn_raw_buffer_store_b128(pack2(acc[mf][2 * np], acc[mf][2 * np + 1], bad), ro, vo + np * 64, so + (unsigned)orow * (NCOL * 2), NAUX);
    }
  }
  STAMP(3);
#undef RWL
#undef RWH
#undef RDX
#undef MMA1
#undef GRP
}

__device__ __forceinline__ bool wait_words(const unsigned* word, bool use, unsigned call, unsigned* tmo) {
  const unsigned long long t0 = __builtin_amdgcn_s_memrealtime();
  for (;;) {
    const unsigned v = ld_sys(word);
    if (__all(!use || v >= call)) return true;
    if (ld_agent(tmo) == call || __builtin_amdgcn_s_memrealtime() - t0 > DEADLINE) { st_agent(tmo, call); return false; }
    __builtin_amdgcn_s_sleep(2);
  }
}

__device__ __forceinline__ void ctrl_wave(const Params& p, const Tile& tl, LAS char* lds, int lane) {
  unsigned* myws = p.ws[p.rank];
  unsigned* tmo = myws + WS_TMO;
  const int b = blockIdx.x;
  const unsigned cnt = ld_agent(myws + WS_CNT + b);
  if (!tl.gathered) BAR();
  const unsigned call = (unsigned)__builtin_amdgcn_readfirstlane((int)cnt) + 1u;
  if (lane == 0) st_agent(myws + WS_CNT + b, call);
  bool ok = true;
  int nb = 1;
  if (tl.gathered) {
    if (lane == 0) lds_wr(lds, C_CALL, call);
    const int s0 = tl.nrow0 / BROWS;
    const int e1 = (tl.nrow0 + NM - 1) / BROWS;
    const int s1 = e1 < 7 ? e1 : 6;
    const unsigned* sflag = myws + WS_XFLAG + ((lane & 16) ? s1 : s0) * 32 + (lane & 15);
    auto decode = [&](unsigned v) -> int {
#pragma unroll
      for (int o = 1; o < 32; o <<= 1) { const unsigned u = (unsigned)__shfl_xor((int)v, o); v = u < v ? u : v; }
      const unsigned vm = (unsigned)__builtin_amdgcn_readfirstlane((int)v);
      const int n = vm > call * 128u ? (int)(vm - call * 128u) : 0;
      return n > NT ? NT : n;
    };
    int seen = 0;
    nb = -1;
    if (ACQ == 3) __builtin_amdgcn_fence(__ATOMIC_ACQUIRE, "");
    while (seen < NT) {
      const int need = nb + 3 < NT ? nb + 3 : NT;
      if (seen < need) {
        const unsigned long long t0 = __builtin_amdgcn_s_memrealtime();
        while ((seen = decode(ld_sys(sflag))) < need) {
          if (ld_agent(tmo) == call || __builtin_amdgcn_s_memrealtime() - t0 > DEADLINE) { st_agent(tmo, call); ok = false; seen = NT; break; }
          __builtin_amdgcn_s_sleep(1);
        }
        if (ACQ == 2 || (ACQ == 1 && nb == -1)) {
          __builtin_amdgcn_fence(__ATOMIC_ACQUIRE, "");
          waitv<0>();
        }
        if (nb == -1) STAMP(5);
        WAITL(0);
        BAR(); ++nb;
      } else if (seen < need + POLL_MARGIN) {
        unsigned v;
        asm volatile("global_load_dword %0, %1, off sc0 sc1" : "=v"(v) : "v"(sflag) : "memory");
        BAR(); ++nb;
        asm volatile("s_waitcnt vmcnt(0)" : "+v"(v) :: "memory");
        const int n = decode(v);
        if (n > seen) {
          if (ACQ == 2) {
            __builtin_amdgcn_fence(__ATOMIC_ACQUIRE, "");
            if (seen >= (nb + 3 < NT ? nb + 3 : NT)) { BAR(); ++nb; }
            waitv<0>();
          }
          seen = n;
        }
      } else {
        BAR(); ++nb;
      }
    }
    STAMP(7);
  } else if (tl.kind == 0) {
    while (nb < NT / 2) { BAR(); ++nb; }
    ok = wait_words(myws + WS_READY + tl.owner * 32, true, call, tmo);
  }
  while (nb < NT - 1) { BAR(); ++nb; }
  if (ld_agent(tmo) == call) ok = false;
  if (lane == 0) lds_wr(lds, C_GO, call * 2u + (ok ? 0u : 1u));
  WAITL(0);
  while (nb <= NT) { BAR(); ++nb; }
}

__device__ __forceinline__ void copy_group(const Params& p, LAS char* lds, int c, int wid, int lane) {
  unsigned* myws = p.ws[p.rank];
  unsigned* tmo = myws + WS_TMO;
  const int b = blockIdx.x;
  const unsigned cnt = ld_agent(myws + WS_CNT + b);
  const int gw = wid * 16 + c;
  const auto rx = __builtin_amdgcn_make_buffer_rsrc((void*)p.x, 0, ML * KD * 2, 0x00020000);
  const unsigned xv = (unsigned)((AROWS + (lane >> 3)) * KD * 2 + (((lane & 7) ^ (((gw & 1) * 4 + (lane >> 4)) & 7)) * 16));
  auto load = [&](int k) { const int u = k * 128 + gw; return __builtin_amdgcn_raw_buffer_load_b128(rx, xv, (unsigned)((u % BRP) * (8 * KD * 2) + (u / BRP) * 128), 0); };
  u32x4 d[NUNIT];
  d[0] = load(0);
  d[1] = load(1);
  char* dbase[NR - 1];
  unsigned dslot[NR - 1];
#pragma unroll
  for (int i = 0; i < NR - 1; ++i) {
    const int j = __builtin_amdgcn_readfirstlane((i + gw) % (NR - 1) + 1);
    dbase[i] = (char*)p.ws[__builtin_amdgcn_readfirstlane((p.rank + NR - j) & 7)];
    dslot[i] = (unsigned)((j - 1) * BROWS * 128);
  }
  if (lane == 0) { lds_wr(lds, wid * 4, (unsigned)(gw / BRP)); if (wid == 0) lds_wr(lds, 64, 0u); }
  WAITL(0);
  const unsigned call = (unsigned)__builtin_amdgcn_readfirstlane((int)cnt) + 1u;
  BAR();
  const unsigned long long tstart = __builtin_amdgcn_s_memrealtime();
  STAMP(1);
  if (wid == 0 && lane == 0) st_agent(myws + WS_CNT + b, call);
  if (gw == 127 && lane < NR && lane != p.rank) st_sys(p.ws[lane] + WS_READY + p.rank * 32, call);
  const size_t xo = WS_XBUF + (call & 1) * XPAR_BYTES;
  unsigned* pflag = p.ws[(p.rank + NR - 1 - (lane < 7 ? lane : 0)) & 7] + WS_XFLAG + (lane < 7 ? lane : 0) * 32 + c;
  auto finish = [&](int k) {
    if (k == 0) STAMP(3);
    const int nxt = k + 1 < NUNIT ? ((k + 1) * 128 + gw) / BRP : NT;
    if (lane == 0) lds_wr(lds, wid * 4, (unsigned)nxt);
    unsigned v = lds_rd(lds, (lane & 7) * 4);
#pragma unroll
    for (int o = 1; o < 8; o <<= 1) { const unsigned u = (unsigned)__shfl_xor((int)v, o); v = u < v ? u : v; }
    const int pre = __builtin_amdgcn_readfirstlane((int)v);
    const int lvl = pre < 4 ? pre : (pre < 12 ? (pre & ~1) : (pre & ~3));
    unsigned old = 0;
    if (lane == 0) old = __hip_atomic_fetch_max((LAS unsigned*)(lds + 64), (unsigned)lvl, __ATOMIC_RELAXED, __HIP_MEMORY_SCOPE_WORKGROUP);
    old = (unsigned)__builtin_amdgcn_readfirstlane((int)old);
    if ((unsigned)lvl > old && lane < 7)
      __hip_atomic_fetch_max(pflag, call * 128u + (unsigned)lvl, __ATOMIC_RELAXED, __HIP_MEMORY_SCOPE_SYSTEM);
  };
  bool pending = false;
  long long tissue = 0;
#pragma unroll
  for (int k = 0; k < NUNIT; ++k) {
    const int u = k * 128 + gw;
    for (;;) {
      const long long now = (long long)(__builtin_amdgcn_s_memrealtime() - tstart) * 10;
      if (now * BRP >= (long long)(u - PUSH_LEAD * BRP) * TILE_NS) break;
      if (pending && now - tissue > ACK_NS) { waitv<0>(); finish(k - 1); pending = false; }
      else __builtin_amdgcn_s_sleep(2);
    }
    const unsigned so = (unsigned)(u / BRP) * (unsigned)XT_BYTES + (unsigned)(u % BRP) * 1024u;
#pragma unroll
    for (int i = 0; i < NR - 1; ++i) {
      const auto rd = __builtin_amdgcn_make_buffer_rsrc(dbase[i] + xo, 0, (int)XPAR_BYTES, 0x00020000);
      __builtin_amdgcn_raw_buffer_store_b128(d[k], rd, (unsigned)(lane * 16), so + dslot[i], 17);
    }
    tissue = (long long)(__builtin_amdgcn_s_memrealtime() - tstart) * 10;
    if (k == 0) STAMP(2);
    if (pending) { waitv<7>(); finish(k - 1); }
    pending = true;
    if (k + 2 < NUNIT) d[k + 2] = load(k + 2);
  }
  waitv<0>();
  finish(NUNIT - 1);
  STAMP(4);
  if (c == 15) {
    if (wid != p.rank) {
      const bool fine = wait_words(myws + WS_DONE + wid * 64 + lane, true, call, tmo);
      __builtin_amdgcn_fence(__ATOMIC_ACQUIRE, "");
      if (!fine && lane == 0) p.out[p.rank][(size_t)(wid * ML) * NCOL] = 0x7fc0;
    }
    STAMP(6);
  }
}

__global__ __launch_bounds__(512, 2) void gemm_a2a(Params p) {
  extern __shared__ __attribute__((aligned(16))) char smem[];
  LAS char* lds = (LAS char*)smem;
  const int wid = __builtin_amdgcn_readfirstlane(threadIdx.x >> 6);
  const int lane = threadIdx.x & 63;
  const int b = blockIdx.x;
  const int xcd = b & 7, idx = b >> 3;
  STAMP(0);
  if (idx < 2) { copy_group(p, lds, xcd * 2 + idx, wid, lane); return; }
  Tile tl;
  constexpr int RMIX = XROWS / NM;
  int rt = 0;
  if (idx < 16) {
    const int a = xcd * 14 + idx - 2;
    tl.kind = 0; tl.owner = (p.rank + (AMAP ? a % 7 : a >> 4) + 1) & 7; tl.col0 = (AMAP ? a / 7 : a & 15) * 128;
    tl.nrow0 = 0; tl.gathered = false; tl.koff = KSKEW * xcd; tl.ai = idx - 2;
  } else {
    const int e = idx - 16;
    rt = (xcd >> 1) * 4 + (e >> 2);
    tl.kind = 1; tl.owner = p.rank; tl.col0 = ((xcd & 1) * 4 + (e & 3)) * 256;
    tl.nrow0 = rt * NM; tl.gathered = rt <= RMIX; tl.koff = 0; tl.ai = 0;
  }
  if (wid < 4) {
    if (tl.kind == 0) mfma_a<MFA>(p, tl, lds, wid, lane);
    else mfma_n(p, tl, lds, wid, lane);
  } else if (wid == 7) {
    ctrl_wave(p, tl, lds, lane);
  } else {
#define LOADERS(MF, NIMG, SW, PAIR, NWS, AUX) do { if (wid == 4) loader<MF, NIMG, SW, PAIR, 0, NWS, AUX>(p, tl, lds, lane); \
    else if (wid == 5) loader<MF, NIMG, SW, PAIR, 1, NWS, AUX>(p, tl, lds, lane); else loader<MF, NIMG, SW, PAIR, 2, NWS, AUX>(p, tl, lds, lane); } while (0)
    if (tl.kind == 0) LOADERS(MFA, 1, SWA, WPAIR != 0, 0, WAUX);
    else if (rt < RMIX) LOADERS(MFN, 2, 3, false, 4 * MFN, 0);
    else if (rt == RMIX) LOADERS(MFN, 2, 3, false, (XROWS - RMIX * NM) / 8, 0);
    else LOADERS(MFN, 2, 3, false, 0, 0);
  }
}

extern "C" void kernel_launch(int rank, int nranks, ncclComm_t comm,
                              hipStream_t stream,
                              void* const* d_in, const int* in_sizes, int n_in,
                              void* d_out, int out_size,
                              const int* in_dtypes, int out_dtype,
                              void* const* d_out_all,
                              void* const* d_ws_all, size_t ws_bytes) {
  if (nranks != NR || (unsigned)rank >= (unsigned)NR || ws_bytes < WS_NEED) { fprintf(stderr, "unexpected configuration\n"); return; }
  Params p;
  p.x = (const unsigned short*)d_in[0];
  p.w = (const unsigned short*)d_in[1];
  for (int i = 0; i < NR; ++i) { p.out[i] = (unsigned short*)d_out_all[i]; p.ws[i] = (unsigned*)d_ws_all[i]; }
  p.rank = rank;
  static bool init[NR] = {};
  if (!init[rank]) {
    HIP_CHECK(hipFuncSetAttribute((const void*)gemm_a2a, hipFuncAttributeMaxDynamicSharedMemorySize, LDS_BYTES));
    init[rank] = true;
  }
  gemm_a2a<<<NBLK, 512, LDS_BYTES, stream>>>(p);
  HIP_CHECK(hipGetLastError());
}
```

```cpp
#include <hip/hip_runtime.h>
#include <rccl/rccl.h>
#include <cstdio>
#include <stdint.h>

#define HIP_CHECK(expr) do { hipError_t _e = (expr); if (_e != hipSuccess) { \
    fprintf(stderr, "HIP error %s at %s:%d\n", hipGetErrorString(_e), __FILE__, __LINE__); \
    return; } } while (0)

#define LAS __attribute__((address_space(3)))
typedef short s16x4 __attribute__((ext_vector_type(4)));
typedef __bf16 bf16x2 __attribute__((ext_vector_type(2)));
typedef __bf16 bf16x8 __attribute__((ext_vector_type(8)));
typedef float f32x4 __attribute__((ext_vector_type(4)));
typedef float f32x2 __attribute__((ext_vector_type(2)));
typedef unsigned u32x4 __attribute__((ext_vector_type(4)));

constexpr int NR = 8, ML = 512, KD = 4096, ND = 16384, NCOL = ND / NR;
constexpr int BK = 64, NT = KD / BK;
constexpr int MFA = 7;
constexpr int MFN = 5;
constexpr int AROWS = MFA * 32, BROWS = ML - AROWS;
constexpr int BRP = BROWS / 8;
constexpr int XROWS = 7 * BROWS;
constexpr int NROWS = XROWS + ML;
constexpr int NM = MFN * 32;
constexpr int WODD = 8192 + 128;
constexpr int WIMG = WODD + 8192;
#define WPAIR 1
constexpr int SWA = WPAIR ? 4 : 3;
constexpr int LDS_CTL = 3 * (2 * WIMG + MFN * 4096);
static_assert(SWA * WIMG + 3 * MFA * 4096 + 3072 <= LDS_CTL, "LDS");
constexpr int C_GO = LDS_CTL;
constexpr int C_CALL = LDS_CTL + 16;
constexpr int LDS_BYTES = LDS_CTL + 64;
static_assert(LDS_BYTES <= 163840, "LDS");
constexpr int NBLK = 256;
constexpr unsigned long long DEADLINE = 50000000ull;
constexpr int WS_TMO = 32, WS_READY = 64  , WS_CNT = 512  , WS_DONE = 1024  ;
constexpr int WS_XFLAG = 2048;
constexpr size_t WS_XBUF = 1u << 20;
constexpr size_t XT_BYTES = (size_t)XROWS * 128;
constexpr size_t XPAR_BYTES = XT_BYTES * NT;
constexpr size_t WS_NEED = WS_XBUF + 2 * XPAR_BYTES;
constexpr int NUNIT = BRP * NT / 128;
#define POLL_MARGIN 4
#define TILE_NS 640
#define PUSH_LEAD 1
#define ACK_NS 800
#define KSKEW 8
#define XPD 4
#define ACQ 2
#define KSTRIDE 11
#define NAUX 16
#define XWARM 1
#define AMAP 1
#define AWIN 2
#define WAUX 2

struct Params {
  const unsigned short* x;
  const unsigned short* w;
  unsigned short* out[NR];
  unsigned* ws[NR];
  int rank;
};

#define WAITL(n) asm volatile("s_waitcnt lgkmcnt(" #n ")" ::: "memory")
template <int N> __device__ __forceinline__ void waitv() { asm volatile("s_waitcnt vmcnt(%0)" :: "n"(N) : "memory"); }
#define BAR() __builtin_amdgcn_s_barrier()
#define SCHED() __builtin_amdgcn_sched_barrier(0)
#define CFENCE() asm volatile("" ::: "memory")
#define STAMP(i) do {} while (0)
#define TRACE_DECL do {} while (0)
#define TRACE(t) do {} while (0)
#define TRACE_OUT do {} while (0)

__device__ __forceinline__ bf16x8 ldx(LAS const char* p) { return *(LAS const bf16x8*)p; }
__device__ __forceinline__ s16x4 ldw(LAS const char* p) { return __builtin_amdgcn_ds_read_tr16_b64_v4i16((LAS s16x4*)p); }
__device__ __forceinline__ unsigned pk(float a, float b) { f32x2 v = {a, b}; return __builtin_bit_cast(unsigned, __builtin_convertvector(v, bf16x2)); }
__device__ __forceinline__ unsigned ld_sys(const unsigned* p) { return __hip_atomic_load(p, __ATOMIC_RELAXED, __HIP_MEMORY_SCOPE_SYSTEM); }
__device__ __forceinline__ unsigned ld_agent(const unsigned* p) { return __hip_atomic_load(p, __ATOMIC_RELAXED, __HIP_MEMORY_SCOPE_AGENT); }
__device__ __forceinline__ void st_sys(unsigned* p, unsigned v) { __hip_atomic_store(p, v, __ATOMIC_RELAXED, __HIP_MEMORY_SCOPE_SYSTEM); }
__device__ __forceinline__ void st_agent(unsigned* p, unsigned v) { __hip_atomic_store(p, v, __ATOMIC_RELAXED, __HIP_MEMORY_SCOPE_AGENT); }
__device__ __forceinline__ unsigned lds_rd(LAS char* lds, int off) { return *(LAS volatile unsigned*)(lds + off); }
__device__ __forceinline__ void lds_wr(LAS char* lds, int off, unsigned v) { *(LAS volatile unsigned*)(lds + off) = v; }
__device__ __forceinline__ bf16x8 wfrag(s16x4 lo, s16x4 hi) { return __builtin_bit_cast(bf16x8, __builtin_shufflevector(lo, hi, 0, 1, 2, 3, 4, 5, 6, 7)); }
__device__ __forceinline__ u32x4 pack2(f32x4 a, f32x4 c, bool bad) {
  const unsigned a0 = pk(a[0], a[1]), a1 = pk(a[2], a[3]), c0 = pk(c[0], c[1]), c1 = pk(c[2], c[3]);
  auto r0 = __builtin_amdgcn_permlane16_swap(a0, c0, false, false);
  auto r1 = __builtin_amdgcn_permlane16_swap(a1, c1, false, false);
  return bad ? (u32x4){0x7fc07fc0u, 0x7fc07fc0u, 0x7fc07fc0u, 0x7fc07fc0u} : (u32x4){r0[0], r1[0], r0[1], r1[1]};
}
__device__ __forceinline__ unsigned out_voff(int lane) { const int fr = lane & 15, fq = lane >> 4; return (unsigned)((fr * NCOL + (fq & 1) * 16 + (fq >> 1) * 8) * 2); }
__host__ __device__ constexpr int wpiece(int pc) { return ((pc >> 1) & 1) * WODD + (pc >> 2) * 2048 + (pc & 1) * 1024; }

struct Tile {
  int kind;
  int owner;
  int col0;
  int nrow0;
  bool gathered;
  int koff;
  int ai;
};

template <int MF, int NIMG, int SW, bool PAIR, int H, int NWS, int AUXW>
__device__ __forceinline__ void loader(const Params& p, const Tile& tl, LAS char* lds, int lane) {
  constexpr int NWP = 16 * NIMG;
  constexpr int WSLOT = NIMG * WIMG, XSLOT = MF * 4096, LX = SW * WSLOT;
  constexpr int NW_ = (NWP - H + 2) / 3;
  constexpr int X0 = (H + 3 * NW_) - NWP;
  constexpr int NX_ = (4 * MF - X0 + 2) / 3;
  constexpr bool XWS = NWS > 0;
  static_assert(2 * NW_ + 2 * NX_ <= 63, "");
  const auto rw = __builtin_amdgcn_make_buffer_rsrc((void*)p.w, 0, KD * ND * 2, 0x00020000);
  const auto rx = __builtin_amdgcn_make_buffer_rsrc((void*)p.x, 0, ML * KD * 2, 0x00020000);
  const unsigned wv = (unsigned)((lane >> 4) * ND * 2 + (((lane & 15) ^ ((lane >> 4) << 1)) * 16));
  unsigned xv[2];
#pragma unroll
  for (int v = 0; v < 2; ++v) xv[v] = (unsigned)((lane >> 3) * KD * 2 + (((lane & 7) ^ ((v * 4 + (lane >> 4)) & 7)) * 16));
  const unsigned iv = (unsigned)(lane * 16);
  const unsigned wbase = (unsigned)((tl.owner * NCOL + tl.col0) * 2);
  const unsigned xbase = tl.kind == 0 ? 0u : (unsigned)((tl.nrow0 - XROWS) * KD * 2);
  int ws_ = 0, xs_ = 0;
  auto kmap = [&](int t) -> unsigned { return PAIR ? (unsigned)(((((t >> 1) * KSTRIDE) << 1) + (t & 1) + tl.koff) & (NT - 1)) : (unsigned)((t + tl.koff) & (NT - 1)); };
  auto issue_w = [&](int t) {
    const unsigned tb = wbase + kmap(t) * (BK * ND * 2);
#pragma unroll
    for (int i = 0; i < NW_; ++i) {
      const int pc = H + 3 * i, img = pc >> 4, q = pc & 15;
      __builtin_amdgcn_raw_ptr_buffer_load_lds(rw, (LAS void*)(lds + ws_ * WSLOT + img * WIMG + wpiece(q)), 16, wv, tb + q * (4 * ND * 2) + img * 256, 0, AUXW);
    }
    ws_ = ws_ == SW - 1 ? 0 : ws_ + 1;
  };
  auto issue_xl = [&](int t) {
    const unsigned tb = xbase + kmap(t) * 128u;
#pragma unroll
    for (int i = 0; i < NX_; ++i) {
      const int pc = X0 + 3 * i;
      if (pc >= NWS) __builtin_amdgcn_raw_ptr_buffer_load_lds(rx, (LAS void*)(lds + LX + xs_ * XSLOT + pc * 1024), 16, xv[pc & 1], tb + pc * (8 * KD * 2), 0, 0);
    }
  };
  issue_w(0);
  issue_xl(0);
  unsigned call = 0;
  if (XWS) {
    BAR();
    call = (unsigned)__builtin_amdgcn_readfirstlane((int)lds_rd(lds, C_CALL));
  }
  const auto ri = __builtin_amdgcn_make_buffer_rsrc((char*)p.ws[p.rank] + WS_XBUF + (call & 1) * XPAR_BYTES, 0, (int)XPAR_BYTES, 0x00020000);
  const unsigned ibase = (unsigned)(tl.nrow0 * 128);
  auto issue_xi = [&](int t) {
#pragma unroll
    for (int i = 0; i < NX_; ++i) {
      const int pc = X0 + 3 * i;
      if (pc < NWS) __builtin_amdgcn_raw_ptr_buffer_load_lds(ri, (LAS void*)(lds + LX + xs_ * XSLOT + pc * 1024), 16, iv, ibase + (unsigned)t * (unsigned)XT_BYTES + pc * 1024, 0, 0);
    }
    xs_ = xs_ == 2 ? 0 : xs_ + 1;
  };
  issue_xi(0);
  issue_w(1);
  issue_xl(1);
  issue_xi(1);
  waitv<NW_ + NX_>();
  BAR();
  if (PAIR) {
    constexpr int NPF = (XPD && H > 0) ? 2 : 0;
    auto prefetch = [&](int t) {
      const unsigned tb = kmap(t) * 128u + (unsigned)tl.ai * (16 * KD * 2);
#pragma unroll
      for (int i = 0; i < NPF; ++i)
        __builtin_amdgcn_raw_ptr_buffer_load_lds(rx, (LAS void*)(lds + LX + 3 * XSLOT + H * 1024), 16, xv[0], tb + i * (8 * KD * 2), 0, 0);
    };
    for (int t = 0; t < NT - 2; t += 2) {
      issue_w(t + 2);
      issue_w(t + 3);
      prefetch(t + XPD + H - 1);
      issue_xl(t + 2);
      issue_xi(t + 2);
      waitv<2 * NW_ + NPF + NX_>();
      BAR();
      issue_xl(t + 3);
      issue_xi(t + 3);
      waitv<NX_>();
      BAR();
    }
  } else {
    for (int t = 0; t < NT - 2; ++t) {
      issue_w(t + 2);
      issue_xl(t + 2);
      issue_xi(t + 2);
      waitv<NW_ + NX_>();
      BAR();
    }
  }
  waitv<0>();
  BAR();
  BAR();
}

#define NOP_ do {} while (0)
template <int MF>
__device__ __forceinline__ void mfma_a(const Params& p, const Tile& tl, LAS char* lds, int wid, int lane) {
  constexpr int XSLOT = MF * 4096, LX = SWA * WIMG;
  const int wr = wid >> 1, wc = wid & 1;
  const int fr = lane & 15, fq = lane >> 4;
  const int q = (lane & 15) >> 2, pp = lane & 3;
  const unsigned xa0 = (unsigned)(LX + (wr * MF * 16 + fr) * 128 + ((fq ^ ((fr >> 1) & 7)) * 16));
  const unsigned xa1 = xa0 ^ 64u;
  unsigned wa[4];
#pragma unroll
  for (int nf = 0; nf < 4; ++nf)
    wa[nf] = (unsigned)((fq & 1) * WODD + (fq >> 1) * 2048 + q * 256 + wc * 128 + ((nf ^ q) * 32) + 8 * pp);
  f32x4 acc[MF][4];
#pragma unroll
  for (int i = 0; i < MF; ++i)
#pragma unroll
    for (int j = 0; j < 4; ++j) acc[i][j] = (f32x4){0.f, 0.f, 0.f, 0.f};
  bf16x8 Xf[MF];
  s16x4 Wl[2][4], Wh[2][4];
#define RDW(bu, sb, ks, i) do { if ((i) & 1) Wh[bu][(i) >> 1] = ldw((sb) + wa[(i) >> 1] + (ks) * 4096 + 1024); \
    else Wl[bu][(i) >> 1] = ldw((sb) + wa[(i) >> 1] + (ks) * 4096); } while (0)
#define MMA1(mf, bu, nf) do { acc[mf][nf] = __builtin_amdgcn_mfma_f32_16x16x32_bf16(wfrag(Wl[bu][nf], Wh[bu][nf]), Xf[mf], acc[mf][nf], 0, 0, 0); SCHED(); } while (0)
#define GRP(mf, bu, F0, F1, F2) do { MMA1(mf, bu, 0); F0; SCHED(); MMA1(mf, bu, 1); F1; SCHED(); MMA1(mf, bu, 2); F2; SCHED(); MMA1(mf, bu, 3); } while (0)
#define RDX(i, sb, xa) Xf[i] = ldx((sb) + (xa) + (i) * 2048)
  BAR();
  STAMP(1);
#pragma unroll
  for (int i = 0; i < 8; ++i) RDW(0, lds, 0, i);
#pragma unroll
  for (int mf = 0; mf < MF - 1; ++mf) RDX(mf, lds, xa0);
  int sw = 0, sx = 0;
  TRACE_DECL;
  for (int t = 0; t < NT; ++t) {
    TRACE(t);
    LAS const char* wb = lds + sw * WIMG;
    LAS const char* xb = lds + sx * XSLOT;
    sw = (sw == SWA - 1) ? 0 : sw + 1;
    sx = (sx == 2) ? 0 : sx + 1;
    LAS const char* wn = lds + sw * WIMG;
    LAS const char* xn = lds + sx * XSLOT;
    GRP(0, 0, RDX(MF - 1, xb, xa0), RDW(1, wb, 1, 0), RDW(1, wb, 1, 1));
    GRP(1, 0, RDX(0, xb, xa1), RDW(1, wb, 1, 2), RDW(1, wb, 1, 3));
    GRP(2, 0, RDX(1, xb, xa1), RDW(1, wb, 1, 4), NOP_);
    GRP(3, 0, RDX(2, xb, xa1), RDW(1, wb, 1, 5), NOP_);
    GRP(4, 0, RDX(3, xb, xa1), RDW(1, wb, 1, 6), NOP_);
    GRP(5, 0, RDX(4, xb, xa1), RDW(1, wb, 1, 7), NOP_);
#pragma unroll
    for (int g = 6; g < MF; ++g) GRP(g, 0, RDX(g - 1, xb, xa1), NOP_, NOP_);
    GRP(0, 1, RDX(MF - 1, xb, xa1), NOP_, NOP_);
    GRP(1, 1, NOP_, NOP_, NOP_);
    WAITL(0);
    BAR();
    CFENCE();
    SCHED();
    GRP(2, 1, RDW(0, wn, 0, 0), RDW(0, wn, 0, 1), RDX(0, xn, xa0));
    GRP(3, 1, RDW(0, wn, 0, 2), RDW(0, wn, 0, 3), RDX(1, xn, xa0));
    GRP(4, 1, RDW(0, wn, 0, 4), RDW(0, wn, 0, 5), RDX(2, xn, xa0));
    GRP(5, 1, RDW(0, wn, 0, 6), RDW(0, wn, 0, 7), RDX(3, xn, xa0));
    GRP(6, 1, RDX(4, xn, xa0), RDX(5, xn, xa0), NOP_);
#pragma unroll
    for (int g = 7; g < MF; ++g) GRP(g, 1, RDX(g - 1, xn, xa0), NOP_, NOP_);
  }
  STAMP(2);
  TRACE_OUT;
  const unsigned go = (unsigned)__builtin_amdgcn_readfirstlane((int)lds_rd(lds, C_GO));
  const bool bad = (go & 1u) != 0u;
  const auto ro = __builtin_amdgcn_make_buffer_rsrc((void*)p.out[tl.owner], 0, ML * NR * NCOL * 2, 0x00020000);
  const unsigned vo = out_voff(lane);
  const unsigned so = (unsigned)(((p.rank * ML + wr * MF * 16) * NCOL + tl.col0 + wc * 64) * 2);
#pragma unroll
  for (int mf = 0; mf < MF; ++mf) {
#pragma unroll
    for (int np = 0; np < 2; ++np)
      __builtin_amdgcn_raw_buffer_store_b128(pack2(acc[mf][2 * np], acc[mf][2 * np + 1], bad), ro, vo + np * 64, so + mf * (16 * NCOL * 2), 17);
    waitv<AWIN>();
  }
  waitv<0>();
  if (lane == 0) st_sys(p.ws[tl.owner] + WS_DONE + p.rank * 64 + (tl.col0 >> 7) * 4 + wid, go >> 1);
  STAMP(3);
#undef RDW
#undef MMA1
#undef GRP
#undef RDX
}

__device__ __forceinline__ void mfma_n(const Params& p, const Tile& tl, LAS char* lds, int wid, int lane) {
  constexpr int MF = MFN;
  constexpr int WSLOT = 2 * WIMG, XSLOT = MF * 4096, LX = 3 * WSLOT;
  const int wr = wid >> 1, wc = wid & 1;
  const int fr = lane & 15, fq = lane >> 4;
  const int q = (lane & 15) >> 2, pp = lane & 3;
  const unsigned xa0 = (unsigned)(LX + (wr * MF * 16 + fr) * 128 + ((fq ^ ((fr >> 1) & 7)) * 16));
  const unsigned xa1 = xa0 ^ 64u;
  unsigned wa[4];
#pragma unroll
  for (int nf = 0; nf < 4; ++nf)
    wa[nf] = (unsigned)(wc * WIMG + (fq & 1) * WODD + (fq >> 1) * 2048 + q * 256 + ((nf ^ q) * 32) + 8 * pp);
  f32x4 acc[MF][8];
#pragma unroll
  for (int i = 0; i < MF; ++i)
#pragma unroll
    for (int j = 0; j < 8; ++j) acc[i][j] = (f32x4){0.f, 0.f, 0.f, 0.f};
  bf16x8 Xf[2][MF];
  s16x4 Wl[8], Wh[8];
#define RWL(nf, sb, ks) Wl[nf] = ldw((sb) + wa[(nf) & 3] + ((nf) >> 2) * 128 + (ks) * 4096)
#define RWH(nf, sb, ks) Wh[nf] = ldw((sb) + wa[(nf) & 3] + ((nf) >> 2) * 128 + (ks) * 4096 + 1024)
#define RDX(bu, i, sb, xa) Xf[bu][i] = ldx((sb) + (xa) + (i) * 2048)
#define MMA1(nf, bu, mf) do { acc[mf][nf] = __builtin_amdgcn_mfma_f32_16x16x32_bf16(wfrag(Wl[nf], Wh[nf]), Xf[bu][mf], acc[mf][nf], 0, 0, 0); SCHED(); } while (0)
#define GRP(nf, bu, F0, F1, F2, F3, F4) do { MMA1(nf, bu, 0); F0; SCHED(); MMA1(nf, bu, 1); F1; SCHED(); MMA1(nf, bu, 2); F2; SCHED(); \
    MMA1(nf, bu, 3); F3; SCHED(); MMA1(nf, bu, 4); F4; SCHED(); } while (0)
  if (tl.gathered) {
    const auto rx = __builtin_amdgcn_make_buffer_rsrc((void*)p.x, 0, ML * KD * 2, 0x00020000);
    const unsigned wo = (unsigned)(((tl.nrow0 / NM) * 8 + (tl.col0 >> 8)) * 4 + wid) * 10240u;
#pragma unroll
    for (int i = 0; i < 10; ++i) { u32x4 v = __builtin_amdgcn_raw_buffer_load_b128(rx, (unsigned)(lane * 16), wo + i * 1024, 0); asm volatile("" :: "v"(v)); }
    BAR();
  }
  BAR();
  STAMP(1);
#pragma unroll
  for (int mf = 0; mf < MF; ++mf) RDX(0, mf, lds, xa0);
#pragma unroll
  for (int nf = 0; nf < 7; ++nf) { RWL(nf, lds, 0); RWH(nf, lds, 0); }
  int st = 0;
  TRACE_DECL;
  for (int t = 0; t < NT; ++t) {
    TRACE(t);
    LAS const char* wb = lds + st * WSLOT;
    LAS const char* xb = lds + st * XSLOT;
    st = (st == 2) ? 0 : st + 1;
    LAS const char* wn = lds + st * WSLOT;
    LAS const char* xn = lds + st * XSLOT;
    GRP(0, 0, RWL(7, wb, 0), RWH(7, wb, 0), RDX(1, 0, xb, xa1), NOP_, NOP_);
    GRP(1, 0, RWL(0, wb, 1), RWH(0, wb, 1), RDX(1, 1, xb, xa1), NOP_, NOP_);
    GRP(2, 0, RWL(1, wb, 1), RWH(1, wb, 1), RDX(1, 2, xb, xa1), NOP_, NOP_);
    GRP(3, 0, RWL(2, wb, 1), RWH(2, wb, 1), RDX(1, 3, xb, xa1), NOP_, NOP_);
    GRP(4, 0, RWL(3, wb, 1), RWH(3, wb, 1), RDX(1, 4, xb, xa1), NOP_, NOP_);
    GRP(5, 0, RWL(4, wb, 1), RWH(4, wb, 1), NOP_, NOP_, NOP_);
    GRP(6, 0, RWL(5, wb, 1), RWH(5, wb, 1), NOP_, NOP_, NOP_);
    GRP(7, 0, RWL(6, wb, 1), RWH(6, wb, 1), NOP_, NOP_, NOP_);
    GRP(0, 1, RWL(7, wb, 1), RWH(7, wb, 1), NOP_, NOP_, NOP_);
    GRP(1, 1, NOP_, NOP_, NOP_, NOP_, NOP_);
    WAITL(0);
    BAR();
    CFENCE();
    SCHED();
    GRP(2, 1, RWL(0, wn, 0), RWH(0, wn, 0), RDX(0, 0, xn, xa0), RWL(1, wn, 0), RWH(1, wn, 0));
    GRP(3, 1, RWL(2, wn, 0), RWH(2, wn, 0), RDX(0, 1, xn, xa0), RDX(0, 2, xn, xa0), NOP_);
    GRP(4, 1, RWL(3, wn, 0), RWH(3, wn, 0), RDX(0, 3, xn, xa0), RDX(0, 4, xn, xa0), NOP_);
    GRP(5, 1, RWL(4, wn, 0), RWH(4, wn, 0), NOP_, NOP_, NOP_);
    GRP(6, 1, RWL(5, wn, 0), RWH(5, wn, 0), NOP_, NOP_, NOP_);
    GRP(7, 1, RWL(6, wn, 0), RWH(6, wn, 0), NOP_, NOP_, NOP_);
  }
  STAMP(2);
  TRACE_OUT;
  const unsigned go = (unsigned)__builtin_amdgcn_readfirstlane((int)lds_rd(lds, C_GO));
  const bool bad = (go & 1u) != 0u;
  const auto ro = __builtin_amdgcn_make_buffer_rsrc((void*)p.out[p.rank], 0, ML * NR * NCOL * 2, 0x00020000);
  const unsigned vo = out_voff(lane);
  const unsigned so = (unsigned)((tl.col0 + wc * 128) * 2);
#pragma unroll
  for (int mf = 0; mf < MF; ++mf) {
    const int nr = tl.nrow0 + wr * MF * 16 + mf * 16;
    if (nr < NROWS) {
      const int orow = nr < XROWS ? ((p.rank + 1 + nr / BROWS) & 7) * ML + AROWS + nr % BROWS : p.rank * ML + nr - XROWS;
#pragma unroll
      for (int np = 0; np < 4; ++np)
        __builtin_amdgcn_raw_buffer_store_b128(pack2(acc[mf][2 * np], acc[mf][2 * np + 1], bad), ro, vo + np * 64, so + (unsigned)orow * (NCOL * 2), NAUX);
    }
  }
  STAMP(3);
#undef RWL
#undef RWH
#undef RDX
#undef MMA1
#undef GRP
}

__device__ __forceinline__ bool wait_words(const unsigned* word, bool use, unsigned call, unsigned* tmo) {
  const unsigned long long t0 = __builtin_amdgcn_s_memrealtime();
  unsigned spins = 0;
  for (;;) {
    const unsigned v = ld_sys(word);
    if (__all(!use || v >= call)) return true;
    if ((++spins & 63u) == 0u && (ld_agent(tmo) == call || __builtin_amdgcn_s_memrealtime() - t0 > DEADLINE)) { st_agent(tmo, call); return false; }
    __builtin_amdgcn_s_sleep(1);
  }
}

__device__ __forceinline__ void ctrl_wave(const Params& p, const Tile& tl, LAS char* lds, int lane) {
  unsigned* myws = p.ws[p.rank];
  unsigned* tmo = myws + WS_TMO;
  const int b = blockIdx.x;
  const unsigned cnt = ld_agent(myws + WS_CNT + b);
  if (!tl.gathered) BAR();
  const unsigned call = (unsigned)__builtin_amdgcn_readfirstlane((int)cnt) + 1u;
  if (lane == 0) st_agent(myws + WS_CNT + b, call);
  bool ok = true;
  int nb = 1;
  if (tl.gathered) {
    if (lane == 0) lds_wr(lds, C_CALL, call);
    const int s0 = tl.nrow0 / BROWS;
    const int e1 = (tl.nrow0 + NM - 1) / BROWS;
    const int s1 = e1 < 7 ? e1 : 6;
    const unsigned* sflag = myws + WS_XFLAG + ((lane & 16) ? s1 : s0) * 32 + (lane & 15);
    auto decode = [&](unsigned v) -> int {
#pragma unroll
      for (int o = 1; o < 32; o <<= 1) { const unsigned u = (unsigned)__shfl_xor((int)v, o); v = u < v ? u : v; }
      const unsigned vm = (unsigned)__builtin_amdgcn_readfirstlane((int)v);
      const int n = vm > call * 128u ? (int)(vm - call * 128u) : 0;
      return n > NT ? NT : n;
    };
    int seen = 0;
    nb = -1;
    if (ACQ == 3) __builtin_amdgcn_fence(__ATOMIC_ACQUIRE, "");
    while (seen < NT) {
      const int need = nb + 3 < NT ? nb + 3 : NT;
      if (seen < need) {
        const unsigned long long t0 = __builtin_amdgcn_s_memrealtime();
        unsigned spins = 0;
        while ((seen = decode(ld_sys(sflag))) < need) {
          if ((++spins & 63u) == 0u && (ld_agent(tmo) == call || __builtin_amdgcn_s_memrealtime() - t0 > DEADLINE)) { st_agent(tmo, call); ok = false; seen = NT; break; }
          __builtin_amdgcn_s_sleep(1);
        }
        if (ACQ == 2 || (ACQ == 1 && nb == -1)) {
          __builtin_amdgcn_fence(__ATOMIC_ACQUIRE, "");
          waitv<0>();
        }
        if (nb == -1) STAMP(5);
        WAITL(0);
        BAR(); ++nb;
      } else if (seen < need + POLL_MARGIN) {
        unsigned v;
        asm volatile("global_load_dword %0, %1, off sc0 sc1" : "=v"(v) : "v"(sflag) : "memory");
        BAR(); ++nb;
        asm volatile("s_waitcnt vmcnt(0)" : "+v"(v) :: "memory");
        const int n = decode(v);
        if (n > seen) {
          if (ACQ == 2) {
            __builtin_amdgcn_fence(__ATOMIC_ACQUIRE, "");
            if (seen >= (nb + 3 < NT ? nb + 3 : NT)) { BAR(); ++nb; }
            waitv<0>();
          }
          seen = n;
        }
      } else {
        BAR(); ++nb;
      }
    }
    STAMP(7);
  } else if (tl.kind == 0) {
    while (nb < NT / 2) { BAR(); ++nb; }
    ok = wait_words(myws + WS_READY + tl.owner * 32, true, call, tmo);
  }
  while (nb < NT - 1) { BAR(); ++nb; }
  if (ld_agent(tmo) == call) ok = false;
  if (lane == 0) lds_wr(lds, C_GO, call * 2u + (ok ? 0u : 1u));
  WAITL(0);
  while (nb <= NT) { BAR(); ++nb; }
}

__device__ __forceinline__ void copy_group(const Params& p, LAS char* lds, int c, int wid, int lane) {
  unsigned* myws = p.ws[p.rank];
  unsigned* tmo = myws + WS_TMO;
  const int b = blockIdx.x;
  const unsigned cnt = ld_agent(myws + WS_CNT + b);
  const int gw = wid * 16 + c;
  const auto rx = __builtin_amdgcn_make_buffer_rsrc((void*)p.x, 0, ML * KD * 2, 0x00020000);
  const unsigned xv = (unsigned)((AROWS + (lane >> 3)) * KD * 2 + (((lane & 7) ^ (((gw & 1) * 4 + (lane >> 4)) & 7)) * 16));
  auto load = [&](int k) { const int u = k * 128 + gw; return __builtin_amdgcn_raw_buffer_load_b128(rx, xv, (unsigned)((u % BRP) * (8 * KD * 2) + (u / BRP) * 128), 0); };
  u32x4 d[NUNIT];
  d[0] = load(0);
  d[1] = load(1);
  char* dbase[NR - 1];
  unsigned dslot[NR - 1];
#pragma unroll
  for (int i = 0; i < NR - 1; ++i) {
    const int j = __builtin_amdgcn_readfirstlane((i + gw) % (NR - 1) + 1);
    dbase[i] = (char*)p.ws[__builtin_amdgcn_readfirstlane((p.rank + NR - j) & 7)];
    dslot[i] = (unsigned)((j - 1) * BROWS * 128);
  }
  if (lane == 0) { lds_wr(lds, wid * 4, (unsigned)(gw / BRP)); if (wid == 0) lds_wr(lds, 64, 0u); }
  WAITL(0);
  const unsigned call = (unsigned)__builtin_amdgcn_readfirstlane((int)cnt) + 1u;
  BAR();
  const unsigned long long tstart = __builtin_amdgcn_s_memrealtime();
  STAMP(1);
  if (wid == 0 && lane == 0) st_agent(myws + WS_CNT + b, call);
  if (gw == 127 && lane < NR && lane != p.rank) st_sys(p.ws[lane] + WS_READY + p.rank * 32, call);
  const size_t xo = WS_XBUF + (call & 1) * XPAR_BYTES;
  unsigned* pflag = p.ws[(p.rank + NR - 1 - (lane < 7 ? lane : 0)) & 7] + WS_XFLAG + (lane < 7 ? lane : 0) * 32 + c;
  auto finish = [&](int k) {
    if (k == 0) STAMP(3);
    const int nxt = k + 1 < NUNIT ? ((k + 1) * 128 + gw) / BRP : NT;
    if (lane == 0) lds_wr(lds, wid * 4, (unsigned)nxt);
    unsigned v = lds_rd(lds, (lane & 7) * 4);
#pragma unroll
    for (int o = 1; o < 8; o <<= 1) { const unsigned u = (unsigned)__shfl_xor((int)v, o); v = u < v ? u : v; }
    const int pre = __builtin_amdgcn_readfirstlane((int)v);
    const int lvl = pre < 4 ? pre : (pre < 12 ? (pre & ~1) : (pre & ~3));
    unsigned old = 0;
    if (lane == 0) old = __hip_atomic_fetch_max((LAS unsigned*)(lds + 64), (unsigned)lvl, __ATOMIC_RELAXED, __HIP_MEMORY_SCOPE_WORKGROUP);
    old = (unsigned)__builtin_amdgcn_readfirstlane((int)old);
    if ((unsigned)lvl > old && lane < 7)
      __hip_atomic_fetch_max(pflag, call * 128u + (unsigned)lvl, __ATOMIC_RELAXED, __HIP_MEMORY_SCOPE_SYSTEM);
  };
  bool pending = false;
  long long tissue = 0;
#pragma unroll
  for (int k = 0; k < NUNIT; ++k) {
    const int u = k * 128 + gw;
    for (;;) {
      const long long now = (long long)(__builtin_amdgcn_s_memrealtime() - tstart) * 10;
      if (now * BRP >= (long long)(u - PUSH_LEAD * BRP) * TILE_NS) break;
      if (pending && now - tissue > ACK_NS) { waitv<0>(); finish(k - 1); pending = false; }
      else __builtin_amdgcn_s_sleep(2);
    }
    const unsigned so = (unsigned)(u / BRP) * (unsigned)XT_BYTES + (unsigned)(u % BRP) * 1024u;
#pragma unroll
    for (int i = 0; i < NR - 1; ++i) {
      const auto rd = __builtin_amdgcn_make_buffer_rsrc(dbase[i] + xo, 0, (int)XPAR_BYTES, 0x00020000);
      __builtin_amdgcn_raw_buffer_store_b128(d[k], rd, (unsigned)(lane * 16), so + dslot[i], 17);
    }
    tissue = (long long)(__builtin_amdgcn_s_memrealtime() - tstart) * 10;
    if (k == 0) STAMP(2);
    if (pending) { waitv<7>(); finish(k - 1); }
    pending = true;
    if (k + 2 < NUNIT) d[k + 2] = load(k + 2);
  }
  waitv<0>();
  finish(NUNIT - 1);
  STAMP(4);
  if (c == 15) {
    if (wid != p.rank) {
      const bool fine = wait_words(myws + WS_DONE + wid * 64 + lane, true, call, tmo);
      __builtin_amdgcn_fence(__ATOMIC_ACQUIRE, "");
      if (!fine && lane == 0) p.out[p.rank][(size_t)(wid * ML) * NCOL] = 0x7fc0;
    }
    STAMP(6);
  }
}

__global__ __launch_bounds__(512, 2) void gemm_a2a(Params p) {
  extern __shared__ __attribute__((aligned(16))) char smem[];
  LAS char* lds = (LAS char*)smem;
  const int wid = __builtin_amdgcn_readfirstlane(threadIdx.x >> 6);
  const int lane = threadIdx.x & 63;
  const int b = blockIdx.x;
  const int xcd = b & 7, idx = b >> 3;
  STAMP(0);
  if (idx < 2) { copy_group(p, lds, xcd * 2 + idx, wid, lane); return; }
  Tile tl;
  constexpr int RMIX = XROWS / NM;
  int rt = 0;
  if (idx < 16) {
    const int a = xcd * 14 + idx - 2;
    tl.kind = 0; tl.owner = (p.rank + (AMAP ? a % 7 : a >> 4) + 1) & 7; tl.col0 = (AMAP ? a / 7 : a & 15) * 128;
    tl.nrow0 = 0; tl.gathered = false; tl.koff = KSKEW * xcd; tl.ai = idx - 2;
  } else {
    const int e = idx - 16;
    rt = (xcd >> 1) * 4 + (e >> 2);
    tl.kind = 1; tl.owner = p.rank; tl.col0 = ((xcd & 1) * 4 + (e & 3)) * 256;
    tl.nrow0 = rt * NM; tl.gathered = rt <= RMIX; tl.koff = 0; tl.ai = 0;
  }
  if (wid < 4) {
    if (tl.kind == 0) mfma_a<MFA>(p, tl, lds, wid, lane);
    else mfma_n(p, tl, lds, wid, lane);
  } else if (wid == 7) {
    ctrl_wave(p, tl, lds, lane);
  } else {
#define LOADERS(MF, NIMG, SW, PAIR, NWS, AUX) do { if (wid == 4) loader<MF, NIMG, SW, PAIR, 0, NWS, AUX>(p, tl, lds, lane); \
    else if (wid == 5) loader<MF, NIMG, SW, PAIR, 1, NWS, AUX>(p, tl, lds, lane); else loader<MF, NIMG, SW, PAIR, 2, NWS, AUX>(p, tl, lds, lane); } while (0)
    if (tl.kind == 0) LOADERS(MFA, 1, SWA, WPAIR != 0, 0, WAUX);
    else if (rt < RMIX) LOADERS(MFN, 2, 3, false, 4 * MFN, 0);
    else if (rt == RMIX) LOADERS(MFN, 2, 3, false, (XROWS - RMIX * NM) / 8, 0);
    else LOADERS(MFN, 2, 3, false, 0, 0);
  }
}

extern "C" void kernel_launch(int rank, int nranks, ncclComm_t comm,
                              hipStream_t stream,
                              void* const* d_in, const int* in_sizes, int n_in,
                              void* d_out, int out_size,
                              const int* in_dtypes, int out_dtype,
                              void* const* d_out_all,
                              void* const* d_ws_all, size_t ws_bytes) {
  if (nranks != NR || (unsigned)rank >= (unsigned)NR || ws_bytes < WS_NEED) { fprintf(stderr, "unexpected configuration\n"); return; }
  Params p;
  p.x = (const unsigned short*)d_in[0];
  p.w = (const unsigned short*)d_in[1];
  for (int i = 0; i < NR; ++i) { p.out[i] = (unsigned short*)d_out_all[i]; p.ws[i] = (unsigned*)d_ws_all[i]; }
  p.rank = rank;
  static bool init[NR] = {};
  if (!init[rank]) {
    HIP_CHECK(hipFuncSetAttribute((const void*)gemm_a2a, hipFuncAttributeMaxDynamicSharedMemorySize, LDS_BYTES));
    init[rank] = true;
  }
  gemm_a2a<<<NBLK, 512, LDS_BYTES, stream>>>(p);
  HIP_CHECK(hipGetLastError());
}
```

```cpp
#include <hip/hip_runtime.h>
#include <rccl/rccl.h>
#include <cstdio>
#include <stdint.h>

#define HIP_CHECK(expr) do { hipError_t _e = (expr); if (_e != hipSuccess) { \
    fprintf(stderr, "HIP error %s at %s:%d\n", hipGetErrorString(_e), __FILE__, __LINE__); \
    return; } } while (0)

#define LAS __attribute__((address_space(3)))
typedef short s16x4 __attribute__((ext_vector_type(4)));
typedef __bf16 bf16x2 __attribute__((ext_vector_type(2)));
typedef __bf16 bf16x8 __attribute__((ext_vector_type(8)));
typedef float f32x4 __attribute__((ext_vector_type(4)));
typedef float f32x2 __attribute__((ext_vector_type(2)));
typedef unsigned u32x4 __attribute__((ext_vector_type(4)));

constexpr int NR = 8, ML = 512, KD = 4096, ND = 16384, NCOL = ND / NR;
constexpr int BK = 64, NT = KD / BK;
constexpr int MFA = 7;
constexpr int MFN = 5;
constexpr int AROWS = MFA * 32, BROWS = ML - AROWS;
constexpr int BRP = BROWS / 8;
constexpr int XROWS = 7 * BROWS;
constexpr int NROWS = XROWS + ML;
constexpr int NM = MFN * 32;
constexpr int WODD = 8192 + 128;
constexpr int WIMG = WODD + 8192;
#define WPAIR 1
constexpr int SWA = WPAIR ? 4 : 3;
constexpr int LDS_CTL = 3 * (2 * WIMG + MFN * 4096);
static_assert(SWA * WIMG + 3 * MFA * 4096 + 3072 <= LDS_CTL, "LDS");
constexpr int C_GO = LDS_CTL;
constexpr int C_CALL = LDS_CTL + 16;
constexpr int LDS_BYTES = LDS_CTL + 64;
static_assert(LDS_BYTES <= 163840, "LDS");
constexpr int NBLK = 256;
constexpr unsigned long long DEADLINE = 50000000ull;
constexpr int WS_TMO = 32, WS_READY = 64  , WS_CNT = 512  , WS_DONE = 1024  ;
constexpr int WS_XFLAG = 2048;
constexpr size_t WS_XBUF = 1u << 20;
constexpr size_t XT_BYTES = (size_t)XROWS * 128;
constexpr size_t XPAR_BYTES = XT_BYTES * NT;
constexpr size_t WS_NEED = WS_XBUF + 2 * XPAR_BYTES;
constexpr int NUNIT = BRP * NT / 128;
#define POLL_MARGIN 4
#define TILE_NS 640
#define PUSH_LEAD 1
#define ACK_NS 800
#define KSKEW 8
#define XPD 4
#define ACQ 3
#define KSTRIDE 11
#define NAUX 16
#define XWARM 1
#define AFINE 0
#define WWARM 0
#define WWARM0 0
#define ENDACQ 0
#define RTT_TICKS 0
#define RTT_SKIP 2
#define NPACE 92
#define NPACE_END 4650
#define NC_N0 10
#define NC_T0 1150
#define NC_SLOPE 125
#define NS_T0 950
#define NS_N0 3
#define SPLX 0
#define NCYC 0
#define AMAP 1
#define AWIN 0
#define WAUX 2

struct Params {
  const unsigned short* x;
  const unsigned short* w;
  unsigned short* out[NR];
  unsigned* ws[NR];
  int rank;
};

#define WAITL(n) asm volatile("s_waitcnt lgkmcnt(" #n ")" ::: "memory")
template <int N> __device__ __forceinline__ void waitv() { asm volatile("s_waitcnt vmcnt(%0)" :: "n"(N) : "memory"); }
#define BAR() __builtin_amdgcn_s_barrier()
#define SCHED() __builtin_amdgcn_sched_barrier(0)
#define CFENCE() asm volatile("" ::: "memory")
#define STAMP(i) do {} while (0)
#define TRACE_DECL do {} while (0)
#define TRACE(t) do {} while (0)
#define TRACE_OUT do {} while (0)

__device__ __forceinline__ bf16x8 ldx(LAS const char* p) { return *(LAS const bf16x8*)p; }
__device__ __forceinline__ s16x4 ldw(LAS const char* p) { return __builtin_amdgcn_ds_read_tr16_b64_v4i16((LAS s16x4*)p); }
__device__ __forceinline__ unsigned pk(float a, float b) { f32x2 v = {a, b}; return __builtin_bit_cast(unsigned, __builtin_convertvector(v, bf16x2)); }
__device__ __forceinline__ unsigned ld_sys(const unsigned* p) { return __hip_atomic_load(p, __ATOMIC_RELAXED, __HIP_MEMORY_SCOPE_SYSTEM); }
__device__ __forceinline__ unsigned ld_agent(const unsigned* p) { return __hip_atomic_load(p, __ATOMIC_RELAXED, __HIP_MEMORY_SCOPE_AGENT); }
__device__ __forceinline__ void st_sys(unsigned* p, unsigned v) { __hip_atomic_store(p, v, __ATOMIC_RELAXED, __HIP_MEMORY_SCOPE_SYSTEM); }
__device__ __forceinline__ void st_agent(unsigned* p, unsigned v) { __hip_atomic_store(p, v, __ATOMIC_RELAXED, __HIP_MEMORY_SCOPE_AGENT); }
__device__ __forceinline__ unsigned lds_rd(LAS char* lds, int off) { return *(LAS volatile unsigned*)(lds + off); }
__device__ __forceinline__ void lds_wr(LAS char* lds, int off, unsigned v) { *(LAS volatile unsigned*)(lds + off) = v; }
__device__ __forceinline__ bf16x8 wfrag(s16x4 lo, s16x4 hi) { return __builtin_bit_cast(bf16x8, __builtin_shufflevector(lo, hi, 0, 1, 2, 3, 4, 5, 6, 7)); }
__device__ __forceinline__ u32x4 pack2(f32x4 a, f32x4 c, bool bad) {
  const unsigned a0 = pk(a[0], a[1]), a1 = pk(a[2], a[3]), c0 = pk(c[0], c[1]), c1 = pk(c[2], c[3]);
  auto r0 = __builtin_amdgcn_permlane16_swap(a0, c0, false, false);
  auto r1 = __builtin_amdgcn_permlane16_swap(a1, c1, false, false);
  return bad ? (u32x4){0x7fc07fc0u, 0x7fc07fc0u, 0x7fc07fc0u, 0x7fc07fc0u} : (u32x4){r0[0], r1[0], r0[1], r1[1]};
}
__device__ __forceinline__ unsigned out_voff(int lane) { const int fr = lane & 15, fq = lane >> 4; return (unsigned)((fr * NCOL + (fq & 1) * 16 + (fq >> 1) * 8) * 2); }
__host__ __device__ constexpr int wpiece(int pc) { return ((pc >> 1) & 1) * WODD + (pc >> 2) * 2048 + (pc & 1) * 1024; }

struct Tile {
  int kind;
  int owner;
  int col0;
  int cgap;
  int slot;
  int nrow0;
  bool gathered;
  int koff;
  int ai;
};

template <int MF, int NIMG, int SW, bool PAIR, int H, int NWS, int AUXW>
__device__ __forceinline__ void loader(const Params& p, const Tile& tl, LAS char* lds, int lane) {
  constexpr int NWP = 16 * NIMG;
  constexpr int WSLOT = NIMG * WIMG, XSLOT = MF * 4096, LX = SW * WSLOT;
  constexpr int NW_ = (NWP - H + 2) / 3;
  constexpr int X0 = (H + 3 * NW_) - NWP;
  constexpr int NX_ = (4 * MF - X0 + 2) / 3;
  constexpr bool XWS = NWS > 0;
  static_assert(2 * NW_ + 2 * NX_ <= 63, "");
  const auto rw = __builtin_amdgcn_make_buffer_rsrc((void*)p.w, 0, KD * ND * 2, 0x00020000);
  const auto rx = __builtin_amdgcn_make_buffer_rsrc((void*)p.x, 0, ML * KD * 2, 0x00020000);
  const int wch = (lane & 15) ^ ((lane >> 4) << 1);
  const unsigned wv = (unsigned)((lane >> 4) * ND * 2 + (wch & 7) * 16 + (wch >> 3) * tl.cgap * 2);
  unsigned xv[2];
#pragma unroll
  for (int v = 0; v < 2; ++v) xv[v] = (unsigned)((lane >> 3) * KD * 2 + (((lane & 7) ^ ((v * 4 + (lane >> 4)) & 7)) * 16));
  const unsigned iv = (unsigned)(lane * 16);
  const unsigned wbase = (unsigned)((tl.owner * NCOL + tl.col0) * 2);
  const unsigned xbase = tl.kind == 0 ? 0u : (unsigned)((tl.nrow0 - XROWS) * KD * 2);
  int ws_ = 0, xs_ = 0;
  auto kmap = [&](int t) -> unsigned { return PAIR ? (unsigned)(((((t >> 1) * KSTRIDE) << 1) + (t & 1) + tl.koff) & (NT - 1)) : (unsigned)((t + tl.koff) & (NT - 1)); };
  auto issue_w = [&](int t) {
    const unsigned tb = wbase + kmap(t) * (BK * ND * 2);
#pragma unroll
    for (int i = 0; i < NW_; ++i) {
      const int pc = H + 3 * i, img = pc >> 4, q = pc & 15;
      __builtin_amdgcn_raw_ptr_buffer_load_lds(rw, (LAS void*)(lds + ws_ * WSLOT + img * WIMG + wpiece(q)), 16, wv, tb + q * (4 * ND * 2) + img * 256, 0, AUXW);
    }
    ws_ = ws_ == SW - 1 ? 0 : ws_ + 1;
  };
  auto issue_xl = [&](int t) {
    const unsigned tb = xbase + kmap(t) * 128u;
#pragma unroll
    for (int i = 0; i < NX_; ++i) {
      const int pc = X0 + 3 * i;
      if (pc >= NWS) __builtin_amdgcn_raw_ptr_buffer_load_lds(rx, (LAS void*)(lds + LX + xs_ * XSLOT + pc * 1024), 16, xv[pc & 1], tb + pc * (8 * KD * 2), 0, 0);
    }
  };
  issue_w(0);
  issue_xl(0);
  unsigned call = 0;
  if (XWS) {
    BAR();
    call = (unsigned)__builtin_amdgcn_readfirstlane((int)lds_rd(lds, C_CALL));
  }
  const auto ri = __builtin_amdgcn_make_buffer_rsrc((char*)p.ws[p.rank] + WS_XBUF + (call & 1) * XPAR_BYTES, 0, (int)XPAR_BYTES, 0x00020000);
  const unsigned ibase = (unsigned)(tl.nrow0 * 128);
  auto issue_xi = [&](int t) {
#pragma unroll
    for (int i = 0; i < NX_; ++i) {
      const int pc = X0 + 3 * i;
      if (pc < NWS) __builtin_amdgcn_raw_ptr_buffer_load_lds(ri, (LAS void*)(lds + LX + xs_ * XSLOT + pc * 1024), 16, iv, ibase + (unsigned)t * (unsigned)XT_BYTES + pc * 1024, 0, 0);
    }
    xs_ = xs_ == 2 ? 0 : xs_ + 1;
  };
  issue_xi(0);
  issue_w(1);
  issue_xl(1);
  issue_xi(1);
  waitv<NW_ + NX_>();
  BAR();
  if (PAIR) {
    constexpr int NPF = (XPD && H > 0) ? 2 : 0;
    auto prefetch = [&](int t) {
      const unsigned tb = kmap(t) * 128u + (unsigned)tl.ai * (16 * KD * 2);
#pragma unroll
      for (int i = 0; i < NPF; ++i)
        __builtin_amdgcn_raw_ptr_buffer_load_lds(rx, (LAS void*)(lds + LX + 3 * XSLOT + H * 1024), 16, xv[0], tb + i * (8 * KD * 2), 0, 0);
    };
    for (int t = 0; t < NT - 2; t += 2) {
      issue_w(t + 2);
      issue_w(t + 3);
      prefetch(t + XPD + H - 1);
      issue_xl(t + 2);
      issue_xi(t + 2);
      waitv<2 * NW_ + NPF + NX_>();
      BAR();
      issue_xl(t + 3);
      issue_xi(t + 3);
      waitv<NX_>();
      BAR();
    }
  } else {
    for (int t = 0; t < NT - 2; ++t) {
      issue_w(t + 2);
      issue_xl(t + 2);
      issue_xi(t + 2);
      waitv<NW_ + NX_>();
      BAR();
    }
  }
  waitv<0>();
  BAR();
  BAR();
}

#define NOP_ do {} while (0)
template <int MF>
__device__ __forceinline__ void mfma_a(const Params& p, const Tile& tl, LAS char* lds, int wid, int lane) {
  constexpr int XSLOT = MF * 4096, LX = SWA * WIMG;
  const int wr = wid >> 1, wc = wid & 1;
  const int fr = lane & 15, fq = lane >> 4;
  const int q = (lane & 15) >> 2, pp = lane & 3;
  const unsigned xa0 = (unsigned)(LX + (wr * MF * 16 + fr) * 128 + ((fq ^ ((fr >> 1) & 7)) * 16));
  const unsigned xa1 = xa0 ^ 64u;
  unsigned wa[4];
#pragma unroll
  for (int nf = 0; nf < 4; ++nf)
    wa[nf] = (unsigned)((fq & 1) * WODD + (fq >> 1) * 2048 + q * 256 + wc * 128 + ((nf ^ q) * 32) + 8 * pp);
  f32x4 acc[MF][4];
#pragma unroll
  for (int i = 0; i < MF; ++i)
#pragma unroll
    for (int j = 0; j < 4; ++j) acc[i][j] = (f32x4){0.f, 0.f, 0.f, 0.f};
  bf16x8 Xf[MF];
  s16x4 Wl[2][4], Wh[2][4];
#define RDW(bu, sb, ks, i) do { if ((i) & 1) Wh[bu][(i) >> 1] = ldw((sb) + wa[(i) >> 1] + (ks) * 4096 + 1024); \
    else Wl[bu][(i) >> 1] = ldw((sb) + wa[(i) >> 1] + (ks) * 4096); } while (0)
#define MMA1(mf, bu, nf) do { acc[mf][nf] = __builtin_amdgcn_mfma_f32_16x16x32_bf16(wfrag(Wl[bu][nf], Wh[bu][nf]), Xf[mf], acc[mf][nf], 0, 0, 0); SCHED(); } while (0)
#define GRP(mf, bu, F0, F1, F2) do { MMA1(mf, bu, 0); F0; SCHED(); MMA1(mf, bu, 1); F1; SCHED(); MMA1(mf, bu, 2); F2; SCHED(); MMA1(mf, bu, 3); } while (0)
#define RDX(i, sb, xa) Xf[i] = ldx((sb) + (xa) + (i) * 2048)
  BAR();
  STAMP(1);
#pragma unroll
  for (int i = 0; i < 8; ++i) RDW(0, lds, 0, i);
#pragma unroll
  for (int mf = 0; mf < MF - 1; ++mf) RDX(mf, lds, xa0);
  int sw = 0, sx = 0;
  TRACE_DECL;
  for (int t = 0; t < NT; ++t) {
    TRACE(t);
    LAS const char* wb = lds + sw * WIMG;
    LAS const char* xb = lds + sx * XSLOT;
    sw = (sw == SWA - 1) ? 0 : sw + 1;
    sx = (sx == 2) ? 0 : sx + 1;
    LAS const char* wn = lds + sw * WIMG;
    LAS const char* xn = lds + sx * XSLOT;
    GRP(0, 0, RDX(MF - 1, xb, xa0), RDW(1, wb, 1, 0), RDW(1, wb, 1, 1));
    GRP(1, 0, RDX(0, xb, xa1), RDW(1, wb, 1, 2), RDW(1, wb, 1, 3));
    GRP(2, 0, RDX(1, xb, xa1), RDW(1, wb, 1, 4), NOP_);
    GRP(3, 0, RDX(2, xb, xa1), RDW(1, wb, 1, 5), NOP_);
    GRP(4, 0, RDX(3, xb, xa1), RDW(1, wb, 1, 6), NOP_);
    GRP(5, 0, RDX(4, xb, xa1), RDW(1, wb, 1, 7), NOP_);
#pragma unroll
    for (int g = 6; g < MF; ++g) GRP(g, 0, RDX(g - 1, xb, xa1), NOP_, NOP_);
    GRP(0, 1, RDX(MF - 1, xb, xa1), NOP_, NOP_);
    GRP(1, 1, NOP_, NOP_, NOP_);
    WAITL(0);
    BAR();
    CFENCE();
    SCHED();
    GRP(2, 1, RDW(0, wn, 0, 0), RDW(0, wn, 0, 1), RDX(0, xn, xa0));
    GRP(3, 1, RDW(0, wn, 0, 2), RDW(0, wn, 0, 3), RDX(1, xn, xa0));
    GRP(4, 1, RDW(0, wn, 0, 4), RDW(0, wn, 0, 5), RDX(2, xn, xa0));
    GRP(5, 1, RDW(0, wn, 0, 6), RDW(0, wn, 0, 7), RDX(3, xn, xa0));
    GRP(6, 1, RDX(4, xn, xa0), RDX(5, xn, xa0), NOP_);
#pragma unroll
    for (int g = 7; g < MF; ++g) GRP(g, 1, RDX(g - 1, xn, xa0), NOP_, NOP_);
  }
  STAMP(2);
  TRACE_OUT;
  const unsigned go = (unsigned)__builtin_amdgcn_readfirstlane((int)lds_rd(lds, C_GO));
  const bool bad = (go & 1u) != 0u;
  const auto ro = __builtin_amdgcn_make_buffer_rsrc((void*)p.out[tl.owner], 0, ML * NR * NCOL * 2, 0x00020000);
  const unsigned vo = out_voff(lane);
  const unsigned so = (unsigned)(((p.rank * ML + wr * MF * 16) * NCOL + tl.col0 + wc * tl.cgap) * 2);
  int skip = 0;
#pragma unroll
  for (int mf = 0; mf < MF; ++mf) {
    const unsigned long long t0 = __builtin_amdgcn_s_memrealtime();
#pragma unroll
    for (int np = 0; np < 2; ++np)
      __builtin_amdgcn_raw_buffer_store_b128(pack2(acc[mf][2 * np], acc[mf][2 * np + 1], bad), ro, vo + np * 64, so + mf * (16 * NCOL * 2), 17);
    if (skip > 0) --skip;
    else {
      waitv<AWIN>();
      if (RTT_TICKS && __builtin_amdgcn_s_memrealtime() - t0 < RTT_TICKS) skip = RTT_SKIP;
    }
  }
  waitv<0>();
  if (lane == 0) st_sys(p.ws[tl.owner] + WS_DONE + p.rank * 64 + tl.slot * 4 + wid, go >> 1);
  STAMP(3);
#undef RDW
#undef MMA1
#undef GRP
#undef RDX
}

__device__ __forceinline__ void mfma_n(const Params& p, const Tile& tl, LAS char* lds, int wid, int lane) {
  constexpr int MF = MFN;
  constexpr int WSLOT = 2 * WIMG, XSLOT = MF * 4096, LX = 3 * WSLOT;
  const int wr = wid >> 1, wc = wid & 1;
  const int fr = lane & 15, fq = lane >> 4;
  const int q = (lane & 15) >> 2, pp = lane & 3;
  const unsigned xa0 = (unsigned)(LX + (wr * MF * 16 + fr) * 128 + ((fq ^ ((fr >> 1) & 7)) * 16));
  const unsigned xa1 = xa0 ^ 64u;
  unsigned wa[4];
#pragma unroll
  for (int nf = 0; nf < 4; ++nf)
    wa[nf] = (unsigned)(wc * WIMG + (fq & 1) * WODD + (fq >> 1) * 2048 + q * 256 + ((nf ^ q) * 32) + 8 * pp);
  f32x4 acc[MF][8];
#pragma unroll
  for (int i = 0; i < MF; ++i)
#pragma unroll
    for (int j = 0; j < 8; ++j) acc[i][j] = (f32x4){0.f, 0.f, 0.f, 0.f};
  bf16x8 Xf[2][MF];
  s16x4 Wl[8], Wh[8];
#define RWL(nf, sb, ks) Wl[nf] = ldw((sb) + wa[(nf) & 3] + ((nf) >> 2) * 128 + (ks) * 4096)
#define RWH(nf, sb, ks) Wh[nf] = ldw((sb) + wa[(nf) & 3] + ((nf) >> 2) * 128 + (ks) * 4096 + 1024)
#define RDX(bu, i, sb, xa) Xf[bu][i] = ldx((sb) + (xa) + (i) * 2048)
#define MMA1(nf, bu, mf) do { acc[mf][nf] = __builtin_amdgcn_mfma_f32_16x16x32_bf16(wfrag(Wl[nf], Wh[nf]), Xf[bu][mf], acc[mf][nf], 0, 0, 0); SCHED(); } while (0)
#define GRP(nf, bu, F0, F1, F2, F3, F4) do { MMA1(nf, bu, 0); F0; SCHED(); MMA1(nf, bu, 1); F1; SCHED(); MMA1(nf, bu, 2); F2; SCHED(); \
    MMA1(nf, bu, 3); F3; SCHED(); MMA1(nf, bu, 4); F4; SCHED(); } while (0)
  if (tl.gathered) {
    const unsigned gwv = (unsigned)(((tl.nrow0 / NM) * 8 + (tl.col0 >> 8)) * 4 + wid);
    LAS void* dummy = (LAS void*)(lds + 2 * WSLOT + wid * 1024);
    {
      const auto rx = __builtin_amdgcn_make_buffer_rsrc((void*)p.x, 0, ML * KD * 2, 0x00020000);
#pragma unroll
      for (int i = 0; i < 10; ++i) __builtin_amdgcn_raw_ptr_buffer_load_lds(rx, dummy, 16, (unsigned)(lane * 16), gwv * 10240u + i * 1024, 0, 0);
    }
    waitv<0>();
    BAR();
  }
  BAR();
  STAMP(1);
#pragma unroll
  for (int mf = 0; mf < MF; ++mf) RDX(0, mf, lds, xa0);
#pragma unroll
  for (int nf = 0; nf < 7; ++nf) { RWL(nf, lds, 0); RWH(nf, lds, 0); }
  int st = 0;
  TRACE_DECL;
  for (int t = 0; t < NT; ++t) {
    TRACE(t);
    LAS const char* wb = lds + st * WSLOT;
    LAS const char* xb = lds + st * XSLOT;
    st = (st == 2) ? 0 : st + 1;
    LAS const char* wn = lds + st * WSLOT;
    LAS const char* xn = lds + st * XSLOT;
    GRP(0, 0, RWL(7, wb, 0), RWH(7, wb, 0), RDX(1, 0, xb, xa1), NOP_, NOP_);
    GRP(1, 0, RWL(0, wb, 1), RWH(0, wb, 1), RDX(1, 1, xb, xa1), NOP_, NOP_);
    GRP(2, 0, RWL(1, wb, 1), RWH(1, wb, 1), RDX(1, 2, xb, xa1), NOP_, NOP_);
    GRP(3, 0, RWL(2, wb, 1), RWH(2, wb, 1), RDX(1, 3, xb, xa1), NOP_, NOP_);
    GRP(4, 0, RWL(3, wb, 1), RWH(3, wb, 1), RDX(1, 4, xb, xa1), NOP_, NOP_);
    GRP(5, 0, RWL(4, wb, 1), RWH(4, wb, 1), NOP_, NOP_, NOP_);
    GRP(6, 0, RWL(5, wb, 1), RWH(5, wb, 1), NOP_, NOP_, NOP_);
    GRP(7, 0, RWL(6, wb, 1), RWH(6, wb, 1), NOP_, NOP_, NOP_);
    GRP(0, 1, RWL(7, wb, 1), RWH(7, wb, 1), NOP_, NOP_, NOP_);
    GRP(1, 1, NOP_, NOP_, NOP_, NOP_, NOP_);
    WAITL(0);
    BAR();
    CFENCE();
    SCHED();
    GRP(2, 1, RWL(0, wn, 0), RWH(0, wn, 0), RDX(0, 0, xn, xa0), RWL(1, wn, 0), RWH(1, wn, 0));
    GRP(3, 1, RWL(2, wn, 0), RWH(2, wn, 0), RDX(0, 1, xn, xa0), RDX(0, 2, xn, xa0), NOP_);
    GRP(4, 1, RWL(3, wn, 0), RWH(3, wn, 0), RDX(0, 3, xn, xa0), RDX(0, 4, xn, xa0), NOP_);
    GRP(5, 1, RWL(4, wn, 0), RWH(4, wn, 0), NOP_, NOP_, NOP_);
    GRP(6, 1, RWL(5, wn, 0), RWH(5, wn, 0), NOP_, NOP_, NOP_);
    GRP(7, 1, RWL(6, wn, 0), RWH(6, wn, 0), NOP_, NOP_, NOP_);
  }
  STAMP(2);
  TRACE_OUT;
  const unsigned go = (unsigned)__builtin_amdgcn_readfirstlane((int)lds_rd(lds, C_GO));
  const bool bad = (go & 1u) != 0u;
  const auto ro = __builtin_amdgcn_make_buffer_rsrc((void*)p.out[p.rank], 0, ML * NR * NCOL * 2, 0x00020000);
  const unsigned vo = out_voff(lane);
  const unsigned so = (unsigned)((tl.col0 + wc * 128) * 2);
#pragma unroll
  for (int mf = 0; mf < MF; ++mf) {
    const int nr = tl.nrow0 + wr * MF * 16 + mf * 16;
    if (nr < NROWS) {
      const int orow = nr < XROWS ? ((p.rank + 1 + nr / BROWS) & 7) * ML + AROWS + nr % BROWS : p.rank * ML + nr - XROWS;
#pragma unroll
      for (int np = 0; np < 4; ++np)
        __builtin_amdgcn_raw_buffer_store_b128(pack2(acc[mf][2 * np], acc[mf][2 * np + 1], bad), ro, vo + np * 64, so + (unsigned)orow * (NCOL * 2), NAUX);
    }
  }
  STAMP(3);
#undef RWL
#undef RWH
#undef RDX
#undef MMA1
#undef GRP
}

__device__ __forceinline__ bool wait_words(const unsigned* word, bool use, unsigned call, unsigned* tmo) {
  const unsigned long long t0 = __builtin_amdgcn_s_memrealtime();
  unsigned spins = 0;
  for (;;) {
    const unsigned v = ld_sys(word);
    if (__all(!use || v >= call)) return true;
    if ((++spins & 63u) == 0u && (ld_agent(tmo) == call || __builtin_amdgcn_s_memrealtime() - t0 > DEADLINE)) { st_agent(tmo, call); return false; }
    __builtin_amdgcn_s_sleep(1);
  }
}

__device__ __forceinline__ void ctrl_wave(const Params& p, const Tile& tl, LAS char* lds, int lane) {
  unsigned* myws = p.ws[p.rank];
  unsigned* tmo = myws + WS_TMO;
  const int b = blockIdx.x;
  const unsigned long long tk0 = NCYC ? __builtin_amdgcn_s_memtime() : __builtin_amdgcn_s_memrealtime();
  const unsigned cnt = ld_agent(myws + WS_CNT + b);
  if (!tl.gathered) BAR();
  const unsigned call = (unsigned)__builtin_amdgcn_readfirstlane((int)cnt) + 1u;
  if (lane == 0) st_agent(myws + WS_CNT + b, call);
  bool ok = true;
  int nb = 1;
#define PCLK() __builtin_amdgcn_s_memrealtime()
#define PSC 1
  auto join = [&]() {
    if (NPACE && tl.kind == 1) {
      long long due = (long long)NS_T0 + (long long)nb * NPACE;
      if (!tl.gathered) { const long long d2 = nb > NC_N0 ? (long long)NC_T0 + (long long)(nb - NC_N0) * NC_SLOPE : 0ll; due = d2 < due ? d2 : due; }
      if (nb < NS_N0) due = 0;
      due = (due < (long long)NPACE_END ? due : (long long)NPACE_END) * PSC;
      while ((long long)(PCLK() - tk0) < due) __builtin_amdgcn_s_sleep(1);
      BAR();
    } else BAR();
    ++nb;
  };
  if (tl.gathered) {
    if (lane == 0) lds_wr(lds, C_CALL, call);
    const int s0 = tl.nrow0 / BROWS;
    const int e1 = (tl.nrow0 + NM - 1) / BROWS;
    const int s1 = e1 < 7 ? e1 : 6;
    const unsigned* sflag = myws + WS_XFLAG + ((lane & 16) ? s1 : s0) * 32 + (lane & 15);
    auto decode = [&](unsigned v) -> int {
#pragma unroll
      for (int o = 1; o < 32; o <<= 1) { const unsigned u = (unsigned)__shfl_xor((int)v, o); v = u < v ? u : v; }
      const unsigned vm = (unsigned)__builtin_amdgcn_readfirstlane((int)v);
      const int n = vm > call * 128u ? (int)(vm - call * 128u) : 0;
      return n > NT ? NT : n;
    };
    int seen = 0;
    nb = -1;
    if (ACQ == 3) __builtin_amdgcn_fence(__ATOMIC_ACQUIRE, "");
    while (seen < NT) {
      const int need = nb + 3 < NT ? nb + 3 : NT;
      if (seen < need) {
        const unsigned long long t0 = __builtin_amdgcn_s_memrealtime();
        unsigned spins = 0;
        while ((seen = decode(ld_sys(sflag))) < need) {
          if ((++spins & 63u) == 0u && (ld_agent(tmo) == call || __builtin_amdgcn_s_memrealtime() - t0 > DEADLINE)) { st_agent(tmo, call); ok = false; seen = NT; break; }
          __builtin_amdgcn_s_sleep(1);
        }
        if (ACQ == 2 || (ACQ == 1 && nb == -1)) {
          __builtin_amdgcn_fence(__ATOMIC_ACQUIRE, "");
          waitv<0>();
        }
        if (nb == -1) STAMP(5);
        WAITL(0);
        join();
      } else if (seen < need + POLL_MARGIN) {
        unsigned v;
        asm volatile("global_load_dword %0, %1, off sc0 sc1" : "=v"(v) : "v"(sflag) : "memory");
        join();
        asm volatile("s_waitcnt vmcnt(0)" : "+v"(v) :: "memory");
        const int n = decode(v);
        if (n > seen) {
          if (ACQ == 2) {
            __builtin_amdgcn_fence(__ATOMIC_ACQUIRE, "");
            if (seen >= (nb + 3 < NT ? nb + 3 : NT)) { join(); }
            waitv<0>();
          }
          seen = n;
        }
      } else {
        join();
      }
    }
    STAMP(7);
  } else if (tl.kind == 0) {
    while (nb < NT / 2) { join(); }
    ok = wait_words(myws + WS_READY + tl.owner * 32, true, call, tmo);
  }
  while (nb < NT - 1) { join(); }
  if (ld_agent(tmo) == call) ok = false;
  if (lane == 0) lds_wr(lds, C_GO, call * 2u + (ok ? 0u : 1u));
  WAITL(0);
  while (nb <= NT) { join(); }
}

__device__ __forceinline__ void copy_group(const Params& p, LAS char* lds, int c, int wid, int lane) {
  unsigned* myws = p.ws[p.rank];
  unsigned* tmo = myws + WS_TMO;
  const int b = blockIdx.x;
  const unsigned cnt = ld_agent(myws + WS_CNT + b);
  const int gw = wid * 16 + c;
  const auto rx = __builtin_amdgcn_make_buffer_rsrc((void*)p.x, 0, ML * KD * 2, 0x00020000);
  const unsigned xv = (unsigned)((AROWS + (lane >> 3)) * KD * 2 + (((lane & 7) ^ (((gw & 1) * 4 + (lane >> 4)) & 7)) * 16));
  auto load = [&](int k) { const int u = k * 128 + gw; return __builtin_amdgcn_raw_buffer_load_b128(rx, xv, (unsigned)((u % BRP) * (8 * KD * 2) + (u / BRP) * 128), 0); };
  u32x4 d[NUNIT];
  d[0] = load(0);
  d[1] = load(1);
  char* dbase[NR - 1];
  unsigned dslot[NR - 1];
#pragma unroll
  for (int i = 0; i < NR - 1; ++i) {
    const int j = __builtin_amdgcn_readfirstlane((i + gw) % (NR - 1) + 1);
    dbase[i] = (char*)p.ws[__builtin_amdgcn_readfirstlane((p.rank + NR - j) & 7)];
    dslot[i] = (unsigned)((j - 1) * BROWS * 128);
  }
  if (lane == 0) { lds_wr(lds, wid * 4, (unsigned)(gw / BRP)); if (wid == 0) lds_wr(lds, 64, 0u); }
  WAITL(0);
  const unsigned call = (unsigned)__builtin_amdgcn_readfirstlane((int)cnt) + 1u;
  BAR();
  const unsigned long long tstart = __builtin_amdgcn_s_memrealtime();
  STAMP(1);
  if (wid == 0 && lane == 0) st_agent(myws + WS_CNT + b, call);
  if (gw == 127 && lane < NR && lane != p.rank) st_sys(p.ws[lane] + WS_READY + p.rank * 32, call);
  const size_t xo = WS_XBUF + (call & 1) * XPAR_BYTES;
  unsigned* pflag = p.ws[(p.rank + NR - 1 - (lane < 7 ? lane : 0)) & 7] + WS_XFLAG + (lane < 7 ? lane : 0) * 32 + c;
  auto finish = [&](int k) {
    if (k == 0) STAMP(3);
    const int nxt = k + 1 < NUNIT ? ((k + 1) * 128 + gw) / BRP : NT;
    if (lane == 0) lds_wr(lds, wid * 4, (unsigned)nxt);
    unsigned v = lds_rd(lds, (lane & 7) * 4);
#pragma unroll
    for (int o = 1; o < 8; o <<= 1) { const unsigned u = (unsigned)__shfl_xor((int)v, o); v = u < v ? u : v; }
    const int pre = __builtin_amdgcn_readfirstlane((int)v);
    const int lvl = pre < 4 ? pre : (pre < 12 ? (pre & ~1) : (pre & ~3));
    unsigned old = 0;
    if (lane == 0) old = __hip_atomic_fetch_max((LAS unsigned*)(lds + 64), (unsigned)lvl, __ATOMIC_RELAXED, __HIP_MEMORY_SCOPE_WORKGROUP);
    old = (unsigned)__builtin_amdgcn_readfirstlane((int)old);
    if ((unsigned)lvl > old && lane < 7)
      __hip_atomic_fetch_max(pflag, call * 128u + (unsigned)lvl, __ATOMIC_RELAXED, __HIP_MEMORY_SCOPE_SYSTEM);
  };
  bool pending = false;
  long long tissue = 0;
#pragma unroll
  for (int k = 0; k < NUNIT; ++k) {
    const int u = k * 128 + gw;
    for (;;) {
      const long long now = (long long)(__builtin_amdgcn_s_memrealtime() - tstart) * 10;
      if (now * BRP >= (long long)(u - PUSH_LEAD * BRP) * TILE_NS) break;
      if (pending && now - tissue > ACK_NS) { waitv<0>(); finish(k - 1); pending = false; }
      else __builtin_amdgcn_s_sleep(2);
    }
    const unsigned so = (unsigned)(u / BRP) * (unsigned)XT_BYTES + (unsigned)(u % BRP) * 1024u;
#pragma unroll
    for (int i = 0; i < NR - 1; ++i) {
      const auto rd = __builtin_amdgcn_make_buffer_rsrc(dbase[i] + xo, 0, (int)XPAR_BYTES, 0x00020000);
      __builtin_amdgcn_raw_buffer_store_b128(d[k], rd, (unsigned)(lane * 16), so + dslot[i], 17);
    }
    tissue = (long long)(__builtin_amdgcn_s_memrealtime() - tstart) * 10;
    if (k == 0) STAMP(2);
    if (pending) { waitv<7>(); finish(k - 1); }
    pending = true;
    if (k + 2 < NUNIT) d[k + 2] = load(k + 2);
  }
  waitv<0>();
  finish(NUNIT - 1);
  STAMP(4);
  if (c == 15) {
    if (wid != p.rank) {
      const bool fine = wait_words(myws + WS_DONE + wid * 64 + lane, true, call, tmo);
      if (!fine && lane == 0) p.out[p.rank][(size_t)(wid * ML) * NCOL] = 0x7fc0;
    }
    STAMP(6);
  }
}

__global__ __launch_bounds__(512, 2) void gemm_a2a(Params p) {
  extern __shared__ __attribute__((aligned(16))) char smem[];
  LAS char* lds = (LAS char*)smem;
  const int wid = __builtin_amdgcn_readfirstlane(threadIdx.x >> 6);
  const int lane = threadIdx.x & 63;
  const int b = blockIdx.x;
  const int xcd = b & 7, idx = b >> 3;
  STAMP(0);
  if (idx < 2) { copy_group(p, lds, xcd * 2 + idx, wid, lane); return; }
  Tile tl;
  constexpr int RMIX = XROWS / NM;
  int rt = 0;
  if (idx < 16) {
    const int a = xcd * 14 + idx - 2;
    tl.kind = 0; tl.owner = (p.rank + (AMAP ? a % 7 : a >> 4) + 1) & 7;
    { const int pn = AMAP ? a / 7 : a & 15; tl.slot = pn;
      if (SPLX) { tl.col0 = (pn & ~SPLX) * 128 + ((pn & SPLX) ? 64 : 0); tl.cgap = SPLX * 128; } else { tl.col0 = pn * 128; tl.cgap = 64; } }
    tl.nrow0 = 0; tl.gathered = false; tl.koff = KSKEW * xcd; tl.ai = idx - 2;
  } else {
    const int e = idx - 16;
    rt = (xcd >> 1) * 4 + (e >> 2);
    tl.kind = 1; tl.owner = p.rank; tl.col0 = ((xcd & 1) * 4 + (e & 3)) * 256; tl.cgap = 64; tl.slot = 0;
    tl.nrow0 = rt * NM; tl.gathered = rt <= RMIX; tl.koff = 0; tl.ai = 0;
  }
  if (wid < 4) {
    if (tl.kind == 0) mfma_a<MFA>(p, tl, lds, wid, lane);
    else mfma_n(p, tl, lds, wid, lane);
  } else if (wid == 7) {
    ctrl_wave(p, tl, lds, lane);
  } else {
#define LOADERS(MF, NIMG, SW, PAIR, NWS, AUX) do { if (wid == 4) loader<MF, NIMG, SW, PAIR, 0, NWS, AUX>(p, tl, lds, lane); \
    else if (wid == 5) loader<MF, NIMG, SW, PAIR, 1, NWS, AUX>(p, tl, lds, lane); else loader<MF, NIMG, SW, PAIR, 2, NWS, AUX>(p, tl, lds, lane); } while (0)
    if (tl.kind == 0) LOADERS(MFA, 1, SWA, WPAIR != 0, 0, WAUX);
    else if (rt < RMIX) LOADERS(MFN, 2, 3, false, 4 * MFN, 0);
    else if (rt == RMIX) LOADERS(MFN, 2, 3, false, (XROWS - RMIX * NM) / 8, 0);
    else LOADERS(MFN, 2, 3, false, 0, 0);
  }
}

extern "C" void kernel_launch(int rank, int nranks, ncclComm_t comm,
                              hipStream_t stream,
                              void* const* d_in, const int* in_sizes, int n_in,
                              void* d_out, int out_size,
                              const int* in_dtypes, int out_dtype,
                              void* const* d_out_all,
                              void* const* d_ws_all, size_t ws_bytes) {
  if (nranks != NR || (unsigned)rank >= (unsigned)NR || ws_bytes < WS_NEED) { fprintf(stderr, "unexpected configuration\n"); return; }
  Params p;
  p.x = (const unsigned short*)d_in[0];
  p.w = (const unsigned short*)d_in[1];
  for (int i = 0; i < NR; ++i) { p.out[i] = (unsigned short*)d_out_all[i]; p.ws[i] = (unsigned*)d_ws_all[i]; }
  p.rank = rank;
  static bool init[NR] = {};
  if (!init[rank]) {
    HIP_CHECK(hipFuncSetAttribute((const void*)gemm_a2a, hipFuncAttributeMaxDynamicSharedMemorySize, LDS_BYTES));
    init[rank] = true;
  }
  gemm_a2a<<<NBLK, 512, LDS_BYTES, stream>>>(p);
  HIP_CHECK(hipGetLastError());
}
```

```cpp
#include <hip/hip_runtime.h>
#include <rccl/rccl.h>
#include <cstdio>
#include <stdint.h>

#define HIP_CHECK(expr) do { hipError_t _e = (expr); if (_e != hipSuccess) { \
    fprintf(stderr, "HIP error %s at %s:%d\n", hipGetErrorString(_e), __FILE__, __LINE__); \
    return; } } while (0)

#define LAS __attribute__((address_space(3)))
typedef short s16x4 __attribute__((ext_vector_type(4)));
typedef __bf16 bf16x2 __attribute__((ext_vector_type(2)));
typedef __bf16 bf16x8 __attribute__((ext_vector_type(8)));
typedef float f32x4 __attribute__((ext_vector_type(4)));
typedef float f32x2 __attribute__((ext_vector_type(2)));
typedef unsigned u32x4 __attribute__((ext_vector_type(4)));

constexpr int NR = 8, ML = 512, KD = 4096, ND = 16384, NCOL = ND / NR;
constexpr int BK = 64, NT = KD / BK;
constexpr int MFA = 7;
constexpr int MFN = 5;
constexpr int AROWS = MFA * 32, BROWS = ML - AROWS;
constexpr int BRP = BROWS / 8;
constexpr int XROWS = 7 * BROWS;
constexpr int NROWS = XROWS + ML;
constexpr int NM = MFN * 32;
constexpr int WODD = 8192 + 128;
constexpr int WIMG = WODD + 8192;
#define WPAIR 1
constexpr int SWA = WPAIR ? 4 : 3;
constexpr int LDS_CTL = 3 * (2 * WIMG + MFN * 4096);
static_assert(SWA * WIMG + 3 * MFA * 4096 + 3072 <= LDS_CTL, "LDS");
constexpr int C_GO = LDS_CTL;
constexpr int C_CALL = LDS_CTL + 16;
constexpr int LDS_BYTES = LDS_CTL + 64;
static_assert(LDS_BYTES <= 163840, "LDS");
constexpr int NBLK = 256;
constexpr unsigned long long DEADLINE = 50000000ull;
constexpr int WS_TMO = 32, WS_READY = 64  , WS_CNT = 512  , WS_DONE = 1024  ;
constexpr int WS_XFLAG = 2048;
constexpr size_t WS_XBUF = 1u << 20;
constexpr size_t XT_BYTES = (size_t)XROWS * 128;
constexpr size_t XPAR_BYTES = XT_BYTES * NT;
constexpr size_t WS_NEED = WS_XBUF + 2 * XPAR_BYTES;
constexpr int NUNIT = BRP * NT / 128;
#define POLL_MARGIN 4
#define TILE_NS 640
#define PUSH_LEAD 1
#define ACK_NS 800
#define KSKEW 8
#define XPD 0
#define ACQ 3
#define KSTRIDE 11
#define NAUX 16
#define XWARM 1
#define AFINE 0
#define WWARM 0
#define WWARM0 0
#define ENDACQ 0
#define RTT_TICKS 0
#define RTT_SKIP 2
#define NPACE 106
#define NPACE_END 4400
#define NC_N0 10
#define NC_T0 1150
#define NC_SLOPE 125
#define NS_T0 800
#define NS_N0 3
#define SPLX 0
#define NCYC 0
#define SNAKE 1
#define ROT 1
#define ASMMFMA 1
#define LATE_TICKS 0
#define LATE_SKIP 2
#define AMAP 1
#define AWIN 0
#define WAUX 2

struct Params {
  const unsigned short* x;
  const unsigned short* w;
  unsigned short* out[NR];
  unsigned* ws[NR];
  int rank;
};

#define WAITL(n) asm volatile("s_waitcnt lgkmcnt(" #n ")" ::: "memory")
template <int N> __device__ __forceinline__ void waitv() { asm volatile("s_waitcnt vmcnt(%0)" :: "n"(N) : "memory"); }
#define BAR() __builtin_amdgcn_s_barrier()
#define SCHED() __builtin_amdgcn_sched_barrier(0)
#define CFENCE() asm volatile("" ::: "memory")
#define STAMP(i) do {} while (0)
#define TRACE_DECL do {} while (0)
#define TRACE(t) do {} while (0)
#define TRACE_OUT do {} while (0)

__device__ __forceinline__ bf16x8 ldx(unsigned a) { return *(LAS const bf16x8*)(size_t)a; }
__device__ __forceinline__ s16x4 ldw(unsigned a) { return __builtin_amdgcn_ds_read_tr16_b64_v4i16((LAS s16x4*)(size_t)a); }
__device__ __forceinline__ void mma(f32x4& c, bf16x8 a, bf16x8 b) { asm volatile("v_mfma_f32_16x16x32_bf16 %0, %1, %2, %0" : "+v"(c) : "v"(a), "v"(b)); }
__device__ __forceinline__ unsigned pk(float a, float b) { f32x2 v = {a, b}; return __builtin_bit_cast(unsigned, __builtin_convertvector(v, bf16x2)); }
__device__ __forceinline__ unsigned ld_sys(const unsigned* p) { return __hip_atomic_load(p, __ATOMIC_RELAXED, __HIP_MEMORY_SCOPE_SYSTEM); }
__device__ __forceinline__ unsigned ld_agent(const unsigned* p) { return __hip_atomic_load(p, __ATOMIC_RELAXED, __HIP_MEMORY_SCOPE_AGENT); }
__device__ __forceinline__ void st_sys(unsigned* p, unsigned v) { __hip_atomic_store(p, v, __ATOMIC_RELAXED, __HIP_MEMORY_SCOPE_SYSTEM); }
__device__ __forceinline__ void st_agent(unsigned* p, unsigned v) { __hip_atomic_store(p, v, __ATOMIC_RELAXED, __HIP_MEMORY_SCOPE_AGENT); }
__device__ __forceinline__ unsigned lds_rd(LAS char* lds, int off) { return *(LAS volatile unsigned*)(lds + off); }
__device__ __forceinline__ void lds_wr(LAS char* lds, int off, unsigned v) { *(LAS volatile unsigned*)(lds + off) = v; }
__device__ __forceinline__ bf16x8 wfrag(s16x4 lo, s16x4 hi) { return __builtin_bit_cast(bf16x8, __builtin_shufflevector(lo, hi, 0, 1, 2, 3, 4, 5, 6, 7)); }
__device__ __forceinline__ u32x4 pack2(f32x4 a, f32x4 c, bool bad) {
  const unsigned a0 = pk(a[0], a[1]), a1 = pk(a[2], a[3]), c0 = pk(c[0], c[1]), c1 = pk(c[2], c[3]);
  auto r0 = __builtin_amdgcn_permlane16_swap(a0, c0, false, false);
  auto r1 = __builtin_amdgcn_permlane16_swap(a1, c1, false, false);
  return bad ? (u32x4){0x7fc07fc0u, 0x7fc07fc0u, 0x7fc07fc0u, 0x7fc07fc0u} : (u32x4){r0[0], r1[0], r0[1], r1[1]};
}
__device__ __forceinline__ unsigned out_voff(int lane) { const int fr = lane & 15, fq = lane >> 4; return (unsigned)((fr * NCOL + (fq & 1) * 16 + (fq >> 1) * 8) * 2); }
__host__ __device__ constexpr int wpiece(int pc) { return ((pc >> 1) & 1) * WODD + (pc >> 2) * 2048 + (pc & 1) * 1024; }

struct Tile {
  int kind;
  int owner;
  int col0;
  int cgap;
  int slot;
  int nrow0;
  bool gathered;
  int koff;
  int ai;
  int qw, qx;
};

template <int MF, int NIMG, int SW, bool PAIR, int H, int NWS, int AUXW>
__device__ __forceinline__ void loader(const Params& p, const Tile& tl, LAS char* lds, int lane) {
  constexpr int NWP = 16 * NIMG;
  constexpr int WSLOT = NIMG * WIMG, XSLOT = MF * 4096, LX = SW * WSLOT;
  constexpr int NW_ = (NWP - H + 2) / 3;
  constexpr int X0 = (H + 3 * NW_) - NWP;
  constexpr int NX_ = (4 * MF - X0 + 2) / 3;
  constexpr bool XWS = NWS > 0;
  static_assert(2 * NW_ + 2 * NX_ <= 63, "");
  const auto rw = __builtin_amdgcn_make_buffer_rsrc((void*)p.w, 0, KD * ND * 2, 0x00020000);
  const auto rx = __builtin_amdgcn_make_buffer_rsrc((void*)p.x, 0, ML * KD * 2, 0x00020000);
  const int wch = (lane & 15) ^ ((lane >> 4) << 1);
  const unsigned wv = (unsigned)((lane >> 4) * ND * 2 + (wch & 7) * 16 + (wch >> 3) * tl.cgap * 2);
  unsigned xv[2];
#pragma unroll
  for (int v = 0; v < 2; ++v) xv[v] = (unsigned)((lane >> 3) * KD * 2 + (((lane & 7) ^ ((v * 4 + (lane >> 4)) & 7)) * 16));
  const unsigned iv = (unsigned)(lane * 16);
  const unsigned wbase = (unsigned)((tl.owner * NCOL + tl.col0) * 2);
  const unsigned xbase = tl.kind == 0 ? 0u : (unsigned)((tl.nrow0 - XROWS) * KD * 2);
  auto xpc = [&](int pc) -> unsigned {
    const bool pad = NIMG == 2 && NWS == 0 && tl.nrow0 + pc * 8 >= NROWS;
    return (unsigned)((pad ? pc - 8 : pc) * (8 * KD * 2));
  };
  int ws_ = 0, xs_ = 0;
  auto kmap = [&](int t) -> unsigned { return PAIR ? (unsigned)(((((t >> 1) * KSTRIDE) << 1) + (t & 1) + tl.koff) & (NT - 1)) : (unsigned)((t + tl.koff) & (NT - 1)); };
  auto issue_w = [&](int t) {
    const unsigned tb = wbase + kmap(t) * (BK * ND * 2);
#pragma unroll
    for (int i = 0; i < NW_; ++i) {
      const int pc = H + 3 * i, img = pc >> 4, q = pc & 15;
      __builtin_amdgcn_raw_ptr_buffer_load_lds(rw, (LAS void*)(lds + ws_ * WSLOT + img * WIMG + wpiece(q)), 16, wv, tb + q * (4 * ND * 2) + img * 256, 0, AUXW);
    }
    ws_ = ws_ == SW - 1 ? 0 : ws_ + 1;
  };
  auto issue_xl = [&](int t) {
    const unsigned tb = xbase + kmap(t) * 128u;
#pragma unroll
    for (int i = 0; i < NX_; ++i) {
      const int pc = X0 + 3 * i;
      if (pc >= NWS) __builtin_amdgcn_raw_ptr_buffer_load_lds(rx, (LAS void*)(lds + LX + xs_ * XSLOT + pc * 1024), 16, xv[pc & 1], tb + xpc(pc), 0, 0);
    }
  };
  issue_w(0);
  issue_xl(0);
  unsigned call = 0;
  if (XWS) {
    BAR();
    call = (unsigned)__builtin_amdgcn_readfirstlane((int)lds_rd(lds, C_CALL));
  }
  const auto ri = __builtin_amdgcn_make_buffer_rsrc((char*)p.ws[p.rank] + WS_XBUF + (call & 1) * XPAR_BYTES, 0, (int)XPAR_BYTES, 0x00020000);
  const unsigned ibase = (unsigned)(tl.nrow0 * 128);
  auto issue_xi = [&](int t) {
#pragma unroll
    for (int i = 0; i < NX_; ++i) {
      const int pc = X0 + 3 * i;
      if (pc < NWS) __builtin_amdgcn_raw_ptr_buffer_load_lds(ri, (LAS void*)(lds + LX + xs_ * XSLOT + pc * 1024), 16, iv, ibase + (unsigned)t * (unsigned)XT_BYTES + pc * 1024, 0, 0);
    }
    xs_ = xs_ == 2 ? 0 : xs_ + 1;
  };
  issue_xi(0);
  issue_w(1);
  issue_xl(1);
  issue_xi(1);
  waitv<NW_ + NX_>();
  BAR();
  if (PAIR) {
    constexpr int NPF = (XPD && H > 0) ? 2 : 0;
    auto prefetch = [&](int t) {
      const unsigned tb = kmap(t) * 128u + (unsigned)tl.ai * (16 * KD * 2);
#pragma unroll
      for (int i = 0; i < NPF; ++i)
        __builtin_amdgcn_raw_ptr_buffer_load_lds(rx, (LAS void*)(lds + LX + 3 * XSLOT + H * 1024), 16, xv[0], tb + i * (8 * KD * 2), 0, 0);
    };
    for (int t = 0; t < NT - 2; t += 2) {
      issue_w(t + 2);
      issue_w(t + 3);
      prefetch(t + XPD + H - 1);
      issue_xl(t + 2);
      issue_xi(t + 2);
      waitv<2 * NW_ + NPF + NX_>();
      BAR();
      issue_xl(t + 3);
      issue_xi(t + 3);
      waitv<NX_>();
      BAR();
    }
  } else if (ROT && (NWS == 0 || NWS == 4 * MF)) {
    constexpr int QW = (NW_ + 3) / 4, QX = (NX_ + 3) / 4;
    unsigned wl[NW_], wg[NW_], xl[NX_], xg[NX_];
#pragma unroll
    for (int i = 0; i < NW_; ++i) {
      int k = i + tl.qw * QW; k = k >= NW_ ? k - NW_ : k;
      const int pc = H + 3 * k, img = pc >> 4, q = pc & 15;
      wl[i] = (unsigned)(img * WIMG + ((q >> 1) & 1) * WODD + (q >> 2) * 2048 + (q & 1) * 1024);
      wg[i] = wv + (unsigned)(q * (4 * ND * 2) + img * 256);
    }
#pragma unroll
    for (int i = 0; i < NX_; ++i) {
      int k = i + tl.qx * QX; k = k >= NX_ ? k - NX_ : k;
      const int pc = X0 + 3 * k;
      xl[i] = (unsigned)(pc * 1024);
      xg[i] = XWS ? iv + (unsigned)(pc * 1024) : ((pc & 1) ? xv[1] : xv[0]) + xpc(pc);
    }
    for (int t = 0; t < NT - 2; ++t) {
      const unsigned tw = wbase + (unsigned)(t + 2) * (BK * ND * 2);
      const unsigned tx = XWS ? ibase + (unsigned)(t + 2) * (unsigned)XT_BYTES : xbase + (unsigned)(t + 2) * 128u;
      LAS char* lw = lds + ws_ * WSLOT;
      LAS char* lx = lds + LX + xs_ * XSLOT;
#pragma unroll
      for (int ph = 0; ph < 2; ++ph) {
#pragma unroll
        for (int i = ph ? QW : 0; i < (ph ? NW_ : QW); ++i)
          __builtin_amdgcn_raw_ptr_buffer_load_lds(rw, (LAS void*)(lw + wl[i]), 16, wg[i], tw, 0, AUXW);
#pragma unroll
        for (int i = ph ? QX : 0; i < (ph ? NX_ : QX); ++i)
          __builtin_amdgcn_raw_ptr_buffer_load_lds(XWS ? ri : rx, (LAS void*)(lx + xl[i]), 16, xg[i], tx, 0, 0);
      }
      ws_ = ws_ == SW - 1 ? 0 : ws_ + 1;
      xs_ = xs_ == 2 ? 0 : xs_ + 1;
      waitv<NW_ + NX_>();
      BAR();
    }
  } else {
    for (int t = 0; t < NT - 2; ++t) {
      issue_w(t + 2);
      issue_xl(t + 2);
      issue_xi(t + 2);
      waitv<NW_ + NX_>();
      BAR();
    }
  }
  waitv<0>();
  BAR();
  BAR();
}

#define NOP_ do {} while (0)
#define STEP(a, d) asm volatile("v_add_u32 %0, %1, %0" : "+v"(a) : "s"(d))
template <int MF>
__device__ __forceinline__ void mfma_a(const Params& p, const Tile& tl, LAS char* lds, int wid, int lane) {
  constexpr int XSLOT = MF * 4096, LX = SWA * WIMG;
  const unsigned long long tk0 = __builtin_amdgcn_s_memrealtime();
  const int wr = wid >> 1, wc = wid & 1;
  const int fr = lane & 15, fq = lane >> 4;
  const int q = (lane & 15) >> 2, pp = lane & 3;
  const unsigned lb = (unsigned)(size_t)lds;
  unsigned xa0 = lb + (unsigned)(LX + (wr * MF * 16 + fr) * 128 + ((fq ^ ((fr >> 1) & 7)) * 16));
  unsigned xa1 = lb + ((unsigned)(LX + (wr * MF * 16 + fr) * 128 + ((fq ^ ((fr >> 1) & 7)) * 16)) ^ 64u);
  unsigned wa[4];
#pragma unroll
  for (int nf = 0; nf < 4; ++nf)
    wa[nf] = lb + (unsigned)((fq & 1) * WODD + (fq >> 1) * 2048 + q * 256 + wc * 128 + ((nf ^ q) * 32) + 8 * pp);
  f32x4 acc[MF][4];
#pragma unroll
  for (int i = 0; i < MF; ++i)
#pragma unroll
    for (int j = 0; j < 4; ++j) acc[i][j] = (f32x4){0.f, 0.f, 0.f, 0.f};
  bf16x8 Xf[MF];
  s16x4 Wl[2][4], Wh[2][4];
#define RDW(bu, sb, ks, i) do { if ((i) & 1) Wh[bu][(i) >> 1] = ldw(wa[(i) >> 1] + (ks) * 4096 + 1024); \
    else Wl[bu][(i) >> 1] = ldw(wa[(i) >> 1] + (ks) * 4096); } while (0)
#define MMA1(mf, bu, nf) do { mma(acc[mf][nf], wfrag(Wl[bu][nf], Wh[bu][nf]), Xf[mf]); SCHED(); } while (0)
#define SN(mf, i) (((SNAKE != 0) & (((mf) & 1) != 0)) ? 3 - (i) : (i))
#define GRP(mf, bu, F0, F1, F2) do { MMA1(mf, bu, SN(mf, 0)); F0; SCHED(); MMA1(mf, bu, SN(mf, 1)); F1; SCHED(); MMA1(mf, bu, SN(mf, 2)); F2; SCHED(); MMA1(mf, bu, SN(mf, 3)); } while (0)
#define RDX(i, sb, xa) Xf[i] = ldx((xa) + (i) * 2048)
  BAR();
  STAMP(1);
#pragma unroll
  for (int i = 0; i < 8; ++i) RDW(0, lds, 0, i);
#pragma unroll
  for (int mf = 0; mf < MF - 1; ++mf) RDX(mf, lds, xa0);
  __builtin_amdgcn_s_waitcnt(0xC07F);
  int sw = 0, sx = 0;
  TRACE_DECL;
  for (int t = 0; t < NT; ++t) {
    TRACE(t);
    const int dw = (sw == SWA - 1) ? -(SWA - 1) * WIMG : WIMG;
    const int dx = (sx == 2) ? -2 * XSLOT : XSLOT;
    sw = (sw == SWA - 1) ? 0 : sw + 1;
    sx = (sx == 2) ? 0 : sx + 1;
    GRP(0, 0, RDX(MF - 1, lds, xa0), RDW(1, lds, 1, 0), RDW(1, lds, 1, 1));
    GRP(1, 0, RDX(0, lds, xa1), RDW(1, lds, 1, 2), RDW(1, lds, 1, 3));
    GRP(2, 0, RDX(1, lds, xa1), RDW(1, lds, 1, 4), STEP(xa0, dx));
    GRP(3, 0, RDX(2, lds, xa1), RDW(1, lds, 1, 5), STEP(wa[0], dw));
    GRP(4, 0, RDX(3, lds, xa1), RDW(1, lds, 1, 6), STEP(wa[1], dw));
    GRP(5, 0, RDX(4, lds, xa1), RDW(1, lds, 1, 7), STEP(wa[2], dw));
    GRP(6, 0, RDX(5, lds, xa1), NOP_, STEP(wa[3], dw));
#pragma unroll
    for (int g = 7; g < MF; ++g) GRP(g, 0, RDX(g - 1, lds, xa1), NOP_, NOP_);
    GRP(0, 1, RDX(MF - 1, lds, xa1), NOP_, NOP_);
    GRP(1, 1, NOP_, STEP(xa1, dx), NOP_);
    __builtin_amdgcn_s_waitcnt(0xC07F);
    CFENCE();
    BAR();
    CFENCE();
    SCHED();
    GRP(2, 1, RDW(0, lds, 0, 0), RDW(0, lds, 0, 1), RDX(0, lds, xa0));
    GRP(3, 1, RDW(0, lds, 0, 2), RDW(0, lds, 0, 3), RDX(1, lds, xa0));
    GRP(4, 1, RDW(0, lds, 0, 4), RDW(0, lds, 0, 5), RDX(2, lds, xa0));
    GRP(5, 1, RDW(0, lds, 0, 6), RDW(0, lds, 0, 7), RDX(3, lds, xa0));
    GRP(6, 1, RDX(4, lds, xa0), RDX(5, lds, xa0), NOP_);
#pragma unroll
    for (int g = 7; g < MF; ++g) GRP(g, 1, RDX(g - 1, lds, xa0), NOP_, NOP_);
  }
  if (ASMMFMA) asm volatile("s_nop 15\n\ts_nop 7" ::: "memory");
  STAMP(2);
  TRACE_OUT;
  const unsigned go = (unsigned)__builtin_amdgcn_readfirstlane((int)lds_rd(lds, C_GO));
  const bool bad = (go & 1u) != 0u;
  const auto ro = __builtin_amdgcn_make_buffer_rsrc((void*)p.out[tl.owner], 0, ML * NR * NCOL * 2, 0x00020000);
  const unsigned vo = out_voff(lane);
  const unsigned so = (unsigned)(((p.rank * ML + wr * MF * 16) * NCOL + tl.col0 + wc * tl.cgap) * 2);
  int skip = 0;
  const bool late = LATE_TICKS && __builtin_amdgcn_s_memrealtime() - tk0 > LATE_TICKS;
  if (late) skip = LATE_SKIP;
#pragma unroll
  for (int mf = 0; mf < MF; ++mf) {
    const unsigned long long t0 = __builtin_amdgcn_s_memrealtime();
#pragma unroll
    for (int np = 0; np < 2; ++np)
      __builtin_amdgcn_raw_buffer_store_b128(pack2(acc[mf][2 * np], acc[mf][2 * np + 1], bad), ro, vo + np * 64, so + mf * (16 * NCOL * 2), 17);
    if (skip > 0) --skip;
    else {
      waitv<AWIN>();
      if (RTT_TICKS && __builtin_amdgcn_s_memrealtime() - t0 < RTT_TICKS) skip = RTT_SKIP;
      if (late) skip = LATE_SKIP;
    }
  }
  waitv<0>();
  if (lane == 0) st_sys(p.ws[tl.owner] + WS_DONE + p.rank * 64 + tl.slot * 4 + wid, go >> 1);
  STAMP(3);
#undef RDW
#undef MMA1
#undef GRP
#undef RDX
}

__device__ __forceinline__ void mfma_n(const Params& p, const Tile& tl, LAS char* lds, int wid, int lane) {
  constexpr int MF = MFN;
  constexpr int WSLOT = 2 * WIMG, XSLOT = MF * 4096, LX = 3 * WSLOT;
  const int wr = wid >> 1, wc = wid & 1;
  const int fr = lane & 15, fq = lane >> 4;
  const int q = (lane & 15) >> 2, pp = lane & 3;
  const unsigned lb = (unsigned)(size_t)lds;
  unsigned xa0 = lb + (unsigned)(LX + (wr * MF * 16 + fr) * 128 + ((fq ^ ((fr >> 1) & 7)) * 16));
  unsigned xa1 = lb + ((unsigned)(LX + (wr * MF * 16 + fr) * 128 + ((fq ^ ((fr >> 1) & 7)) * 16)) ^ 64u);
  unsigned wa[4];
#pragma unroll
  for (int nf = 0; nf < 4; ++nf)
    wa[nf] = lb + (unsigned)(wc * WIMG + (fq & 1) * WODD + (fq >> 1) * 2048 + q * 256 + ((nf ^ q) * 32) + 8 * pp);
  f32x4 acc[MF][8];
#pragma unroll
  for (int i = 0; i < MF; ++i)
#pragma unroll
    for (int j = 0; j < 8; ++j) acc[i][j] = (f32x4){0.f, 0.f, 0.f, 0.f};
  bf16x8 Xf[2][MF];
  s16x4 Wl[8], Wh[8];
#define RWL(nf, sb, ks) Wl[nf] = ldw(wa[(nf) & 3] + ((nf) >> 2) * 128 + (ks) * 4096)
#define RWH(nf, sb, ks) Wh[nf] = ldw(wa[(nf) & 3] + ((nf) >> 2) * 128 + (ks) * 4096 + 1024)
#define RDX(bu, i, sb, xa) Xf[bu][i] = ldx((xa) + (i) * 2048)
#define MMA1(nf, bu, mf) do { mma(acc[mf][nf], wfrag(Wl[nf], Wh[nf]), Xf[bu][mf]); SCHED(); } while (0)
#define SN5(nf, i) (((SNAKE != 0) & (((nf) & 1) != 0)) ? 4 - (i) : (i))
#define GRP(nf, bu, F0, F1, F2, F3, F4) do { MMA1(nf, bu, SN5(nf, 0)); F0; SCHED(); MMA1(nf, bu, SN5(nf, 1)); F1; SCHED(); MMA1(nf, bu, SN5(nf, 2)); F2; SCHED(); \
    MMA1(nf, bu, SN5(nf, 3)); F3; SCHED(); MMA1(nf, bu, SN5(nf, 4)); F4; SCHED(); } while (0)
  if (tl.gathered) {
    const unsigned gwv = (unsigned)(((tl.nrow0 / NM) * 8 + (tl.col0 >> 8)) * 4 + wid);
    LAS void* dummy = (LAS void*)(lds + 2 * WSLOT + wid * 1024);
    {
      const auto rx = __builtin_amdgcn_make_buffer_rsrc((void*)p.x, 0, ML * KD * 2, 0x00020000);
#pragma unroll
      for (int i = 0; i < 10; ++i) __builtin_amdgcn_raw_ptr_buffer_load_lds(rx, dummy, 16, (unsigned)(lane * 16), (gwv * 10240u + i * 1024u) & (unsigned)(ML * KD * 2 - 1), 0, 0);
    }
    waitv<0>();
    BAR();
  }
  BAR();
  STAMP(1);
#pragma unroll
  for (int mf = 0; mf < MF; ++mf) RDX(0, mf, lds, xa0);
#pragma unroll
  for (int nf = 0; nf < 7; ++nf) { RWL(nf, lds, 0); RWH(nf, lds, 0); }
  __builtin_amdgcn_s_waitcnt(0xC07F);
  int st = 0;
  TRACE_DECL;
  for (int t = 0; t < NT; ++t) {
    TRACE(t);
    const int dw = (st == 2) ? -2 * WSLOT : WSLOT;
    const int dx = (st == 2) ? -2 * XSLOT : XSLOT;
    st = (st == 2) ? 0 : st + 1;
    GRP(0, 0, RWL(7, lds, 0), RWH(7, lds, 0), RDX(1, 0, lds, xa1), STEP(xa0, dx), NOP_);
    GRP(1, 0, RWL(0, lds, 1), RWH(0, lds, 1), RDX(1, 1, lds, xa1), NOP_, NOP_);
    GRP(2, 0, RWL(1, lds, 1), RWH(1, lds, 1), RDX(1, 2, lds, xa1), NOP_, NOP_);
    GRP(3, 0, RWL(2, lds, 1), RWH(2, lds, 1), RDX(1, 3, lds, xa1), NOP_, NOP_);
    GRP(4, 0, RWL(3, lds, 1), RWH(3, lds, 1), RDX(1, 4, lds, xa1), NOP_, NOP_);
    GRP(5, 0, RWL(4, lds, 1), RWH(4, lds, 1), NOP_, STEP(wa[0], dw), STEP(xa1, dx));
    GRP(6, 0, RWL(5, lds, 1), RWH(5, lds, 1), NOP_, STEP(wa[1], dw), NOP_);
    GRP(7, 0, RWL(6, lds, 1), RWH(6, lds, 1), NOP_, STEP(wa[2], dw), NOP_);
    GRP(0, 1, RWL(7, lds, 1), RWH(7, lds, 1), NOP_, STEP(wa[3], dw), NOP_);
    GRP(1, 1, NOP_, NOP_, NOP_, NOP_, NOP_);
    __builtin_amdgcn_s_waitcnt(0xC07F);
    CFENCE();
    BAR();
    CFENCE();
    SCHED();
    GRP(2, 1, RWL(0, lds, 0), RWH(0, lds, 0), RDX(0, 0, lds, xa0), RWL(1, lds, 0), RWH(1, lds, 0));
    GRP(3, 1, RWL(2, lds, 0), RWH(2, lds, 0), RDX(0, 1, lds, xa0), RDX(0, 2, lds, xa0), NOP_);
    GRP(4, 1, RWL(3, lds, 0), RWH(3, lds, 0), RDX(0, 3, lds, xa0), RDX(0, 4, lds, xa0), NOP_);
    GRP(5, 1, RWL(4, lds, 0), RWH(4, lds, 0), NOP_, NOP_, NOP_);
    GRP(6, 1, RWL(5, lds, 0), RWH(5, lds, 0), NOP_, NOP_, NOP_);
    GRP(7, 1, RWL(6, lds, 0), RWH(6, lds, 0), NOP_, NOP_, NOP_);
  }
  if (ASMMFMA) asm volatile("s_nop 15\n\ts_nop 7" ::: "memory");
  STAMP(2);
  TRACE_OUT;
  const unsigned go = (unsigned)__builtin_amdgcn_readfirstlane((int)lds_rd(lds, C_GO));
  const bool bad = (go & 1u) != 0u;
  const auto ro = __builtin_amdgcn_make_buffer_rsrc((void*)p.out[p.rank], 0, ML * NR * NCOL * 2, 0x00020000);
  const unsigned vo = out_voff(lane);
  const unsigned so = (unsigned)((tl.col0 + wc * 128) * 2);
#pragma unroll
  for (int mf = 0; mf < MF; ++mf) {
    const int nr = tl.nrow0 + wr * MF * 16 + mf * 16;
    if (nr < NROWS) {
      const int orow = nr < XROWS ? ((p.rank + 1 + nr / BROWS) & 7) * ML + AROWS + nr % BROWS : p.rank * ML + nr - XROWS;
#pragma unroll
      for (int np = 0; np < 4; ++np)
        __builtin_amdgcn_raw_buffer_store_b128(pack2(acc[mf][2 * np], acc[mf][2 * np + 1], bad), ro, vo + np * 64, so + (unsigned)orow * (NCOL * 2), NAUX);
    }
  }
  STAMP(3);
#undef RWL
#undef RWH
#undef RDX
#undef MMA1
#undef GRP
}

__device__ __forceinline__ bool wait_words(const unsigned* word, bool use, unsigned call, unsigned* tmo) {
  const unsigned long long t0 = __builtin_amdgcn_s_memrealtime();
  unsigned spins = 0;
  for (;;) {
    const unsigned v = ld_sys(word);
    if (__all(!use || v >= call)) return true;
    if ((++spins & 63u) == 0u && (ld_agent(tmo) == call || __builtin_amdgcn_s_memrealtime() - t0 > DEADLINE)) { st_agent(tmo, call); return false; }
    __builtin_amdgcn_s_sleep(1);
  }
}

__device__ __forceinline__ void ctrl_wave(const Params& p, const Tile& tl, LAS char* lds, int lane) {
  unsigned* myws = p.ws[p.rank];
  unsigned* tmo = myws + WS_TMO;
  const int b = blockIdx.x;
  const unsigned long long tk0 = NCYC ? __builtin_amdgcn_s_memtime() : __builtin_amdgcn_s_memrealtime();
  const unsigned cnt = ld_agent(myws + WS_CNT + b);
  if (!tl.gathered) BAR();
  const unsigned call = (unsigned)__builtin_amdgcn_readfirstlane((int)cnt) + 1u;
  if (lane == 0) st_agent(myws + WS_CNT + b, call);
  bool ok = true;
  int nb = 1;
#define PCLK() __builtin_amdgcn_s_memrealtime()
#define PSC 1
  auto join = [&]() {
    if (NPACE && tl.kind == 1) {
      long long due = (long long)NS_T0 + (long long)nb * NPACE;
      if (!tl.gathered) { const long long d2 = nb > NC_N0 ? (long long)NC_T0 + (long long)(nb - NC_N0) * NC_SLOPE : 0ll; due = d2 < due ? d2 : due; }
      if (nb < NS_N0) due = 0;
      due = (due < (long long)NPACE_END ? due : (long long)NPACE_END) * PSC;
      while ((long long)(PCLK() - tk0) < due) __builtin_amdgcn_s_sleep(1);
      BAR();
    } else BAR();
    ++nb;
  };
  if (tl.gathered) {
    if (lane == 0) lds_wr(lds, C_CALL, call);
    const int s0 = tl.nrow0 / BROWS;
    const int e1 = (tl.nrow0 + NM - 1) / BROWS;
    const int s1 = e1 < 7 ? e1 : 6;
    const unsigned* sflag = myws + WS_XFLAG + ((lane & 16) ? s1 : s0) * 32 + (lane & 15);
    auto decode = [&](unsigned v) -> int {
#pragma unroll
      for (int o = 1; o < 32; o <<= 1) { const unsigned u = (unsigned)__shfl_xor((int)v, o); v = u < v ? u : v; }
      const unsigned vm = (unsigned)__builtin_amdgcn_readfirstlane((int)v);
      const int n = vm > call * 128u ? (int)(vm - call * 128u) : 0;
      return n > NT ? NT : n;
    };
    int seen = 0;
    nb = -1;
    if (ACQ == 3) __builtin_amdgcn_fence(__ATOMIC_ACQUIRE, "");
    while (seen < NT) {
      const int need = nb + 3 < NT ? nb + 3 : NT;
      if (seen < need) {
        const unsigned long long t0 = __builtin_amdgcn_s_memrealtime();
        unsigned spins = 0;
        while ((seen = decode(ld_sys(sflag))) < need) {
          if ((++spins & 63u) == 0u && (ld_agent(tmo) == call || __builtin_amdgcn_s_memrealtime() - t0 > DEADLINE)) { st_agent(tmo, call); ok = false; seen = NT; break; }
          __builtin_amdgcn_s_sleep(1);
        }
        if (ACQ == 2 || (ACQ == 1 && nb == -1)) {
          __builtin_amdgcn_fence(__ATOMIC_ACQUIRE, "");
          waitv<0>();
        }
        if (nb == -1) STAMP(5);
        WAITL(0);
        join();
      } else if (seen < need + POLL_MARGIN) {
        unsigned v;
        asm volatile("global_load_dword %0, %1, off sc0 sc1" : "=v"(v) : "v"(sflag) : "memory");
        join();
        asm volatile("s_waitcnt vmcnt(0)" : "+v"(v) :: "memory");
        const int n = decode(v);
        if (n > seen) {
          if (ACQ == 2) {
            __builtin_amdgcn_fence(__ATOMIC_ACQUIRE, "");
            if (seen >= (nb + 3 < NT ? nb + 3 : NT)) { join(); }
            waitv<0>();
          }
          seen = n;
        }
      } else {
        join();
      }
    }
    STAMP(7);
  } else if (tl.kind == 0) {
    while (nb < NT / 2) { join(); }
    ok = wait_words(myws + WS_READY + tl.owner * 32, true, call, tmo);
  }
  while (nb < NT - 1) { join(); }
  if (ld_agent(tmo) == call) ok = false;
  if (lane == 0) lds_wr(lds, C_GO, call * 2u + (ok ? 0u : 1u));
  WAITL(0);
  while (nb <= NT) { join(); }
}

__device__ __forceinline__ void copy_group(const Params& p, LAS char* lds, int c, int wid, int lane) {
  unsigned* myws = p.ws[p.rank];
  unsigned* tmo = myws + WS_TMO;
  const int b = blockIdx.x;
  const unsigned cnt = ld_agent(myws + WS_CNT + b);
  const int gw = wid * 16 + c;
  const auto rx = __builtin_amdgcn_make_buffer_rsrc((void*)p.x, 0, ML * KD * 2, 0x00020000);
  const unsigned xv = (unsigned)((AROWS + (lane >> 3)) * KD * 2 + (((lane & 7) ^ (((gw & 1) * 4 + (lane >> 4)) & 7)) * 16));
  auto load = [&](int k) { const int u = k * 128 + gw; return __builtin_amdgcn_raw_buffer_load_b128(rx, xv, (unsigned)((u % BRP) * (8 * KD * 2) + (u / BRP) * 128), 0); };
  u32x4 d[NUNIT];
  d[0] = load(0);
  d[1] = load(1);
  char* dbase[NR - 1];
  unsigned dslot[NR - 1];
#pragma unroll
  for (int i = 0; i < NR - 1; ++i) {
    const int j = __builtin_amdgcn_readfirstlane((i + gw) % (NR - 1) + 1);
    dbase[i] = (char*)p.ws[__builtin_amdgcn_readfirstlane((p.rank + NR - j) & 7)];
    dslot[i] = (unsigned)((j - 1) * BROWS * 128);
  }
  if (lane == 0) { lds_wr(lds, wid * 4, (unsigned)(gw / BRP)); if (wid == 0) lds_wr(lds, 64, 0u); }
  WAITL(0);
  const unsigned call = (unsigned)__builtin_amdgcn_readfirstlane((int)cnt) + 1u;
  BAR();
  const unsigned long long tstart = __builtin_amdgcn_s_memrealtime();
  STAMP(1);
  if (wid == 0 && lane == 0) st_agent(myws + WS_CNT + b, call);
  if (gw == 127 && lane < NR && lane != p.rank) st_sys(p.ws[lane] + WS_READY + p.rank * 32, call);
  const size_t xo = WS_XBUF + (call & 1) * XPAR_BYTES;
  unsigned* pflag = p.ws[(p.rank + NR - 1 - (lane < 7 ? lane : 0)) & 7] + WS_XFLAG + (lane < 7 ? lane : 0) * 32 + c;
  auto finish = [&](int k) {
    if (k == 0) STAMP(3);
    const int nxt = k + 1 < NUNIT ? ((k + 1) * 128 + gw) / BRP : NT;
    if (lane == 0) lds_wr(lds, wid * 4, (unsigned)nxt);
    unsigned v = lds_rd(lds, (lane & 7) * 4);
#pragma unroll
    for (int o = 1; o < 8; o <<= 1) { const unsigned u = (unsigned)__shfl_xor((int)v, o); v = u < v ? u : v; }
    const int pre = __builtin_amdgcn_readfirstlane((int)v);
    const int lvl = pre < 4 ? pre : (pre < 12 ? (pre & ~1) : (pre & ~3));
    unsigned old = 0;
    if (lane == 0) old = __hip_atomic_fetch_max((LAS unsigned*)(lds + 64), (unsigned)lvl, __ATOMIC_RELAXED, __HIP_MEMORY_SCOPE_WORKGROUP);
    old = (unsigned)__builtin_amdgcn_readfirstlane((int)old);
    if ((unsigned)lvl > old && lane < 7)
      __hip_atomic_fetch_max(pflag, call * 128u + (unsigned)lvl, __ATOMIC_RELAXED, __HIP_MEMORY_SCOPE_SYSTEM);
  };
  bool pending = false;
  long long tissue = 0;
#pragma unroll
  for (int k = 0; k < NUNIT; ++k) {
    const int u = k * 128 + gw;
    for (;;) {
      const long long now = (long long)(__builtin_amdgcn_s_memrealtime() - tstart) * 10;
      if (now * BRP >= (long long)(u - PUSH_LEAD * BRP) * TILE_NS) break;
      if (pending && now - tissue > ACK_NS) { waitv<0>(); finish(k - 1); pending = false; }
      else __builtin_amdgcn_s_sleep(2);
    }
    const unsigned so = (unsigned)(u / BRP) * (unsigned)XT_BYTES + (unsigned)(u % BRP) * 1024u;
#pragma unroll
    for (int i = 0; i < NR - 1; ++i) {
      const auto rd = __builtin_amdgcn_make_buffer_rsrc(dbase[i] + xo, 0, (int)XPAR_BYTES, 0x00020000);
      __builtin_amdgcn_raw_buffer_store_b128(d[k], rd, (unsigned)(lane * 16), so + dslot[i], 17);
    }
    tissue = (long long)(__builtin_amdgcn_s_memrealtime() - tstart) * 10;
    if (k == 0) STAMP(2);
    if (pending) { waitv<7>(); finish(k - 1); }
    pending = true;
    if (k + 2 < NUNIT) d[k + 2] = load(k + 2);
  }
  waitv<0>();
  finish(NUNIT - 1);
  STAMP(4);
  if (c == 15) {
    if (wid != p.rank) {
      const bool fine = wait_words(myws + WS_DONE + wid * 64 + lane, true, call, tmo);
      if (!fine && lane == 0) p.out[p.rank][(size_t)(wid * ML) * NCOL] = 0x7fc0;
    }
    STAMP(6);
  }
}

__global__ __launch_bounds__(512, 2) void gemm_a2a(Params p) {
  extern __shared__ __attribute__((aligned(16))) char smem[];
  LAS char* lds = (LAS char*)smem;
  const int wid = __builtin_amdgcn_readfirstlane(threadIdx.x >> 6);
  const int lane = threadIdx.x & 63;
  const int b = blockIdx.x;
  const int xcd = b & 7, idx = b >> 3;
  STAMP(0);
  if (idx < 2) { copy_group(p, lds, xcd * 2 + idx, wid, lane); return; }
  Tile tl;
  constexpr int RMIX = XROWS / NM;
  int rt = 0;
  if (idx < 16) {
    const int a = xcd * 14 + idx - 2;
    tl.kind = 0; tl.owner = (p.rank + (AMAP ? a % 7 : a >> 4) + 1) & 7;
    { const int pn = AMAP ? a / 7 : a & 15; tl.slot = pn;
      if (SPLX) { tl.col0 = (pn & ~SPLX) * 128 + ((pn & SPLX) ? 64 : 0); tl.cgap = SPLX * 128; } else { tl.col0 = pn * 128; tl.cgap = 64; } }
    tl.nrow0 = 0; tl.gathered = false; tl.koff = KSKEW * xcd; tl.ai = idx - 2; tl.qw = tl.qx = 0;
  } else {
    const int e = idx - 16;
    rt = (xcd >> 1) * 4 + (e >> 2);
    tl.kind = 1; tl.owner = p.rank; tl.col0 = ((xcd & 1) * 4 + (e & 3)) * 256; tl.cgap = 64; tl.slot = 0;
    tl.nrow0 = rt * NM; tl.gathered = rt <= RMIX; tl.koff = 0; tl.ai = 0; tl.qw = e >> 2; tl.qx = e & 3;
  }
  if (wid < 4) {
    if (tl.kind == 0) mfma_a<MFA>(p, tl, lds, wid, lane);
    else mfma_n(p, tl, lds, wid, lane);
  } else if (wid == 7) {
    ctrl_wave(p, tl, lds, lane);
  } else {
#define LOADERS(MF, NIMG, SW, PAIR, NWS, AUX) do { if (wid == 4) loader<MF, NIMG, SW, PAIR, 0, NWS, AUX>(p, tl, lds, lane); \
    else if (wid == 5) loader<MF, NIMG, SW, PAIR, 1, NWS, AUX>(p, tl, lds, lane); else loader<MF, NIMG, SW, PAIR, 2, NWS, AUX>(p, tl, lds, lane); } while (0)
    if (tl.kind == 0) LOADERS(MFA, 1, SWA, WPAIR != 0, 0, WAUX);
    else if (rt < RMIX) LOADERS(MFN, 2, 3, false, 4 * MFN, 0);
    else if (rt == RMIX) LOADERS(MFN, 2, 3, false, (XROWS - RMIX * NM) / 8, 0);
    else LOADERS(MFN, 2, 3, false, 0, 0);
  }
}

extern "C" void kernel_launch(int rank, int nranks, ncclComm_t comm,
                              hipStream_t stream,
                              void* const* d_in, const int* in_sizes, int n_in,
                              void* d_out, int out_size,
                              const int* in_dtypes, int out_dtype,
                              void* const* d_out_all,
                              void* const* d_ws_all, size_t ws_bytes) {
  if (nranks != NR || (unsigned)rank >= (unsigned)NR || ws_bytes < WS_NEED) { fprintf(stderr, "unexpected configuration\n"); return; }
  Params p;
  p.x = (const unsigned short*)d_in[0];
  p.w = (const unsigned short*)d_in[1];
  for (int i = 0; i < NR; ++i) { p.out[i] = (unsigned short*)d_out_all[i]; p.ws[i] = (unsigned*)d_ws_all[i]; }
  p.rank = rank;
  static bool init[NR] = {};
  if (!init[rank]) {
    HIP_CHECK(hipFuncSetAttribute((const void*)gemm_a2a, hipFuncAttributeMaxDynamicSharedMemorySize, LDS_BYTES));
    init[rank] = true;
  }
  gemm_a2a<<<NBLK, 512, LDS_BYTES, stream>>>(p);
  HIP_CHECK(hipGetLastError());
}
```

```cpp
#include <hip/hip_runtime.h>
#include <rccl/rccl.h>
#include <cstdio>
#include <stdint.h>

#define HIP_CHECK(expr) do { hipError_t _e = (expr); if (_e != hipSuccess) { \
    fprintf(stderr, "HIP error %s at %s:%d\n", hipGetErrorString(_e), __FILE__, __LINE__); \
    return; } } while (0)

#define LAS __attribute__((address_space(3)))
typedef short s16x4 __attribute__((ext_vector_type(4)));
typedef __bf16 bf16x2 __attribute__((ext_vector_type(2)));
typedef __bf16 bf16x8 __attribute__((ext_vector_type(8)));
typedef float f32x4 __attribute__((ext_vector_type(4)));
typedef float f32x2 __attribute__((ext_vector_type(2)));
typedef unsigned u32x4 __attribute__((ext_vector_type(4)));

constexpr int NR = 8, ML = 512, KD = 4096, ND = 16384, NCOL = ND / NR;
constexpr int BK = 64, NT = KD / BK;
constexpr int MFA = 7;
constexpr int MFN = 5;
constexpr int AROWS = MFA * 32, BROWS = ML - AROWS;
constexpr int BRP = BROWS / 8;
constexpr int XROWS = 7 * BROWS;
constexpr int NROWS = XROWS + ML;
constexpr int NM = MFN * 32;
constexpr int RMIX = XROWS / NM;
constexpr int WODD = 8192 + 128;
constexpr int WIMG = WODD + 8192;
constexpr int SWA = 4;
constexpr int LDS_CTL = 3 * (2 * WIMG + MFN * 4096);
static_assert(SWA * WIMG + 3 * MFA * 4096 <= LDS_CTL, "LDS");
constexpr int C_GO = LDS_CTL;
constexpr int C_CALL = LDS_CTL + 16;
constexpr int LDS_BYTES = LDS_CTL + 64;
static_assert(LDS_BYTES <= 163840, "LDS");
constexpr int NBLK = 256;
constexpr unsigned long long DEADLINE = 50000000ull;
constexpr int WS_TMO = 32, WS_READY = 64  , WS_CNT = 512  , WS_DONE = 1024  ;
constexpr int WS_XFLAG = 2048;
constexpr size_t WS_XBUF = 1u << 20;
constexpr size_t XT_BYTES = (size_t)XROWS * 128;
constexpr size_t XPAR_BYTES = XT_BYTES * NT;
constexpr size_t WS_NEED = WS_XBUF + 2 * XPAR_BYTES;
constexpr int NUNIT = BRP * NT / 128;
constexpr int TILE_NS = 640;
constexpr int PUSH_LEAD = 1;
constexpr int ACK_NS = 800;
constexpr int POLL_MARGIN = 4;
constexpr int KSKEW = 8, KSTRIDE = 11;
constexpr int NS_T0 = 800, NPACE = 106, NPACE_END = 4400, NS_N0 = 3;
constexpr int NC_N0 = 10, NC_T0 = 1150, NC_SLOPE = 125;
constexpr int AUX_NT = 2, AUX_WT_AGENT = 16, AUX_WT_SYS = 17;

struct Params {
  const unsigned short* x;
  const unsigned short* w;
  unsigned short* out[NR];
  unsigned* ws[NR];
  int rank;
};

#define WAITL0() asm volatile("s_waitcnt lgkmcnt(0)" ::: "memory")
template <int N> __device__ __forceinline__ void waitv() { asm volatile("s_waitcnt vmcnt(%0)" :: "n"(N) : "memory"); }
#define BAR() __builtin_amdgcn_s_barrier()
#define SCHED() __builtin_amdgcn_sched_barrier(0)
#define CFENCE() asm volatile("" ::: "memory")
#define NOP_ do {} while (0)
#define STEP(a, d) asm volatile("v_add_u32 %0, %1, %0" : "+v"(a) : "s"(d))
#define NEXT1(o, d) asm volatile("s_add_i32 %0, %0, %1" : "+s"(o) : "s"(d) : "scc")
#define NEXT2(o, d, last, slot) asm volatile("s_cmp_eq_u32 %1, %2\n\ts_cselect_b32 %0, %3, %4" : "=s"(d) : "s"(o), "s"(last), "s"(-(last)), "s"(slot) : "scc")

__device__ __forceinline__ bf16x8 ldx(unsigned a) { return *(LAS const bf16x8*)(size_t)a; }
__device__ __forceinline__ s16x4 ldw(unsigned a) { return __builtin_amdgcn_ds_read_tr16_b64_v4i16((LAS s16x4*)(size_t)a); }
__device__ __forceinline__ void mma(f32x4& c, bf16x8 a, bf16x8 b) { asm volatile("v_mfma_f32_16x16x32_bf16 %0, %1, %2, %0" : "+v"(c) : "v"(a), "v"(b)); }
__device__ __forceinline__ void mma_drain() {
  __builtin_amdgcn_sched_barrier(0);
  asm volatile("s_nop 15\n\ts_nop 7" ::: "memory");
  __builtin_amdgcn_sched_barrier(0);
}
__device__ __forceinline__ unsigned pk(float a, float b) { f32x2 v = {a, b}; return __builtin_bit_cast(unsigned, __builtin_convertvector(v, bf16x2)); }
__device__ __forceinline__ unsigned ld_sys(const unsigned* p) { return __hip_atomic_load(p, __ATOMIC_RELAXED, __HIP_MEMORY_SCOPE_SYSTEM); }
__device__ __forceinline__ unsigned ld_agent(const unsigned* p) { return __hip_atomic_load(p, __ATOMIC_RELAXED, __HIP_MEMORY_SCOPE_AGENT); }
__device__ __forceinline__ void st_sys(unsigned* p, unsigned v) { __hip_atomic_store(p, v, __ATOMIC_RELAXED, __HIP_MEMORY_SCOPE_SYSTEM); }
__device__ __forceinline__ void st_agent(unsigned* p, unsigned v) { __hip_atomic_store(p, v, __ATOMIC_RELAXED, __HIP_MEMORY_SCOPE_AGENT); }
__device__ __forceinline__ unsigned lds_rd(LAS char* lds, int off) { return *(LAS volatile unsigned*)(lds + off); }
__device__ __forceinline__ void lds_wr(LAS char* lds, int off, unsigned v) { *(LAS volatile unsigned*)(lds + off) = v; }
__device__ __forceinline__ bf16x8 wfrag(s16x4 lo, s16x4 hi) { return __builtin_bit_cast(bf16x8, __builtin_shufflevector(lo, hi, 0, 1, 2, 3, 4, 5, 6, 7)); }
__device__ __forceinline__ u32x4 pack2(f32x4 a, f32x4 c, bool bad) {
  const unsigned a0 = pk(a[0], a[1]), a1 = pk(a[2], a[3]), c0 = pk(c[0], c[1]), c1 = pk(c[2], c[3]);
  auto r0 = __builtin_amdgcn_permlane16_swap(a0, c0, false, false);
  auto r1 = __builtin_amdgcn_permlane16_swap(a1, c1, false, false);
  return bad ? (u32x4){0x7fc07fc0u, 0x7fc07fc0u, 0x7fc07fc0u, 0x7fc07fc0u} : (u32x4){r0[0], r1[0], r0[1], r1[1]};
}
__device__ __forceinline__ unsigned out_voff(int lane) { const int fr = lane & 15, fq = lane >> 4; return (unsigned)((fr * NCOL + (fq & 1) * 16 + (fq >> 1) * 8) * 2); }
__host__ __device__ constexpr int wpiece(int pc) { return ((pc >> 1) & 1) * WODD + (pc >> 2) * 2048 + (pc & 1) * 1024; }

struct Tile {
  int kind;
  int owner;
  int col0;
  int nrow0;
  bool gathered;
  int koff;
};

template <int MF, int NIMG, int SW, bool PAIR, int H, int NWS, int AUXW>
__device__ __forceinline__ void loader(const Params& p, const Tile& tl, LAS char* lds, int lane) {
  constexpr int NWP = 16 * NIMG;
  constexpr int WSLOT = NIMG * WIMG, XSLOT = MF * 4096, LX = SW * WSLOT;
  constexpr int NW_ = (NWP - H + 2) / 3;
  constexpr int X0 = (H + 3 * NW_) - NWP;
  constexpr int NX_ = (4 * MF - X0 + 2) / 3;
  constexpr bool XWS = NWS > 0;
  static_assert(2 * NW_ + 2 * NX_ <= 63, "vmcnt");
  const auto rw = __builtin_amdgcn_make_buffer_rsrc((void*)p.w, 0, KD * ND * 2, 0x00020000);
  const auto rx = __builtin_amdgcn_make_buffer_rsrc((void*)p.x, 0, ML * KD * 2, 0x00020000);
  const unsigned wv = (unsigned)((lane >> 4) * ND * 2 + (((lane & 15) ^ ((lane >> 4) << 1)) * 16));
  unsigned xv[2];
#pragma unroll
  for (int v = 0; v < 2; ++v) xv[v] = (unsigned)((lane >> 3) * KD * 2 + (((lane & 7) ^ ((v * 4 + (lane >> 4)) & 7)) * 16));
  const unsigned iv = (unsigned)(lane * 16);
  const unsigned wbase = (unsigned)((tl.owner * NCOL + tl.col0) * 2);
  const unsigned xbase = tl.kind == 0 ? 0u : (unsigned)((tl.nrow0 - XROWS) * KD * 2);
  auto xpc = [&](int pc) -> unsigned {
    const bool pad = NIMG == 2 && NWS == 0 && tl.nrow0 + pc * 8 >= NROWS;
    return (unsigned)((pad ? pc - 8 : pc) * (8 * KD * 2));
  };
  auto kmap = [&](int t) -> unsigned {
    return PAIR ? (unsigned)(((((t >> 1) * KSTRIDE) << 1) + (t & 1) + tl.koff) & (NT - 1)) : (unsigned)t;
  };
  int ws_ = 0, xs_ = 0;
  auto issue_w = [&](int t) {
    const unsigned tb = wbase + kmap(t) * (BK * ND * 2);
#pragma unroll
    for (int i = 0; i < NW_; ++i) {
      const int pc = H + 3 * i, img = pc >> 4, q = pc & 15;
      __builtin_amdgcn_raw_ptr_buffer_load_lds(rw, (LAS void*)(lds + ws_ * WSLOT + img * WIMG + wpiece(q)), 16, wv, tb + q * (4 * ND * 2) + img * 256, 0, AUXW);
    }
    ws_ = ws_ == SW - 1 ? 0 : ws_ + 1;
  };
  auto issue_xl = [&](int t) {
    const unsigned tb = xbase + kmap(t) * 128u;
#pragma unroll
    for (int i = 0; i < NX_; ++i) {
      const int pc = X0 + 3 * i;
      if (pc >= NWS) __builtin_amdgcn_raw_ptr_buffer_load_lds(rx, (LAS void*)(lds + LX + xs_ * XSLOT + pc * 1024), 16, xv[pc & 1], tb + xpc(pc), 0, 0);
    }
  };
  issue_w(0);
  issue_xl(0);
  unsigned call = 0;
  if (XWS) {
    BAR();
    call = (unsigned)__builtin_amdgcn_readfirstlane((int)lds_rd(lds, C_CALL));
  }
  const auto ri = __builtin_amdgcn_make_buffer_rsrc((char*)p.ws[p.rank] + WS_XBUF + (call & 1) * XPAR_BYTES, 0, (int)XPAR_BYTES, 0x00020000);
  const unsigned ibase = (unsigned)(tl.nrow0 * 128);
  auto issue_xi = [&](int t) {
#pragma unroll
    for (int i = 0; i < NX_; ++i) {
      const int pc = X0 + 3 * i;
      if (pc < NWS) __builtin_amdgcn_raw_ptr_buffer_load_lds(ri, (LAS void*)(lds + LX + xs_ * XSLOT + pc * 1024), 16, iv, ibase + (unsigned)t * (unsigned)XT_BYTES + pc * 1024, 0, 0);
    }
    xs_ = xs_ == 2 ? 0 : xs_ + 1;
  };
  issue_xi(0);
  issue_w(1);
  issue_xl(1);
  issue_xi(1);
  waitv<NW_ + NX_>();
  BAR();
  if (PAIR) {
    for (int t = 0; t < NT - 2; t += 2) {
      issue_w(t + 2);
      issue_w(t + 3);
      issue_xl(t + 2);
      issue_xi(t + 2);
      waitv<2 * NW_ + NX_>();
      BAR();
      issue_xl(t + 3);
      issue_xi(t + 3);
      waitv<NX_>();
      BAR();
    }
  } else {
    for (int t = 0; t < NT - 2; ++t) {
      issue_w(t + 2);
      issue_xl(t + 2);
      issue_xi(t + 2);
      waitv<NW_ + NX_>();
      BAR();
    }
  }
  waitv<0>();
  BAR();
  BAR();
}

__device__ __forceinline__ void mfma_a(const Params& p, const Tile& tl, LAS char* lds, int wid, int lane) {
  constexpr int MF = MFA;
  static_assert(MF == 7, "the schedule below is written for 7 fragments");
  constexpr int XSLOT = MF * 4096, LX = SWA * WIMG;
  const int wr = wid >> 1, wc = wid & 1;
  const int fr = lane & 15, fq = lane >> 4;
  const int q = (lane & 15) >> 2, pp = lane & 3;
  const unsigned lb = (unsigned)(size_t)lds;
  const unsigned xo = (unsigned)(LX + (wr * MF * 16 + fr) * 128 + ((fq ^ ((fr >> 1) & 7)) * 16));
  unsigned xa0 = lb + xo, xa1 = lb + (xo ^ 64u);
  unsigned wa[4];
#pragma unroll
  for (int nf = 0; nf < 4; ++nf)
    wa[nf] = lb + (unsigned)((fq & 1) * WODD + (fq >> 1) * 2048 + q * 256 + wc * 128 + ((nf ^ q) * 32) + 8 * pp);
  f32x4 acc[MF][4];
#pragma unroll
  for (int i = 0; i < MF; ++i)
#pragma unroll
    for (int j = 0; j < 4; ++j) { acc[i][j] = (f32x4){0.f, 0.f, 0.f, 0.f}; asm volatile("" : "+v"(acc[i][j])); }
  bf16x8 Xf[MF];
  s16x4 Wl[2][4], Wh[2][4];
#define RDW(bu, ks, i) do { if ((i) & 1) Wh[bu][(i) >> 1] = ldw(wa[(i) >> 1] + (ks) * 4096 + 1024); \
    else Wl[bu][(i) >> 1] = ldw(wa[(i) >> 1] + (ks) * 4096); } while (0)
#define RDX(i, xa) Xf[i] = ldx((xa) + (i) * 2048)
#define MMA1(mf, bu, nf) do { mma(acc[mf][nf], wfrag(Wl[bu][nf], Wh[bu][nf]), Xf[mf]); SCHED(); } while (0)
#define SN(mf, i) (((mf) & 1) ? 3 - (i) : (i))
#define GRP(mf, bu, F0, F1, F2) do { MMA1(mf, bu, SN(mf, 0)); F0; SCHED(); MMA1(mf, bu, SN(mf, 1)); F1; SCHED(); \
    MMA1(mf, bu, SN(mf, 2)); F2; SCHED(); MMA1(mf, bu, SN(mf, 3)); } while (0)
  BAR();
#pragma unroll
  for (int i = 0; i < 8; ++i) RDW(0, 0, i);
#pragma unroll
  for (int mf = 0; mf < MF - 1; ++mf) RDX(mf, xa0);
  __builtin_amdgcn_s_waitcnt(0xC07F);
  int ow = 0, ox = 0, dw = WIMG, dx = XSLOT;
  for (int t = 0; t < NT; ++t) {
    GRP(0, 0, RDX(6, xa0), RDW(1, 1, 0), RDW(1, 1, 1));
    GRP(1, 0, RDX(0, xa1), RDW(1, 1, 2), RDW(1, 1, 3));
    GRP(2, 0, RDX(1, xa1), RDW(1, 1, 4), STEP(xa0, dx));
    GRP(3, 0, RDX(2, xa1), RDW(1, 1, 5), STEP(wa[0], dw));
    GRP(4, 0, RDX(3, xa1), RDW(1, 1, 6), STEP(wa[1], dw));
    GRP(5, 0, RDX(4, xa1), RDW(1, 1, 7), STEP(wa[2], dw));
    GRP(6, 0, RDX(5, xa1), NOP_, STEP(wa[3], dw));
    GRP(0, 1, RDX(6, xa1), NEXT1(ow, dw), NEXT2(ow, dw, (SWA - 1) * WIMG, WIMG));
    GRP(1, 1, STEP(xa1, dx), NEXT1(ox, dx), NEXT2(ox, dx, 2 * XSLOT, XSLOT));
    __builtin_amdgcn_s_waitcnt(0xC07F);
    CFENCE();
    BAR();
    CFENCE();
    SCHED();
    GRP(2, 1, RDW(0, 0, 0), RDW(0, 0, 1), RDX(0, xa0));
    GRP(3, 1, RDW(0, 0, 2), RDW(0, 0, 3), RDX(1, xa0));
    GRP(4, 1, RDW(0, 0, 4), RDW(0, 0, 5), RDX(2, xa0));
    GRP(5, 1, RDW(0, 0, 6), RDW(0, 0, 7), RDX(3, xa0));
    GRP(6, 1, RDX(4, xa0), RDX(5, xa0), NOP_);
  }
  mma_drain();
  const unsigned go = (unsigned)__builtin_amdgcn_readfirstlane((int)lds_rd(lds, C_GO));
  const bool bad = (go & 1u) != 0u;
  const auto ro = __builtin_amdgcn_make_buffer_rsrc((void*)p.out[tl.owner], 0, ML * NR * NCOL * 2, 0x00020000);
  const unsigned vo = out_voff(lane);
  const unsigned so = (unsigned)(((p.rank * ML + wr * MF * 16) * NCOL + tl.col0 + wc * 64) * 2);
#pragma unroll
  for (int mf = 0; mf < MF; ++mf) {
#pragma unroll
    for (int np = 0; np < 2; ++np)
      __builtin_amdgcn_raw_buffer_store_b128(pack2(acc[mf][2 * np], acc[mf][2 * np + 1], bad), ro, vo + np * 64, so + mf * (16 * NCOL * 2), AUX_WT_SYS);
    waitv<0>();
  }
  if (lane == 0) st_sys(p.ws[tl.owner] + WS_DONE + p.rank * 64 + (tl.col0 >> 7) * 4 + wid, go >> 1);
#undef RDW
#undef RDX
#undef MMA1
#undef GRP
}

__device__ __forceinline__ void mfma_n(const Params& p, const Tile& tl, LAS char* lds, int wid, int lane) {
  constexpr int MF = MFN;
  static_assert(MF == 5, "the schedule below is written for 5 fragments");
  constexpr int WSLOT = 2 * WIMG, XSLOT = MF * 4096, LX = 3 * WSLOT;
  const int wr = wid >> 1, wc = wid & 1;
  const int fr = lane & 15, fq = lane >> 4;
  const int q = (lane & 15) >> 2, pp = lane & 3;
  const unsigned lb = (unsigned)(size_t)lds;
  const unsigned xo = (unsigned)(LX + (wr * MF * 16 + fr) * 128 + ((fq ^ ((fr >> 1) & 7)) * 16));
  unsigned xa0 = lb + xo, xa1 = lb + (xo ^ 64u);
  unsigned wa[4];
#pragma unroll
  for (int nf = 0; nf < 4; ++nf)
    wa[nf] = lb + (unsigned)(wc * WIMG + (fq & 1) * WODD + (fq >> 1) * 2048 + q * 256 + ((nf ^ q) * 32) + 8 * pp);
  f32x4 acc[MF][8];
#pragma unroll
  for (int i = 0; i < MF; ++i)
#pragma unroll
    for (int j = 0; j < 8; ++j) { acc[i][j] = (f32x4){0.f, 0.f, 0.f, 0.f}; asm volatile("" : "+v"(acc[i][j])); }
  bf16x8 Xf[2][MF];
  s16x4 Wl[8], Wh[8];
#define RWL(nf, ks) Wl[nf] = ldw(wa[(nf) & 3] + ((nf) >> 2) * 128 + (ks) * 4096)
#define RWH(nf, ks) Wh[nf] = ldw(wa[(nf) & 3] + ((nf) >> 2) * 128 + (ks) * 4096 + 1024)
#define RDX(bu, i, xa) Xf[bu][i] = ldx((xa) + (i) * 2048)
#define MMA1(nf, bu, mf) do { mma(acc[mf][nf], wfrag(Wl[nf], Wh[nf]), Xf[bu][mf]); SCHED(); } while (0)
#define SN5(nf, i) (((nf) & 1) ? 4 - (i) : (i))
#define GRP(nf, bu, F0, F1, F2, F3, F4) do { MMA1(nf, bu, SN5(nf, 0)); F0; SCHED(); MMA1(nf, bu, SN5(nf, 1)); F1; SCHED(); \
    MMA1(nf, bu, SN5(nf, 2)); F2; SCHED(); MMA1(nf, bu, SN5(nf, 3)); F3; SCHED(); MMA1(nf, bu, SN5(nf, 4)); F4; SCHED(); } while (0)
  if (tl.gathered) {
    const unsigned gwv = (unsigned)(((tl.nrow0 / NM) * 8 + (tl.col0 >> 8)) * 4 + wid);
    LAS void* dummy = (LAS void*)(lds + 2 * WSLOT + wid * 1024);
    const auto rx = __builtin_amdgcn_make_buffer_rsrc((void*)p.x, 0, ML * KD * 2, 0x00020000);
#pragma unroll
    for (int i = 0; i < 10; ++i)
      __builtin_amdgcn_raw_ptr_buffer_load_lds(rx, dummy, 16, (unsigned)(lane * 16), (gwv * 10240u + i * 1024u) & (unsigned)(ML * KD * 2 - 1), 0, 0);
    waitv<0>();
    BAR();
  }
  BAR();
#pragma unroll
  for (int mf = 0; mf < MF; ++mf) RDX(0, mf, xa0);
#pragma unroll
  for (int nf = 0; nf < 7; ++nf) { RWL(nf, 0); RWH(nf, 0); }
  __builtin_amdgcn_s_waitcnt(0xC07F);
  int ow = 0, ox = 0, dw = WSLOT, dx = XSLOT;
  for (int t = 0; t < NT; ++t) {
    GRP(0, 0, RWL(7, 0), RWH(7, 0), RDX(1, 0, xa1), STEP(xa0, dx), NOP_);
    GRP(1, 0, RWL(0, 1), RWH(0, 1), RDX(1, 1, xa1), NOP_, NOP_);
    GRP(2, 0, RWL(1, 1), RWH(1, 1), RDX(1, 2, xa1), NOP_, NOP_);
    GRP(3, 0, RWL(2, 1), RWH(2, 1), RDX(1, 3, xa1), NOP_, NOP_);
    GRP(4, 0, RWL(3, 1), RWH(3, 1), RDX(1, 4, xa1), NOP_, NOP_);
    GRP(5, 0, RWL(4, 1), RWH(4, 1), NOP_, STEP(wa[0], dw), STEP(xa1, dx));
    GRP(6, 0, RWL(5, 1), RWH(5, 1), NOP_, STEP(wa[1], dw), NOP_);
    GRP(7, 0, RWL(6, 1), RWH(6, 1), NOP_, STEP(wa[2], dw), NOP_);
    GRP(0, 1, RWL(7, 1), RWH(7, 1), NOP_, STEP(wa[3], dw), NOP_);
    GRP(1, 1, NEXT1(ow, dw), NEXT2(ow, dw, 2 * WSLOT, WSLOT), NEXT1(ox, dx), NEXT2(ox, dx, 2 * XSLOT, XSLOT), NOP_);
    __builtin_amdgcn_s_waitcnt(0xC07F);
    CFENCE();
    BAR();
    CFENCE();
    SCHED();
    GRP(2, 1, RWL(0, 0), RWH(0, 0), RDX(0, 0, xa0), RWL(1, 0), RWH(1, 0));
    GRP(3, 1, RWL(2, 0), RWH(2, 0), RDX(0, 1, xa0), RDX(0, 2, xa0), NOP_);
    GRP(4, 1, RWL(3, 0), RWH(3, 0), RDX(0, 3, xa0), RDX(0, 4, xa0), NOP_);
    GRP(5, 1, RWL(4, 0), RWH(4, 0), NOP_, NOP_, NOP_);
    GRP(6, 1, RWL(5, 0), RWH(5, 0), NOP_, NOP_, NOP_);
    GRP(7, 1, RWL(6, 0), RWH(6, 0), NOP_, NOP_, NOP_);
  }
  mma_drain();
  const unsigned go = (unsigned)__builtin_amdgcn_readfirstlane((int)lds_rd(lds, C_GO));
  const bool bad = (go & 1u) != 0u;
  const auto ro = __builtin_amdgcn_make_buffer_rsrc((void*)p.out[p.rank], 0, ML * NR * NCOL * 2, 0x00020000);
  const unsigned vo = out_voff(lane);
  const unsigned so = (unsigned)((tl.col0 + wc * 128) * 2);
#pragma unroll
  for (int mf = 0; mf < MF; ++mf) {
    const int nr = tl.nrow0 + wr * MF * 16 + mf * 16;
    if (nr < NROWS) {
      const int orow = nr < XROWS ? ((p.rank + 1 + nr / BROWS) & 7) * ML + AROWS + nr % BROWS : p.rank * ML + nr - XROWS;
#pragma unroll
      for (int np = 0; np < 4; ++np)
        __builtin_amdgcn_raw_buffer_store_b128(pack2(acc[mf][2 * np], acc[mf][2 * np + 1], bad), ro, vo + np * 64, so + (unsigned)orow * (NCOL * 2), AUX_WT_AGENT);
    }
  }
#undef RWL
#undef RWH
#undef RDX
#undef MMA1
#undef GRP
}

__device__ __forceinline__ bool wait_words(const unsigned* word, unsigned call, unsigned* tmo) {
  const unsigned long long t0 = __builtin_amdgcn_s_memrealtime();
  unsigned spins = 0;
  for (;;) {
    if (__all(ld_sys(word) >= call)) return true;
    if ((++spins & 63u) == 0u && (ld_agent(tmo) == call || __builtin_amdgcn_s_memrealtime() - t0 > DEADLINE)) { st_agent(tmo, call); return false; }
    __builtin_amdgcn_s_sleep(1);
  }
}

__device__ __forceinline__ void ctrl_wave(const Params& p, const Tile& tl, LAS char* lds, int lane) {
  unsigned* myws = p.ws[p.rank];
  unsigned* tmo = myws + WS_TMO;
  const int b = blockIdx.x;
  const unsigned long long tk0 = __builtin_amdgcn_s_memrealtime();
  const unsigned cnt = ld_agent(myws + WS_CNT + b);
  if (!tl.gathered) BAR();
  const unsigned call = (unsigned)__builtin_amdgcn_readfirstlane((int)cnt) + 1u;
  if (lane == 0) st_agent(myws + WS_CNT + b, call);
  bool ok = true;
  int nb = 1;
  auto join = [&]() {
    if (tl.kind == 1) {
      int due = NS_T0 + nb * NPACE;
      if (!tl.gathered) { const int d2 = nb > NC_N0 ? NC_T0 + (nb - NC_N0) * NC_SLOPE : 0; due = d2 < due ? d2 : due; }
      if (nb < NS_N0) due = 0;
      due = due < NPACE_END ? due : NPACE_END;
      while ((long long)(__builtin_amdgcn_s_memrealtime() - tk0) < (long long)due) __builtin_amdgcn_s_sleep(1);
    }
    BAR();
    ++nb;
  };
  if (tl.gathered) {
    if (lane == 0) lds_wr(lds, C_CALL, call);
    const int s0 = tl.nrow0 / BROWS;
    const int e1 = (tl.nrow0 + NM - 1) / BROWS;
    const int s1 = e1 < 7 ? e1 : 6;
    const unsigned* sflag = myws + WS_XFLAG + ((lane & 16) ? s1 : s0) * 32 + (lane & 15);
    auto decode = [&](unsigned v) -> int {
#pragma unroll
      for (int o = 1; o < 32; o <<= 1) { const unsigned u = (unsigned)__shfl_xor((int)v, o); v = u < v ? u : v; }
      const unsigned vm = (unsigned)__builtin_amdgcn_readfirstlane((int)v);
      const int n = vm > call * 128u ? (int)(vm - call * 128u) : 0;
      return n > NT ? NT : n;
    };
    __builtin_amdgcn_fence(__ATOMIC_ACQUIRE, "");
    int seen = 0;
    nb = -1;
    while (seen < NT) {
      const int need = nb + 3 < NT ? nb + 3 : NT;
      if (seen < need) {
        const unsigned long long t0 = __builtin_amdgcn_s_memrealtime();
        unsigned spins = 0;
        while ((seen = decode(ld_sys(sflag))) < need) {
          if ((++spins & 63u) == 0u && (ld_agent(tmo) == call || __builtin_amdgcn_s_memrealtime() - t0 > DEADLINE)) { st_agent(tmo, call); ok = false; seen = NT; break; }
          __builtin_amdgcn_s_sleep(1);
        }
        waitv<0>();
        WAITL0();
        join();
      } else if (seen < need + POLL_MARGIN) {
        unsigned v;
        asm volatile("global_load_dword %0, %1, off sc0 sc1" : "=v"(v) : "v"(sflag) : "memory");
        join();
        asm volatile("s_waitcnt vmcnt(0)" : "+v"(v) :: "memory");
        const int n = decode(v);
        seen = n > seen ? n : seen;
      } else {
        join();
      }
    }
  } else if (tl.kind == 0) {
    while (nb < NT / 2) join();
    ok = wait_words(myws + WS_READY + tl.owner * 32, call, tmo);
  }
  while (nb < NT - 1) join();
  if (ld_agent(tmo) == call) ok = false;
  if (lane == 0) lds_wr(lds, C_GO, call * 2u + (ok ? 0u : 1u));
  WAITL0();
  while (nb <= NT) join();
}

__device__ __forceinline__ void copy_group(const Params& p, LAS char* lds, int c, int wid, int lane) {
  unsigned* myws = p.ws[p.rank];
  unsigned* tmo = myws + WS_TMO;
  const int b = blockIdx.x;
  const unsigned cnt = ld_agent(myws + WS_CNT + b);
  const int gw = wid * 16 + c;
  const auto rx = __builtin_amdgcn_make_buffer_rsrc((void*)p.x, 0, ML * KD * 2, 0x00020000);
  const unsigned xv = (unsigned)((AROWS + (lane >> 3)) * KD * 2 + (((lane & 7) ^ (((gw & 1) * 4 + (lane >> 4)) & 7)) * 16));
  auto load = [&](int k) { const int u = k * 128 + gw; return __builtin_amdgcn_raw_buffer_load_b128(rx, xv, (unsigned)((u % BRP) * (8 * KD * 2) + (u / BRP) * 128), 0); };
  u32x4 d[NUNIT];
  d[0] = load(0);
  d[1] = load(1);
  char* dbase[NR - 1];
  unsigned dslot[NR - 1];
#pragma unroll
  for (int i = 0; i < NR - 1; ++i) {
    const int j = __builtin_amdgcn_readfirstlane((i + gw) % (NR - 1) + 1);
    dbase[i] = (char*)p.ws[__builtin_amdgcn_readfirstlane((p.rank + NR - j) & 7)];
    dslot[i] = (unsigned)((j - 1) * BROWS * 128);
  }
  if (lane == 0) { lds_wr(lds, wid * 4, (unsigned)(gw / BRP)); if (wid == 0) lds_wr(lds, 64, 0u); }
  WAITL0();
  const unsigned call = (unsigned)__builtin_amdgcn_readfirstlane((int)cnt) + 1u;
  BAR();
  const unsigned long long tstart = __builtin_amdgcn_s_memrealtime();
  if (wid == 0 && lane == 0) st_agent(myws + WS_CNT + b, call);
  if (gw == 127 && lane < NR && lane != p.rank) st_sys(p.ws[lane] + WS_READY + p.rank * 32, call);
  const size_t xo = WS_XBUF + (call & 1) * XPAR_BYTES;
  unsigned* pflag = p.ws[(p.rank + NR - 1 - (lane < 7 ? lane : 0)) & 7] + WS_XFLAG + (lane < 7 ? lane : 0) * 32 + c;
  auto finish = [&](int k) {
    const int nxt = k + 1 < NUNIT ? ((k + 1) * 128 + gw) / BRP : NT;
    if (lane == 0) lds_wr(lds, wid * 4, (unsigned)nxt);
    unsigned v = lds_rd(lds, (lane & 7) * 4);
#pragma unroll
    for (int o = 1; o < 8; o <<= 1) { const unsigned u = (unsigned)__shfl_xor((int)v, o); v = u < v ? u : v; }
    const int pre = __builtin_amdgcn_readfirstlane((int)v);
    const int lvl = pre < 4 ? pre : (pre < 12 ? (pre & ~1) : (pre & ~3));
    unsigned old = 0;
    if (lane == 0) old = __hip_atomic_fetch_max((LAS unsigned*)(lds + 64), (unsigned)lvl, __ATOMIC_RELAXED, __HIP_MEMORY_SCOPE_WORKGROUP);
    old = (unsigned)__builtin_amdgcn_readfirstlane((int)old);
    if ((unsigned)lvl > old && lane < 7)
      __hip_atomic_fetch_max(pflag, call * 128u + (unsigned)lvl, __ATOMIC_RELAXED, __HIP_MEMORY_SCOPE_SYSTEM);
  };
  bool pending = false;
  long long tissue = 0;
#pragma unroll
  for (int k = 0; k < NUNIT; ++k) {
    const int u = k * 128 + gw;
    for (;;) {
      const long long now = (long long)(__builtin_amdgcn_s_memrealtime() - tstart) * 10;
      if (now * BRP >= (long long)(u - PUSH_LEAD * BRP) * TILE_NS) break;
      if (pending && now - tissue > ACK_NS) { waitv<0>(); finish(k - 1); pending = false; }
      else __builtin_amdgcn_s_sleep(2);
    }
    const unsigned so = (unsigned)(u / BRP) * (unsigned)XT_BYTES + (unsigned)(u % BRP) * 1024u;
#pragma unroll
    for (int i = 0; i < NR - 1; ++i) {
      const auto rd = __builtin_amdgcn_make_buffer_rsrc(dbase[i] + xo, 0, (int)XPAR_BYTES, 0x00020000);
      __builtin_amdgcn_raw_buffer_store_b128(d[k], rd, (unsigned)(lane * 16), so + dslot[i], AUX_WT_SYS);
    }
    tissue = (long long)(__builtin_amdgcn_s_memrealtime() - tstart) * 10;
    if (pending) { waitv<NR - 1>(); finish(k - 1); }
    pending = true;
    if (k + 2 < NUNIT) d[k + 2] = load(k + 2);
  }
  waitv<0>();
  finish(NUNIT - 1);
  if (c == 15 && wid != p.rank) {
    const bool fine = wait_words(myws + WS_DONE + wid * 64 + lane, call, tmo);
    if (!fine && lane == 0) p.out[p.rank][(size_t)(p.rank * ML) * NCOL] = 0x7fc0;
  }
}

__global__ __launch_bounds__(512, 2) void gemm_a2a(Params p) {
  extern __shared__ __attribute__((aligned(16))) char smem[];
  LAS char* lds = (LAS char*)smem;
  const int wid = __builtin_amdgcn_readfirstlane(threadIdx.x >> 6);
  const int lane = threadIdx.x & 63;
  const int b = blockIdx.x;
  const int xcd = b & 7, idx = b >> 3;
  if (idx < 2) { copy_group(p, lds, xcd * 2 + idx, wid, lane); return; }
  Tile tl;
  int rt = 0;
  if (idx < 16) {
    const int a = xcd * 14 + idx - 2;
    tl.kind = 0; tl.owner = (p.rank + a % 7 + 1) & 7; tl.col0 = (a / 7) * 128;
    tl.nrow0 = 0; tl.gathered = false; tl.koff = KSKEW * xcd;
  } else {
    const int e = idx - 16;
    rt = (xcd >> 1) * 4 + (e >> 2);
    tl.kind = 1; tl.owner = p.rank; tl.col0 = ((xcd & 1) * 4 + (e & 3)) * 256;
    tl.nrow0 = rt * NM; tl.gathered = rt <= RMIX; tl.koff = 0;
  }
  if (wid < 4) {
    if (tl.kind == 0) mfma_a(p, tl, lds, wid, lane);
    else mfma_n(p, tl, lds, wid, lane);
  } else if (wid == 7) {
    ctrl_wave(p, tl, lds, lane);
  } else {
#define LOADERS(MF, NIMG, SW, PAIR, NWS, AUX) do { if (wid == 4) loader<MF, NIMG, SW, PAIR, 0, NWS, AUX>(p, tl, lds, lane); \
    else if (wid == 5) loader<MF, NIMG, SW, PAIR, 1, NWS, AUX>(p, tl, lds, lane); else loader<MF, NIMG, SW, PAIR, 2, NWS, AUX>(p, tl, lds, lane); } while (0)
    if (tl.kind == 0) LOADERS(MFA, 1, SWA, true, 0, AUX_NT);
    else if (rt < RMIX) LOADERS(MFN, 2, 3, false, 4 * MFN, 0);
    else if (rt == RMIX) LOADERS(MFN, 2, 3, false, (XROWS - RMIX * NM) / 8, 0);
    else LOADERS(MFN, 2, 3, false, 0, 0);
  }
}

extern "C" void kernel_launch(int rank, int nranks, ncclComm_t comm,
                              hipStream_t stream,
                              void* const* d_in, const int* in_sizes, int n_in,
                              void* d_out, int out_size,
                              const int* in_dtypes, int out_dtype,
                              void* const* d_out_all,
                              void* const* d_ws_all, size_t ws_bytes) {
  if (nranks != NR || (unsigned)rank >= (unsigned)NR || ws_bytes < WS_NEED || n_in < 2 ||
      in_sizes[0] != ML * KD || in_sizes[1] != KD * ND || out_size != NR * ML * NCOL) {
    fprintf(stderr, "unexpected configuration\n");
    return;
  }
  Params p;
  p.x = (const unsigned short*)d_in[0];
  p.w = (const unsigned short*)d_in[1];
  for (int i = 0; i < NR; ++i) { p.out[i] = (unsigned short*)d_out_all[i]; p.ws[i] = (unsigned*)d_ws_all[i]; }
  p.rank = rank;
  static bool init[NR] = {};
  if (!init[rank]) {
    HIP_CHECK(hipFuncSetAttribute((const void*)gemm_a2a, hipFuncAttributeMaxDynamicSharedMemorySize, LDS_BYTES));
    init[rank] = true;
  }
  gemm_a2a<<<NBLK, 512, LDS_BYTES, stream>>>(p);
  HIP_CHECK(hipGetLastError());
}
```

```cpp
#include <hip/hip_runtime.h>
#include <rccl/rccl.h>
#include <cstdio>
#include <stdint.h>

#define HIP_CHECK(expr) do { hipError_t _e = (expr); if (_e != hipSuccess) { \
    fprintf(stderr, "HIP error %s at %s:%d\n", hipGetErrorString(_e), __FILE__, __LINE__); \
    return; } } while (0)

#define LAS __attribute__((address_space(3)))
typedef short s16x4 __attribute__((ext_vector_type(4)));
typedef __bf16 bf16x2 __attribute__((ext_vector_type(2)));
typedef __bf16 bf16x8 __attribute__((ext_vector_type(8)));
typedef float f32x4 __attribute__((ext_vector_type(4)));
typedef float f32x2 __attribute__((ext_vector_type(2)));
typedef unsigned u32x4 __attribute__((ext_vector_type(4)));

constexpr int NR = 8, ML = 512, KD = 4096, ND = 16384, NCOL = ND / NR;
constexpr int BK = 64, NT = KD / BK;
constexpr int MFA = 7;
constexpr int MFN = 5;
constexpr int AROWS = MFA * 32, BROWS = ML - AROWS;
constexpr int BRP = BROWS / 8;
constexpr int XROWS = 7 * BROWS;
constexpr int NROWS = XROWS + ML;
constexpr int NM = MFN * 32;
constexpr int RMIX = XROWS / NM;
constexpr int WODD = 8192 + 128;
constexpr int WIMG = WODD + 8192;
constexpr int SWA = 4;
constexpr int LDS_CTL = 3 * (2 * WIMG + MFN * 4096);
static_assert(SWA * WIMG + 3 * MFA * 4096 <= LDS_CTL, "LDS");
constexpr int C_GO = LDS_CTL;
constexpr int C_CALL = LDS_CTL + 16;
constexpr int LDS_BYTES = LDS_CTL + 64;
static_assert(LDS_BYTES <= 163840, "LDS");
constexpr int NBLK = 256;
constexpr unsigned long long DEADLINE = 50000000ull;
constexpr int WS_TMO = 32, WS_READY = 64  , WS_CNT = 512  , WS_DONE = 1024  ;
constexpr int WS_XFLAG = 2048;
constexpr size_t WS_XBUF = 1u << 20;
constexpr size_t XT_BYTES = (size_t)XROWS * 128;
constexpr size_t XPAR_BYTES = XT_BYTES * NT;
constexpr size_t WS_NEED = WS_XBUF + 2 * XPAR_BYTES;
constexpr int NUNIT = BRP * NT / 128;
constexpr int TILE_NS = 640;
constexpr int PUSH_LEAD = 1;
constexpr int ACK_NS = 800;
constexpr int POLL_MARGIN = 4;
constexpr int KSKEW = 8, KSTRIDE = 11;
constexpr int NS_T0 = 800, NPACE = 106, NPACE_END = 4400, NS_N0 = 3;
constexpr int NC_N0 = 10, NC_T0 = 1150, NC_SLOPE = 125;
constexpr int AUX_NT = 2, AUX_WT_AGENT = 16, AUX_WT_SYS = 17;

struct Params {
  const unsigned short* x;
  const unsigned short* w;
  unsigned short* out[NR];
  unsigned* ws[NR];
  int rank;
};

#define WAITL0() asm volatile("s_waitcnt lgkmcnt(0)" ::: "memory")
template <int N> __device__ __forceinline__ void waitv() { asm volatile("s_waitcnt vmcnt(%0)" :: "n"(N) : "memory"); }
#define BAR() __builtin_amdgcn_s_barrier()
#define SCHED() __builtin_amdgcn_sched_barrier(0)
#define CFENCE() asm volatile("" ::: "memory")
#define NOP_ do {} while (0)
#define STEP(a, d) asm volatile("v_add_u32 %0, %1, %0" : "+v"(a) : "s"(d))
#define NEXT1(o, d) asm volatile("s_add_i32 %0, %0, %1" : "+s"(o) : "s"(d) : "scc")
#define NEXT2(o, d, last, slot) asm volatile("s_cmp_eq_u32 %1, %2\n\ts_cselect_b32 %0, %3, %4" : "=s"(d) : "s"(o), "s"(last), "s"(-(last)), "s"(slot) : "scc")

__device__ __forceinline__ bf16x8 ldx(unsigned a) { return *(LAS const bf16x8*)(size_t)a; }
__device__ __forceinline__ s16x4 ldw(unsigned a) { return __builtin_amdgcn_ds_read_tr16_b64_v4i16((LAS s16x4*)(size_t)a); }
__device__ __forceinline__ void mma(f32x4& c, bf16x8 a, bf16x8 b) { asm volatile("v_mfma_f32_16x16x32_bf16 %0, %1, %2, %0" : "+v"(c) : "v"(a), "v"(b)); }
__device__ __forceinline__ void mma_drain() {
  __builtin_amdgcn_sched_barrier(0);
  asm volatile("s_nop 15\n\ts_nop 7" ::: "memory");
  __builtin_amdgcn_sched_barrier(0);
}
__device__ __forceinline__ unsigned pk(float a, float b) { f32x2 v = {a, b}; return __builtin_bit_cast(unsigned, __builtin_convertvector(v, bf16x2)); }
__device__ __forceinline__ unsigned ld_sys(const unsigned* p) { return __hip_atomic_load(p, __ATOMIC_RELAXED, __HIP_MEMORY_SCOPE_SYSTEM); }
__device__ __forceinline__ unsigned ld_agent(const unsigned* p) { return __hip_atomic_load(p, __ATOMIC_RELAXED, __HIP_MEMORY_SCOPE_AGENT); }
__device__ __forceinline__ void st_sys(unsigned* p, unsigned v) { __hip_atomic_store(p, v, __ATOMIC_RELAXED, __HIP_MEMORY_SCOPE_SYSTEM); }
__device__ __forceinline__ void st_agent(unsigned* p, unsigned v) { __hip_atomic_store(p, v, __ATOMIC_RELAXED, __HIP_MEMORY_SCOPE_AGENT); }
__device__ __forceinline__ unsigned lds_rd(LAS char* lds, int off) { return *(LAS volatile unsigned*)(lds + off); }
__device__ __forceinline__ void lds_wr(LAS char* lds, int off, unsigned v) { *(LAS volatile unsigned*)(lds + off) = v; }
__device__ __forceinline__ bf16x8 wfrag(s16x4 lo, s16x4 hi) { return __builtin_bit_cast(bf16x8, __builtin_shufflevector(lo, hi, 0, 1, 2, 3, 4, 5, 6, 7)); }
__device__ __forceinline__ u32x4 pack2(f32x4 a, f32x4 c, bool bad) {
  const unsigned a0 = pk(a[0], a[1]), a1 = pk(a[2], a[3]), c0 = pk(c[0], c[1]), c1 = pk(c[2], c[3]);
  auto r0 = __builtin_amdgcn_permlane16_swap(a0, c0, false, false);
  auto r1 = __builtin_amdgcn_permlane16_swap(a1, c1, false, false);
  return bad ? (u32x4){0x7fc07fc0u, 0x7fc07fc0u, 0x7fc07fc0u, 0x7fc07fc0u} : (u32x4){r0[0], r1[0], r0[1], r1[1]};
}
__device__ __forceinline__ unsigned out_voff(int lane) { const int fr = lane & 15, fq = lane >> 4; return (unsigned)((fr * NCOL + (fq & 1) * 16 + (fq >> 1) * 8) * 2); }
__host__ __device__ constexpr int wpiece(int pc) { return ((pc >> 1) & 1) * WODD + (pc >> 2) * 2048 + (pc & 1) * 1024; }

struct Tile {
  int kind;
  int owner;
  int col0;
  int nrow0;
  bool gathered;
  int koff;
  int qw, qx;
};

template <int MF, int NIMG, int SW, bool PAIR, int H, int NWS, int AUXW>
__device__ __forceinline__ void loader(const Params& p, const Tile& tl, LAS char* lds, int lane) {
  constexpr int NWP = 16 * NIMG;
  constexpr int WSLOT = NIMG * WIMG, XSLOT = MF * 4096, LX = SW * WSLOT;
  constexpr int NW_ = (NWP - H + 2) / 3;
  constexpr int X0 = (H + 3 * NW_) - NWP;
  constexpr int NX_ = (4 * MF - X0 + 2) / 3;
  constexpr bool XWS = NWS > 0;
  static_assert(2 * NW_ + 2 * NX_ <= 63, "vmcnt");
  const auto rw = __builtin_amdgcn_make_buffer_rsrc((void*)p.w, 0, KD * ND * 2, 0x00020000);
  const auto rx = __builtin_amdgcn_make_buffer_rsrc((void*)p.x, 0, ML * KD * 2, 0x00020000);
  const unsigned wv = (unsigned)((lane >> 4) * ND * 2 + (((lane & 15) ^ ((lane >> 4) << 1)) * 16));
  unsigned xv[2];
#pragma unroll
  for (int v = 0; v < 2; ++v) xv[v] = (unsigned)((lane >> 3) * KD * 2 + (((lane & 7) ^ ((v * 4 + (lane >> 4)) & 7)) * 16));
  const unsigned iv = (unsigned)(lane * 16);
  const unsigned wbase = (unsigned)((tl.owner * NCOL + tl.col0) * 2);
  const unsigned xbase = tl.kind == 0 ? 0u : (unsigned)((tl.nrow0 - XROWS) * KD * 2);
  auto xpc = [&](int pc) -> unsigned {
    const bool pad = NIMG == 2 && NWS == 0 && tl.nrow0 + pc * 8 >= NROWS;
    return (unsigned)((pad ? pc - 8 : pc) * (8 * KD * 2));
  };
  auto kmap = [&](int t) -> unsigned {
    return PAIR ? (unsigned)(((((t >> 1) * KSTRIDE) << 1) + (t & 1) + tl.koff) & (NT - 1)) : (unsigned)t;
  };
  int ws_ = 0, xs_ = 0;
  auto issue_w = [&](int t) {
    const unsigned tb = wbase + kmap(t) * (BK * ND * 2);
#pragma unroll
    for (int i = 0; i < NW_; ++i) {
      const int pc = H + 3 * i, img = pc >> 4, q = pc & 15;
      __builtin_amdgcn_raw_ptr_buffer_load_lds(rw, (LAS void*)(lds + ws_ * WSLOT + img * WIMG + wpiece(q)), 16, wv, tb + q * (4 * ND * 2) + img * 256, 0, AUXW);
    }
    ws_ = ws_ == SW - 1 ? 0 : ws_ + 1;
  };
  auto issue_xl = [&](int t) {
    const unsigned tb = xbase + kmap(t) * 128u;
#pragma unroll
    for (int i = 0; i < NX_; ++i) {
      const int pc = X0 + 3 * i;
      if (pc >= NWS) __builtin_amdgcn_raw_ptr_buffer_load_lds(rx, (LAS void*)(lds + LX + xs_ * XSLOT + pc * 1024), 16, xv[pc & 1], tb + xpc(pc), 0, 0);
    }
  };
  issue_w(0);
  issue_xl(0);
  unsigned call = 0;
  if (XWS) {
    BAR();
    call = (unsigned)__builtin_amdgcn_readfirstlane((int)lds_rd(lds, C_CALL));
  }
  const auto ri = __builtin_amdgcn_make_buffer_rsrc((char*)p.ws[p.rank] + WS_XBUF + (call & 1) * XPAR_BYTES, 0, (int)XPAR_BYTES, 0x00020000);
  const unsigned ibase = (unsigned)(tl.nrow0 * 128);
  auto issue_xi = [&](int t) {
#pragma unroll
    for (int i = 0; i < NX_; ++i) {
      const int pc = X0 + 3 * i;
      if (pc < NWS) __builtin_amdgcn_raw_ptr_buffer_load_lds(ri, (LAS void*)(lds + LX + xs_ * XSLOT + pc * 1024), 16, iv, ibase + (unsigned)t * (unsigned)XT_BYTES + pc * 1024, 0, 0);
    }
    xs_ = xs_ == 2 ? 0 : xs_ + 1;
  };
  issue_xi(0);
  issue_w(1);
  issue_xl(1);
  issue_xi(1);
  waitv<NW_ + NX_>();
  BAR();
  if (PAIR) {
    for (int t = 0; t < NT - 2; t += 2) {
      issue_w(t + 2);
      issue_w(t + 3);
      issue_xl(t + 2);
      issue_xi(t + 2);
      waitv<2 * NW_ + NX_>();
      BAR();
      issue_xl(t + 3);
      issue_xi(t + 3);
      waitv<NX_>();
      BAR();
    }
  } else if (NWS == 0 || NWS == 4 * MF) {
    constexpr int QW = (NW_ + 3) / 4, QX = (NX_ + 3) / 4;
    unsigned wl[NW_], wg[NW_], xl[NX_], xg[NX_];
#pragma unroll
    for (int i = 0; i < NW_; ++i) {
      int k = i + tl.qw * QW; k = k >= NW_ ? k - NW_ : k;
      const int pc = H + 3 * k, img = pc >> 4, q = pc & 15;
      wl[i] = (unsigned)(img * WIMG + wpiece(q));
      wg[i] = wv + (unsigned)(q * (4 * ND * 2) + img * 256);
    }
#pragma unroll
    for (int i = 0; i < NX_; ++i) {
      int k = i + tl.qx * QX; k = k >= NX_ ? k - NX_ : k;
      const int pc = X0 + 3 * k;
      xl[i] = (unsigned)(pc * 1024);
      xg[i] = XWS ? iv + (unsigned)(pc * 1024) : ((pc & 1) ? xv[1] : xv[0]) + xpc(pc);
    }
    for (int t = 0; t < NT - 2; ++t) {
      const unsigned tw = wbase + (unsigned)(t + 2) * (BK * ND * 2);
      const unsigned tx = XWS ? ibase + (unsigned)(t + 2) * (unsigned)XT_BYTES : xbase + (unsigned)(t + 2) * 128u;
      LAS char* lw = lds + ws_ * WSLOT;
      LAS char* lx = lds + LX + xs_ * XSLOT;
#pragma unroll
      for (int ph = 0; ph < 2; ++ph) {
#pragma unroll
        for (int i = ph ? QW : 0; i < (ph ? NW_ : QW); ++i)
          __builtin_amdgcn_raw_ptr_buffer_load_lds(rw, (LAS void*)(lw + wl[i]), 16, wg[i], tw, 0, AUXW);
#pragma unroll
        for (int i = ph ? QX : 0; i < (ph ? NX_ : QX); ++i)
          __builtin_amdgcn_raw_ptr_buffer_load_lds(XWS ? ri : rx, (LAS void*)(lx + xl[i]), 16, xg[i], tx, 0, 0);
      }
      ws_ = ws_ == SW - 1 ? 0 : ws_ + 1;
      xs_ = xs_ == 2 ? 0 : xs_ + 1;
      waitv<NW_ + NX_>();
      BAR();
    }
  } else {
    for (int t = 0; t < NT - 2; ++t) {
      issue_w(t + 2);
      issue_xl(t + 2);
      issue_xi(t + 2);
      waitv<NW_ + NX_>();
      BAR();
    }
  }
  waitv<0>();
  BAR();
  BAR();
}

__device__ __forceinline__ void mfma_a(const Params& p, const Tile& tl, LAS char* lds, int wid, int lane) {
  constexpr int MF = MFA;
  static_assert(MF == 7, "the schedule below is written for 7 fragments");
  constexpr int XSLOT = MF * 4096, LX = SWA * WIMG;
  const int wr = wid >> 1, wc = wid & 1;
  const int fr = lane & 15, fq = lane >> 4;
  const int q = (lane & 15) >> 2, pp = lane & 3;
  const unsigned lb = (unsigned)(size_t)lds;
  const unsigned xo = (unsigned)(LX + (wr * MF * 16 + fr) * 128 + ((fq ^ ((fr >> 1) & 7)) * 16));
  unsigned xa0 = lb + xo, xa1 = lb + (xo ^ 64u);
  unsigned wa[4];
#pragma unroll
  for (int nf = 0; nf < 4; ++nf)
    wa[nf] = lb + (unsigned)((fq & 1) * WODD + (fq >> 1) * 2048 + q * 256 + wc * 128 + ((nf ^ q) * 32) + 8 * pp);
  f32x4 acc[MF][4];
#pragma unroll
  for (int i = 0; i < MF; ++i)
#pragma unroll
    for (int j = 0; j < 4; ++j) { acc[i][j] = (f32x4){0.f, 0.f, 0.f, 0.f}; asm volatile("" : "+v"(acc[i][j])); }
  bf16x8 Xf[MF];
  s16x4 Wl[2][4], Wh[2][4];
#define RDW(bu, ks, i) do { if ((i) & 1) Wh[bu][(i) >> 1] = ldw(wa[(i) >> 1] + (ks) * 4096 + 1024); \
    else Wl[bu][(i) >> 1] = ldw(wa[(i) >> 1] + (ks) * 4096); } while (0)
#define RDX(i, xa) Xf[i] = ldx((xa) + (i) * 2048)
#define MMA1(mf, bu, nf) do { mma(acc[mf][nf], wfrag(Wl[bu][nf], Wh[bu][nf]), Xf[mf]); SCHED(); } while (0)
#define SN(mf, i) (((mf) & 1) ? 3 - (i) : (i))
#define GRP(mf, bu, F0, F1, F2) do { MMA1(mf, bu, SN(mf, 0)); F0; SCHED(); MMA1(mf, bu, SN(mf, 1)); F1; SCHED(); \
    MMA1(mf, bu, SN(mf, 2)); F2; SCHED(); MMA1(mf, bu, SN(mf, 3)); } while (0)
  BAR();
#pragma unroll
  for (int i = 0; i < 8; ++i) RDW(0, 0, i);
#pragma unroll
  for (int mf = 0; mf < MF - 1; ++mf) RDX(mf, xa0);
  __builtin_amdgcn_s_waitcnt(0xC07F);
  int ow = 0, ox = 0, dw = WIMG, dx = XSLOT;
  for (int t = 0; t < NT; ++t) {
    GRP(0, 0, RDX(6, xa0), RDW(1, 1, 0), RDW(1, 1, 1));
    GRP(1, 0, RDX(0, xa1), RDW(1, 1, 2), RDW(1, 1, 3));
    GRP(2, 0, RDX(1, xa1), RDW(1, 1, 4), STEP(xa0, dx));
    GRP(3, 0, RDX(2, xa1), RDW(1, 1, 5), STEP(wa[0], dw));
    GRP(4, 0, RDX(3, xa1), RDW(1, 1, 6), STEP(wa[1], dw));
    GRP(5, 0, RDX(4, xa1), RDW(1, 1, 7), STEP(wa[2], dw));
    GRP(6, 0, RDX(5, xa1), NOP_, STEP(wa[3], dw));
    GRP(0, 1, RDX(6, xa1), NEXT1(ow, dw), NEXT2(ow, dw, (SWA - 1) * WIMG, WIMG));
    GRP(1, 1, STEP(xa1, dx), NEXT1(ox, dx), NEXT2(ox, dx, 2 * XSLOT, XSLOT));
    __builtin_amdgcn_s_waitcnt(0xC07F);
    CFENCE();
    BAR();
    CFENCE();
    SCHED();
    GRP(2, 1, RDW(0, 0, 0), RDW(0, 0, 1), RDX(0, xa0));
    GRP(3, 1, RDW(0, 0, 2), RDW(0, 0, 3), RDX(1, xa0));
    GRP(4, 1, RDW(0, 0, 4), RDW(0, 0, 5), RDX(2, xa0));
    GRP(5, 1, RDW(0, 0, 6), RDW(0, 0, 7), RDX(3, xa0));
    GRP(6, 1, RDX(4, xa0), RDX(5, xa0), NOP_);
  }
  mma_drain();
  const unsigned go = (unsigned)__builtin_amdgcn_readfirstlane((int)lds_rd(lds, C_GO));
  const bool bad = (go & 1u) != 0u;
  const auto ro = __builtin_amdgcn_make_buffer_rsrc((void*)p.out[tl.owner], 0, ML * NR * NCOL * 2, 0x00020000);
  const unsigned vo = out_voff(lane);
  const unsigned so = (unsigned)(((p.rank * ML + wr * MF * 16) * NCOL + tl.col0 + wc * 64) * 2);
#pragma unroll
  for (int mf = 0; mf < MF; ++mf) {
#pragma unroll
    for (int np = 0; np < 2; ++np)
      __builtin_amdgcn_raw_buffer_store_b128(pack2(acc[mf][2 * np], acc[mf][2 * np + 1], bad), ro, vo + np * 64, so + mf * (16 * NCOL * 2), AUX_WT_SYS);
    waitv<0>();
  }
  if (lane == 0) st_sys(p.ws[tl.owner] + WS_DONE + p.rank * 64 + (tl.col0 >> 7) * 4 + wid, go >> 1);
#undef RDW
#undef RDX
#undef MMA1
#undef GRP
}

__device__ __forceinline__ void mfma_n(const Params& p, const Tile& tl, LAS char* lds, int wid, int lane) {
  constexpr int MF = MFN;
  static_assert(MF == 5, "the schedule below is written for 5 fragments");
  constexpr int WSLOT = 2 * WIMG, XSLOT = MF * 4096, LX = 3 * WSLOT;
  const int wr = wid >> 1, wc = wid & 1;
  const int fr = lane & 15, fq = lane >> 4;
  const int q = (lane & 15) >> 2, pp = lane & 3;
  const unsigned lb = (unsigned)(size_t)lds;
  const unsigned xo = (unsigned)(LX + (wr * MF * 16 + fr) * 128 + ((fq ^ ((fr >> 1) & 7)) * 16));
  unsigned xa0 = lb + xo, xa1 = lb + (xo ^ 64u);
  unsigned wa[4];
#pragma unroll
  for (int nf = 0; nf < 4; ++nf)
    wa[nf] = lb + (unsigned)(wc * WIMG + (fq & 1) * WODD + (fq >> 1) * 2048 + q * 256 + ((nf ^ q) * 32) + 8 * pp);
  f32x4 acc[MF][8];
#pragma unroll
  for (int i = 0; i < MF; ++i)
#pragma unroll
    for (int j = 0; j < 8; ++j) { acc[i][j] = (f32x4){0.f, 0.f, 0.f, 0.f}; asm volatile("" : "+v"(acc[i][j])); }
  bf16x8 Xf[2][MF];
  s16x4 Wl[8], Wh[8];
#define RWL(nf, ks) Wl[nf] = ldw(wa[(nf) & 3] + ((nf) >> 2) * 128 + (ks) * 4096)
#define RWH(nf, ks) Wh[nf] = ldw(wa[(nf) & 3] + ((nf) >> 2) * 128 + (ks) * 4096 + 1024)
#define RDX(bu, i, xa) Xf[bu][i] = ldx((xa) + (i) * 2048)
#define MMA1(nf, bu, mf) do { mma(acc[mf][nf], wfrag(Wl[nf], Wh[nf]), Xf[bu][mf]); SCHED(); } while (0)
#define SN5(nf, i) (((nf) & 1) ? 4 - (i) : (i))
#define GRP(nf, bu, F0, F1, F2, F3, F4) do { MMA1(nf, bu, SN5(nf, 0)); F0; SCHED(); MMA1(nf, bu, SN5(nf, 1)); F1; SCHED(); \
    MMA1(nf, bu, SN5(nf, 2)); F2; SCHED(); MMA1(nf, bu, SN5(nf, 3)); F3; SCHED(); MMA1(nf, bu, SN5(nf, 4)); F4; SCHED(); } while (0)
  if (tl.gathered) {
    const unsigned gwv = (unsigned)(((tl.nrow0 / NM) * 8 + (tl.col0 >> 8)) * 4 + wid);
    LAS void* dummy = (LAS void*)(lds + 2 * WSLOT + wid * 1024);
    const auto rx = __builtin_amdgcn_make_buffer_rsrc((void*)p.x, 0, ML * KD * 2, 0x00020000);
#pragma unroll
    for (int i = 0; i < 10; ++i)
      __builtin_amdgcn_raw_ptr_buffer_load_lds(rx, dummy, 16, (unsigned)(lane * 16), (gwv * 10240u + i * 1024u) & (unsigned)(ML * KD * 2 - 1), 0, 0);
    waitv<0>();
    BAR();
  }
  BAR();
#pragma unroll
  for (int mf = 0; mf < MF; ++mf) RDX(0, mf, xa0);
#pragma unroll
  for (int nf = 0; nf < 7; ++nf) { RWL(nf, 0); RWH(nf, 0); }
  __builtin_amdgcn_s_waitcnt(0xC07F);
  int ow = 0, ox = 0, dw = WSLOT, dx = XSLOT;
  for (int t = 0; t < NT; ++t) {
    GRP(0, 0, RWL(7, 0), RWH(7, 0), RDX(1, 0, xa1), STEP(xa0, dx), NOP_);
    GRP(1, 0, RWL(0, 1), RWH(0, 1), RDX(1, 1, xa1), NOP_, NOP_);
    GRP(2, 0, RWL(1, 1), RWH(1, 1), RDX(1, 2, xa1), NOP_, NOP_);
    GRP(3, 0, RWL(2, 1), RWH(2, 1), RDX(1, 3, xa1), NOP_, NOP_);
    GRP(4, 0, RWL(3, 1), RWH(3, 1), RDX(1, 4, xa1), NOP_, NOP_);
    GRP(5, 0, RWL(4, 1), RWH(4, 1), NOP_, STEP(wa[0], dw), STEP(xa1, dx));
    GRP(6, 0, RWL(5, 1), RWH(5, 1), NOP_, STEP(wa[1], dw), NOP_);
    GRP(7, 0, RWL(6, 1), RWH(6, 1), NOP_, STEP(wa[2], dw), NOP_);
    GRP(0, 1, RWL(7, 1), RWH(7, 1), NOP_, STEP(wa[3], dw), NOP_);
    GRP(1, 1, NEXT1(ow, dw), NEXT2(ow, dw, 2 * WSLOT, WSLOT), NEXT1(ox, dx), NEXT2(ox, dx, 2 * XSLOT, XSLOT), NOP_);
    __builtin_amdgcn_s_waitcnt(0xC07F);
    CFENCE();
    BAR();
    CFENCE();
    SCHED();
    GRP(2, 1, RWL(0, 0), RWH(0, 0), RDX(0, 0, xa0), RWL(1, 0), RWH(1, 0));
    GRP(3, 1, RWL(2, 0), RWH(2, 0), RDX(0, 1, xa0), RDX(0, 2, xa0), NOP_);
    GRP(4, 1, RWL(3, 0), RWH(3, 0), RDX(0, 3, xa0), RDX(0, 4, xa0), NOP_);
    GRP(5, 1, RWL(4, 0), RWH(4, 0), NOP_, NOP_, NOP_);
    GRP(6, 1, RWL(5, 0), RWH(5, 0), NOP_, NOP_, NOP_);
    GRP(7, 1, RWL(6, 0), RWH(6, 0), NOP_, NOP_, NOP_);
  }
  mma_drain();
  const unsigned go = (unsigned)__builtin_amdgcn_readfirstlane((int)lds_rd(lds, C_GO));
  const bool bad = (go & 1u) != 0u;
  const auto ro = __builtin_amdgcn_make_buffer_rsrc((void*)p.out[p.rank], 0, ML * NR * NCOL * 2, 0x00020000);
  const unsigned vo = out_voff(lane);
  const unsigned so = (unsigned)((tl.col0 + wc * 128) * 2);
#pragma unroll
  for (int mf = 0; mf < MF; ++mf) {
    const int nr = tl.nrow0 + wr * MF * 16 + mf * 16;
    if (nr < NROWS) {
      const int orow = nr < XROWS ? ((p.rank + 1 + nr / BROWS) & 7) * ML + AROWS + nr % BROWS : p.rank * ML + nr - XROWS;
#pragma unroll
      for (int np = 0; np < 4; ++np)
        __builtin_amdgcn_raw_buffer_store_b128(pack2(acc[mf][2 * np], acc[mf][2 * np + 1], bad), ro, vo + np * 64, so + (unsigned)orow * (NCOL * 2), AUX_WT_AGENT);
    }
  }
#undef RWL
#undef RWH
#undef RDX
#undef MMA1
#undef GRP
}

__device__ __forceinline__ bool wait_words(const unsigned* word, unsigned call, unsigned* tmo) {
  const unsigned long long t0 = __builtin_amdgcn_s_memrealtime();
  unsigned spins = 0;
  for (;;) {
    if (__all(ld_sys(word) >= call)) return true;
    if ((++spins & 63u) == 0u && (ld_agent(tmo) == call || __builtin_amdgcn_s_memrealtime() - t0 > DEADLINE)) { st_agent(tmo, call); return false; }
    __builtin_amdgcn_s_sleep(1);
  }
}

__device__ __forceinline__ void ctrl_wave(const Params& p, const Tile& tl, LAS char* lds, int lane) {
  unsigned* myws = p.ws[p.rank];
  unsigned* tmo = myws + WS_TMO;
  const int b = blockIdx.x;
  const unsigned long long tk0 = __builtin_amdgcn_s_memrealtime();
  const unsigned cnt = ld_agent(myws + WS_CNT + b);
  if (!tl.gathered) BAR();
  const unsigned call = (unsigned)__builtin_amdgcn_readfirstlane((int)cnt) + 1u;
  if (lane == 0) st_agent(myws + WS_CNT + b, call);
  bool ok = true;
  int nb = 1;
  auto join = [&]() {
    if (tl.kind == 1) {
      int due = NS_T0 + nb * NPACE;
      if (!tl.gathered) { const int d2 = nb > NC_N0 ? NC_T0 + (nb - NC_N0) * NC_SLOPE : 0; due = d2 < due ? d2 : due; }
      if (nb < NS_N0) due = 0;
      due = due < NPACE_END ? due : NPACE_END;
      while ((long long)(__builtin_amdgcn_s_memrealtime() - tk0) < (long long)due) __builtin_amdgcn_s_sleep(1);
    }
    BAR();
    ++nb;
  };
  if (tl.gathered) {
    if (lane == 0) lds_wr(lds, C_CALL, call);
    const int s0 = tl.nrow0 / BROWS;
    const int e1 = (tl.nrow0 + NM - 1) / BROWS;
    const int s1 = e1 < 7 ? e1 : 6;
    const unsigned* sflag = myws + WS_XFLAG + ((lane & 16) ? s1 : s0) * 32 + (lane & 15);
    auto decode = [&](unsigned v) -> int {
#pragma unroll
      for (int o = 1; o < 32; o <<= 1) { const unsigned u = (unsigned)__shfl_xor((int)v, o); v = u < v ? u : v; }
      const unsigned vm = (unsigned)__builtin_amdgcn_readfirstlane((int)v);
      const int n = vm > call * 128u ? (int)(vm - call * 128u) : 0;
      return n > NT ? NT : n;
    };
    __builtin_amdgcn_fence(__ATOMIC_ACQUIRE, "");
    int seen = 0;
    nb = -1;
    while (seen < NT) {
      const int need = nb + 3 < NT ? nb + 3 : NT;
      if (seen < need) {
        const unsigned long long t0 = __builtin_amdgcn_s_memrealtime();
        unsigned spins = 0;
        while ((seen = decode(ld_sys(sflag))) < need) {
          if ((++spins & 63u) == 0u && (ld_agent(tmo) == call || __builtin_amdgcn_s_memrealtime() - t0 > DEADLINE)) { st_agent(tmo, call); ok = false; seen = NT; break; }
          __builtin_amdgcn_s_sleep(1);
        }
        waitv<0>();
        WAITL0();
        join();
      } else if (seen < need + POLL_MARGIN) {
        unsigned v;
        asm volatile("global_load_dword %0, %1, off sc0 sc1" : "=v"(v) : "v"(sflag) : "memory");
        join();
        asm volatile("s_waitcnt vmcnt(0)" : "+v"(v) :: "memory");
        const int n = decode(v);
        seen = n > seen ? n : seen;
      } else {
        join();
      }
    }
  } else if (tl.kind == 0) {
    while (nb < NT / 2) join();
    ok = wait_words(myws + WS_READY + tl.owner * 32, call, tmo);
  }
  while (nb < NT - 1) join();
  if (ld_agent(tmo) == call) ok = false;
  if (lane == 0) lds_wr(lds, C_GO, call * 2u + (ok ? 0u : 1u));
  WAITL0();
  while (nb <= NT) join();
}

__device__ __forceinline__ void copy_group(const Params& p, LAS char* lds, int c, int wid, int lane) {
  unsigned* myws = p.ws[p.rank];
  unsigned* tmo = myws + WS_TMO;
  const int b = blockIdx.x;
  const unsigned cnt = ld_agent(myws + WS_CNT + b);
  const int gw = wid * 16 + c;
  const auto rx = __builtin_amdgcn_make_buffer_rsrc((void*)p.x, 0, ML * KD * 2, 0x00020000);
  const unsigned xv = (unsigned)((AROWS + (lane >> 3)) * KD * 2 + (((lane & 7) ^ (((gw & 1) * 4 + (lane >> 4)) & 7)) * 16));
  auto load = [&](int k) { const int u = k * 128 + gw; return __builtin_amdgcn_raw_buffer_load_b128(rx, xv, (unsigned)((u % BRP) * (8 * KD * 2) + (u / BRP) * 128), 0); };
  u32x4 d[NUNIT];
  d[0] = load(0);
  d[1] = load(1);
  char* dbase[NR - 1];
  unsigned dslot[NR - 1];
#pragma unroll
  for (int i = 0; i < NR - 1; ++i) {
    const int j = __builtin_amdgcn_readfirstlane((i + gw) % (NR - 1) + 1);
    dbase[i] = (char*)p.ws[__builtin_amdgcn_readfirstlane((p.rank + NR - j) & 7)];
    dslot[i] = (unsigned)((j - 1) * BROWS * 128);
  }
  if (lane == 0) { lds_wr(lds, wid * 4, (unsigned)(gw / BRP)); if (wid == 0) lds_wr(lds, 64, 0u); }
  WAITL0();
  const unsigned call = (unsigned)__builtin_amdgcn_readfirstlane((int)cnt) + 1u;
  BAR();
  const unsigned long long tstart = __builtin_amdgcn_s_memrealtime();
  if (wid == 0 && lane == 0) st_agent(myws + WS_CNT + b, call);
  if (gw == 127 && lane < NR && lane != p.rank) st_sys(p.ws[lane] + WS_READY + p.rank * 32, call);
  const size_t xo = WS_XBUF + (call & 1) * XPAR_BYTES;
  unsigned* pflag = p.ws[(p.rank + NR - 1 - (lane < 7 ? lane : 0)) & 7] + WS_XFLAG + (lane < 7 ? lane : 0) * 32 + c;
  auto finish = [&](int k) {
    const int nxt = k + 1 < NUNIT ? ((k + 1) * 128 + gw) / BRP : NT;
    if (lane == 0) lds_wr(lds, wid * 4, (unsigned)nxt);
    unsigned v = lds_rd(lds, (lane & 7) * 4);
#pragma unroll
    for (int o = 1; o < 8; o <<= 1) { const unsigned u = (unsigned)__shfl_xor((int)v, o); v = u < v ? u : v; }
    const int pre = __builtin_amdgcn_readfirstlane((int)v);
    const int lvl = pre < 4 ? pre : (pre < 12 ? (pre & ~1) : (pre & ~3));
    unsigned old = 0;
    if (lane == 0) old = __hip_atomic_fetch_max((LAS unsigned*)(lds + 64), (unsigned)lvl, __ATOMIC_RELAXED, __HIP_MEMORY_SCOPE_WORKGROUP);
    old = (unsigned)__builtin_amdgcn_readfirstlane((int)old);
    if ((unsigned)lvl > old && lane < 7)
      __hip_atomic_fetch_max(pflag, call * 128u + (unsigned)lvl, __ATOMIC_RELAXED, __HIP_MEMORY_SCOPE_SYSTEM);
  };
  bool pending = false;
  long long tissue = 0;
#pragma unroll
  for (int k = 0; k < NUNIT; ++k) {
    const int u = k * 128 + gw;
    for (;;) {
      const long long now = (long long)(__builtin_amdgcn_s_memrealtime() - tstart) * 10;
      if (now * BRP >= (long long)(u - PUSH_LEAD * BRP) * TILE_NS) break;
      if (pending && now - tissue > ACK_NS) { waitv<0>(); finish(k - 1); pending = false; }
      else __builtin_amdgcn_s_sleep(2);
    }
    const unsigned so = (unsigned)(u / BRP) * (unsigned)XT_BYTES + (unsigned)(u % BRP) * 1024u;
#pragma unroll
    for (int i = 0; i < NR - 1; ++i) {
      const auto rd = __builtin_amdgcn_make_buffer_rsrc(dbase[i] + xo, 0, (int)XPAR_BYTES, 0x00020000);
      __builtin_amdgcn_raw_buffer_store_b128(d[k], rd, (unsigned)(lane * 16), so + dslot[i], AUX_WT_SYS);
    }
    tissue = (long long)(__builtin_amdgcn_s_memrealtime() - tstart) * 10;
    if (pending) { waitv<NR - 1>(); finish(k - 1); }
    pending = true;
    if (k + 2 < NUNIT) d[k + 2] = load(k + 2);
  }
  waitv<0>();
  finish(NUNIT - 1);
  if (c == 15 && wid != p.rank) {
    const bool fine = wait_words(myws + WS_DONE + wid * 64 + lane, call, tmo);
    if (!fine && lane == 0) p.out[p.rank][(size_t)(p.rank * ML) * NCOL] = 0x7fc0;
  }
}

__global__ __launch_bounds__(512, 2) void gemm_a2a(Params p) {
  extern __shared__ __attribute__((aligned(16))) char smem[];
  LAS char* lds = (LAS char*)smem;
  const int wid = __builtin_amdgcn_readfirstlane(threadIdx.x >> 6);
  const int lane = threadIdx.x & 63;
  const int b = blockIdx.x;
  const int xcd = b & 7, idx = b >> 3;
  if (idx < 2) { copy_group(p, lds, xcd * 2 + idx, wid, lane); return; }
  Tile tl;
  int rt = 0;
  if (idx < 16) {
    const int a = xcd * 14 + idx - 2;
    tl.kind = 0; tl.owner = (p.rank + a % 7 + 1) & 7; tl.col0 = (a / 7) * 128;
    tl.nrow0 = 0; tl.gathered = false; tl.koff = KSKEW * xcd; tl.qw = tl.qx = 0;
  } else {
    const int e = idx - 16;
    rt = (xcd >> 1) * 4 + (e >> 2);
    tl.kind = 1; tl.owner = p.rank; tl.col0 = ((xcd & 1) * 4 + (e & 3)) * 256;
    tl.nrow0 = rt * NM; tl.gathered = rt <= RMIX; tl.koff = 0; tl.qw = e >> 2; tl.qx = e & 3;
  }
  if (wid < 4) {
    if (tl.kind == 0) mfma_a(p, tl, lds, wid, lane);
    else mfma_n(p, tl, lds, wid, lane);
  } else if (wid == 7) {
    ctrl_wave(p, tl, lds, lane);
  } else {
#define LOADERS(MF, NIMG, SW, PAIR, NWS, AUX) do { if (wid == 4) loader<MF, NIMG, SW, PAIR, 0, NWS, AUX>(p, tl, lds, lane); \
    else if (wid == 5) loader<MF, NIMG, SW, PAIR, 1, NWS, AUX>(p, tl, lds, lane); else loader<MF, NIMG, SW, PAIR, 2, NWS, AUX>(p, tl, lds, lane); } while (0)
    if (tl.kind == 0) LOADERS(MFA, 1, SWA, true, 0, AUX_NT);
    else if (rt < RMIX) LOADERS(MFN, 2, 3, false, 4 * MFN, 0);
    else if (rt == RMIX) LOADERS(MFN, 2, 3, false, (XROWS - RMIX * NM) / 8, 0);
    else LOADERS(MFN, 2, 3, false, 0, 0);
  }
}

extern "C" void kernel_launch(int rank, int nranks, ncclComm_t comm,
                              hipStream_t stream,
                              void* const* d_in, const int* in_sizes, int n_in,
                              void* d_out, int out_size,
                              const int* in_dtypes, int out_dtype,
                              void* const* d_out_all,
                              void* const* d_ws_all, size_t ws_bytes) {
  if (nranks != NR || (unsigned)rank >= (unsigned)NR || ws_bytes < WS_NEED || n_in < 2 ||
      in_sizes[0] != ML * KD || in_sizes[1] != KD * ND || out_size != NR * ML * NCOL) {
    fprintf(stderr, "unexpected configuration\n");
    return;
  }
  Params p;
  p.x = (const unsigned short*)d_in[0];
  p.w = (const unsigned short*)d_in[1];
  for (int i = 0; i < NR; ++i) { p.out[i] = (unsigned short*)d_out_all[i]; p.ws[i] = (unsigned*)d_ws_all[i]; }
  p.rank = rank;
  static bool init[NR] = {};
  if (!init[rank]) {
    HIP_CHECK(hipFuncSetAttribute((const void*)gemm_a2a, hipFuncAttributeMaxDynamicSharedMemorySize, LDS_BYTES));
    init[rank] = true;
  }
  gemm_a2a<<<NBLK, 512, LDS_BYTES, stream>>>(p);
  HIP_CHECK(hipGetLastError());
}
```

```cpp
#include <hip/hip_runtime.h>
#include <rccl/rccl.h>
#include <cstdio>
#include <stdint.h>

#define HIP_CHECK(expr) do { hipError_t _e = (expr); if (_e != hipSuccess) { \
    fprintf(stderr, "HIP error %s at %s:%d\n", hipGetErrorString(_e), __FILE__, __LINE__); \
    return; } } while (0)

#define LAS __attribute__((address_space(3)))
typedef short s16x4 __attribute__((ext_vector_type(4)));
typedef __bf16 bf16x2 __attribute__((ext_vector_type(2)));
typedef __bf16 bf16x8 __attribute__((ext_vector_type(8)));
typedef float f32x4 __attribute__((ext_vector_type(4)));
typedef float f32x2 __attribute__((ext_vector_type(2)));
typedef unsigned u32x4 __attribute__((ext_vector_type(4)));

constexpr int NR = 8, ML = 512, KD = 4096, ND = 16384, NCOL = ND / NR;
constexpr int BK = 64, NT = KD / BK;
constexpr int MFA = 7;
constexpr int MFN = 5;
constexpr int AROWS = MFA * 32, BROWS = ML - AROWS;
constexpr int BRP = BROWS / 8;
constexpr int XROWS = 7 * BROWS;
constexpr int NROWS = XROWS + ML;
constexpr int NM = MFN * 32;
constexpr int RMIX = XROWS / NM;
constexpr int WODD = 8192 + 128;
constexpr int WIMG = WODD + 8192;
constexpr int SWA = 4;
constexpr int LDS_CTL = 3 * (2 * WIMG + MFN * 4096);
static_assert(SWA * WIMG + 3 * MFA * 4096 <= LDS_CTL, "LDS");
constexpr int C_GO = LDS_CTL;
constexpr int C_CALL = LDS_CTL + 16;
constexpr int LDS_BYTES = LDS_CTL + 64;
static_assert(LDS_BYTES <= 163840, "LDS");
constexpr int NBLK = 256;
constexpr unsigned long long DEADLINE = 50000000ull;
constexpr int WS_TMO = 32, WS_READY = 64  , WS_CNT = 512  , WS_DONE = 1024  ;
constexpr int WS_XFLAG = 2048;
constexpr size_t WS_XBUF = 1u << 20;
constexpr size_t XT_BYTES = (size_t)XROWS * 128;
constexpr size_t XPAR_BYTES = XT_BYTES * NT;
constexpr size_t WS_NEED = WS_XBUF + 2 * XPAR_BYTES;
constexpr int NUNIT = BRP * NT / 128;
constexpr int TILE_NS = 640;
constexpr int PUSH_LEAD = 1;
constexpr int ACK_NS = 800;
constexpr int POLL_MARGIN = 4;
constexpr int KSKEW = 8, KSTRIDE = 11;
constexpr int NS_T0 = 800, NPACE = 106, NPACE_END = 4400, NS_N0 = 3;
constexpr int NC_N0 = 10, NC_T0 = 1150, NC_SLOPE = 125;
constexpr int AUX_NT = 2, AUX_WT_AGENT = 16, AUX_WT_SYS = 17;

struct Params {
  const unsigned short* x;
  const unsigned short* w;
  unsigned short* out[NR];
  unsigned* ws[NR];
  int rank;
};

#define WAITL0() asm volatile("s_waitcnt lgkmcnt(0)" ::: "memory")
template <int N> __device__ __forceinline__ void waitv() { asm volatile("s_waitcnt vmcnt(%0)" :: "n"(N) : "memory"); }
#define BAR() __builtin_amdgcn_s_barrier()
#define SCHED() __builtin_amdgcn_sched_barrier(0)
#define CFENCE() asm volatile("" ::: "memory")
#define NOP_ do {} while (0)
#define STEP(a, d) asm volatile("v_add_u32 %0, %1, %0" : "+v"(a) : "s"(d))
#define NEXT1(o, d) asm volatile("s_add_i32 %0, %0, %1" : "+s"(o) : "s"(d) : "scc")
#define NEXT2(o, d, last, slot) asm volatile("s_cmp_eq_u32 %1, %2\n\ts_cselect_b32 %0, %3, %4" : "=s"(d) : "s"(o), "s"(last), "s"(-(last)), "s"(slot) : "scc")

__device__ __forceinline__ bf16x8 ldx(unsigned a) { return *(LAS const bf16x8*)(size_t)a; }
__device__ __forceinline__ s16x4 ldw(unsigned a) { return __builtin_amdgcn_ds_read_tr16_b64_v4i16((LAS s16x4*)(size_t)a); }
__device__ __forceinline__ void mma(f32x4& c, bf16x8 a, bf16x8 b) { asm volatile("v_mfma_f32_16x16x32_bf16 %0, %1, %2, %0" : "+v"(c) : "v"(a), "v"(b)); }
__device__ __forceinline__ void mma_drain() {
  __builtin_amdgcn_sched_barrier(0);
  asm volatile("s_nop 15\n\ts_nop 7" ::: "memory");
  __builtin_amdgcn_sched_barrier(0);
}
__device__ __forceinline__ unsigned pk(float a, float b) { f32x2 v = {a, b}; return __builtin_bit_cast(unsigned, __builtin_convertvector(v, bf16x2)); }
__device__ __forceinline__ unsigned ld_sys(const unsigned* p) { return __hip_atomic_load(p, __ATOMIC_RELAXED, __HIP_MEMORY_SCOPE_SYSTEM); }
__device__ __forceinline__ unsigned ld_agent(const unsigned* p) { return __hip_atomic_load(p, __ATOMIC_RELAXED, __HIP_MEMORY_SCOPE_AGENT); }
__device__ __forceinline__ void st_sys(unsigned* p, unsigned v) { __hip_atomic_store(p, v, __ATOMIC_RELAXED, __HIP_MEMORY_SCOPE_SYSTEM); }
__device__ __forceinline__ void st_agent(unsigned* p, unsigned v) { __hip_atomic_store(p, v, __ATOMIC_RELAXED, __HIP_MEMORY_SCOPE_AGENT); }
__device__ __forceinline__ unsigned lds_rd(LAS char* lds, int off) { return *(LAS volatile unsigned*)(lds + off); }
__device__ __forceinline__ void lds_wr(LAS char* lds, int off, unsigned v) { *(LAS volatile unsigned*)(lds + off) = v; }
__device__ __forceinline__ bf16x8 wfrag(s16x4 lo, s16x4 hi) { return __builtin_bit_cast(bf16x8, __builtin_shufflevector(lo, hi, 0, 1, 2, 3, 4, 5, 6, 7)); }
__device__ __forceinline__ u32x4 pack2(f32x4 a, f32x4 c, bool bad) {
  const unsigned a0 = pk(a[0], a[1]), a1 = pk(a[2], a[3]), c0 = pk(c[0], c[1]), c1 = pk(c[2], c[3]);
  auto r0 = __builtin_amdgcn_permlane16_swap(a0, c0, false, false);
  auto r1 = __builtin_amdgcn_permlane16_swap(a1, c1, false, false);
  return bad ? (u32x4){0x7fc07fc0u, 0x7fc07fc0u, 0x7fc07fc0u, 0x7fc07fc0u} : (u32x4){r0[0], r1[0], r0[1], r1[1]};
}
__device__ __forceinline__ unsigned out_voff(int lane) { const int fr = lane & 15, fq = lane >> 4; return (unsigned)((fr * NCOL + (fq & 1) * 16 + (fq >> 1) * 8) * 2); }
__host__ __device__ constexpr int wpiece(int pc) { return ((pc >> 1) & 1) * WODD + (pc >> 2) * 2048 + (pc & 1) * 1024; }

struct Tile {
  int kind;
  int owner;
  int col0;
  int nrow0;
  bool gathered;
  int koff;
  int qw, qx;
};

template <int MF, int NIMG, int SW, bool PAIR, int H, int NWS, int AUXW>
__device__ __forceinline__ void loader(const Params& p, const Tile& tl, LAS char* lds, int lane) {
  constexpr int NWP = 16 * NIMG;
  constexpr int WSLOT = NIMG * WIMG, XSLOT = MF * 4096, LX = SW * WSLOT;
  constexpr int NW_ = (NWP - H + 2) / 3;
  constexpr int X0 = (H + 3 * NW_) - NWP;
  constexpr int NX_ = (4 * MF - X0 + 2) / 3;
  constexpr bool XWS = NWS > 0;
  static_assert(2 * NW_ + 2 * NX_ <= 63, "vmcnt");
  const auto rw = __builtin_amdgcn_make_buffer_rsrc((void*)p.w, 0, KD * ND * 2, 0x00020000);
  const auto rx = __builtin_amdgcn_make_buffer_rsrc((void*)p.x, 0, ML * KD * 2, 0x00020000);
  const unsigned wv = (unsigned)((lane >> 4) * ND * 2 + (((lane & 15) ^ ((lane >> 4) << 1)) * 16));
  unsigned xv[2];
#pragma unroll
  for (int v = 0; v < 2; ++v) xv[v] = (unsigned)((lane >> 3) * KD * 2 + (((lane & 7) ^ ((v * 4 + (lane >> 4)) & 7)) * 16));
  const unsigned iv = (unsigned)(lane * 16);
  const unsigned wbase = (unsigned)((tl.owner * NCOL + tl.col0) * 2);
  const unsigned xbase = tl.kind == 0 ? 0u : (unsigned)((tl.nrow0 - XROWS) * KD * 2);
  auto xpc = [&](int pc) -> unsigned {
    const bool pad = NIMG == 2 && NWS == 0 && tl.nrow0 + pc * 8 >= NROWS;
    return (unsigned)((pad ? pc - 8 : pc) * (8 * KD * 2));
  };
  auto kmap = [&](int t) -> unsigned {
    return PAIR ? (unsigned)(((((t >> 1) * KSTRIDE) << 1) + (t & 1) + tl.koff) & (NT - 1)) : (unsigned)t;
  };
  int ws_ = 0, xs_ = 0;
  auto issue_w = [&](int t) {
    const unsigned tb = wbase + kmap(t) * (BK * ND * 2);
#pragma unroll
    for (int i = 0; i < NW_; ++i) {
      const int pc = H + 3 * i, img = pc >> 4, q = pc & 15;
      __builtin_amdgcn_raw_ptr_buffer_load_lds(rw, (LAS void*)(lds + ws_ * WSLOT + img * WIMG + wpiece(q)), 16, wv, tb + q * (4 * ND * 2) + img * 256, 0, AUXW);
    }
    ws_ = ws_ == SW - 1 ? 0 : ws_ + 1;
  };
  auto issue_xl = [&](int t) {
    const unsigned tb = xbase + kmap(t) * 128u;
#pragma unroll
    for (int i = 0; i < NX_; ++i) {
      const int pc = X0 + 3 * i;
      if (pc >= NWS) __builtin_amdgcn_raw_ptr_buffer_load_lds(rx, (LAS void*)(lds + LX + xs_ * XSLOT + pc * 1024), 16, xv[pc & 1], tb + xpc(pc), 0, 0);
    }
  };
  issue_w(0);
  issue_xl(0);
  unsigned call = 0;
  if (XWS) {
    BAR();
    call = (unsigned)__builtin_amdgcn_readfirstlane((int)lds_rd(lds, C_CALL));
  }
  const auto ri = __builtin_amdgcn_make_buffer_rsrc((char*)p.ws[p.rank] + WS_XBUF + (call & 1) * XPAR_BYTES, 0, (int)XPAR_BYTES, 0x00020000);
  const unsigned ibase = (unsigned)(tl.nrow0 * 128);
  auto issue_xi = [&](int t) {
#pragma unroll
    for (int i = 0; i < NX_; ++i) {
      const int pc = X0 + 3 * i;
      if (pc < NWS) __builtin_amdgcn_raw_ptr_buffer_load_lds(ri, (LAS void*)(lds + LX + xs_ * XSLOT + pc * 1024), 16, iv, ibase + (unsigned)t * (unsigned)XT_BYTES + pc * 1024, 0, 0);
    }
    xs_ = xs_ == 2 ? 0 : xs_ + 1;
  };
  issue_xi(0);
  issue_w(1);
  issue_xl(1);
  issue_xi(1);
  waitv<NW_ + NX_>();
  BAR();
  if (PAIR) {
    unsigned xl[NX_], xg[NX_];
#pragma unroll
    for (int i = 0; i < NX_; ++i) {
      int k = i + (tl.qx * NX_) / 14; k = k >= NX_ ? k - NX_ : k;
      const int pc = X0 + 3 * k;
      xl[i] = (unsigned)(pc * 1024);
      xg[i] = ((pc & 1) ? xv[1] : xv[0]) + (unsigned)(pc * (8 * KD * 2));
    }
    auto issue_x = [&](int t) {
      const unsigned tx = kmap(t) * 128u;
      LAS char* lx = lds + LX + xs_ * XSLOT;
#pragma unroll
      for (int i = 0; i < NX_; ++i) __builtin_amdgcn_raw_ptr_buffer_load_lds(rx, (LAS void*)(lx + xl[i]), 16, xg[i], tx, 0, 0);
      xs_ = xs_ == 2 ? 0 : xs_ + 1;
    };
    for (int t = 0; t < NT - 2; t += 2) {
      issue_w(t + 2);
      issue_w(t + 3);
      issue_x(t + 2);
      waitv<2 * NW_ + NX_>();
      BAR();
      issue_x(t + 3);
      waitv<NX_>();
      BAR();
    }
  } else if (NWS == 0 || NWS == 4 * MF) {
    constexpr int QW = (NW_ + 3) / 4, QX = (NX_ + 3) / 4;
    unsigned wl[NW_], wg[NW_], xl[NX_], xg[NX_];
#pragma unroll
    for (int i = 0; i < NW_; ++i) {
      int k = i + tl.qw * QW; k = k >= NW_ ? k - NW_ : k;
      const int pc = H + 3 * k, img = pc >> 4, q = pc & 15;
      wl[i] = (unsigned)(img * WIMG + wpiece(q));
      wg[i] = wv + (unsigned)(q * (4 * ND * 2) + img * 256);
    }
#pragma unroll
    for (int i = 0; i < NX_; ++i) {
      int k = i + tl.qx * QX; k = k >= NX_ ? k - NX_ : k;
      const int pc = X0 + 3 * k;
      xl[i] = (unsigned)(pc * 1024);
      xg[i] = XWS ? iv + (unsigned)(pc * 1024) : ((pc & 1) ? xv[1] : xv[0]) + xpc(pc);
    }
    for (int t = 0; t < NT - 2; ++t) {
      const unsigned tw = wbase + (unsigned)(t + 2) * (BK * ND * 2);
      const unsigned tx = XWS ? ibase + (unsigned)(t + 2) * (unsigned)XT_BYTES : xbase + (unsigned)(t + 2) * 128u;
      LAS char* lw = lds + ws_ * WSLOT;
      LAS char* lx = lds + LX + xs_ * XSLOT;
#pragma unroll
      for (int ph = 0; ph < 2; ++ph) {
#pragma unroll
        for (int i = ph ? QW : 0; i < (ph ? NW_ : QW); ++i)
          __builtin_amdgcn_raw_ptr_buffer_load_lds(rw, (LAS void*)(lw + wl[i]), 16, wg[i], tw, 0, AUXW);
#pragma unroll
        for (int i = ph ? QX : 0; i < (ph ? NX_ : QX); ++i)
          __builtin_amdgcn_raw_ptr_buffer_load_lds(XWS ? ri : rx, (LAS void*)(lx + xl[i]), 16, xg[i], tx, 0, 0);
      }
      ws_ = ws_ == SW - 1 ? 0 : ws_ + 1;
      xs_ = xs_ == 2 ? 0 : xs_ + 1;
      waitv<NW_ + NX_>();
      BAR();
    }
  } else {
    for (int t = 0; t < NT - 2; ++t) {
      issue_w(t + 2);
      issue_xl(t + 2);
      issue_xi(t + 2);
      waitv<NW_ + NX_>();
      BAR();
    }
  }
  waitv<0>();
  BAR();
  BAR();
}

__device__ __forceinline__ void mfma_a(const Params& p, const Tile& tl, LAS char* lds, int wid, int lane) {
  constexpr int MF = MFA;
  static_assert(MF == 7, "the schedule below is written for 7 fragments");
  constexpr int XSLOT = MF * 4096, LX = SWA * WIMG;
  const int wr = wid >> 1, wc = wid & 1;
  const int fr = lane & 15, fq = lane >> 4;
  const int q = (lane & 15) >> 2, pp = lane & 3;
  const unsigned lb = (unsigned)(size_t)lds;
  const unsigned xo = (unsigned)(LX + (wr * MF * 16 + fr) * 128 + ((fq ^ ((fr >> 1) & 7)) * 16));
  unsigned xa0 = lb + xo, xa1 = lb + (xo ^ 64u);
  unsigned wa[4];
#pragma unroll
  for (int nf = 0; nf < 4; ++nf)
    wa[nf] = lb + (unsigned)((fq & 1) * WODD + (fq >> 1) * 2048 + q * 256 + wc * 128 + ((nf ^ q) * 32) + 8 * pp);
  f32x4 acc[MF][4];
#pragma unroll
  for (int i = 0; i < MF; ++i)
#pragma unroll
    for (int j = 0; j < 4; ++j) { acc[i][j] = (f32x4){0.f, 0.f, 0.f, 0.f}; asm volatile("" : "+v"(acc[i][j])); }
  bf16x8 Xf[MF];
  s16x4 Wl[2][4], Wh[2][4];
#define RDW(bu, ks, i) do { if ((i) & 1) Wh[bu][(i) >> 1] = ldw(wa[(i) >> 1] + (ks) * 4096 + 1024); \
    else Wl[bu][(i) >> 1] = ldw(wa[(i) >> 1] + (ks) * 4096); } while (0)
#define RDX(i, xa) Xf[i] = ldx((xa) + (i) * 2048)
#define MMA1(mf, bu, nf) do { mma(acc[mf][nf], wfrag(Wl[bu][nf], Wh[bu][nf]), Xf[mf]); SCHED(); } while (0)
#define SN(mf, i) (((mf) & 1) ? 3 - (i) : (i))
#define GRP(mf, bu, F0, F1, F2) do { MMA1(mf, bu, SN(mf, 0)); F0; SCHED(); MMA1(mf, bu, SN(mf, 1)); F1; SCHED(); \
    MMA1(mf, bu, SN(mf, 2)); F2; SCHED(); MMA1(mf, bu, SN(mf, 3)); } while (0)
  BAR();
#pragma unroll
  for (int i = 0; i < 8; ++i) RDW(0, 0, i);
#pragma unroll
  for (int mf = 0; mf < MF - 1; ++mf) RDX(mf, xa0);
  __builtin_amdgcn_s_waitcnt(0xC07F);
  int ow = 0, ox = 0, dw = WIMG, dx = XSLOT;
  for (int t = 0; t < NT; ++t) {
    GRP(0, 0, RDX(6, xa0), RDW(1, 1, 0), RDW(1, 1, 1));
    GRP(1, 0, RDX(0, xa1), RDW(1, 1, 2), RDW(1, 1, 3));
    GRP(2, 0, RDX(1, xa1), RDW(1, 1, 4), STEP(xa0, dx));
    GRP(3, 0, RDX(2, xa1), RDW(1, 1, 5), STEP(wa[0], dw));
    GRP(4, 0, RDX(3, xa1), RDW(1, 1, 6), STEP(wa[1], dw));
    GRP(5, 0, RDX(4, xa1), RDW(1, 1, 7), STEP(wa[2], dw));
    GRP(6, 0, RDX(5, xa1), NOP_, STEP(wa[3], dw));
    GRP(0, 1, RDX(6, xa1), NEXT1(ow, dw), NEXT2(ow, dw, (SWA - 1) * WIMG, WIMG));
    GRP(1, 1, STEP(xa1, dx), NEXT1(ox, dx), NEXT2(ox, dx, 2 * XSLOT, XSLOT));
    __builtin_amdgcn_s_waitcnt(0xC07F);
    CFENCE();
    BAR();
    CFENCE();
    SCHED();
    GRP(2, 1, RDW(0, 0, 0), RDW(0, 0, 1), RDX(0, xa0));
    GRP(3, 1, RDW(0, 0, 2), RDW(0, 0, 3), RDX(1, xa0));
    GRP(4, 1, RDW(0, 0, 4), RDW(0, 0, 5), RDX(2, xa0));
    GRP(5, 1, RDW(0, 0, 6), RDW(0, 0, 7), RDX(3, xa0));
    GRP(6, 1, RDX(4, xa0), RDX(5, xa0), NOP_);
  }
  mma_drain();
  const unsigned go = (unsigned)__builtin_amdgcn_readfirstlane((int)lds_rd(lds, C_GO));
  const bool bad = (go & 1u) != 0u;
  const auto ro = __builtin_amdgcn_make_buffer_rsrc((void*)p.out[tl.owner], 0, ML * NR * NCOL * 2, 0x00020000);
  const unsigned vo = out_voff(lane);
  const unsigned so = (unsigned)(((p.rank * ML + wr * MF * 16) * NCOL + tl.col0 + wc * 64) * 2);
#pragma unroll
  for (int mf = 0; mf < MF; ++mf) {
#pragma unroll
    for (int np = 0; np < 2; ++np)
      __builtin_amdgcn_raw_buffer_store_b128(pack2(acc[mf][2 * np], acc[mf][2 * np + 1], bad), ro, vo + np * 64, so + mf * (16 * NCOL * 2), AUX_WT_SYS);
    waitv<0>();
  }
  if (lane == 0) st_sys(p.ws[tl.owner] + WS_DONE + p.rank * 64 + (tl.col0 >> 7) * 4 + wid, go >> 1);
#undef RDW
#undef RDX
#undef MMA1
#undef GRP
}

__device__ __forceinline__ void mfma_n(const Params& p, const Tile& tl, LAS char* lds, int wid, int lane) {
  constexpr int MF = MFN;
  static_assert(MF == 5, "the schedule below is written for 5 fragments");
  constexpr int WSLOT = 2 * WIMG, XSLOT = MF * 4096, LX = 3 * WSLOT;
  const int wr = wid >> 1, wc = wid & 1;
  const int fr = lane & 15, fq = lane >> 4;
  const int q = (lane & 15) >> 2, pp = lane & 3;
  const unsigned lb = (unsigned)(size_t)lds;
  const unsigned xo = (unsigned)(LX + (wr * MF * 16 + fr) * 128 + ((fq ^ ((fr >> 1) & 7)) * 16));
  unsigned xa0 = lb + xo, xa1 = lb + (xo ^ 64u);
  unsigned wa[4];
#pragma unroll
  for (int nf = 0; nf < 4; ++nf)
    wa[nf] = lb + (unsigned)(wc * WIMG + (fq & 1) * WODD + (fq >> 1) * 2048 + q * 256 + ((nf ^ q) * 32) + 8 * pp);
  f32x4 acc[MF][8];
#pragma unroll
  for (int i = 0; i < MF; ++i)
#pragma unroll
    for (int j = 0; j < 8; ++j) { acc[i][j] = (f32x4){0.f, 0.f, 0.f, 0.f}; asm volatile("" : "+v"(acc[i][j])); }
  bf16x8 Xf[2][MF];
  s16x4 Wl[8], Wh[8];
#define RWL(nf, ks) Wl[nf] = ldw(wa[(nf) & 3] + ((nf) >> 2) * 128 + (ks) * 4096)
#define RWH(nf, ks) Wh[nf] = ldw(wa[(nf) & 3] + ((nf) >> 2) * 128 + (ks) * 4096 + 1024)
#define RDX(bu, i, xa) Xf[bu][i] = ldx((xa) + (i) * 2048)
#define MMA1(nf, bu, mf) do { mma(acc[mf][nf], wfrag(Wl[nf], Wh[nf]), Xf[bu][mf]); SCHED(); } while (0)
#define SN5(nf, i) (((nf) & 1) ? 4 - (i) : (i))
#define GRP(nf, bu, F0, F1, F2, F3, F4) do { MMA1(nf, bu, SN5(nf, 0)); F0; SCHED(); MMA1(nf, bu, SN5(nf, 1)); F1; SCHED(); \
    MMA1(nf, bu, SN5(nf, 2)); F2; SCHED(); MMA1(nf, bu, SN5(nf, 3)); F3; SCHED(); MMA1(nf, bu, SN5(nf, 4)); F4; SCHED(); } while (0)
  if (tl.gathered) {
    const unsigned gwv = (unsigned)(((tl.nrow0 / NM) * 8 + (tl.col0 >> 8)) * 4 + wid);
    LAS void* dummy = (LAS void*)(lds + 2 * WSLOT + wid * 1024);
    const auto rx = __builtin_amdgcn_make_buffer_rsrc((void*)p.x, 0, ML * KD * 2, 0x00020000);
#pragma unroll
    for (int i = 0; i < 10; ++i)
      __builtin_amdgcn_raw_ptr_buffer_load_lds(rx, dummy, 16, (unsigned)(lane * 16), (gwv * 10240u + i * 1024u) & (unsigned)(ML * KD * 2 - 1), 0, 0);
    waitv<0>();
    BAR();
  }
  BAR();
#pragma unroll
  for (int mf = 0; mf < MF; ++mf) RDX(0, mf, xa0);
#pragma unroll
  for (int nf = 0; nf < 7; ++nf) { RWL(nf, 0); RWH(nf, 0); }
  __builtin_amdgcn_s_waitcnt(0xC07F);
  int ow = 0, ox = 0, dw = WSLOT, dx = XSLOT;
  for (int t = 0; t < NT; ++t) {
    GRP(0, 0, RWL(7, 0), RWH(7, 0), RDX(1, 0, xa1), STEP(xa0, dx), NOP_);
    GRP(1, 0, RWL(0, 1), RWH(0, 1), RDX(1, 1, xa1), NOP_, NOP_);
    GRP(2, 0, RWL(1, 1), RWH(1, 1), RDX(1, 2, xa1), NOP_, NOP_);
    GRP(3, 0, RWL(2, 1), RWH(2, 1), RDX(1, 3, xa1), NOP_, NOP_);
    GRP(4, 0, RWL(3, 1), RWH(3, 1), RDX(1, 4, xa1), NOP_, NOP_);
    GRP(5, 0, RWL(4, 1), RWH(4, 1), NOP_, STEP(wa[0], dw), STEP(xa1, dx));
    GRP(6, 0, RWL(5, 1), RWH(5, 1), NOP_, STEP(wa[1], dw), NOP_);
    GRP(7, 0, RWL(6, 1), RWH(6, 1), NOP_, STEP(wa[2], dw), NOP_);
    GRP(0, 1, RWL(7, 1), RWH(7, 1), NOP_, STEP(wa[3], dw), NOP_);
    GRP(1, 1, NEXT1(ow, dw), NEXT2(ow, dw, 2 * WSLOT, WSLOT), NEXT1(ox, dx), NEXT2(ox, dx, 2 * XSLOT, XSLOT), NOP_);
    __builtin_amdgcn_s_waitcnt(0xC07F);
    CFENCE();
    BAR();
    CFENCE();
    SCHED();
    GRP(2, 1, RWL(0, 0), RWH(0, 0), RDX(0, 0, xa0), RWL(1, 0), RWH(1, 0));
    GRP(3, 1, RWL(2, 0), RWH(2, 0), RDX(0, 1, xa0), RDX(0, 2, xa0), NOP_);
    GRP(4, 1, RWL(3, 0), RWH(3, 0), RDX(0, 3, xa0), RDX(0, 4, xa0), NOP_);
    GRP(5, 1, RWL(4, 0), RWH(4, 0), NOP_, NOP_, NOP_);
    GRP(6, 1, RWL(5, 0), RWH(5, 0), NOP_, NOP_, NOP_);
    GRP(7, 1, RWL(6, 0), RWH(6, 0), NOP_, NOP_, NOP_);
  }
  mma_drain();
  const unsigned go = (unsigned)__builtin_amdgcn_readfirstlane((int)lds_rd(lds, C_GO));
  const bool bad = (go & 1u) != 0u;
  const auto ro = __builtin_amdgcn_make_buffer_rsrc((void*)p.out[p.rank], 0, ML * NR * NCOL * 2, 0x00020000);
  const unsigned vo = out_voff(lane);
  const unsigned so = (unsigned)((tl.col0 + wc * 128) * 2);
#pragma unroll
  for (int mf = 0; mf < MF; ++mf) {
    const int nr = tl.nrow0 + wr * MF * 16 + mf * 16;
    if (nr < NROWS) {
      const int orow = nr < XROWS ? ((p.rank + 1 + nr / BROWS) & 7) * ML + AROWS + nr % BROWS : p.rank * ML + nr - XROWS;
#pragma unroll
      for (int np = 0; np < 4; ++np)
        __builtin_amdgcn_raw_buffer_store_b128(pack2(acc[mf][2 * np], acc[mf][2 * np + 1], bad), ro, vo + np * 64, so + (unsigned)orow * (NCOL * 2), AUX_WT_AGENT);
    }
  }
#undef RWL
#undef RWH
#undef RDX
#undef MMA1
#undef GRP
}

__device__ __forceinline__ bool wait_words(const unsigned* word, unsigned call, unsigned* tmo) {
  const unsigned long long t0 = __builtin_amdgcn_s_memrealtime();
  unsigned spins = 0;
  for (;;) {
    if (__all(ld_sys(word) >= call)) return true;
    if ((++spins & 63u) == 0u && (ld_agent(tmo) == call || __builtin_amdgcn_s_memrealtime() - t0 > DEADLINE)) { st_agent(tmo, call); return false; }
    __builtin_amdgcn_s_sleep(1);
  }
}

__device__ __forceinline__ void ctrl_wave(const Params& p, const Tile& tl, LAS char* lds, int lane) {
  unsigned* myws = p.ws[p.rank];
  unsigned* tmo = myws + WS_TMO;
  const int b = blockIdx.x;
  const unsigned long long tk0 = __builtin_amdgcn_s_memrealtime();
  const unsigned cnt = ld_agent(myws + WS_CNT + b);
  if (!tl.gathered) BAR();
  const unsigned call = (unsigned)__builtin_amdgcn_readfirstlane((int)cnt) + 1u;
  if (lane == 0) st_agent(myws + WS_CNT + b, call);
  bool ok = true;
  int nb = 1;
  auto join = [&]() {
    if (tl.kind == 1) {
      int due = NS_T0 + nb * NPACE;
      if (!tl.gathered) { const int d2 = nb > NC_N0 ? NC_T0 + (nb - NC_N0) * NC_SLOPE : 0; due = d2 < due ? d2 : due; }
      if (nb < NS_N0) due = 0;
      due = due < NPACE_END ? due : NPACE_END;
      while ((long long)(__builtin_amdgcn_s_memrealtime() - tk0) < (long long)due) __builtin_amdgcn_s_sleep(1);
    }
    BAR();
    ++nb;
  };
  if (tl.gathered) {
    if (lane == 0) lds_wr(lds, C_CALL, call);
    const int s0 = tl.nrow0 / BROWS;
    const int e1 = (tl.nrow0 + NM - 1) / BROWS;
    const int s1 = e1 < 7 ? e1 : 6;
    const unsigned* sflag = myws + WS_XFLAG + ((lane & 16) ? s1 : s0) * 32 + (lane & 15);
    auto decode = [&](unsigned v) -> int {
#pragma unroll
      for (int o = 1; o < 32; o <<= 1) { const unsigned u = (unsigned)__shfl_xor((int)v, o); v = u < v ? u : v; }
      const unsigned vm = (unsigned)__builtin_amdgcn_readfirstlane((int)v);
      const int n = vm > call * 128u ? (int)(vm - call * 128u) : 0;
      return n > NT ? NT : n;
    };
    __builtin_amdgcn_fence(__ATOMIC_ACQUIRE, "");
    int seen = 0;
    nb = -1;
    while (seen < NT) {
      const int need = nb + 3 < NT ? nb + 3 : NT;
      if (seen < need) {
        const unsigned long long t0 = __builtin_amdgcn_s_memrealtime();
        unsigned spins = 0;
        while ((seen = decode(ld_sys(sflag))) < need) {
          if ((++spins & 63u) == 0u && (ld_agent(tmo) == call || __builtin_amdgcn_s_memrealtime() - t0 > DEADLINE)) { st_agent(tmo, call); ok = false; seen = NT; break; }
          __builtin_amdgcn_s_sleep(1);
        }
        waitv<0>();
        WAITL0();
        join();
      } else if (seen < need + POLL_MARGIN) {
        unsigned v;
        asm volatile("global_load_dword %0, %1, off sc0 sc1" : "=v"(v) : "v"(sflag) : "memory");
        join();
        asm volatile("s_waitcnt vmcnt(0)" : "+v"(v) :: "memory");
        const int n = decode(v);
        seen = n > seen ? n : seen;
      } else {
        join();
      }
    }
  } else if (tl.kind == 0) {
    while (nb < NT / 2) join();
    ok = wait_words(myws + WS_READY + tl.owner * 32, call, tmo);
  }
  while (nb < NT - 1) join();
  if (ld_agent(tmo) == call) ok = false;
  if (lane == 0) lds_wr(lds, C_GO, call * 2u + (ok ? 0u : 1u));
  WAITL0();
  while (nb <= NT) join();
}

__device__ __forceinline__ void copy_group(const Params& p, LAS char* lds, int c, int wid, int lane) {
  unsigned* myws = p.ws[p.rank];
  unsigned* tmo = myws + WS_TMO;
  const int b = blockIdx.x;
  const unsigned cnt = ld_agent(myws + WS_CNT + b);
  const int gw = wid * 16 + c;
  const auto rx = __builtin_amdgcn_make_buffer_rsrc((void*)p.x, 0, ML * KD * 2, 0x00020000);
  const unsigned xv = (unsigned)((AROWS + (lane >> 3)) * KD * 2 + (((lane & 7) ^ (((gw & 1) * 4 + (lane >> 4)) & 7)) * 16));
  auto load = [&](int k) { const int u = k * 128 + gw; return __builtin_amdgcn_raw_buffer_load_b128(rx, xv, (unsigned)((u % BRP) * (8 * KD * 2) + (u / BRP) * 128), 0); };
  u32x4 d[NUNIT];
  d[0] = load(0);
  d[1] = load(1);
  char* dbase[NR - 1];
  unsigned dslot[NR - 1];
#pragma unroll
  for (int i = 0; i < NR - 1; ++i) {
    const int j = __builtin_amdgcn_readfirstlane((i + gw) % (NR - 1) + 1);
    dbase[i] = (char*)p.ws[__builtin_amdgcn_readfirstlane((p.rank + NR - j) & 7)];
    dslot[i] = (unsigned)((j - 1) * BROWS * 128);
  }
  if (lane == 0) { lds_wr(lds, wid * 4, (unsigned)(gw / BRP)); if (wid == 0) lds_wr(lds, 64, 0u); }
  WAITL0();
  const unsigned call = (unsigned)__builtin_amdgcn_readfirstlane((int)cnt) + 1u;
  BAR();
  const unsigned long long tstart = __builtin_amdgcn_s_memrealtime();
  if (wid == 0 && lane == 0) st_agent(myws + WS_CNT + b, call);
  if (gw == 127 && lane < NR && lane != p.rank) st_sys(p.ws[lane] + WS_READY + p.rank * 32, call);
  const size_t xo = WS_XBUF + (call & 1) * XPAR_BYTES;
  unsigned* pflag = p.ws[(p.rank + NR - 1 - (lane < 7 ? lane : 0)) & 7] + WS_XFLAG + (lane < 7 ? lane : 0) * 32 + c;
  auto finish = [&](int k) {
    const int nxt = k + 1 < NUNIT ? ((k + 1) * 128 + gw) / BRP : NT;
    if (lane == 0) lds_wr(lds, wid * 4, (unsigned)nxt);
    unsigned v = lds_rd(lds, (lane & 7) * 4);
#pragma unroll
    for (int o = 1; o < 8; o <<= 1) { const unsigned u = (unsigned)__shfl_xor((int)v, o); v = u < v ? u : v; }
    const int pre = __builtin_amdgcn_readfirstlane((int)v);
    const int lvl = pre < 4 ? pre : (pre < 12 ? (pre & ~1) : (pre & ~3));
    unsigned old = 0;
    if (lane == 0) old = __hip_atomic_fetch_max((LAS unsigned*)(lds + 64), (unsigned)lvl, __ATOMIC_RELAXED, __HIP_MEMORY_SCOPE_WORKGROUP);
    old = (unsigned)__builtin_amdgcn_readfirstlane((int)old);
    if ((unsigned)lvl > old && lane < 7)
      __hip_atomic_fetch_max(pflag, call * 128u + (unsigned)lvl, __ATOMIC_RELAXED, __HIP_MEMORY_SCOPE_SYSTEM);
  };
  bool pending = false;
  long long tissue = 0;
#pragma unroll
  for (int k = 0; k < NUNIT; ++k) {
    const int u = k * 128 + gw;
    for (;;) {
      const long long now = (long long)(__builtin_amdgcn_s_memrealtime() - tstart) * 10;
      if (now * BRP >= (long long)(u - PUSH_LEAD * BRP) * TILE_NS) break;
      if (pending && now - tissue > ACK_NS) { waitv<0>(); finish(k - 1); pending = false; }
      else __builtin_amdgcn_s_sleep(2);
    }
    const unsigned so = (unsigned)(u / BRP) * (unsigned)XT_BYTES + (unsigned)(u % BRP) * 1024u;
#pragma unroll
    for (int i = 0; i < NR - 1; ++i) {
      const auto rd = __builtin_amdgcn_make_buffer_rsrc(dbase[i] + xo, 0, (int)XPAR_BYTES, 0x00020000);
      __builtin_amdgcn_raw_buffer_store_b128(d[k], rd, (unsigned)(lane * 16), so + dslot[i], AUX_WT_SYS);
    }
    tissue = (long long)(__builtin_amdgcn_s_memrealtime() - tstart) * 10;
    if (pending) { waitv<NR - 1>(); finish(k - 1); }
    pending = true;
    if (k + 2 < NUNIT) d[k + 2] = load(k + 2);
  }
  waitv<0>();
  finish(NUNIT - 1);
  if (c == 15 && wid != p.rank) {
    const bool fine = wait_words(myws + WS_DONE + wid * 64 + lane, call, tmo);
    if (!fine && lane == 0) p.out[p.rank][(size_t)(p.rank * ML) * NCOL] = 0x7fc0;
  }
}

__global__ __launch_bounds__(512, 2) void gemm_a2a(Params p) {
  extern __shared__ __attribute__((aligned(16))) char smem[];
  LAS char* lds = (LAS char*)smem;
  const int wid = __builtin_amdgcn_readfirstlane(threadIdx.x >> 6);
  const int lane = threadIdx.x & 63;
  const int b = blockIdx.x;
  const int xcd = b & 7, idx = b >> 3;
  if (idx < 2) { copy_group(p, lds, xcd * 2 + idx, wid, lane); return; }
  Tile tl;
  int rt = 0;
  if (idx < 16) {
    const int a = xcd * 14 + idx - 2;
    tl.kind = 0; tl.owner = (p.rank + a % 7 + 1) & 7; tl.col0 = (a / 7) * 128;
    tl.nrow0 = 0; tl.gathered = false; tl.koff = KSKEW * xcd; tl.qw = 0; tl.qx = idx - 2;
  } else {
    const int e = idx - 16;
    rt = (xcd >> 1) * 4 + (e >> 2);
    tl.kind = 1; tl.owner = p.rank; tl.col0 = ((xcd & 1) * 4 + (e & 3)) * 256;
    tl.nrow0 = rt * NM; tl.gathered = rt <= RMIX; tl.koff = 0; tl.qw = e >> 2; tl.qx = e & 3;
  }
  if (wid < 4) {
    if (tl.kind == 0) mfma_a(p, tl, lds, wid, lane);
    else mfma_n(p, tl, lds, wid, lane);
  } else if (wid == 7) {
    ctrl_wave(p, tl, lds, lane);
  } else {
#define LOADERS(MF, NIMG, SW, PAIR, NWS, AUX) do { if (wid == 4) loader<MF, NIMG, SW, PAIR, 0, NWS, AUX>(p, tl, lds, lane); \
    else if (wid == 5) loader<MF, NIMG, SW, PAIR, 1, NWS, AUX>(p, tl, lds, lane); else loader<MF, NIMG, SW, PAIR, 2, NWS, AUX>(p, tl, lds, lane); } while (0)
    if (tl.kind == 0) LOADERS(MFA, 1, SWA, true, 0, AUX_NT);
    else if (rt < RMIX) LOADERS(MFN, 2, 3, false, 4 * MFN, 0);
    else if (rt == RMIX) LOADERS(MFN, 2, 3, false, (XROWS - RMIX * NM) / 8, 0);
    else LOADERS(MFN, 2, 3, false, 0, 0);
  }
}

extern "C" void kernel_launch(int rank, int nranks, ncclComm_t comm,
                              hipStream_t stream,
                              void* const* d_in, const int* in_sizes, int n_in,
                              void* d_out, int out_size,
                              const int* in_dtypes, int out_dtype,
                              void* const* d_out_all,
                              void* const* d_ws_all, size_t ws_bytes) {
  if (nranks != NR || (unsigned)rank >= (unsigned)NR || ws_bytes < WS_NEED || n_in < 2 ||
      in_sizes[0] != ML * KD || in_sizes[1] != KD * ND || out_size != NR * ML * NCOL) {
    fprintf(stderr, "unexpected configuration\n");
    return;
  }
  Params p;
  p.x = (const unsigned short*)d_in[0];
  p.w = (const unsigned short*)d_in[1];
  for (int i = 0; i < NR; ++i) { p.out[i] = (unsigned short*)d_out_all[i]; p.ws[i] = (unsigned*)d_ws_all[i]; }
  p.rank = rank;
  static bool init[NR] = {};
  if (!init[rank]) {
    HIP_CHECK(hipFuncSetAttribute((const void*)gemm_a2a, hipFuncAttributeMaxDynamicSharedMemorySize, LDS_BYTES));
    init[rank] = true;
  }
  gemm_a2a<<<NBLK, 512, LDS_BYTES, stream>>>(p);
  HIP_CHECK(hipGetLastError());
}
```

```cpp
#include <hip/hip_runtime.h>
#include <rccl/rccl.h>
#include <cstdio>
#include <stdint.h>

#define HIP_CHECK(expr) do { hipError_t _e = (expr); if (_e != hipSuccess) { \
    fprintf(stderr, "HIP error %s at %s:%d\n", hipGetErrorString(_e), __FILE__, __LINE__); \
    return; } } while (0)

#define LAS __attribute__((address_space(3)))
typedef short s16x4 __attribute__((ext_vector_type(4)));
typedef __bf16 bf16x2 __attribute__((ext_vector_type(2)));
typedef __bf16 bf16x8 __attribute__((ext_vector_type(8)));
typedef float f32x4 __attribute__((ext_vector_type(4)));
typedef float f32x2 __attribute__((ext_vector_type(2)));
typedef unsigned u32x4 __attribute__((ext_vector_type(4)));

constexpr int NR = 8, ML = 512, KD = 4096, ND = 16384, NCOL = ND / NR;
constexpr int BK = 64, NT = KD / BK;
constexpr int MFA = 7;
constexpr int MFN = 5;
constexpr int AROWS = MFA * 32, BROWS = ML - AROWS;
constexpr int BRP = BROWS / 8;
constexpr int XROWS = 7 * BROWS;
constexpr int NROWS = XROWS + ML;
constexpr int NM = MFN * 32;
constexpr int RMIX = XROWS / NM;
constexpr int WODD = 8192 + 128;
constexpr int WIMG = WODD + 8192;
constexpr int SWA = 4;
constexpr int LDS_CTL = 3 * (2 * WIMG + MFN * 4096);
static_assert(SWA * WIMG + 3 * MFA * 4096 <= LDS_CTL, "LDS");
constexpr int C_GO = LDS_CTL;
constexpr int C_CALL = LDS_CTL + 16;
constexpr int C_POLL = LDS_CTL + 64;
constexpr int LDS_BYTES = C_POLL + 256;
static_assert(LDS_BYTES <= 163840, "LDS");
constexpr int NBLK = 256;
constexpr unsigned long long DEADLINE = 50000000ull;
constexpr int WS_TMO = 32, WS_READY = 64  , WS_CNT = 512  , WS_DONE = 1024  ;
constexpr int WS_XFLAG = 2048;
constexpr size_t WS_XBUF = 1u << 20;
constexpr size_t XT_BYTES = (size_t)XROWS * 128;
constexpr size_t XPAR_BYTES = XT_BYTES * NT;
constexpr size_t WS_NEED = WS_XBUF + 2 * XPAR_BYTES;
constexpr int NUNIT = BRP * NT / 128;
constexpr int TILE_NS = 640;
constexpr int PUSH_LEAD = 1;
constexpr int ACK_NS = 800;
constexpr int POLL_MARGIN = 4;
constexpr int KSKEW = 8, KSTRIDE = 11;
constexpr int NS_T0 = 800, NPACE = 106, NPACE_END = 4300, NS_N0 = 3;
constexpr int NC_N0 = 10, NC_T0 = 1150, NC_SLOPE = 125;
constexpr int AUX_NT = 2, AUX_WT_AGENT = 16, AUX_WT_SYS = 17;

struct Params {
  const unsigned short* x;
  const unsigned short* w;
  unsigned short* out[NR];
  unsigned* ws[NR];
  int rank;
};

#define WAITL0() asm volatile("s_waitcnt lgkmcnt(0)" ::: "memory")
template <int N> __device__ __forceinline__ void waitv() { asm volatile("s_waitcnt vmcnt(%0)" :: "n"(N) : "memory"); }
#define BAR() __builtin_amdgcn_s_barrier()
#define SCHED() __builtin_amdgcn_sched_barrier(0)
#define CFENCE() asm volatile("" ::: "memory")
#define NOP_ do {} while (0)
#define STEP(a, d) asm volatile("v_add_u32 %0, %1, %0" : "+v"(a) : "s"(d))
#define NEXT1(o, d) asm volatile("s_add_i32 %0, %0, %1" : "+s"(o) : "s"(d) : "scc")
#define NEXT2(o, d, last, slot) asm volatile("s_cmp_eq_u32 %1, %2\n\ts_cselect_b32 %0, %3, %4" : "=s"(d) : "s"(o), "s"(last), "s"(-(last)), "s"(slot) : "scc")

__device__ __forceinline__ bf16x8 ldx(unsigned a) { return *(LAS const bf16x8*)(size_t)a; }
__device__ __forceinline__ s16x4 ldw(unsigned a) { return __builtin_amdgcn_ds_read_tr16_b64_v4i16((LAS s16x4*)(size_t)a); }
__device__ __forceinline__ void mma(f32x4& c, bf16x8 a, bf16x8 b) { asm volatile("v_mfma_f32_16x16x32_bf16 %0, %1, %2, %0" : "+v"(c) : "v"(a), "v"(b)); }
__device__ __forceinline__ void mma_drain() {
  __builtin_amdgcn_sched_barrier(0);
  asm volatile("s_nop 15\n\ts_nop 7" ::: "memory");
  __builtin_amdgcn_sched_barrier(0);
}
__device__ __forceinline__ unsigned pk(float a, float b) { f32x2 v = {a, b}; return __builtin_bit_cast(unsigned, __builtin_convertvector(v, bf16x2)); }
__device__ __forceinline__ unsigned ld_sys(const unsigned* p) { return __hip_atomic_load(p, __ATOMIC_RELAXED, __HIP_MEMORY_SCOPE_SYSTEM); }
__device__ __forceinline__ unsigned ld_agent(const unsigned* p) { return __hip_atomic_load(p, __ATOMIC_RELAXED, __HIP_MEMORY_SCOPE_AGENT); }
__device__ __forceinline__ void st_sys(unsigned* p, unsigned v) { __hip_atomic_store(p, v, __ATOMIC_RELAXED, __HIP_MEMORY_SCOPE_SYSTEM); }
__device__ __forceinline__ void st_agent(unsigned* p, unsigned v) { __hip_atomic_store(p, v, __ATOMIC_RELAXED, __HIP_MEMORY_SCOPE_AGENT); }
__device__ __forceinline__ unsigned lds_rd(LAS char* lds, int off) { return *(LAS volatile unsigned*)(lds + off); }
__device__ __forceinline__ void lds_wr(LAS char* lds, int off, unsigned v) { *(LAS volatile unsigned*)(lds + off) = v; }
__device__ __forceinline__ bf16x8 wfrag(s16x4 lo, s16x4 hi) { return __builtin_bit_cast(bf16x8, __builtin_shufflevector(lo, hi, 0, 1, 2, 3, 4, 5, 6, 7)); }
__device__ __forceinline__ u32x4 pack2(f32x4 a, f32x4 c, bool bad) {
  const unsigned a0 = pk(a[0], a[1]), a1 = pk(a[2], a[3]), c0 = pk(c[0], c[1]), c1 = pk(c[2], c[3]);
  auto r0 = __builtin_amdgcn_permlane16_swap(a0, c0, false, false);
  auto r1 = __builtin_amdgcn_permlane16_swap(a1, c1, false, false);
  return bad ? (u32x4){0x7fc07fc0u, 0x7fc07fc0u, 0x7fc07fc0u, 0x7fc07fc0u} : (u32x4){r0[0], r1[0], r0[1], r1[1]};
}
__device__ __forceinline__ unsigned out_voff(int lane) { const int fr = lane & 15, fq = lane >> 4; return (unsigned)((fr * NCOL + (fq & 1) * 16 + (fq >> 1) * 8) * 2); }
__host__ __device__ constexpr int wpiece(int pc) { return ((pc >> 1) & 1) * WODD + (pc >> 2) * 2048 + (pc & 1) * 1024; }

struct Tile {
  int kind;
  int owner;
  int col0;
  int nrow0;
  bool gathered;
  int koff;
  int qw, qx;
};

template <int MF, int NIMG, int SW, bool PAIR, int H, int NWS, int AUXW>
__device__ __forceinline__ void loader(const Params& p, const Tile& tl, LAS char* lds, int lane) {
  constexpr int NWP = 16 * NIMG;
  constexpr int WSLOT = NIMG * WIMG, XSLOT = MF * 4096, LX = SW * WSLOT;
  constexpr int NW_ = (NWP - H + 2) / 3;
  constexpr int X0 = (H + 3 * NW_) - NWP;
  constexpr int NX_ = (4 * MF - X0 + 2) / 3;
  constexpr bool XWS = NWS > 0;
  static_assert(2 * NW_ + 2 * NX_ <= 63, "vmcnt");
  const auto rw = __builtin_amdgcn_make_buffer_rsrc((void*)p.w, 0, KD * ND * 2, 0x00020000);
  const auto rx = __builtin_amdgcn_make_buffer_rsrc((void*)p.x, 0, ML * KD * 2, 0x00020000);
  const unsigned wv = (unsigned)((lane >> 4) * ND * 2 + (((lane & 15) ^ ((lane >> 4) << 1)) * 16));
  unsigned xv[2];
#pragma unroll
  for (int v = 0; v < 2; ++v) xv[v] = (unsigned)((lane >> 3) * KD * 2 + (((lane & 7) ^ ((v * 4 + (lane >> 4)) & 7)) * 16));
  const unsigned iv = (unsigned)(lane * 16);
  const unsigned wbase = (unsigned)((tl.owner * NCOL + tl.col0) * 2);
  const unsigned xbase = tl.kind == 0 ? 0u : (unsigned)((tl.nrow0 - XROWS) * KD * 2);
  auto xpc = [&](int pc) -> unsigned {
    const bool pad = NIMG == 2 && NWS == 0 && tl.nrow0 + pc * 8 >= NROWS;
    return (unsigned)((pad ? pc - 8 : pc) * (8 * KD * 2));
  };
  auto kmap = [&](int t) -> unsigned {
    return PAIR ? (unsigned)(((((t >> 1) * KSTRIDE) << 1) + (t & 1) + tl.koff) & (NT - 1)) : (unsigned)t;
  };
  int ws_ = 0, xs_ = 0;
  auto issue_w = [&](int t) {
    const unsigned tb = wbase + kmap(t) * (BK * ND * 2);
#pragma unroll
    for (int i = 0; i < NW_; ++i) {
      const int pc = H + 3 * i, img = pc >> 4, q = pc & 15;
      __builtin_amdgcn_raw_ptr_buffer_load_lds(rw, (LAS void*)(lds + ws_ * WSLOT + img * WIMG + wpiece(q)), 16, wv, tb + q * (4 * ND * 2) + img * 256, 0, AUXW);
    }
    ws_ = ws_ == SW - 1 ? 0 : ws_ + 1;
  };
  auto issue_xl = [&](int t) {
    const unsigned tb = xbase + kmap(t) * 128u;
#pragma unroll
    for (int i = 0; i < NX_; ++i) {
      const int pc = X0 + 3 * i;
      if (pc >= NWS) __builtin_amdgcn_raw_ptr_buffer_load_lds(rx, (LAS void*)(lds + LX + xs_ * XSLOT + pc * 1024), 16, xv[pc & 1], tb + xpc(pc), 0, 0);
    }
  };
  issue_w(0);
  issue_xl(0);
  unsigned call = 0;
  if (XWS) {
    BAR();
    call = (unsigned)__builtin_amdgcn_readfirstlane((int)lds_rd(lds, C_CALL));
  }
  const auto ri = __builtin_amdgcn_make_buffer_rsrc((char*)p.ws[p.rank] + WS_XBUF + (call & 1) * XPAR_BYTES, 0, (int)XPAR_BYTES, 0x00020000);
  const unsigned ibase = (unsigned)(tl.nrow0 * 128);
  auto issue_xi = [&](int t) {
#pragma unroll
    for (int i = 0; i < NX_; ++i) {
      const int pc = X0 + 3 * i;
      if (pc < NWS) __builtin_amdgcn_raw_ptr_buffer_load_lds(ri, (LAS void*)(lds + LX + xs_ * XSLOT + pc * 1024), 16, iv, ibase + (unsigned)t * (unsigned)XT_BYTES + pc * 1024, 0, 0);
    }
    xs_ = xs_ == 2 ? 0 : xs_ + 1;
  };
  issue_xi(0);
  issue_w(1);
  issue_xl(1);
  issue_xi(1);
  waitv<NW_ + NX_>();
  BAR();
  if (PAIR) {
    for (int t = 0; t < NT - 2; t += 2) {
      issue_w(t + 2);
      issue_w(t + 3);
      issue_xl(t + 2);
      issue_xi(t + 2);
      waitv<2 * NW_ + NX_>();
      BAR();
      issue_xl(t + 3);
      issue_xi(t + 3);
      waitv<NX_>();
      BAR();
    }
  } else if (NWS == 0 || NWS == 4 * MF) {
    constexpr int QW = (NW_ + 3) / 4, QX = (NX_ + 3) / 4;
    unsigned wl[NW_], wg[NW_], xl[NX_], xg[NX_];
#pragma unroll
    for (int i = 0; i < NW_; ++i) {
      int k = i + tl.qw * QW; k = k >= NW_ ? k - NW_ : k;
      const int pc = H + 3 * k, img = pc >> 4, q = pc & 15;
      wl[i] = (unsigned)(img * WIMG + wpiece(q));
      wg[i] = wv + (unsigned)(q * (4 * ND * 2) + img * 256);
    }
#pragma unroll
    for (int i = 0; i < NX_; ++i) {
      int k = i + tl.qx * QX; k = k >= NX_ ? k - NX_ : k;
      const int pc = X0 + 3 * k;
      xl[i] = (unsigned)(pc * 1024);
      xg[i] = XWS ? iv + (unsigned)(pc * 1024) : ((pc & 1) ? xv[1] : xv[0]) + xpc(pc);
    }
    for (int t = 0; t < NT - 2; ++t) {
      const unsigned tw = wbase + (unsigned)(t + 2) * (BK * ND * 2);
      const unsigned tx = XWS ? ibase + (unsigned)(t + 2) * (unsigned)XT_BYTES : xbase + (unsigned)(t + 2) * 128u;
      LAS char* lw = lds + ws_ * WSLOT;
      LAS char* lx = lds + LX + xs_ * XSLOT;
#pragma unroll
      for (int ph = 0; ph < 2; ++ph) {
#pragma unroll
        for (int i = ph ? QW : 0; i < (ph ? NW_ : QW); ++i)
          __builtin_amdgcn_raw_ptr_buffer_load_lds(rw, (LAS void*)(lw + wl[i]), 16, wg[i], tw, 0, AUXW);
#pragma unroll
        for (int i = ph ? QX : 0; i < (ph ? NX_ : QX); ++i)
          __builtin_amdgcn_raw_ptr_buffer_load_lds(XWS ? ri : rx, (LAS void*)(lx + xl[i]), 16, xg[i], tx, 0, 0);
      }
      ws_ = ws_ == SW - 1 ? 0 : ws_ + 1;
      xs_ = xs_ == 2 ? 0 : xs_ + 1;
      waitv<NW_ + NX_>();
      BAR();
    }
  } else {
    for (int t = 0; t < NT - 2; ++t) {
      issue_w(t + 2);
      issue_xl(t + 2);
      issue_xi(t + 2);
      waitv<NW_ + NX_>();
      BAR();
    }
  }
  waitv<0>();
  BAR();
  BAR();
}

__device__ __forceinline__ void mfma_a(const Params& p, const Tile& tl, LAS char* lds, int wid, int lane) {
  constexpr int MF = MFA;
  static_assert(MF == 7, "the schedule below is written for 7 fragments");
  constexpr int XSLOT = MF * 4096, LX = SWA * WIMG;
  const int wr = wid >> 1, wc = wid & 1;
  const int fr = lane & 15, fq = lane >> 4;
  const int q = (lane & 15) >> 2, pp = lane & 3;
  const unsigned lb = (unsigned)(size_t)lds;
  const unsigned xo = (unsigned)(LX + (wr * MF * 16 + fr) * 128 + ((fq ^ ((fr >> 1) & 7)) * 16));
  unsigned xa0 = lb + xo, xa1 = lb + (xo ^ 64u);
  unsigned wa[4];
#pragma unroll
  for (int nf = 0; nf < 4; ++nf)
    wa[nf] = lb + (unsigned)((fq & 1) * WODD + (fq >> 1) * 2048 + q * 256 + wc * 128 + ((nf ^ q) * 32) + 8 * pp);
  f32x4 acc[MF][4];
#pragma unroll
  for (int i = 0; i < MF; ++i)
#pragma unroll
    for (int j = 0; j < 4; ++j) { acc[i][j] = (f32x4){0.f, 0.f, 0.f, 0.f}; asm volatile("" : "+v"(acc[i][j])); }
  bf16x8 Xf[MF];
  s16x4 Wl[2][4], Wh[2][4];
#define RDW(bu, ks, i) do { if ((i) & 1) Wh[bu][(i) >> 1] = ldw(wa[(i) >> 1] + (ks) * 4096 + 1024); \
    else Wl[bu][(i) >> 1] = ldw(wa[(i) >> 1] + (ks) * 4096); } while (0)
#define RDX(i, xa) Xf[i] = ldx((xa) + (i) * 2048)
#define MMA1(mf, bu, nf) do { mma(acc[mf][nf], wfrag(Wl[bu][nf], Wh[bu][nf]), Xf[mf]); SCHED(); } while (0)
#define SN(mf, i) (((mf) & 1) ? 3 - (i) : (i))
#define GRP(mf, bu, F0, F1, F2) do { MMA1(mf, bu, SN(mf, 0)); F0; SCHED(); MMA1(mf, bu, SN(mf, 1)); F1; SCHED(); \
    MMA1(mf, bu, SN(mf, 2)); F2; SCHED(); MMA1(mf, bu, SN(mf, 3)); } while (0)
  BAR();
#pragma unroll
  for (int i = 0; i < 8; ++i) RDW(0, 0, i);
#pragma unroll
  for (int mf = 0; mf < MF - 1; ++mf) RDX(mf, xa0);
  __builtin_amdgcn_s_waitcnt(0xC07F);
  int ow = 0, ox = 0, dw = WIMG, dx = XSLOT;
  for (int t = 0; t < NT; ++t) {
    GRP(0, 0, RDX(6, xa0), RDW(1, 1, 0), RDW(1, 1, 1));
    GRP(1, 0, RDX(0, xa1), RDW(1, 1, 2), RDW(1, 1, 3));
    GRP(2, 0, RDX(1, xa1), RDW(1, 1, 4), STEP(xa0, dx));
    GRP(3, 0, RDX(2, xa1), RDW(1, 1, 5), STEP(wa[0], dw));
    GRP(4, 0, RDX(3, xa1), RDW(1, 1, 6), STEP(wa[1], dw));
    GRP(5, 0, RDX(4, xa1), RDW(1, 1, 7), STEP(wa[2], dw));
    GRP(6, 0, RDX(5, xa1), NOP_, STEP(wa[3], dw));
    GRP(0, 1, RDX(6, xa1), NEXT1(ow, dw), NEXT2(ow, dw, (SWA - 1) * WIMG, WIMG));
    GRP(1, 1, STEP(xa1, dx), NEXT1(ox, dx), NEXT2(ox, dx, 2 * XSLOT, XSLOT));
    __builtin_amdgcn_s_waitcnt(0xC07F);
    CFENCE();
    BAR();
    CFENCE();
    SCHED();
    GRP(2, 1, RDW(0, 0, 0), RDW(0, 0, 1), RDX(0, xa0));
    GRP(3, 1, RDW(0, 0, 2), RDW(0, 0, 3), RDX(1, xa0));
    GRP(4, 1, RDW(0, 0, 4), RDW(0, 0, 5), RDX(2, xa0));
    GRP(5, 1, RDW(0, 0, 6), RDW(0, 0, 7), RDX(3, xa0));
    GRP(6, 1, RDX(4, xa0), RDX(5, xa0), NOP_);
  }
  mma_drain();
  const unsigned go = (unsigned)__builtin_amdgcn_readfirstlane((int)lds_rd(lds, C_GO));
  const bool bad = (go & 1u) != 0u;
  const auto ro = __builtin_amdgcn_make_buffer_rsrc((void*)p.out[tl.owner], 0, ML * NR * NCOL * 2, 0x00020000);
  const unsigned vo = out_voff(lane);
  const unsigned so = (unsigned)(((p.rank * ML + wr * MF * 16) * NCOL + tl.col0 + wc * 64) * 2);
#pragma unroll
  for (int mf = 0; mf < MF; ++mf) {
#pragma unroll
    for (int np = 0; np < 2; ++np)
      __builtin_amdgcn_raw_buffer_store_b128(pack2(acc[mf][2 * np], acc[mf][2 * np + 1], bad), ro, vo + np * 64, so + mf * (16 * NCOL * 2), AUX_WT_SYS);
    waitv<0>();
  }
  if (lane == 0) st_sys(p.ws[tl.owner] + WS_DONE + p.rank * 64 + (tl.col0 >> 7) * 4 + wid, go >> 1);
#undef RDW
#undef RDX
#undef MMA1
#undef GRP
}

__device__ __forceinline__ void mfma_n(const Params& p, const Tile& tl, LAS char* lds, int wid, int lane) {
  constexpr int MF = MFN;
  static_assert(MF == 5, "the schedule below is written for 5 fragments");
  constexpr int WSLOT = 2 * WIMG, XSLOT = MF * 4096, LX = 3 * WSLOT;
  const int wr = wid >> 1, wc = wid & 1;
  const int fr = lane & 15, fq = lane >> 4;
  const int q = (lane & 15) >> 2, pp = lane & 3;
  const unsigned lb = (unsigned)(size_t)lds;
  const unsigned xo = (unsigned)(LX + (wr * MF * 16 + fr) * 128 + ((fq ^ ((fr >> 1) & 7)) * 16));
  unsigned xa0 = lb + xo, xa1 = lb + (xo ^ 64u);
  unsigned wa[4];
#pragma unroll
  for (int nf = 0; nf < 4; ++nf)
    wa[nf] = lb + (unsigned)(wc * WIMG + (fq & 1) * WODD + (fq >> 1) * 2048 + q * 256 + ((nf ^ q) * 32) + 8 * pp);
  f32x4 acc[MF][8];
#pragma unroll
  for (int i = 0; i < MF; ++i)
#pragma unroll
    for (int j = 0; j < 8; ++j) { acc[i][j] = (f32x4){0.f, 0.f, 0.f, 0.f}; asm volatile("" : "+v"(acc[i][j])); }
  bf16x8 Xf[2][MF];
  s16x4 Wl[8], Wh[8];
#define RWL(nf, ks) Wl[nf] = ldw(wa[(nf) & 3] + ((nf) >> 2) * 128 + (ks) * 4096)
#define RWH(nf, ks) Wh[nf] = ldw(wa[(nf) & 3] + ((nf) >> 2) * 128 + (ks) * 4096 + 1024)
#define RDX(bu, i, xa) Xf[bu][i] = ldx((xa) + (i) * 2048)
#define MMA1(nf, bu, mf) do { mma(acc[mf][nf], wfrag(Wl[nf], Wh[nf]), Xf[bu][mf]); SCHED(); } while (0)
#define SN5(nf, i) (((nf) & 1) ? 4 - (i) : (i))
#define GRP(nf, bu, F0, F1, F2, F3, F4) do { MMA1(nf, bu, SN5(nf, 0)); F0; SCHED(); MMA1(nf, bu, SN5(nf, 1)); F1; SCHED(); \
    MMA1(nf, bu, SN5(nf, 2)); F2; SCHED(); MMA1(nf, bu, SN5(nf, 3)); F3; SCHED(); MMA1(nf, bu, SN5(nf, 4)); F4; SCHED(); } while (0)
  if (tl.gathered) {
    const unsigned gwv = (unsigned)(((tl.nrow0 / NM) * 8 + (tl.col0 >> 8)) * 4 + wid);
    LAS void* dummy = (LAS void*)(lds + 2 * WSLOT + wid * 1024);
    const auto rx = __builtin_amdgcn_make_buffer_rsrc((void*)p.x, 0, ML * KD * 2, 0x00020000);
    const auto rw = __builtin_amdgcn_make_buffer_rsrc((void*)p.w, 0, KD * ND * 2, 0x00020000);
#pragma unroll
    for (int i = 0; i < 10; ++i)
      __builtin_amdgcn_raw_ptr_buffer_load_lds(rx, dummy, 16, (unsigned)(lane * 16), (gwv * 10240u + i * 1024u) & (unsigned)(ML * KD * 2 - 1), 0, 0);
    for (unsigned i0 = gwv; i0 < KD * 4u; i0 += 416u * 8u) {
#pragma unroll
      for (int i = 0; i < 8; ++i) {
        const unsigned pi = (i0 + i * 416u) & (KD * 4u - 1u);
        __builtin_amdgcn_raw_ptr_buffer_load_lds(rw, dummy, 16, (unsigned)(lane * 16), (pi >> 2) * (ND * 2) + (unsigned)p.rank * (NCOL * 2) + (pi & 3u) * 1024u, 0, 0);
      }
      waitv<8>();
    }
    waitv<0>();
    BAR();
  }
  BAR();
#pragma unroll
  for (int mf = 0; mf < MF; ++mf) RDX(0, mf, xa0);
#pragma unroll
  for (int nf = 0; nf < 7; ++nf) { RWL(nf, 0); RWH(nf, 0); }
  __builtin_amdgcn_s_waitcnt(0xC07F);
  int ow = 0, ox = 0, dw = WSLOT, dx = XSLOT;
  for (int t = 0; t < NT; ++t) {
    GRP(0, 0, RWL(7, 0), RWH(7, 0), RDX(1, 0, xa1), STEP(xa0, dx), NOP_);
    GRP(1, 0, RWL(0, 1), RWH(0, 1), RDX(1, 1, xa1), NOP_, NOP_);
    GRP(2, 0, RWL(1, 1), RWH(1, 1), RDX(1, 2, xa1), NOP_, NOP_);
    GRP(3, 0, RWL(2, 1), RWH(2, 1), RDX(1, 3, xa1), NOP_, NOP_);
    GRP(4, 0, RWL(3, 1), RWH(3, 1), RDX(1, 4, xa1), NOP_, NOP_);
    GRP(5, 0, RWL(4, 1), RWH(4, 1), NOP_, STEP(wa[0], dw), STEP(xa1, dx));
    GRP(6, 0, RWL(5, 1), RWH(5, 1), NOP_, STEP(wa[1], dw), NOP_);
    GRP(7, 0, RWL(6, 1), RWH(6, 1), NOP_, STEP(wa[2], dw), NOP_);
    GRP(0, 1, RWL(7, 1), RWH(7, 1), NOP_, STEP(wa[3], dw), NOP_);
    GRP(1, 1, NEXT1(ow, dw), NEXT2(ow, dw, 2 * WSLOT, WSLOT), NEXT1(ox, dx), NEXT2(ox, dx, 2 * XSLOT, XSLOT), NOP_);
    __builtin_amdgcn_s_waitcnt(0xC07F);
    CFENCE();
    BAR();
    CFENCE();
    SCHED();
    GRP(2, 1, RWL(0, 0), RWH(0, 0), RDX(0, 0, xa0), RWL(1, 0), RWH(1, 0));
    GRP(3, 1, RWL(2, 0), RWH(2, 0), RDX(0, 1, xa0), RDX(0, 2, xa0), NOP_);
    GRP(4, 1, RWL(3, 0), RWH(3, 0), RDX(0, 3, xa0), RDX(0, 4, xa0), NOP_);
    GRP(5, 1, RWL(4, 0), RWH(4, 0), NOP_, NOP_, NOP_);
    GRP(6, 1, RWL(5, 0), RWH(5, 0), NOP_, NOP_, NOP_);
    GRP(7, 1, RWL(6, 0), RWH(6, 0), NOP_, NOP_, NOP_);
  }
  mma_drain();
  const unsigned go = (unsigned)__builtin_amdgcn_readfirstlane((int)lds_rd(lds, C_GO));
  const bool bad = (go & 1u) != 0u;
  const auto ro = __builtin_amdgcn_make_buffer_rsrc((void*)p.out[p.rank], 0, ML * NR * NCOL * 2, 0x00020000);
  const unsigned vo = out_voff(lane);
  const unsigned so = (unsigned)((tl.col0 + wc * 128) * 2);
#pragma unroll
  for (int mf = 0; mf < MF; ++mf) {
    const int nr = tl.nrow0 + wr * MF * 16 + mf * 16;
    if (nr < NROWS) {
      const int orow = nr < XROWS ? ((p.rank + 1 + nr / BROWS) & 7) * ML + AROWS + nr % BROWS : p.rank * ML + nr - XROWS;
#pragma unroll
      for (int np = 0; np < 4; ++np)
        __builtin_amdgcn_raw_buffer_store_b128(pack2(acc[mf][2 * np], acc[mf][2 * np + 1], bad), ro, vo + np * 64, so + (unsigned)orow * (NCOL * 2), AUX_WT_AGENT);
    }
  }
#undef RWL
#undef RWH
#undef RDX
#undef MMA1
#undef GRP
}

__device__ __forceinline__ bool wait_words(const unsigned* word, unsigned call, unsigned* tmo) {
  const unsigned long long t0 = __builtin_amdgcn_s_memrealtime();
  unsigned spins = 0;
  for (;;) {
    if (__all(ld_sys(word) >= call)) return true;
    if ((++spins & 63u) == 0u && (ld_agent(tmo) == call || __builtin_amdgcn_s_memrealtime() - t0 > DEADLINE)) { st_agent(tmo, call); return false; }
    __builtin_amdgcn_s_sleep(1);
  }
}

__device__ __forceinline__ void ctrl_wave(const Params& p, const Tile& tl, LAS char* lds, int lane) {
  unsigned* myws = p.ws[p.rank];
  unsigned* tmo = myws + WS_TMO;
  const int b = blockIdx.x;
  const unsigned long long tk0 = __builtin_amdgcn_s_memrealtime();
  const unsigned cnt = ld_agent(myws + WS_CNT + b);
  if (!tl.gathered) BAR();
  const unsigned call = (unsigned)__builtin_amdgcn_readfirstlane((int)cnt) + 1u;
  if (lane == 0) st_agent(myws + WS_CNT + b, call);
  bool ok = true;
  int nb = 1;
  auto join = [&]() {
    if (tl.kind == 1) {
      int due = NS_T0 + nb * NPACE;
      if (!tl.gathered) { const int d2 = nb > NC_N0 ? NC_T0 + (nb - NC_N0) * NC_SLOPE : 0; due = d2 < due ? d2 : due; }
      if (nb < NS_N0) due = 0;
      due = due < NPACE_END ? due : NPACE_END;
      while ((long long)(__builtin_amdgcn_s_memrealtime() - tk0) < (long long)due) __builtin_amdgcn_s_sleep(1);
    }
    BAR();
    ++nb;
  };
  if (tl.gathered) {
    if (lane == 0) lds_wr(lds, C_CALL, call);
    const int s0 = tl.nrow0 / BROWS;
    const int e1 = (tl.nrow0 + NM - 1) / BROWS;
    const int s1 = e1 < 7 ? e1 : 6;
    const unsigned* sflag = myws + WS_XFLAG + ((lane & 16) ? s1 : s0) * 32 + (lane & 15);
    auto decode = [&](unsigned v) -> int {
#pragma unroll
      for (int o = 1; o < 32; o <<= 1) { const unsigned u = (unsigned)__shfl_xor((int)v, o); v = u < v ? u : v; }
      const unsigned vm = (unsigned)__builtin_amdgcn_readfirstlane((int)v);
      const int n = vm > call * 128u ? (int)(vm - call * 128u) : 0;
      return n > NT ? NT : n;
    };
    __builtin_amdgcn_fence(__ATOMIC_ACQUIRE, "");
    int seen = 0;
    nb = -1;
    while (seen < NT) {
      const int need = nb + 3 < NT ? nb + 3 : NT;
      if (seen < need) {
        const unsigned long long t0 = __builtin_amdgcn_s_memrealtime();
        unsigned spins = 0;
        while ((seen = decode(ld_sys(sflag))) < need) {
          if ((++spins & 63u) == 0u && (ld_agent(tmo) == call || __builtin_amdgcn_s_memrealtime() - t0 > DEADLINE)) { st_agent(tmo, call); ok = false; seen = NT; break; }
          __builtin_amdgcn_s_sleep(1);
        }
        waitv<0>();
        WAITL0();
        join();
      } else if (seen < need + POLL_MARGIN) {
        __builtin_amdgcn_global_load_lds((const __attribute__((address_space(1))) unsigned*)sflag, (LAS unsigned*)(lds + C_POLL), 4, 0, AUX_WT_SYS);
        join();
        waitv<0>();
        const int n = decode(lds_rd(lds, C_POLL + lane * 4));
        seen = n > seen ? n : seen;
      } else {
        join();
      }
    }
  } else if (tl.kind == 0) {
    while (nb < NT / 2) join();
    ok = wait_words(myws + WS_READY + tl.owner * 32, call, tmo);
  }
  while (nb < NT - 1) join();
  if (ld_agent(tmo) == call) ok = false;
  if (lane == 0) lds_wr(lds, C_GO, call * 2u + (ok ? 0u : 1u));
  WAITL0();
  while (nb <= NT) join();
}

__device__ __forceinline__ void copy_group(const Params& p, LAS char* lds, int c, int wid, int lane) {
  unsigned* myws = p.ws[p.rank];
  unsigned* tmo = myws + WS_TMO;
  const int b = blockIdx.x;
  const unsigned cnt = ld_agent(myws + WS_CNT + b);
  const int gw = wid * 16 + c;
  const auto rx = __builtin_amdgcn_make_buffer_rsrc((void*)p.x, 0, ML * KD * 2, 0x00020000);
  const unsigned xv = (unsigned)((AROWS + (lane >> 3)) * KD * 2 + (((lane & 7) ^ (((gw & 1) * 4 + (lane >> 4)) & 7)) * 16));
  auto load = [&](int k) { const int u = k * 128 + gw; return __builtin_amdgcn_raw_buffer_load_b128(rx, xv, (unsigned)((u % BRP) * (8 * KD * 2) + (u / BRP) * 128), 0); };
  u32x4 d[NUNIT];
  d[0] = load(0);
  d[1] = load(1);
  char* dbase[NR - 1];
  unsigned dslot[NR - 1];
#pragma unroll
  for (int i = 0; i < NR - 1; ++i) {
    const int j = __builtin_amdgcn_readfirstlane((i + gw) % (NR - 1) + 1);
    dbase[i] = (char*)p.ws[__builtin_amdgcn_readfirstlane((p.rank + NR - j) & 7)];
    dslot[i] = (unsigned)((j - 1) * BROWS * 128);
  }
  if (lane == 0) { lds_wr(lds, wid * 4, (unsigned)(gw / BRP)); if (wid == 0) lds_wr(lds, 64, 0u); }
  WAITL0();
  const unsigned call = (unsigned)__builtin_amdgcn_readfirstlane((int)cnt) + 1u;
  BAR();
  const unsigned long long tstart = __builtin_amdgcn_s_memrealtime();
  if (wid == 0 && lane == 0) st_agent(myws + WS_CNT + b, call);
  if (gw == 127 && lane < NR && lane != p.rank) st_sys(p.ws[lane] + WS_READY + p.rank * 32, call);
  const size_t xo = WS_XBUF + (call & 1) * XPAR_BYTES;
  unsigned* pflag = p.ws[(p.rank + NR - 1 - (lane < 7 ? lane : 0)) & 7] + WS_XFLAG + (lane < 7 ? lane : 0) * 32 + c;
  auto finish = [&](int k) {
    const int nxt = k + 1 < NUNIT ? ((k + 1) * 128 + gw) / BRP : NT;
    if (lane == 0) lds_wr(lds, wid * 4, (unsigned)nxt);
    unsigned v = lds_rd(lds, (lane & 7) * 4);
#pragma unroll
    for (int o = 1; o < 8; o <<= 1) { const unsigned u = (unsigned)__shfl_xor((int)v, o); v = u < v ? u : v; }
    const int pre = __builtin_amdgcn_readfirstlane((int)v);
    const int lvl = pre < 4 ? pre : (pre < 12 ? (pre & ~1) : (pre & ~3));
    unsigned old = 0;
    if (lane == 0) old = __hip_atomic_fetch_max((LAS unsigned*)(lds + 64), (unsigned)lvl, __ATOMIC_RELAXED, __HIP_MEMORY_SCOPE_WORKGROUP);
    old = (unsigned)__builtin_amdgcn_readfirstlane((int)old);
    if ((unsigned)lvl > old && lane < 7)
      __hip_atomic_fetch_max(pflag, call * 128u + (unsigned)lvl, __ATOMIC_RELAXED, __HIP_MEMORY_SCOPE_SYSTEM);
  };
  bool pending = false;
  long long tissue = 0;
#pragma unroll
  for (int k = 0; k < NUNIT; ++k) {
    const int u = k * 128 + gw;
    for (;;) {
      const long long now = (long long)(__builtin_amdgcn_s_memrealtime() - tstart) * 10;
      if (now * BRP >= (long long)(u - PUSH_LEAD * BRP) * TILE_NS) break;
      if (pending && now - tissue > ACK_NS) { waitv<0>(); finish(k - 1); pending = false; }
      else __builtin_amdgcn_s_sleep(2);
    }
    const unsigned so = (unsigned)(u / BRP) * (unsigned)XT_BYTES + (unsigned)(u % BRP) * 1024u;
#pragma unroll
    for (int i = 0; i < NR - 1; ++i) {
      const auto rd = __builtin_amdgcn_make_buffer_rsrc(dbase[i] + xo, 0, (int)XPAR_BYTES, 0x00020000);
      __builtin_amdgcn_raw_buffer_store_b128(d[k], rd, (unsigned)(lane * 16), so + dslot[i], AUX_WT_SYS);
    }
    tissue = (long long)(__builtin_amdgcn_s_memrealtime() - tstart) * 10;
    if (pending) { waitv<NR - 1>(); finish(k - 1); }
    pending = true;
    if (k + 2 < NUNIT) d[k + 2] = load(k + 2);
  }
  waitv<0>();
  finish(NUNIT - 1);
  if (c == 15 && wid != p.rank) {
    const bool fine = wait_words(myws + WS_DONE + wid * 64 + lane, call, tmo);
    if (!fine && lane == 0) p.out[p.rank][(size_t)(p.rank * ML) * NCOL] = 0x7fc0;
  }
}

__global__ __launch_bounds__(512, 2) void gemm_a2a(Params p) {
  extern __shared__ __attribute__((aligned(16))) char smem[];
  LAS char* lds = (LAS char*)smem;
  const int wid = __builtin_amdgcn_readfirstlane(threadIdx.x >> 6);
  const int lane = threadIdx.x & 63;
  const int b = blockIdx.x;
  const int xcd = b & 7, idx = b >> 3;
  if (idx < 2) { copy_group(p, lds, xcd * 2 + idx, wid, lane); return; }
  Tile tl;
  int rt = 0;
  if (idx < 16) {
    const int a = xcd * 14 + idx - 2;
    tl.kind = 0; tl.owner = (p.rank + a % 7 + 1) & 7; tl.col0 = (a / 7) * 128;
    tl.nrow0 = 0; tl.gathered = false; tl.koff = KSKEW * xcd; tl.qw = tl.qx = 0;
  } else {
    const int e = idx - 16;
    rt = (xcd >> 1) * 4 + (e >> 2);
    tl.kind = 1; tl.owner = p.rank; tl.col0 = ((xcd & 1) * 4 + (e & 3)) * 256;
    tl.nrow0 = rt * NM; tl.gathered = rt <= RMIX; tl.koff = 0; tl.qw = e >> 2; tl.qx = e & 3;
  }
  if (wid < 4) {
    if (tl.kind == 0) mfma_a(p, tl, lds, wid, lane);
    else mfma_n(p, tl, lds, wid, lane);
  } else if (wid == 7) {
    ctrl_wave(p, tl, lds, lane);
  } else {
#define LOADERS(MF, NIMG, SW, PAIR, NWS, AUX) do { if (wid == 4) loader<MF, NIMG, SW, PAIR, 0, NWS, AUX>(p, tl, lds, lane); \
    else if (wid == 5) loader<MF, NIMG, SW, PAIR, 1, NWS, AUX>(p, tl, lds, lane); else loader<MF, NIMG, SW, PAIR, 2, NWS, AUX>(p, tl, lds, lane); } while (0)
    if (tl.kind == 0) LOADERS(MFA, 1, SWA, true, 0, AUX_NT);
    else if (rt < RMIX) LOADERS(MFN, 2, 3, false, 4 * MFN, 0);
    else if (rt == RMIX) LOADERS(MFN, 2, 3, false, (XROWS - RMIX * NM) / 8, 0);
    else LOADERS(MFN, 2, 3, false, 0, 0);
  }
}

extern "C" void kernel_launch(int rank, int nranks, ncclComm_t comm,
                              hipStream_t stream,
                              void* const* d_in, const int* in_sizes, int n_in,
                              void* d_out, int out_size,
                              const int* in_dtypes, int out_dtype,
                              void* const* d_out_all,
                              void* const* d_ws_all, size_t ws_bytes) {
  if (nranks != NR || (unsigned)rank >= (unsigned)NR || ws_bytes < WS_NEED || n_in < 2 ||
      in_sizes[0] != ML * KD || in_sizes[1] != KD * ND || out_size != NR * ML * NCOL) {
    fprintf(stderr, "unexpected configuration\n");
    return;
  }
  Params p;
  p.x = (const unsigned short*)d_in[0];
  p.w = (const unsigned short*)d_in[1];
  for (int i = 0; i < NR; ++i) { p.out[i] = (unsigned short*)d_out_all[i]; p.ws[i] = (unsigned*)d_ws_all[i]; }
  p.rank = rank;
  static bool init[NR] = {};
  if (!init[rank]) {
    HIP_CHECK(hipFuncSetAttribute((const void*)gemm_a2a, hipFuncAttributeMaxDynamicSharedMemorySize, LDS_BYTES));
    init[rank] = true;
  }
  gemm_a2a<<<NBLK, 512, LDS_BYTES, stream>>>(p);
  HIP_CHECK(hipGetLastError());
}
```

```cpp
#include <hip/hip_runtime.h>
#include <rccl/rccl.h>
#include <cstdio>
#include <stdint.h>

#define HIP_CHECK(expr) do { hipError_t _e = (expr); if (_e != hipSuccess) { \
    fprintf(stderr, "HIP error %s at %s:%d\n", hipGetErrorString(_e), __FILE__, __LINE__); \
    return; } } while (0)

#define LAS __attribute__((address_space(3)))
typedef short s16x4 __attribute__((ext_vector_type(4)));
typedef __bf16 bf16x2 __attribute__((ext_vector_type(2)));
typedef __bf16 bf16x8 __attribute__((ext_vector_type(8)));
typedef float f32x4 __attribute__((ext_vector_type(4)));
typedef float f32x2 __attribute__((ext_vector_type(2)));
typedef unsigned u32x4 __attribute__((ext_vector_type(4)));

constexpr int NR = 8, ML = 512, KD = 4096, ND = 16384, NCOL = ND / NR;
constexpr int BK = 64, NT = KD / BK;
constexpr int MFA = 7;
constexpr int MFN = 5;
constexpr int AROWS = MFA * 32, BROWS = ML - AROWS;
constexpr int BRP = BROWS / 8;
constexpr int XROWS = 7 * BROWS;
constexpr int NROWS = XROWS + ML;
constexpr int NM = MFN * 32;
constexpr int RMIX = XROWS / NM;
constexpr int WODD = 8192 + 128;
constexpr int WIMG = WODD + 8192;
constexpr int SWA = 4;
constexpr int LDS_CTL = 3 * (2 * WIMG + MFN * 4096);
static_assert(SWA * WIMG + 3 * MFA * 4096 <= LDS_CTL, "LDS");
constexpr int C_GO = LDS_CTL;
constexpr int C_CALL = LDS_CTL + 16;
constexpr int C_POLL = LDS_CTL + 64;
constexpr int LDS_BYTES = C_POLL + 256;
static_assert(LDS_BYTES <= 163840, "LDS");
constexpr int NBLK = 256;
constexpr unsigned long long DEADLINE = 50000000ull;
constexpr int WS_TMO = 32, WS_READY = 64  , WS_CNT = 512  , WS_DONE = 1024  ;
constexpr int WS_XFLAG = 2048;
constexpr size_t WS_XBUF = 1u << 20;
constexpr size_t XT_BYTES = (size_t)XROWS * 128;
constexpr size_t XPAR_BYTES = XT_BYTES * NT;
constexpr size_t WS_NEED = WS_XBUF + 2 * XPAR_BYTES;
constexpr int NUNIT = BRP * NT / 128;
constexpr int TILE_NS = 640;
constexpr int PUSH_LEAD = 1;
constexpr int ACK_NS = 800;
constexpr int POLL_MARGIN = 4;
constexpr int KSKEW = 8, KSTRIDE = 11;
constexpr int NS_T0 = 800, NPACE = 106, NPACE_END = 4300, NS_N0 = 3;
constexpr int NC_N0 = 10, NC_T0 = 1150, NC_SLOPE = 125;
constexpr int AUX_NT = 2, AUX_WT_AGENT = 16, AUX_WT_SYS = 17;

struct Params {
  const unsigned short* x;
  const unsigned short* w;
  unsigned short* out[NR];
  unsigned* ws[NR];
  int rank;
};

#define WAITL0() asm volatile("s_waitcnt lgkmcnt(0)" ::: "memory")
template <int N> __device__ __forceinline__ void waitv() { asm volatile("s_waitcnt vmcnt(%0)" :: "n"(N) : "memory"); }
#define BAR() __builtin_amdgcn_s_barrier()
#define SCHED() __builtin_amdgcn_sched_barrier(0)
#define CFENCE() asm volatile("" ::: "memory")
#define NOP_ do {} while (0)
#define STEP(a, d) asm volatile("v_add_u32 %0, %1, %0" : "+v"(a) : "s"(d))
#define NEXT1(o, d) asm volatile("s_add_i32 %0, %0, %1" : "+s"(o) : "s"(d) : "scc")
#define NEXT2(o, d, last, slot) asm volatile("s_cmp_eq_u32 %1, %2\n\ts_cselect_b32 %0, %3, %4" : "=s"(d) : "s"(o), "s"(last), "s"(-(last)), "s"(slot) : "scc")

__device__ __forceinline__ bf16x8 ldx(unsigned a) { return *(LAS const bf16x8*)(size_t)a; }
__device__ __forceinline__ s16x4 ldw(unsigned a) { return __builtin_amdgcn_ds_read_tr16_b64_v4i16((LAS s16x4*)(size_t)a); }
__device__ __forceinline__ void mma(f32x4& c, bf16x8 a, bf16x8 b) { c = __builtin_amdgcn_mfma_f32_16x16x32_bf16(a, b, c, 0, 0, 0); }
__device__ __forceinline__ unsigned pk(float a, float b) { f32x2 v = {a, b}; return __builtin_bit_cast(unsigned, __builtin_convertvector(v, bf16x2)); }
__device__ __forceinline__ unsigned ld_sys(const unsigned* p) { return __hip_atomic_load(p, __ATOMIC_RELAXED, __HIP_MEMORY_SCOPE_SYSTEM); }
__device__ __forceinline__ unsigned ld_agent(const unsigned* p) { return __hip_atomic_load(p, __ATOMIC_RELAXED, __HIP_MEMORY_SCOPE_AGENT); }
__device__ __forceinline__ void st_sys(unsigned* p, unsigned v) { __hip_atomic_store(p, v, __ATOMIC_RELAXED, __HIP_MEMORY_SCOPE_SYSTEM); }
__device__ __forceinline__ void st_agent(unsigned* p, unsigned v) { __hip_atomic_store(p, v, __ATOMIC_RELAXED, __HIP_MEMORY_SCOPE_AGENT); }
__device__ __forceinline__ unsigned lds_rd(LAS char* lds, int off) { return *(LAS volatile unsigned*)(lds + off); }
__device__ __forceinline__ void lds_wr(LAS char* lds, int off, unsigned v) { *(LAS volatile unsigned*)(lds + off) = v; }
__device__ __forceinline__ bf16x8 wfrag(s16x4 lo, s16x4 hi) { return __builtin_bit_cast(bf16x8, __builtin_shufflevector(lo, hi, 0, 1, 2, 3, 4, 5, 6, 7)); }
__device__ __forceinline__ u32x4 pack2(f32x4 a, f32x4 c, bool bad) {
  const unsigned a0 = pk(a[0], a[1]), a1 = pk(a[2], a[3]), c0 = pk(c[0], c[1]), c1 = pk(c[2], c[3]);
  auto r0 = __builtin_amdgcn_permlane16_swap(a0, c0, false, false);
  auto r1 = __builtin_amdgcn_permlane16_swap(a1, c1, false, false);
  return bad ? (u32x4){0x7fc07fc0u, 0x7fc07fc0u, 0x7fc07fc0u, 0x7fc07fc0u} : (u32x4){r0[0], r1[0], r0[1], r1[1]};
}
__device__ __forceinline__ unsigned out_voff(int lane) { const int fr = lane & 15, fq = lane >> 4; return (unsigned)((fr * NCOL + (fq & 1) * 16 + (fq >> 1) * 8) * 2); }
__host__ __device__ constexpr int wpiece(int pc) { return ((pc >> 1) & 1) * WODD + (pc >> 2) * 2048 + (pc & 1) * 1024; }

struct Tile {
  int kind;
  int owner;
  int col0;
  int nrow0;
  bool gathered;
  int koff;
  int qw, qx;
};

template <int MF, int NIMG, int SW, bool PAIR, int H, int NWS, int AUXW>
__device__ __forceinline__ void loader(const Params& p, const Tile& tl, LAS char* lds, int lane) {
  constexpr int NWP = 16 * NIMG;
  constexpr int WSLOT = NIMG * WIMG, XSLOT = MF * 4096, LX = SW * WSLOT;
  constexpr int NW_ = (NWP - H + 2) / 3;
  constexpr int X0 = (H + 3 * NW_) - NWP;
  constexpr int NX_ = (4 * MF - X0 + 2) / 3;
  constexpr bool XWS = NWS > 0;
  static_assert(2 * NW_ + 2 * NX_ <= 63, "vmcnt");
  const auto rw = __builtin_amdgcn_make_buffer_rsrc((void*)p.w, 0, KD * ND * 2, 0x00020000);
  const auto rx = __builtin_amdgcn_make_buffer_rsrc((void*)p.x, 0, ML * KD * 2, 0x00020000);
  const unsigned wv = (unsigned)((lane >> 4) * ND * 2 + (((lane & 15) ^ ((lane >> 4) << 1)) * 16));
  unsigned xv[2];
#pragma unroll
  for (int v = 0; v < 2; ++v) xv[v] = (unsigned)((lane >> 3) * KD * 2 + (((lane & 7) ^ ((v * 4 + (lane >> 4)) & 7)) * 16));
  const unsigned iv = (unsigned)(lane * 16);
  const unsigned wbase = (unsigned)((tl.owner * NCOL + tl.col0) * 2);
  const unsigned xbase = tl.kind == 0 ? 0u : (unsigned)((tl.nrow0 - XROWS) * KD * 2);
  auto xpc = [&](int pc) -> unsigned {
    const bool pad = NIMG == 2 && NWS == 0 && tl.nrow0 + pc * 8 >= NROWS;
    return (unsigned)((pad ? pc - 8 : pc) * (8 * KD * 2));
  };
  auto kmap = [&](int t) -> unsigned {
    return PAIR ? (unsigned)(((((t >> 1) * KSTRIDE) << 1) + (t & 1) + tl.koff) & (NT - 1)) : (unsigned)t;
  };
  int ws_ = 0, xs_ = 0;
  auto issue_w = [&](int t) {
    const unsigned tb = wbase + kmap(t) * (BK * ND * 2);
#pragma unroll
    for (int i = 0; i < NW_; ++i) {
      const int pc = H + 3 * i, img = pc >> 4, q = pc & 15;
      __builtin_amdgcn_raw_ptr_buffer_load_lds(rw, (LAS void*)(lds + ws_ * WSLOT + img * WIMG + wpiece(q)), 16, wv, tb + q * (4 * ND * 2) + img * 256, 0, AUXW);
    }
    ws_ = ws_ == SW - 1 ? 0 : ws_ + 1;
  };
  auto issue_xl = [&](int t) {
    const unsigned tb = xbase + kmap(t) * 128u;
#pragma unroll
    for (int i = 0; i < NX_; ++i) {
      const int pc = X0 + 3 * i;
      if (pc >= NWS) __builtin_amdgcn_raw_ptr_buffer_load_lds(rx, (LAS void*)(lds + LX + xs_ * XSLOT + pc * 1024), 16, xv[pc & 1], tb + xpc(pc), 0, 0);
    }
  };
  issue_w(0);
  issue_xl(0);
  unsigned call = 0;
  if (XWS) {
    BAR();
    call = (unsigned)__builtin_amdgcn_readfirstlane((int)lds_rd(lds, C_CALL));
  }
  const auto ri = __builtin_amdgcn_make_buffer_rsrc((char*)p.ws[p.rank] + WS_XBUF + (call & 1) * XPAR_BYTES, 0, (int)XPAR_BYTES, 0x00020000);
  const unsigned ibase = (unsigned)(tl.nrow0 * 128);
  auto issue_xi = [&](int t) {
#pragma unroll
    for (int i = 0; i < NX_; ++i) {
      const int pc = X0 + 3 * i;
      if (pc < NWS) __builtin_amdgcn_raw_ptr_buffer_load_lds(ri, (LAS void*)(lds + LX + xs_ * XSLOT + pc * 1024), 16, iv, ibase + (unsigned)t * (unsigned)XT_BYTES + pc * 1024, 0, 0);
    }
    xs_ = xs_ == 2 ? 0 : xs_ + 1;
  };
  issue_xi(0);
  issue_w(1);
  issue_xl(1);
  issue_xi(1);
  waitv<NW_ + NX_>();
  BAR();
  if (PAIR) {
    for (int t = 0; t < NT - 2; t += 2) {
      issue_w(t + 2);
      issue_w(t + 3);
      issue_xl(t + 2);
      issue_xi(t + 2);
      waitv<2 * NW_ + NX_>();
      BAR();
      issue_xl(t + 3);
      issue_xi(t + 3);
      waitv<NX_>();
      BAR();
    }
  } else if (NWS == 0 || NWS == 4 * MF) {
    constexpr int QW = (NW_ + 3) / 4, QX = (NX_ + 3) / 4;
    unsigned wl[NW_], wg[NW_], xl[NX_], xg[NX_];
#pragma unroll
    for (int i = 0; i < NW_; ++i) {
      int k = i + tl.qw * QW; k = k >= NW_ ? k - NW_ : k;
      const int pc = H + 3 * k, img = pc >> 4, q = pc & 15;
      wl[i] = (unsigned)(img * WIMG + wpiece(q));
      wg[i] = wv + (unsigned)(q * (4 * ND * 2) + img * 256);
    }
#pragma unroll
    for (int i = 0; i < NX_; ++i) {
      int k = i + tl.qx * QX; k = k >= NX_ ? k - NX_ : k;
      const int pc = X0 + 3 * k;
      xl[i] = (unsigned)(pc * 1024);
      xg[i] = XWS ? iv + (unsigned)(pc * 1024) : ((pc & 1) ? xv[1] : xv[0]) + xpc(pc);
    }
    for (int t = 0; t < NT - 2; ++t) {
      const unsigned tw = wbase + (unsigned)(t + 2) * (BK * ND * 2);
      const unsigned tx = XWS ? ibase + (unsigned)(t + 2) * (unsigned)XT_BYTES : xbase + (unsigned)(t + 2) * 128u;
      LAS char* lw = lds + ws_ * WSLOT;
      LAS char* lx = lds + LX + xs_ * XSLOT;
#pragma unroll
      for (int ph = 0; ph < 2; ++ph) {
#pragma unroll
        for (int i = ph ? QW : 0; i < (ph ? NW_ : QW); ++i)
          __builtin_amdgcn_raw_ptr_buffer_load_lds(rw, (LAS void*)(lw + wl[i]), 16, wg[i], tw, 0, AUXW);
#pragma unroll
        for (int i = ph ? QX : 0; i < (ph ? NX_ : QX); ++i)
          __builtin_amdgcn_raw_ptr_buffer_load_lds(XWS ? ri : rx, (LAS void*)(lx + xl[i]), 16, xg[i], tx, 0, 0);
      }
      ws_ = ws_ == SW - 1 ? 0 : ws_ + 1;
      xs_ = xs_ == 2 ? 0 : xs_ + 1;
      waitv<NW_ + NX_>();
      BAR();
    }
  } else {
    for (int t = 0; t < NT - 2; ++t) {
      issue_w(t + 2);
      issue_xl(t + 2);
      issue_xi(t + 2);
      waitv<NW_ + NX_>();
      BAR();
    }
  }
  waitv<0>();
  BAR();
  BAR();
}

__device__ __forceinline__ void mfma_a(const Params& p, const Tile& tl, LAS char* lds, int wid, int lane) {
  constexpr int MF = MFA;
  static_assert(MF == 7, "the schedule below is written for 7 fragments");
  constexpr int XSLOT = MF * 4096, LX = SWA * WIMG;
  const int wr = wid >> 1, wc = wid & 1;
  const int fr = lane & 15, fq = lane >> 4;
  const int q = (lane & 15) >> 2, pp = lane & 3;
  const unsigned lb = (unsigned)(size_t)lds;
  const unsigned xo = (unsigned)(LX + (wr * MF * 16 + fr) * 128 + ((fq ^ ((fr >> 1) & 7)) * 16));
  unsigned xa0 = lb + xo, xa1 = lb + (xo ^ 64u);
  unsigned wa[4];
#pragma unroll
  for (int nf = 0; nf < 4; ++nf)
    wa[nf] = lb + (unsigned)((fq & 1) * WODD + (fq >> 1) * 2048 + q * 256 + wc * 128 + ((nf ^ q) * 32) + 8 * pp);
  f32x4 acc[MF][4];
#pragma unroll
  for (int i = 0; i < MF; ++i)
#pragma unroll
    for (int j = 0; j < 4; ++j) { acc[i][j] = (f32x4){0.f, 0.f, 0.f, 0.f}; asm volatile("" : "+v"(acc[i][j])); }
  bf16x8 Xf[MF];
  s16x4 Wl[2][4], Wh[2][4];
#define RDW(bu, ks, i) do { if ((i) & 1) Wh[bu][(i) >> 1] = ldw(wa[(i) >> 1] + (ks) * 4096 + 1024); \
    else Wl[bu][(i) >> 1] = ldw(wa[(i) >> 1] + (ks) * 4096); } while (0)
#define RDX(i, xa) Xf[i] = ldx((xa) + (i) * 2048)
#define MMA1(mf, bu, nf) do { mma(acc[mf][nf], wfrag(Wl[bu][nf], Wh[bu][nf]), Xf[mf]); SCHED(); } while (0)
#define SN(mf, i) (((mf) & 1) ? 3 - (i) : (i))
#define GRP(mf, bu, F0, F1, F2) do { MMA1(mf, bu, SN(mf, 0)); F0; SCHED(); MMA1(mf, bu, SN(mf, 1)); F1; SCHED(); \
    MMA1(mf, bu, SN(mf, 2)); F2; SCHED(); MMA1(mf, bu, SN(mf, 3)); } while (0)
  BAR();
#pragma unroll
  for (int i = 0; i < 8; ++i) RDW(0, 0, i);
#pragma unroll
  for (int mf = 0; mf < MF - 1; ++mf) RDX(mf, xa0);
  __builtin_amdgcn_s_waitcnt(0xC07F);
  int ow = 0, ox = 0, dw = WIMG, dx = XSLOT;
  for (int t = 0; t < NT; ++t) {
    GRP(0, 0, RDX(6, xa0), RDW(1, 1, 0), RDW(1, 1, 1));
    GRP(1, 0, RDX(0, xa1), RDW(1, 1, 2), RDW(1, 1, 3));
    GRP(2, 0, RDX(1, xa1), RDW(1, 1, 4), STEP(xa0, dx));
    GRP(3, 0, RDX(2, xa1), RDW(1, 1, 5), STEP(wa[0], dw));
    GRP(4, 0, RDX(3, xa1), RDW(1, 1, 6), STEP(wa[1], dw));
    GRP(5, 0, RDX(4, xa1), RDW(1, 1, 7), STEP(wa[2], dw));
    GRP(6, 0, RDX(5, xa1), NOP_, STEP(wa[3], dw));
    GRP(0, 1, RDX(6, xa1), NEXT1(ow, dw), NEXT2(ow, dw, (SWA - 1) * WIMG, WIMG));
    GRP(1, 1, STEP(xa1, dx), NEXT1(ox, dx), NEXT2(ox, dx, 2 * XSLOT, XSLOT));
    __builtin_amdgcn_s_waitcnt(0xC07F);
    CFENCE();
    BAR();
    CFENCE();
    SCHED();
    GRP(2, 1, RDW(0, 0, 0), RDW(0, 0, 1), RDX(0, xa0));
    GRP(3, 1, RDW(0, 0, 2), RDW(0, 0, 3), RDX(1, xa0));
    GRP(4, 1, RDW(0, 0, 4), RDW(0, 0, 5), RDX(2, xa0));
    GRP(5, 1, RDW(0, 0, 6), RDW(0, 0, 7), RDX(3, xa0));
    GRP(6, 1, RDX(4, xa0), RDX(5, xa0), NOP_);
  }
  const unsigned go = (unsigned)__builtin_amdgcn_readfirstlane((int)lds_rd(lds, C_GO));
  const bool bad = (go & 1u) != 0u;
  const auto ro = __builtin_amdgcn_make_buffer_rsrc((void*)p.out[tl.owner], 0, ML * NR * NCOL * 2, 0x00020000);
  const unsigned vo = out_voff(lane);
  const unsigned so = (unsigned)(((p.rank * ML + wr * MF * 16) * NCOL + tl.col0 + wc * 64) * 2);
#pragma unroll
  for (int mf = 0; mf < MF; ++mf) {
#pragma unroll
    for (int np = 0; np < 2; ++np)
      __builtin_amdgcn_raw_buffer_store_b128(pack2(acc[mf][2 * np], acc[mf][2 * np + 1], bad), ro, vo + np * 64, so + mf * (16 * NCOL * 2), AUX_WT_SYS);
    waitv<0>();
  }
  if (lane == 0) st_sys(p.ws[tl.owner] + WS_DONE + p.rank * 64 + (tl.col0 >> 7) * 4 + wid, go >> 1);
#undef RDW
#undef RDX
#undef MMA1
#undef GRP
}

__device__ __forceinline__ void mfma_n(const Params& p, const Tile& tl, LAS char* lds, int wid, int lane) {
  constexpr int MF = MFN;
  static_assert(MF == 5, "the schedule below is written for 5 fragments");
  constexpr int WSLOT = 2 * WIMG, XSLOT = MF * 4096, LX = 3 * WSLOT;
  const int wr = wid >> 1, wc = wid & 1;
  const int fr = lane & 15, fq = lane >> 4;
  const int q = (lane & 15) >> 2, pp = lane & 3;
  const unsigned lb = (unsigned)(size_t)lds;
  const unsigned xo = (unsigned)(LX + (wr * MF * 16 + fr) * 128 + ((fq ^ ((fr >> 1) & 7)) * 16));
  unsigned xa0 = lb + xo, xa1 = lb + (xo ^ 64u);
  unsigned wa[4];
#pragma unroll
  for (int nf = 0; nf < 4; ++nf)
    wa[nf] = lb + (unsigned)(wc * WIMG + (fq & 1) * WODD + (fq >> 1) * 2048 + q * 256 + ((nf ^ q) * 32) + 8 * pp);
  f32x4 acc[MF][8];
#pragma unroll
  for (int i = 0; i < MF; ++i)
#pragma unroll
    for (int j = 0; j < 8; ++j) { acc[i][j] = (f32x4){0.f, 0.f, 0.f, 0.f}; asm volatile("" : "+v"(acc[i][j])); }
  bf16x8 Xf[2][MF];
  s16x4 Wl[8], Wh[8];
#define RWL(nf, ks) Wl[nf] = ldw(wa[(nf) & 3] + ((nf) >> 2) * 128 + (ks) * 4096)
#define RWH(nf, ks) Wh[nf] = ldw(wa[(nf) & 3] + ((nf) >> 2) * 128 + (ks) * 4096 + 1024)
#define RDX(bu, i, xa) Xf[bu][i] = ldx((xa) + (i) * 2048)
#define MMA1(nf, bu, mf) do { mma(acc[mf][nf], wfrag(Wl[nf], Wh[nf]), Xf[bu][mf]); SCHED(); } while (0)
#define SN5(nf, i) (((nf) & 1) ? 4 - (i) : (i))
#define GRP(nf, bu, F0, F1, F2, F3, F4) do { MMA1(nf, bu, SN5(nf, 0)); F0; SCHED(); MMA1(nf, bu, SN5(nf, 1)); F1; SCHED(); \
    MMA1(nf, bu, SN5(nf, 2)); F2; SCHED(); MMA1(nf, bu, SN5(nf, 3)); F3; SCHED(); MMA1(nf, bu, SN5(nf, 4)); F4; SCHED(); } while (0)
  if (tl.gathered) {
    const unsigned gwv = (unsigned)(((tl.nrow0 / NM) * 8 + (tl.col0 >> 8)) * 4 + wid);
    LAS void* dummy = (LAS void*)(lds + 2 * WSLOT + wid * 1024);
    const auto rx = __builtin_amdgcn_make_buffer_rsrc((void*)p.x, 0, ML * KD * 2, 0x00020000);
    const auto rw = __builtin_amdgcn_make_buffer_rsrc((void*)p.w, 0, KD * ND * 2, 0x00020000);
#pragma unroll
    for (int i = 0; i < 10; ++i)
      __builtin_amdgcn_raw_ptr_buffer_load_lds(rx, dummy, 16, (unsigned)(lane * 16), (gwv * 10240u + i * 1024u) & (unsigned)(ML * KD * 2 - 1), 0, 0);
    for (unsigned i0 = gwv; i0 < KD * 4u; i0 += 416u * 8u) {
#pragma unroll
      for (int i = 0; i < 8; ++i) {
        const unsigned pi = (i0 + i * 416u) & (KD * 4u - 1u);
        __builtin_amdgcn_raw_ptr_buffer_load_lds(rw, dummy, 16, (unsigned)(lane * 16), (pi >> 2) * (ND * 2) + (unsigned)p.rank * (NCOL * 2) + (pi & 3u) * 1024u, 0, 0);
      }
      waitv<8>();
    }
    waitv<0>();
    BAR();
  }
  BAR();
#pragma unroll
  for (int mf = 0; mf < MF; ++mf) RDX(0, mf, xa0);
#pragma unroll
  for (int nf = 0; nf < 7; ++nf) { RWL(nf, 0); RWH(nf, 0); }
  __builtin_amdgcn_s_waitcnt(0xC07F);
  int ow = 0, ox = 0, dw = WSLOT, dx = XSLOT;
  for (int t = 0; t < NT; ++t) {
    GRP(0, 0, RWL(7, 0), RWH(7, 0), RDX(1, 0, xa1), STEP(xa0, dx), NOP_);
    GRP(1, 0, RWL(0, 1), RWH(0, 1), RDX(1, 1, xa1), NOP_, NOP_);
    GRP(2, 0, RWL(1, 1), RWH(1, 1), RDX(1, 2, xa1), NOP_, NOP_);
    GRP(3, 0, RWL(2, 1), RWH(2, 1), RDX(1, 3, xa1), NOP_, NOP_);
    GRP(4, 0, RWL(3, 1), RWH(3, 1), RDX(1, 4, xa1), NOP_, NOP_);
    GRP(5, 0, RWL(4, 1), RWH(4, 1), NOP_, STEP(wa[0], dw), STEP(xa1, dx));
    GRP(6, 0, RWL(5, 1), RWH(5, 1), NOP_, STEP(wa[1], dw), NOP_);
    GRP(7, 0, RWL(6, 1), RWH(6, 1), NOP_, STEP(wa[2], dw), NOP_);
    GRP(0, 1, RWL(7, 1), RWH(7, 1), NOP_, STEP(wa[3], dw), NOP_);
    GRP(1, 1, NEXT1(ow, dw), NEXT2(ow, dw, 2 * WSLOT, WSLOT), NEXT1(ox, dx), NEXT2(ox, dx, 2 * XSLOT, XSLOT), NOP_);
    __builtin_amdgcn_s_waitcnt(0xC07F);
    CFENCE();
    BAR();
    CFENCE();
    SCHED();
    GRP(2, 1, RWL(0, 0), RWH(0, 0), RDX(0, 0, xa0), RWL(1, 0), RWH(1, 0));
    GRP(3, 1, RWL(2, 0), RWH(2, 0), RDX(0, 1, xa0), RDX(0, 2, xa0), NOP_);
    GRP(4, 1, RWL(3, 0), RWH(3, 0), RDX(0, 3, xa0), RDX(0, 4, xa0), NOP_);
    GRP(5, 1, RWL(4, 0), RWH(4, 0), NOP_, NOP_, NOP_);
    GRP(6, 1, RWL(5, 0), RWH(5, 0), NOP_, NOP_, NOP_);
    GRP(7, 1, RWL(6, 0), RWH(6, 0), NOP_, NOP_, NOP_);
  }
  const unsigned go = (unsigned)__builtin_amdgcn_readfirstlane((int)lds_rd(lds, C_GO));
  const bool bad = (go & 1u) != 0u;
  const auto ro = __builtin_amdgcn_make_buffer_rsrc((void*)p.out[p.rank], 0, ML * NR * NCOL * 2, 0x00020000);
  const unsigned vo = out_voff(lane);
  const unsigned so = (unsigned)((tl.col0 + wc * 128) * 2);
#pragma unroll
  for (int mf = 0; mf < MF; ++mf) {
    const int nr = tl.nrow0 + wr * MF * 16 + mf * 16;
    if (nr < NROWS) {
      const int orow = nr < XROWS ? ((p.rank + 1 + nr / BROWS) & 7) * ML + AROWS + nr % BROWS : p.rank * ML + nr - XROWS;
#pragma unroll
      for (int np = 0; np < 4; ++np)
        __builtin_amdgcn_raw_buffer_store_b128(pack2(acc[mf][2 * np], acc[mf][2 * np + 1], bad), ro, vo + np * 64, so + (unsigned)orow * (NCOL * 2), AUX_WT_AGENT);
    }
  }
#undef RWL
#undef RWH
#undef RDX
#undef MMA1
#undef GRP
}

__device__ __forceinline__ bool wait_words(const unsigned* word, unsigned call, unsigned* tmo) {
  const unsigned long long t0 = __builtin_amdgcn_s_memrealtime();
  unsigned spins = 0;
  for (;;) {
    if (__all(ld_sys(word) >= call)) return true;
    if ((++spins & 63u) == 0u && (ld_agent(tmo) == call || __builtin_amdgcn_s_memrealtime() - t0 > DEADLINE)) { st_agent(tmo, call); return false; }
    __builtin_amdgcn_s_sleep(1);
  }
}

__device__ __forceinline__ void ctrl_wave(const Params& p, const Tile& tl, LAS char* lds, int lane) {
  unsigned* myws = p.ws[p.rank];
  unsigned* tmo = myws + WS_TMO;
  const int b = blockIdx.x;
  const unsigned long long tk0 = __builtin_amdgcn_s_memrealtime();
  const unsigned cnt = ld_agent(myws + WS_CNT + b);
  if (!tl.gathered) BAR();
  const unsigned call = (unsigned)__builtin_amdgcn_readfirstlane((int)cnt) + 1u;
  if (lane == 0) st_agent(myws + WS_CNT + b, call);
  bool ok = true;
  int nb = 1;
  auto join = [&]() {
    if (tl.kind == 1) {
      int due = NS_T0 + nb * NPACE;
      if (!tl.gathered) { const int d2 = nb > NC_N0 ? NC_T0 + (nb - NC_N0) * NC_SLOPE : 0; due = d2 < due ? d2 : due; }
      if (nb < NS_N0) due = 0;
      due = due < NPACE_END ? due : NPACE_END;
      while ((long long)(__builtin_amdgcn_s_memrealtime() - tk0) < (long long)due) __builtin_amdgcn_s_sleep(1);
    }
    BAR();
    ++nb;
  };
  if (tl.gathered) {
    if (lane == 0) lds_wr(lds, C_CALL, call);
    const int s0 = tl.nrow0 / BROWS;
    const int e1 = (tl.nrow0 + NM - 1) / BROWS;
    const int s1 = e1 < 7 ? e1 : 6;
    const unsigned* sflag = myws + WS_XFLAG + ((lane & 16) ? s1 : s0) * 32 + (lane & 15);
    auto decode = [&](unsigned v) -> int {
#pragma unroll
      for (int o = 1; o < 32; o <<= 1) { const unsigned u = (unsigned)__shfl_xor((int)v, o); v = u < v ? u : v; }
      const unsigned vm = (unsigned)__builtin_amdgcn_readfirstlane((int)v);
      const int n = vm > call * 128u ? (int)(vm - call * 128u) : 0;
      return n > NT ? NT : n;
    };
    __builtin_amdgcn_fence(__ATOMIC_ACQUIRE, "");
    int seen = 0;
    nb = -1;
    while (seen < NT) {
      const int need = nb + 3 < NT ? nb + 3 : NT;
      if (seen < need) {
        const unsigned long long t0 = __builtin_amdgcn_s_memrealtime();
        unsigned spins = 0;
        while ((seen = decode(ld_sys(sflag))) < need) {
          if ((++spins & 63u) == 0u && (ld_agent(tmo) == call || __builtin_amdgcn_s_memrealtime() - t0 > DEADLINE)) { st_agent(tmo, call); ok = false; seen = NT; break; }
          __builtin_amdgcn_s_sleep(1);
        }
        waitv<0>();
        WAITL0();
        join();
      } else if (seen < need + POLL_MARGIN) {
        __builtin_amdgcn_global_load_lds((const __attribute__((address_space(1))) unsigned*)sflag, (LAS unsigned*)(lds + C_POLL), 4, 0, AUX_WT_SYS);
        join();
        waitv<0>();
        const int n = decode(lds_rd(lds, C_POLL + lane * 4));
        seen = n > seen ? n : seen;
      } else {
        join();
      }
    }
  } else if (tl.kind == 0) {
    while (nb < NT / 2) join();
    ok = wait_words(myws + WS_READY + tl.owner * 32, call, tmo);
  }
  while (nb < NT - 1) join();
  if (ld_agent(tmo) == call) ok = false;
  if (lane == 0) lds_wr(lds, C_GO, call * 2u + (ok ? 0u : 1u));
  WAITL0();
  while (nb <= NT) join();
}

__device__ __forceinline__ void copy_group(const Params& p, LAS char* lds, int c, int wid, int lane) {
  unsigned* myws = p.ws[p.rank];
  unsigned* tmo = myws + WS_TMO;
  const int b = blockIdx.x;
  const unsigned cnt = ld_agent(myws + WS_CNT + b);
  const int gw = wid * 16 + c;
  const auto rx = __builtin_amdgcn_make_buffer_rsrc((void*)p.x, 0, ML * KD * 2, 0x00020000);
  const unsigned xv = (unsigned)((AROWS + (lane >> 3)) * KD * 2 + (((lane & 7) ^ (((gw & 1) * 4 + (lane >> 4)) & 7)) * 16));
  auto load = [&](int k) { const int u = k * 128 + gw; return __builtin_amdgcn_raw_buffer_load_b128(rx, xv, (unsigned)((u % BRP) * (8 * KD * 2) + (u / BRP) * 128), 0); };
  u32x4 d[NUNIT];
  d[0] = load(0);
  d[1] = load(1);
  char* dbase[NR - 1];
  unsigned dslot[NR - 1];
#pragma unroll
  for (int i = 0; i < NR - 1; ++i) {
    const int j = __builtin_amdgcn_readfirstlane((i + gw) % (NR - 1) + 1);
    dbase[i] = (char*)p.ws[__builtin_amdgcn_readfirstlane((p.rank + NR - j) & 7)];
    dslot[i] = (unsigned)((j - 1) * BROWS * 128);
  }
  if (lane == 0) { lds_wr(lds, wid * 4, (unsigned)(gw / BRP)); if (wid == 0) lds_wr(lds, 64, 0u); }
  WAITL0();
  const unsigned call = (unsigned)__builtin_amdgcn_readfirstlane((int)cnt) + 1u;
  BAR();
  const unsigned long long tstart = __builtin_amdgcn_s_memrealtime();
  if (wid == 0 && lane == 0) st_agent(myws + WS_CNT + b, call);
  if (gw == 127 && lane < NR && lane != p.rank) st_sys(p.ws[lane] + WS_READY + p.rank * 32, call);
  const size_t xo = WS_XBUF + (call & 1) * XPAR_BYTES;
  unsigned* pflag = p.ws[(p.rank + NR - 1 - (lane < 7 ? lane : 0)) & 7] + WS_XFLAG + (lane < 7 ? lane : 0) * 32 + c;
  auto finish = [&](int k) {
    const int nxt = k + 1 < NUNIT ? ((k + 1) * 128 + gw) / BRP : NT;
    if (lane == 0) lds_wr(lds, wid * 4, (unsigned)nxt);
    unsigned v = lds_rd(lds, (lane & 7) * 4);
#pragma unroll
    for (int o = 1; o < 8; o <<= 1) { const unsigned u = (unsigned)__shfl_xor((int)v, o); v = u < v ? u : v; }
    const int pre = __builtin_amdgcn_readfirstlane((int)v);
    const int lvl = pre < 4 ? pre : (pre < 12 ? (pre & ~1) : (pre & ~3));
    unsigned old = 0;
    if (lane == 0) old = __hip_atomic_fetch_max((LAS unsigned*)(lds + 64), (unsigned)lvl, __ATOMIC_RELAXED, __HIP_MEMORY_SCOPE_WORKGROUP);
    old = (unsigned)__builtin_amdgcn_readfirstlane((int)old);
    if ((unsigned)lvl > old && lane < 7)
      __hip_atomic_fetch_max(pflag, call * 128u + (unsigned)lvl, __ATOMIC_RELAXED, __HIP_MEMORY_SCOPE_SYSTEM);
  };
  bool pending = false;
  long long tissue = 0;
#pragma unroll
  for (int k = 0; k < NUNIT; ++k) {
    const int u = k * 128 + gw;
    for (;;) {
      const long long now = (long long)(__builtin_amdgcn_s_memrealtime() - tstart) * 10;
      if (now * BRP >= (long long)(u - PUSH_LEAD * BRP) * TILE_NS) break;
      if (pending && now - tissue > ACK_NS) { waitv<0>(); finish(k - 1); pending = false; }
      else __builtin_amdgcn_s_sleep(2);
    }
    const unsigned so = (unsigned)(u / BRP) * (unsigned)XT_BYTES + (unsigned)(u % BRP) * 1024u;
#pragma unroll
    for (int i = 0; i < NR - 1; ++i) {
      const auto rd = __builtin_amdgcn_make_buffer_rsrc(dbase[i] + xo, 0, (int)XPAR_BYTES, 0x00020000);
      __builtin_amdgcn_raw_buffer_store_b128(d[k], rd, (unsigned)(lane * 16), so + dslot[i], AUX_WT_SYS);
    }
    tissue = (long long)(__builtin_amdgcn_s_memrealtime() - tstart) * 10;
    if (pending) { waitv<NR - 1>(); finish(k - 1); }
    pending = true;
    if (k + 2 < NUNIT) d[k + 2] = load(k + 2);
  }
  waitv<0>();
  finish(NUNIT - 1);
  if (c == 15 && wid != p.rank) {
    const bool fine = wait_words(myws + WS_DONE + wid * 64 + lane, call, tmo);
    if (!fine && lane == 0) p.out[p.rank][(size_t)(p.rank * ML) * NCOL] = 0x7fc0;
  }
}

__global__ __launch_bounds__(512, 2) void gemm_a2a(Params p) {
  extern __shared__ __attribute__((aligned(16))) char smem[];
  LAS char* lds = (LAS char*)smem;
  const int wid = __builtin_amdgcn_readfirstlane(threadIdx.x >> 6);
  const int lane = threadIdx.x & 63;
  const int b = blockIdx.x;
  const int xcd = b & 7, idx = b >> 3;
  if (idx < 2) { copy_group(p, lds, xcd * 2 + idx, wid, lane); return; }
  Tile tl;
  int rt = 0;
  if (idx < 16) {
    const int a = xcd * 14 + idx - 2;
    tl.kind = 0; tl.owner = (p.rank + a % 7 + 1) & 7; tl.col0 = (a / 7) * 128;
    tl.nrow0 = 0; tl.gathered = false; tl.koff = KSKEW * xcd; tl.qw = tl.qx = 0;
  } else {
    const int e = idx - 16;
    rt = (xcd >> 1) * 4 + (e >> 2);
    tl.kind = 1; tl.owner = p.rank; tl.col0 = ((xcd & 1) * 4 + (e & 3)) * 256;
    tl.nrow0 = rt * NM; tl.gathered = rt <= RMIX; tl.koff = 0; tl.qw = e >> 2; tl.qx = e & 3;
  }
  if (wid < 4) {
    if (tl.kind == 0) mfma_a(p, tl, lds, wid, lane);
    else mfma_n(p, tl, lds, wid, lane);
  } else if (wid == 7) {
    ctrl_wave(p, tl, lds, lane);
  } else {
#define LOADERS(MF, NIMG, SW, PAIR, NWS, AUX) do { if (wid == 4) loader<MF, NIMG, SW, PAIR, 0, NWS, AUX>(p, tl, lds, lane); \
    else if (wid == 5) loader<MF, NIMG, SW, PAIR, 1, NWS, AUX>(p, tl, lds, lane); else loader<MF, NIMG, SW, PAIR, 2, NWS, AUX>(p, tl, lds, lane); } while (0)
    if (tl.kind == 0) LOADERS(MFA, 1, SWA, true, 0, AUX_NT);
    else if (rt < RMIX) LOADERS(MFN, 2, 3, false, 4 * MFN, 0);
    else if (rt == RMIX) LOADERS(MFN, 2, 3, false, (XROWS - RMIX * NM) / 8, 0);
    else LOADERS(MFN, 2, 3, false, 0, 0);
  }
}

extern "C" void kernel_launch(int rank, int nranks, ncclComm_t comm,
                              hipStream_t stream,
                              void* const* d_in, const int* in_sizes, int n_in,
                              void* d_out, int out_size,
                              const int* in_dtypes, int out_dtype,
                              void* const* d_out_all,
                              void* const* d_ws_all, size_t ws_bytes) {
  if (nranks != NR || (unsigned)rank >= (unsigned)NR || ws_bytes < WS_NEED || n_in < 2 ||
      in_sizes[0] != ML * KD || in_sizes[1] != KD * ND || out_size != NR * ML * NCOL) {
    fprintf(stderr, "unexpected configuration\n");
    return;
  }
  Params p;
  p.x = (const unsigned short*)d_in[0];
  p.w = (const unsigned short*)d_in[1];
  for (int i = 0; i < NR; ++i) { p.out[i] = (unsigned short*)d_out_all[i]; p.ws[i] = (unsigned*)d_ws_all[i]; }
  p.rank = rank;
  static bool init[NR] = {};
  if (!init[rank]) {
    HIP_CHECK(hipFuncSetAttribute((const void*)gemm_a2a, hipFuncAttributeMaxDynamicSharedMemorySize, LDS_BYTES));
    init[rank] = true;
  }
  gemm_a2a<<<NBLK, 512, LDS_BYTES, stream>>>(p);
  HIP_CHECK(hipGetLastError());
}
```
